```python
import numpy as np

kernel_name = "gpu_gemm_ar_m4096_k4096_n16384_bf16_q8"


ABSMAX_RTOL = 0.02

COMPUTE_DTYPE = "bfloat16"
EPILOGUE = "quant_int8"
IN_QUANT = None
OUT_QUANT = "int8"
IN_DTYPES = {"x": "bfloat16", "w_mat": "bfloat16"}
WORLD_SIZE = 8

def _to_bf16(x):
    u = np.ascontiguousarray(x, dtype=np.float32).view(np.uint32)
    return ((u + 0x7FFF + ((u >> 16) & 1)) & 0xFFFF0000).view(np.float32)

def setup_inputs():

    x = _to_bf16(np.random.randn(4096, 4096).astype(np.float32))
    w_mat = _to_bf16((np.random.randn(4096, 16384) / np.sqrt(4096)).astype(np.float32))
    return {"x": x, "w_mat": w_mat}

def _quant_dequant(y, amax):
    scale = amax / 127.0
    q = np.clip(np.round(y / scale), -127.0, 127.0)
    return (q * scale).astype(np.float32)
def _epilogue(y):
    return _quant_dequant(y, np.abs(y).max())

def reference(x, w_mat):
    y = (x @ w_mat).astype(np.float32)
    return _epilogue(y)
```

```cpp
#include <hip/hip_runtime.h>
#include <rccl/rccl.h>
#include <cstdio>
#include <cstdint>

#define HIP_CHECK(expr) do { hipError_t _e = (expr); if (_e != hipSuccess) { \
    fprintf(stderr, "HIP error %s at %s:%d\n", hipGetErrorString(_e), __FILE__, __LINE__); \
    return; } } while (0)

#define LAS __attribute__((address_space(3)))
typedef unsigned int u32;
typedef unsigned long long u64;
typedef u32 u32x4 __attribute__((ext_vector_type(4)));
typedef u32 u32x2 __attribute__((ext_vector_type(2)));
typedef short bf16x8 __attribute__((ext_vector_type(8)));
typedef float f32x4 __attribute__((ext_vector_type(4)));

#define NCOMM1 24
#define NCOMM2 24
#define DMA_AUX 0

constexpr int NR = 8;
constexpr u32 MM = 4096, NN = 16384, KSH = 512;
constexpr u32 X_BYTES = MM * KSH * 2, W_BYTES = KSH * NN * 2, OUT_BYTES = MM * NN;
constexpr u32 XP = 32768, WP = 16384;
constexpr u32 STEP_BYTES = 16 * XP + 16 * WP;
constexpr u32 STAGE_OFF = 1u << 20;
constexpr u32 WS_NEED = STAGE_OFF + 64 * STEP_BYTES;
constexpr u32 OFF_SEQ = 0, OFF_TMO = 256;
constexpr u32 OFF_XF = 4096;
constexpr u32 OFF_WF = OFF_XF + 16384;
constexpr u32 OFF_AMX = OFF_WF + 8192;
constexpr u32 OFF_GAM = OFF_AMX + 2048;
constexpr u32 OFF_TF = OFF_GAM + 1024;
constexpr u32 OFF_DONE = OFF_TF + 4096;
constexpr u32 OFF_STAMP = 65536;
static_assert(OFF_DONE + 1024 <= OFF_STAMP && OFF_STAMP + 256 * 8 * 16 * 8 <= STAGE_OFF, "flag region");

constexpr u32 SLOT_BYTES = XP + WP;
constexpr u32 CTRL = 3 * SLOT_BYTES;
constexpr u32 L_FULL = CTRL, L_FREE = CTRL + 16, L_NCL = CTRL + 32, L_AVAIL = CTRL + 48,
              L_AMAX = CTRL + 64, L_ACNT = CTRL + 68, L_BAD = CTRL + 72, L_SEQ = CTRL + 76,
              L_SCALE = CTRL + 80;
constexpr u32 LDS_BYTES = CTRL + 128;
constexpr u64 DEADLINE = 100000000ull;
constexpr int SYS = 17, AGT = 16;

struct Params {
    const char* x;
    const char* w;
    char* out[NR];
    char* ws[NR];
    int rank;
};

typedef __amdgpu_buffer_rsrc_t rsrc_t;
__device__ __forceinline__ rsrc_t mk_rsrc(const void* p, u32 bytes) {
    return __builtin_amdgcn_make_buffer_rsrc((void*)p, 0, bytes, 0x00020000);
}
__device__ __forceinline__ u64 now() { return __builtin_amdgcn_s_memrealtime(); }
#define VMWAIT(n) asm volatile("s_waitcnt vmcnt(" #n ")" ::: "memory")
#define CBAR() asm volatile("" ::: "memory")
#define ST128(v, rs, voff, soff, aux) do { const u32x4 d_ = (v); \
    __builtin_amdgcn_raw_buffer_store_b128(d_, rs, voff, soff, aux); \
    asm volatile("s_nop 1" :: "v"(d_) : "memory"); } while (0)
#define MFMA(a, b, c) __builtin_amdgcn_mfma_f32_16x16x32_bf16(a, b, c, 0, 0, 0)

#define STAMP(i) do { } while (0)

__device__ __forceinline__ u32 wave_max_u32(u32 v) {
    v = max(v, (u32)__builtin_amdgcn_update_dpp(0, (int)v, 0x111, 0xf, 0xf, true));
    v = max(v, (u32)__builtin_amdgcn_update_dpp(0, (int)v, 0x112, 0xf, 0xf, true));
    v = max(v, (u32)__builtin_amdgcn_update_dpp(0, (int)v, 0x114, 0xf, 0xf, true));
    v = max(v, (u32)__builtin_amdgcn_update_dpp(0, (int)v, 0x118, 0xf, 0xf, true));
    v = max(v, (u32)__builtin_amdgcn_update_dpp(0, (int)v, 0x142, 0xa, 0xf, false));
    v = max(v, (u32)__builtin_amdgcn_update_dpp(0, (int)v, 0x143, 0xc, 0xf, false));
    return (u32)__builtin_amdgcn_readlane((int)v, 63);
}

__global__ void __launch_bounds__(512, 2) gemm_ar_q8(Params p) {
    extern __shared__ __attribute__((aligned(16))) char smem[];
    LAS char* L = (LAS char*)smem;
    const u32 lane = threadIdx.x & 63;
    const u32 wid = __builtin_amdgcn_readfirstlane(threadIdx.x >> 6);
    const u32 rank = (u32)p.rank & 7u;
    const u32 q = blockIdx.x & 7u, idx = blockIdx.x >> 3;
    const u32 mb = 4 * (q >> 1) + (idx & 3), nb = 8 * (q & 1) + (idx >> 2);
    const u32 dst = (rank + q) & 7u;
    char* wsl = p.ws[rank];
    const rsrc_t rs_ws = mk_rsrc(wsl, WS_NEED);
    volatile LAS u32* LW = (volatile LAS u32*)L;

    STAMP(0);
    if (threadIdx.x < 32) {
        u32 v = 0;
        if (threadIdx.x == (L_SEQ - CTRL) / 4) v = __builtin_amdgcn_raw_buffer_load_b32(rs_ws, OFF_SEQ, 0, AGT) + 1u;
        LW[CTRL / 4 + threadIdx.x] = v;
    }
    if (wid == 4) { __builtin_amdgcn_fence(__ATOMIC_ACQUIRE, ""); VMWAIT(0); }
    __syncthreads();
    const u32 seq = __builtin_amdgcn_readfirstlane(LW[L_SEQ / 4]);

    u32 spins = 0;
    u64 t0 = 0;
    auto expired = [&]() -> bool {
        if ((++spins & 63u) != 0u) return false;
        if (LW[L_BAD / 4] != 0u) return true;
        const u64 t = now();
        if (t0 == 0) t0 = t;
        if (t - t0 > DEADLINE || __builtin_amdgcn_raw_buffer_load_b32(rs_ws, OFF_TMO, 0, AGT) == seq) {
            __builtin_amdgcn_raw_buffer_store_b32(seq, rs_ws, OFF_TMO, 0, AGT);
            LW[L_BAD / 4] = 1u;
            return true;
        }
        return false;
    };

    if (wid < 4) {
        const u32 c = lane & 15u, g = lane >> 4;
        const u32 xa0 = (64u * wid + c) * 128u + ((g ^ (c >> 1)) << 4);
        const u32 wb0 = XP + (c & 1u) * 4096u + g * 256u + ((8u * (c & 1u) + (c >> 1)) << 4);
        f32x4 acc[8][4];
#pragma unroll
        for (int j = 0; j < 8; ++j)
#pragma unroll
            for (int m = 0; m < 4; ++m) acc[j][m] = f32x4{0.f, 0.f, 0.f, 0.f};
        bf16x8 wf0[8], xf0[4], wf1[8], xf1[4];
#define LDFR(WF, XF, SB, KH) do { \
            const LAS char* xb_ = L + (SB) + ((KH) ? (xa0 ^ 64u) : xa0); \
            const LAS char* wl_ = L + (SB) + wb0 + (KH) * 8192u; \
            const LAS char* wh_ = L + (SB) + (wb0 ^ 128u) + (KH) * 8192u; \
            _Pragma("unroll") for (int m = 0; m < 4; ++m) XF[m] = *(const LAS bf16x8*)(xb_ + m * 2048); \
            _Pragma("unroll") for (int j = 0; j < 4; ++j) WF[j] = *(const LAS bf16x8*)(wl_ + j * 1024); \
            _Pragma("unroll") for (int j = 0; j < 4; ++j) WF[4 + j] = *(const LAS bf16x8*)(wh_ + j * 1024); \
        } while (0)
#define MMAS(WF, XF) do { \
            _Pragma("unroll") for (int j = 0; j < 8; ++j) \
            _Pragma("unroll") for (int m = 0; m < 4; ++m) acc[j][m] = MFMA(WF[j], XF[m], acc[j][m]); \
        } while (0)
#define INTERLEAVE() do { \
            _Pragma("unroll") for (int z = 0; z < 12; ++z) { \
                __builtin_amdgcn_sched_group_barrier(0x100, 1, 0); \
                __builtin_amdgcn_sched_group_barrier(0x008, 2, 0); } \
            __builtin_amdgcn_sched_group_barrier(0x008, 8, 0); \
        } while (0)

        while (LW[L_FULL / 4] < 1u) { if (expired()) break; }
        CBAR();
        STAMP(1);
        LDFR(wf0, xf0, 0u, 0);
        u32 slot = 0, fill = 1;
        for (u32 i = 0; i < 64; ++i) {
            const u32 sb = slot * SLOT_BYTES;
            u32 nslot = slot + 1, nfill = fill;
            if (nslot == 3) { nslot = 0; nfill = fill + 1; }
            __builtin_amdgcn_sched_barrier(0);
            LDFR(wf1, xf1, sb, 1);
            MMAS(wf0, xf0);
            INTERLEAVE();
            __builtin_amdgcn_sched_barrier(0);
            CBAR();
            LW[L_FREE / 4 + wid] = i + 1;
            if (i < 63) {
                while (LW[L_FULL / 4 + nslot] < nfill) { if (expired()) break; }
            }
            CBAR();
            __builtin_amdgcn_sched_barrier(0);
            LDFR(wf0, xf0, nslot * SLOT_BYTES, 0);
            MMAS(wf1, xf1);
            INTERLEAVE();
            __builtin_amdgcn_sched_barrier(0);
            slot = nslot; fill = nfill;
        }
        STAMP(2);
        float mx = 0.f;
#pragma unroll
        for (int j = 0; j < 8; ++j)
#pragma unroll
            for (int m = 0; m < 4; ++m) {
                mx = fmaxf(fmaxf(mx, fabsf(acc[j][m][0])), fabsf(acc[j][m][1]));
                mx = fmaxf(fmaxf(mx, fabsf(acc[j][m][2])), fabsf(acc[j][m][3]));
            }
        const u32 wm = wave_max_u32(__float_as_uint(mx));
        if (lane == 0) {
            __hip_atomic_fetch_max((LAS u32*)(L + L_AMAX), wm, __ATOMIC_RELAXED, __HIP_MEMORY_SCOPE_WORKGROUP);
            const u32 old = __hip_atomic_fetch_add((LAS u32*)(L + L_ACNT), 1u, __ATOMIC_RELAXED, __HIP_MEMORY_SCOPE_WORKGROUP);
            if (old == 3u) {
                u32 a = LW[L_AMAX / 4];
                if (LW[L_BAD / 4] != 0u) a = 0xFFFFFFFFu;
                __builtin_amdgcn_raw_buffer_store_b64(u32x2{a, seq}, rs_ws, OFF_AMX + blockIdx.x * 8u, 0, AGT);
            }
        }
        STAMP(3);
        while (LW[L_SCALE / 4] != 1u) { if (expired()) break; }
        CBAR();
        const u32 gmax = LW[L_SCALE / 4 + 1];
        const bool bad = (gmax == 0xFFFFFFFFu) || (LW[L_BAD / 4] != 0u);
        const float amax = __uint_as_float(gmax);
        const float inv = amax > 0.f ? 127.0f / amax : 0.f;
        STAMP(4);
        const float MAGIC = 12582912.0f;
        u32x4 qv[8];
#pragma unroll
        for (int m = 0; m < 4; ++m)
#pragma unroll
            for (int h = 0; h < 2; ++h) {
                u32x4 o;
#pragma unroll
                for (int r = 0; r < 4; ++r) {
                    u32 b[4];
#pragma unroll
                    for (int jj = 0; jj < 4; ++jj)
                        b[jj] = __float_as_uint(__builtin_amdgcn_fmed3f(__builtin_fmaf(acc[4 * h + jj][m][r], inv, MAGIC),
                                                                        MAGIC - 127.0f, MAGIC + 127.0f));
                    const u32 lo = __builtin_amdgcn_perm(b[1], b[0], 0x0c0c0400u);
                    const u32 hi = __builtin_amdgcn_perm(b[3], b[2], 0x04000c0cu);
                    o[r] = lo | hi;
                }
                if (bad) o = u32x4{0x80808080u, 0x80808080u, 0x80808080u, 0x80808080u};
                qv[2 * m + h] = o;
            }
        const u32 ovoff = (64u * wid + c) * NN + g * 16u;
        const u32 osoff = mb * 256u * NN + rank * 2048u + nb * 128u;
        const rsrc_t rs_ol = mk_rsrc(p.out[rank], OUT_BYTES);
#pragma unroll
        for (int m = 0; m < 4; ++m)
#pragma unroll
            for (int h = 0; h < 2; ++h)
                ST128(qv[2 * m + h], rs_ol, ovoff + h * 64u, osoff + m * 16u * NN, AGT);
        if (q != 0u) {
            const rsrc_t rs_or = mk_rsrc(p.out[dst], OUT_BYTES);
            ST128(qv[0], rs_or, ovoff, osoff, SYS);
            VMWAIT(1);
            if (lane == 0) __builtin_amdgcn_raw_buffer_store_b32(seq, rs_ws, OFF_TF + (blockIdx.x * 4u + wid) * 4u, 0, AGT);
#pragma unroll
            for (int v = 1; v < 8; ++v) {
                VMWAIT(1);
                ST128(qv[v], rs_or, ovoff + (v & 1) * 64u, osoff + (v >> 1) * 16u * NN, SYS);
            }
        } else {
            VMWAIT(0);
            if (lane == 0) __builtin_amdgcn_raw_buffer_store_b32(seq, rs_ws, OFF_TF + (blockIdx.x * 4u + wid) * 4u, 0, AGT);
        }
        STAMP(5);
    } else if (wid < 7) {
        const u32 j = wid - 4;
        const char* stage = wsl + STAGE_OFF + lane * 16u;
        u32 fills = 0;
        for (u32 pos = j; pos < 64; pos += 3) {
            while (LW[L_NCL / 4] != pos) { if (expired()) break; __builtin_amdgcn_s_sleep(1); }
            CBAR();
            u32x4 at = *(volatile LAS u32x4*)(L + L_AVAIL);
            u64 avail = ((u64)__builtin_amdgcn_readfirstlane(at[1]) << 32) | (u32)__builtin_amdgcn_readfirstlane(at[0]);
            u64 taken = ((u64)__builtin_amdgcn_readfirstlane(at[3]) << 32) | (u32)__builtin_amdgcn_readfirstlane(at[2]);
            while (avail == 0) {
                const u32x4 fx = __builtin_amdgcn_raw_buffer_load_b128(rs_ws, OFF_XF + mb * 1024u + lane * 16u, 0, SYS);
                const u32x2 fw = __builtin_amdgcn_raw_buffer_load_b64(rs_ws, OFF_WF + nb * 512u + lane * 8u, 0, SYS);
                const bool ok = fx[0] >= seq && fx[1] >= seq && fx[2] >= seq && fx[3] >= seq && fw[0] >= seq && fw[1] >= seq;
                avail = __builtin_amdgcn_ballot_w64(ok) & ~taken;
                if (avail == 0) {
                    if (expired()) { avail = ~taken; break; }
                    __builtin_amdgcn_s_sleep(2);
                }
            }
            const u32 step = (u32)__builtin_ctzll(avail) & 63u;
            const u64 bit = 1ull << step;
            avail &= ~bit; taken |= bit;
            *(volatile LAS u32x4*)(L + L_AVAIL) = u32x4{(u32)avail, (u32)(avail >> 32), (u32)taken, (u32)(taken >> 32)};
            LW[L_NCL / 4] = pos + 1;
            if (pos >= 3) {
                for (;;) {
                    const u32x4 fr = *(volatile LAS u32x4*)(L + L_FREE);
                    if (min(min(fr[0], fr[1]), min(fr[2], fr[3])) >= pos - 2) break;
                    if (expired()) break;
                    __builtin_amdgcn_s_sleep(1);
                }
            }
            CBAR();
            const char* sx = stage + step * STEP_BYTES + mb * XP;
            const char* sw = stage + step * STEP_BYTES + 16u * XP + nb * WP;
            char* ldst = smem + j * SLOT_BYTES;
#pragma unroll
            for (int i = 0; i < 16; ++i)
                __builtin_amdgcn_global_load_lds((const u32*)(sw + i * 1024), (u32*)(ldst + XP + i * 1024), 16, 0, DMA_AUX);
#pragma unroll
            for (int i = 0; i < 32; ++i)
                __builtin_amdgcn_global_load_lds((const u32*)(sx + i * 1024), (u32*)(ldst + i * 1024), 16, 0, DMA_AUX);
            VMWAIT(0);
            LW[L_FULL / 4 + j] = ++fills;
        }
        STAMP(1);
        if (wid == 4 && blockIdx.x == 0) {
            u32 a = 0;
            for (;;) {
                const u32x4 g0 = __builtin_amdgcn_raw_buffer_load_b128(rs_ws, OFF_AMX + lane * 32u, 0, AGT);
                const u32x4 g1 = __builtin_amdgcn_raw_buffer_load_b128(rs_ws, OFF_AMX + lane * 32u + 16u, 0, AGT);
                a = max(max(g0[0], g0[2]), max(g1[0], g1[2]));
                const bool ok = g0[1] == seq && g0[3] == seq && g1[1] == seq && g1[3] == seq;
                if (__builtin_amdgcn_ballot_w64(ok) == ~0ull) break;
                if (expired()) { a = 0xFFFFFFFFu; break; }
            }
            a = wave_max_u32(a);
            u64* gp = (u64*)(p.ws[lane & 7u] + OFF_GAM + (lane >> 3) * 128u + rank * 8u);
            __hip_atomic_store(gp, ((u64)seq << 32) | a, __ATOMIC_RELAXED, __HIP_MEMORY_SCOPE_SYSTEM);
            if (lane == 0) __builtin_amdgcn_raw_buffer_store_b32(seq, rs_ws, OFF_SEQ, 0, AGT);
            STAMP(2);
        }
        if (wid == 5) {
            u32 a = 0;
            for (;;) {
                const u32x2 gm = __builtin_amdgcn_raw_buffer_load_b64(rs_ws, OFF_GAM + q * 128u + (lane & 7u) * 8u, 0, SYS);
                a = gm[0];
                if (__builtin_amdgcn_ballot_w64(gm[1] == seq) == ~0ull) break;
                if (expired()) { a = 0xFFFFFFFFu; break; }
                __builtin_amdgcn_s_sleep(1);
            }
            a = wave_max_u32(a);
            LW[L_SCALE / 4 + 1] = a;
            LW[L_SCALE / 4] = 1u;
            STAMP(2);
        }
    } else {
        if (idx < NCOMM1) {
            const rsrc_t rs_x = mk_rsrc(p.x, X_BYTES), rs_w = mk_rsrc(p.w, W_BYTES);
            const rsrc_t rs_d = mk_rsrc(p.ws[dst], WS_NEED);
            const u32 vx0 = (lane >> 3) * 1024u + (((lane & 7u) ^ (lane >> 4)) << 4), vx1 = vx0 ^ 64u;
            const u32 vw = (lane >> 4) * 8u * 32768u + (lane & 15u) * 16u;
            const u32 vd0 = lane * 16u, vd1 = (lane ^ 8u) * 16u;
            u32x4 v[8];
            auto load_unit = [&](u32 u) {
                const u32 ch = u / 96u, r = u % 96u;
                if (r < 32u) {
                    const u32 so = (ch * 64u + (r & 1u) * 32u) * 32768u + (dst * 2048u + (r >> 1) * 128u) * 2u;
#pragma unroll
                    for (int i = 0; i < 8; ++i) v[i] = __builtin_amdgcn_raw_buffer_load_b128(rs_w, vw, so + i * 32768u, 0);
                } else {
                    const u32 t = r - 32u, m = ((t >> 2) + 2u * q) & 15u;
                    const u32 so = (m * 256u + (t & 3u) * 64u) * 1024u + ch * 128u;
#pragma unroll
                    for (int i = 0; i < 8; ++i) v[i] = __builtin_amdgcn_raw_buffer_load_b128(rs_x, (i & 1) ? vx1 : vx0, so + i * 8192u, 0);
                }
            };
            u32 pflag = 0xFFFFFFFFu;
            load_unit(idx);
            for (u32 u = idx; u < 768u; u += NCOMM1) {
                const u32 ch = u / 96u, r = u % 96u, step = ch * 8u + q;
                VMWAIT(0);
                if (pflag != 0xFFFFFFFFu && lane == 0) __builtin_amdgcn_raw_buffer_store_b32(seq, rs_d, pflag, 0, SYS);
                if (r < 32u) {
                    const u32 so = STAGE_OFF + step * STEP_BYTES + 16u * XP + (r >> 1) * WP + (r & 1u) * 8192u;
#pragma unroll
                    for (int e = 0; e < 8; ++e) {
                        const u32 sel = (e & 1) ? 0x07060302u : 0x05040100u;
                        u32x4 o;
#pragma unroll
                        for (int k = 0; k < 4; ++k) o[k] = __builtin_amdgcn_perm(v[2 * k + 1][e >> 1], v[2 * k][e >> 1], sel);
                        ST128(o, rs_d, e >= 4 ? vd1 : vd0, so + e * 1024u, SYS);
                    }
                    pflag = OFF_WF + (r >> 1) * 512u + step * 8u + (r & 1u) * 4u;
                } else {
                    const u32 t = r - 32u, m = ((t >> 2) + 2u * q) & 15u;
                    const u32 so = STAGE_OFF + step * STEP_BYTES + m * XP + (t & 3u) * 8192u;
#pragma unroll
                    for (int i = 0; i < 8; ++i) ST128(v[i], rs_d, vd0, so + i * 1024u, SYS);
                    pflag = OFF_XF + m * 1024u + step * 16u + (t & 3u) * 4u;
                }
                if (u + NCOMM1 < 768u) load_unit(u + NCOMM1);
            }
            VMWAIT(0);
            if (lane == 0) __builtin_amdgcn_raw_buffer_store_b32(seq, rs_d, pflag, 0, SYS);
        }
        STAMP(1);
        if (q != 0u && idx < NCOMM2) {
            while (LW[L_SCALE / 4] != 1u) { if (expired()) break; __builtin_amdgcn_s_sleep(4); }
            for (;;) {
                bool ok = true;
#pragma unroll
                for (int i = 0; i < 4; ++i) {
                    const u32x4 f = __builtin_amdgcn_raw_buffer_load_b128(rs_ws, OFF_TF + i * 1024u + lane * 16u, 0, AGT);
                    ok = ok && f[0] >= seq && f[1] >= seq && f[2] >= seq && f[3] >= seq;
                }
                if (__builtin_amdgcn_ballot_w64(ok) == ~0ull) break;
                if (expired()) break;
                __builtin_amdgcn_s_sleep(4);
            }
            STAMP(2);
            const rsrc_t rs_ol = mk_rsrc(p.out[rank], OUT_BYTES), rs_or = mk_rsrc(p.out[dst], OUT_BYTES);
            const u32 vo = lane * 16u;
            u32x4 v[8];
            auto unit_off = [&](u32 u) -> u32 {
                const u32 r = (q + 1u + (u >> 7)) & 7u;
                return ((r >> 1) * 1024u + (u & 127u) * 8u) * NN + rank * 2048u + (r & 1u) * 1024u;
            };
            u32 so = unit_off(idx);
#pragma unroll
            for (int i = 0; i < 8; ++i) v[i] = __builtin_amdgcn_raw_buffer_load_b128(rs_ol, vo, so + i * NN, AGT);
            for (u32 u = idx; u < 896u; u += NCOMM2) {
#pragma unroll
                for (int i = 0; i < 8; ++i) ST128(v[i], rs_or, vo, so + i * NN, SYS);
                if (u + NCOMM2 < 896u) {
                    so = unit_off(u + NCOMM2);
#pragma unroll
                    for (int i = 0; i < 8; ++i) v[i] = __builtin_amdgcn_raw_buffer_load_b128(rs_ol, vo, so + i * NN, AGT);
                }
            }
        }
    }
    VMWAIT(0);
    STAMP(6);
    __syncthreads();
    if (q != 0u && threadIdx.x == 0)
        __builtin_amdgcn_raw_buffer_store_b32(seq, mk_rsrc(p.ws[dst], WS_NEED), OFF_DONE + rank * 128u + idx * 4u, 0, SYS);
    if (blockIdx.x == 0 && wid == 0) {
        bool give_up = false;
        for (;;) {
            const u32x4 f = __builtin_amdgcn_raw_buffer_load_b128(rs_ws, OFF_DONE + lane * 16u, 0, SYS);
            const bool ok = (lane >> 3) == rank || (f[0] >= seq && f[1] >= seq && f[2] >= seq && f[3] >= seq);
            if (__builtin_amdgcn_ballot_w64(ok) == ~0ull) break;
            if (expired()) { give_up = true; break; }
            __builtin_amdgcn_s_sleep(1);
        }
        if (give_up || __builtin_amdgcn_raw_buffer_load_b32(rs_ws, OFF_TMO, 0, AGT) == seq)
            if (lane == 0) p.out[rank][rank * 2048u] = (char)-128;
        __builtin_amdgcn_fence(__ATOMIC_ACQUIRE, "");
        STAMP(7);
    }
}

extern "C" void kernel_launch(int rank, int nranks, ncclComm_t comm, hipStream_t stream,
                              void* const* d_in, const int* in_sizes, int n_in,
                              void* d_out, int out_size, const int* in_dtypes, int out_dtype,
                              void* const* d_out_all, void* const* d_ws_all, size_t ws_bytes) {
    if (nranks != NR || (unsigned)rank >= (unsigned)NR || n_in < 2 || ws_bytes < WS_NEED ||
        in_sizes[0] != (int)(MM * KSH) || in_sizes[1] != (int)(KSH * NN) || out_size != (int)OUT_BYTES) {
        fprintf(stderr, "kernel_launch: unexpected problem shape\n");
        return;
    }
    static bool init = false;
    if (!init) {
        HIP_CHECK(hipFuncSetAttribute((const void*)gemm_ar_q8, hipFuncAttributeMaxDynamicSharedMemorySize, LDS_BYTES));
        init = true;
    }
    Params p;
    p.x = (const char*)d_in[0];
    p.w = (const char*)d_in[1];
    for (int r = 0; r < NR; ++r) { p.out[r] = (char*)d_out_all[r]; p.ws[r] = (char*)d_ws_all[r]; }
    p.rank = rank;
    hipLaunchKernelGGL(gemm_ar_q8, dim3(256), dim3(512), LDS_BYTES, stream, p);
    HIP_CHECK(hipGetLastError());
}
```

```cpp
#include <hip/hip_runtime.h>
#include <rccl/rccl.h>
#include <cstdio>
#include <cstdint>

#define HIP_CHECK(expr) do { hipError_t _e = (expr); if (_e != hipSuccess) { \
    fprintf(stderr, "HIP error %s at %s:%d\n", hipGetErrorString(_e), __FILE__, __LINE__); \
    return; } } while (0)

#define LAS __attribute__((address_space(3)))
typedef unsigned int u32;
typedef unsigned long long u64;
typedef u32 u32x4 __attribute__((ext_vector_type(4)));
typedef u32 u32x2 __attribute__((ext_vector_type(2)));
typedef short bf16x8 __attribute__((ext_vector_type(8)));
typedef float f32x4 __attribute__((ext_vector_type(4)));

#define NCOMM1 10
#define NCOMM2 10
#define THR1 17
#define THR2 16
#define NDIR 1
#define DIRECT_WIN 2
#define CSLEEP 1
#define POLL_SLEEP 2

constexpr int NR = 8;
constexpr u32 MM = 4096, NN = 16384, KSH = 512;
constexpr u32 X_BYTES = MM * KSH * 2, W_BYTES = KSH * NN * 2, OUT_BYTES = MM * NN;
constexpr u32 PIECE = 32768;
constexpr u32 STEP_BYTES = 24 * PIECE;
constexpr u32 STAGE_OFF = 1u << 20;
constexpr u32 WS_NEED = STAGE_OFF + 64 * STEP_BYTES;
constexpr u32 OFF_SEQ = 0, OFF_TMO = 256;
constexpr u32 OFF_XF = 4096;
constexpr u32 OFF_WF = OFF_XF + 16384;
constexpr u32 OFF_AMX = OFF_WF + 8192;
constexpr u32 OFF_GAM = OFF_AMX + 2048;
constexpr u32 OFF_TF = OFF_GAM + 1024;
constexpr u32 OFF_DONE = OFF_TF + 4096;
constexpr u32 OFF_CTR = OFF_DONE + 1024;
constexpr u32 OFF_STAMP = 65536;
static_assert(OFF_CTR + 2048 <= OFF_STAMP && OFF_STAMP + 256 * 12 * 16 * 8 <= STAGE_OFF, "flag region");

constexpr u32 SLOT_BYTES = 32768;
constexpr u32 CTRL = 4 * SLOT_BYTES;
constexpr u32 L_FULL = CTRL, L_FREE = CTRL + 32, L_NRDY = CTRL + 64, L_AMAX = CTRL + 68, L_ACNT = CTRL + 72,
              L_BAD = CTRL + 76, L_SCALE = CTRL + 80, L_STEPS = CTRL + 128;
constexpr u32 LDS_BYTES = CTRL + 128 + 256;
constexpr u64 DEADLINE = 100000000ull;
constexpr u32 OOB = WS_NEED;
constexpr int SYS = 17, AGT = 16;
constexpr u32 AUX_WG = 248;

struct Params {
    const char* x;
    const char* w;
    char* out[NR];
    char* ws[NR];
    int rank;
};

typedef __amdgpu_buffer_rsrc_t rsrc_t;
__device__ __forceinline__ rsrc_t mk_rsrc(const void* p, u32 bytes) {
    return __builtin_amdgcn_make_buffer_rsrc((void*)p, 0, bytes, 0x00020000);
}
__device__ __forceinline__ u64 now() { return __builtin_amdgcn_s_memrealtime(); }
#define VMWAIT(n) asm volatile("s_waitcnt vmcnt(" #n ")" ::: "memory")
#define CBAR() asm volatile("" ::: "memory")
#define ST128(v, rs, voff, soff, aux) do { const u32x4 d_ = (v); \
    __builtin_amdgcn_raw_buffer_store_b128(d_, rs, voff, soff, aux); \
    asm volatile("s_nop 1" :: "v"(d_) : "memory"); } while (0)
#define MFMA(a, b, c) __builtin_amdgcn_mfma_f32_16x16x32_bf16(a, b, c, 0, 0, 0)

__device__ __forceinline__ u32x4 mk_srd(const void* p, u32 bytes) {
    const u64 a = (u64)p;
    return u32x4{(u32)a, (u32)(a >> 32) & 0xFFFFu, bytes, 0x00020000u};
}
#define STR_(x) #x
#define STR(x) STR_(x)
#define V8OUT(v) "=&v"(v[0]), "=&v"(v[1]), "=&v"(v[2]), "=&v"(v[3]), "=&v"(v[4]), "=&v"(v[5]), "=&v"(v[6]), "=&v"(v[7])
#define V8IN(v) "v"(v[0]), "v"(v[1]), "v"(v[2]), "v"(v[3]), "v"(v[4]), "v"(v[5]), "v"(v[6]), "v"(v[7])
#define LOAD8(v, rs, vo, so, st, POL) do { u32 t_; asm volatile( \
    "s_nop 4\n" \
    "buffer_load_dwordx4 %0, %9, %10, %11 offen" POL "\n s_add_u32 %8, %11, %12\n" \
    "buffer_load_dwordx4 %1, %9, %10, %8 offen" POL "\n s_add_u32 %8, %8, %12\n" \
    "buffer_load_dwordx4 %2, %9, %10, %8 offen" POL "\n s_add_u32 %8, %8, %12\n" \
    "buffer_load_dwordx4 %3, %9, %10, %8 offen" POL "\n s_add_u32 %8, %8, %12\n" \
    "buffer_load_dwordx4 %4, %9, %10, %8 offen" POL "\n s_add_u32 %8, %8, %12\n" \
    "buffer_load_dwordx4 %5, %9, %10, %8 offen" POL "\n s_add_u32 %8, %8, %12\n" \
    "buffer_load_dwordx4 %6, %9, %10, %8 offen" POL "\n s_add_u32 %8, %8, %12\n" \
    "buffer_load_dwordx4 %7, %9, %10, %8 offen" POL "\n" \
    : V8OUT(v), "=&s"(t_) : "v"(vo), "s"(rs), "s"(so), "s"(st) : "memory", "scc"); } while (0)
#define LOAD4X2(v, rs, vo, so, st) do { u32 t_; asm volatile( \
    "s_nop 4\n" \
    "buffer_load_dwordx4 %0, %9, %10, %11 offen\n buffer_load_dwordx4 %4, %9, %10, %11 offen offset:64\n s_add_u32 %8, %11, %12\n" \
    "buffer_load_dwordx4 %1, %9, %10, %8 offen\n buffer_load_dwordx4 %5, %9, %10, %8 offen offset:64\n s_add_u32 %8, %8, %12\n" \
    "buffer_load_dwordx4 %2, %9, %10, %8 offen\n buffer_load_dwordx4 %6, %9, %10, %8 offen offset:64\n s_add_u32 %8, %8, %12\n" \
    "buffer_load_dwordx4 %3, %9, %10, %8 offen\n buffer_load_dwordx4 %7, %9, %10, %8 offen offset:64\n" \
    : V8OUT(v), "=&s"(t_) : "v"(vo), "s"(rs), "s"(so), "s"(st) : "memory", "scc"); } while (0)
#define WT(N) "s_waitcnt vmcnt(" STR(N) ")\n"
#define STORE8(v, rs, vd, so, st, N) do { u32 t_; asm volatile( \
    "s_nop 4\n" \
    WT(N) "buffer_store_dwordx4 %1, %9, %10, %11 offen sc0 sc1\n s_add_u32 %0, %11, %12\n" \
    WT(N) "buffer_store_dwordx4 %2, %9, %10, %0 offen sc0 sc1\n s_add_u32 %0, %0, %12\n" \
    WT(N) "buffer_store_dwordx4 %3, %9, %10, %0 offen sc0 sc1\n s_add_u32 %0, %0, %12\n" \
    WT(N) "buffer_store_dwordx4 %4, %9, %10, %0 offen sc0 sc1\n s_add_u32 %0, %0, %12\n" \
    WT(N) "buffer_store_dwordx4 %5, %9, %10, %0 offen sc0 sc1\n s_add_u32 %0, %0, %12\n" \
    WT(N) "buffer_store_dwordx4 %6, %9, %10, %0 offen sc0 sc1\n s_add_u32 %0, %0, %12\n" \
    WT(N) "buffer_store_dwordx4 %7, %9, %10, %0 offen sc0 sc1\n s_add_u32 %0, %0, %12\n" \
    WT(N) "buffer_store_dwordx4 %8, %9, %10, %0 offen sc0 sc1\n s_nop 1\n" \
    : "=&s"(t_) : V8IN(v), "v"(vd), "s"(rs), "s"(so), "s"(st) : "memory", "scc"); } while (0)
#define STORE4X2(v, rs, vd, sa, sb, N) asm volatile( \
    "s_nop 4\n" \
    WT(N) "buffer_store_dwordx4 %0, %8, %9, %10 offen sc0 sc1\n" \
    WT(N) "buffer_store_dwordx4 %1, %8, %9, %10 offen offset:1024 sc0 sc1\n" \
    WT(N) "buffer_store_dwordx4 %2, %8, %9, %10 offen offset:2048 sc0 sc1\n" \
    WT(N) "buffer_store_dwordx4 %3, %8, %9, %10 offen offset:3072 sc0 sc1\n" \
    WT(N) "buffer_store_dwordx4 %4, %8, %9, %11 offen sc0 sc1\n" \
    WT(N) "buffer_store_dwordx4 %5, %8, %9, %11 offen offset:1024 sc0 sc1\n" \
    WT(N) "buffer_store_dwordx4 %6, %8, %9, %11 offen offset:2048 sc0 sc1\n" \
    WT(N) "buffer_store_dwordx4 %7, %8, %9, %11 offen offset:3072 sc0 sc1\n s_nop 1\n" \
    :: V8IN(v), "v"(vd), "s"(rs), "s"(sa), "s"(sb) : "memory")
#define STORE1(d, rs, vd, so, N) asm volatile( \
    WT(N) "buffer_store_dwordx4 %0, %1, %2, %3 offen sc0 sc1\n s_nop 1\n" \
    :: "v"(d), "v"(vd), "s"(rs), "s"(so) : "memory")
#define WAIT8(v, N) asm volatile("s_nop 4\n" WT(N) \
    : "+v"(v[0]), "+v"(v[1]), "+v"(v[2]), "+v"(v[3]), "+v"(v[4]), "+v"(v[5]), "+v"(v[6]), "+v"(v[7]) :: "memory")
#define CLAIM(cl, rs, vone, so) asm volatile( \
    "v_mov_b32 %0, 1\n s_nop 4\n buffer_atomic_add %0, %1, %2, %3 offen sc0 sc1\n" \
    : "=&v"(cl) : "v"(vone), "s"(rs), "s"(so) : "memory")
#define CLAIMED(cl, N) asm volatile(WT(N) : "+v"(cl) :: "memory")
#define FLAGST(val, rs, vone, so) asm volatile( \
    "s_nop 4\n buffer_store_dword %0, %1, %2, %3 offen sc0 sc1\n" :: "v"(val), "v"(vone), "s"(rs), "s"(so) : "memory")

#define STAMPV(i, v) do { } while (0)
#define STAMP(i) STAMPV(i, now())

__device__ __forceinline__ u32 wave_max_u32(u32 v) {
    v = max(v, (u32)__builtin_amdgcn_update_dpp(0, (int)v, 0x111, 0xf, 0xf, true));
    v = max(v, (u32)__builtin_amdgcn_update_dpp(0, (int)v, 0x112, 0xf, 0xf, true));
    v = max(v, (u32)__builtin_amdgcn_update_dpp(0, (int)v, 0x114, 0xf, 0xf, true));
    v = max(v, (u32)__builtin_amdgcn_update_dpp(0, (int)v, 0x118, 0xf, 0xf, true));
    v = max(v, (u32)__builtin_amdgcn_update_dpp(0, (int)v, 0x142, 0xa, 0xf, false));
    v = max(v, (u32)__builtin_amdgcn_update_dpp(0, (int)v, 0x143, 0xc, 0xf, false));
    return (u32)__builtin_amdgcn_readlane((int)v, 63);
}

__global__ void __launch_bounds__(768, 3) gemm_ar_q8(Params p) {
    extern __shared__ __attribute__((aligned(16))) char smem[];
    LAS char* L = (LAS char*)smem;
    u32 lane = threadIdx.x & 63;
    const u32 wid = __builtin_amdgcn_readfirstlane(threadIdx.x >> 6);
    const u32 rank = (u32)p.rank & 7u;
    const u32 q = blockIdx.x & 7u, idx = blockIdx.x >> 3;
    const u32 mb = 4 * (q >> 1) + (idx & 3), nb = 4 * (q & 1) + ((idx >> 2) & 3);
    const u32 dst = (rank + q) & 7u;
    char* wsl = p.ws[rank];
    const rsrc_t rs_ws = mk_rsrc(wsl, WS_NEED);
    volatile LAS u32* LW = (volatile LAS u32*)L;

    STAMP(0);
    if (threadIdx.x < 32) LW[CTRL / 4 + threadIdx.x] = 0u;
    asm volatile("s_waitcnt lgkmcnt(0)" ::: "memory");
    __builtin_amdgcn_s_barrier();
    CBAR();
    const u32 seq = __builtin_amdgcn_readfirstlane(__builtin_amdgcn_raw_buffer_load_b32(rs_ws, OFF_SEQ, 0, AGT)) + 1u;

    u32 spins = 0;
    u64 t0 = 0;
    auto expired = [&]() -> bool {
        if ((++spins & 63u) != 0u) return false;
        if (LW[L_BAD / 4] != 0u) return true;
        const u64 t = now();
        if (t0 == 0) t0 = t;
        if (t - t0 > DEADLINE || __builtin_amdgcn_raw_buffer_load_b32(rs_ws, OFF_TMO, 0, AGT) == seq) {
            u32 sq = seq;
            asm volatile("" : "+s"(sq));
            __builtin_amdgcn_raw_buffer_store_b32(sq, rs_ws, OFF_TMO, 0, AGT);
            LW[L_BAD / 4] = 1u;
            return true;
        }
        return false;
    };

    auto arrive = [&](u32 wm, u32 n) {
        if (lane == 0) {
            __hip_atomic_fetch_max((LAS u32*)(L + L_AMAX), wm, __ATOMIC_RELAXED, __HIP_MEMORY_SCOPE_WORKGROUP);
            const u32 old = __hip_atomic_fetch_add((LAS u32*)(L + L_ACNT), 1u, __ATOMIC_RELAXED, __HIP_MEMORY_SCOPE_WORKGROUP);
            if (old == n - 1u) {
                u32 a = LW[L_AMAX / 4];
                if (LW[L_BAD / 4] != 0u) a = 0xFFFFFFFFu;
                __builtin_amdgcn_raw_buffer_store_b64(u32x2{a, seq}, rs_ws, OFF_AMX + blockIdx.x * 8u, 0, AGT);
            }
        }
    };

    if (idx < 16u && wid < 8u) {
        u32 c = lane & 15u, g = lane >> 4;
        const u32 mw = wid & 3u, nh = wid >> 2;
        u32 xa = mw * 4096u + (c * 4u + (g ^ ((c >> 2) & 2u))) * 16u;
        u32 wa = 16384u + nh * 8192u + (c & 1u) * 4096u + g * 256u + ((8u * (c & 1u) + (c >> 1)) << 4);
        f32x4 acc[8][4];
#pragma unroll
        for (int j = 0; j < 8; ++j)
#pragma unroll
            for (int m = 0; m < 4; ++m) acc[j][m] = f32x4{0.f, 0.f, 0.f, 0.f};
        bf16x8 wl[4], wh[4], xf[4];
#define LDS16(a) (*(const LAS bf16x8*)(L + (a)))
        while (LW[L_FULL / 4] < 1u) { if (expired()) break; __builtin_amdgcn_s_sleep(CSLEEP); }
        CBAR();
        STAMP(1);
#pragma unroll
        for (int m = 0; m < 4; ++m) xf[m] = LDS16(xa + m * 1024);
#pragma unroll
        for (int j = 0; j < 4; ++j) wl[j] = LDS16(wa + j * 1024);
        for (u32 f = 0; f < 128; ++f) {
            __builtin_amdgcn_sched_barrier(0);
#pragma unroll
            for (int j = 0; j < 4; ++j) {
#pragma unroll
                for (int m = 0; m < 4; ++m) acc[j][m] = MFMA(wl[j], xf[m], acc[j][m]);
                wh[j] = LDS16((wa ^ 128u) + j * 1024);
                __builtin_amdgcn_sched_barrier(0);
            }
            CBAR();
            u32 fa = L_FREE + wid * 4u;
            asm volatile("" : "+s"(fa));
            *(volatile LAS u32*)(L + fa) = f + 1;
            if (f < 127) {
                while (LW[L_FULL / 4 + ((f + 1) & 3u)] < ((f + 1) >> 2) + 1u) { if (expired()) break; __builtin_amdgcn_s_sleep(CSLEEP); }
            }
            CBAR();
            xa = (xa + SLOT_BYTES) & (CTRL - 1u);
            wa = (wa + SLOT_BYTES) & (CTRL - 1u);
            __builtin_amdgcn_sched_barrier(0);
#pragma unroll
            for (int m = 0; m < 3; ++m) {
#pragma unroll
                for (int j = 0; j < 4; ++j) acc[4 + j][m] = MFMA(wh[j], xf[m], acc[4 + j][m]);
                xf[m] = LDS16(xa + m * 1024);
                __builtin_amdgcn_sched_barrier(0);
            }
#pragma unroll
            for (int j = 0; j < 4; ++j) {
                acc[4 + j][3] = MFMA(wh[j], xf[3], acc[4 + j][3]);
                wl[j] = LDS16(wa + j * 1024);
                __builtin_amdgcn_sched_barrier(0);
            }
            xf[3] = LDS16(xa + 3 * 1024);
        }
        asm volatile("v_mbcnt_lo_u32_b32 %0, -1, 0\n v_mbcnt_hi_u32_b32 %0, -1, %0" : "=v"(lane));
        c = lane & 15u; g = lane >> 4;
        STAMP(2);
        float mx = 0.f;
#pragma unroll
        for (int j = 0; j < 8; ++j)
#pragma unroll
            for (int m = 0; m < 4; ++m) {
                mx = fmaxf(fmaxf(mx, fabsf(acc[j][m][0])), fabsf(acc[j][m][1]));
                mx = fmaxf(fmaxf(mx, fabsf(acc[j][m][2])), fabsf(acc[j][m][3]));
            }
        arrive(wave_max_u32(__float_as_uint(mx)), 8u);
        STAMP(3);
        while (LW[L_SCALE / 4] != 1u) { if (expired()) break; __builtin_amdgcn_s_sleep(CSLEEP); }
        CBAR();
        const u32 gmax = LW[L_SCALE / 4 + 1];
        const bool bad = (gmax == 0xFFFFFFFFu) || (LW[L_BAD / 4] != 0u);
        const float amax = __uint_as_float(gmax);
        const float inv = amax > 0.f ? 127.0f / amax : 0.f;
        STAMP(4);
        const float MAGIC = 12582912.0f;
        const u32 ovoff = (64u * mw + c) * NN + g * 16u;
        const u32 osoff = mb * 256u * NN + rank * 2048u + nb * 256u + nh * 128u;
        const rsrc_t rs_ol = mk_rsrc(p.out[rank], OUT_BYTES);
        const bool direct = q != 0u && (idx & 3u) < NDIR;
        const rsrc_t rs_or = mk_rsrc(p.out[dst], direct ? OUT_BYTES : 0u);
#pragma unroll
        for (int v = 0; v < 8; ++v) {
            const int m = v >> 1, h = v & 1;
            u32x4 o;
#pragma unroll
            for (int r = 0; r < 4; ++r) {
                u32 b[4];
#pragma unroll
                for (int jj = 0; jj < 4; ++jj)
                    b[jj] = __float_as_uint(__builtin_amdgcn_fmed3f(__builtin_fmaf(acc[4 * h + jj][m][r], inv, MAGIC),
                                                                    MAGIC - 127.0f, MAGIC + 127.0f));
                const u32 lo = __builtin_amdgcn_perm(b[1], b[0], 0x0c0c0400u);
                const u32 hi = __builtin_amdgcn_perm(b[3], b[2], 0x04000c0cu);
                o[r] = lo | hi;
            }
            if (bad) o = u32x4{0x80808080u, 0x80808080u, 0x80808080u, 0x80808080u};
            if (direct) {
                if (v >= DIRECT_WIN) asm volatile("s_waitcnt vmcnt(%0)" :: "n"(2 * DIRECT_WIN - 1) : "memory");
                ST128(o, rs_or, ovoff + h * 64u, osoff + m * 16u * NN, SYS);
            }
            ST128(o, rs_ol, ovoff + h * 64u, osoff + m * 16u * NN, AGT);
        }
        VMWAIT(0);
        if (lane == 0)
            __builtin_amdgcn_raw_buffer_store_b32(seq, rs_ws, OFF_TF + ((((idx & 3u) * 8u + q) * 4u + mw) * 8u + (idx >> 2) * 2u + nh) * 4u, 0, AGT);
        STAMP(5);
    } else if (idx < 16u && wid == 8u) {
        u64 taken = 0;
        u32 n = 0;
        while (n < 64u) {
            const u64 want = ~taken & ((taken << 16) | 0xFFFFull);
            const u32 vo = ((want >> lane) & 1ull) ? lane * 16u : OOB;
            const u32x4 fx = __builtin_amdgcn_raw_buffer_load_b128(rs_ws, vo, OFF_XF + mb * 1024u, SYS);
            const u32x4 fw = __builtin_amdgcn_raw_buffer_load_b128(rs_ws, vo, OFF_WF + nb * 1024u, SYS);
            const u32 mn = min(min(min(fx[0], fx[1]), min(fx[2], fx[3])), min(min(fw[0], fw[1]), min(fw[2], fw[3])));
            u64 avail = __builtin_amdgcn_ballot_w64(mn >= seq);
            if (avail == 0) {
                if (!expired()) { __builtin_amdgcn_s_sleep(POLL_SLEEP); continue; }
                avail = ~taken;
            }
            if ((avail >> lane) & 1ull)
                LW[L_STEPS / 4 + n + __builtin_amdgcn_mbcnt_hi((u32)(avail >> 32), __builtin_amdgcn_mbcnt_lo((u32)avail, 0u))] = lane;
            n += (u32)__builtin_popcountll(avail);
            taken |= avail;
            LW[L_NRDY / 4] = n;
        }
        STAMP(1);
        u32 a = 0;
        for (;;) {
            const u32x2 gm = __builtin_amdgcn_raw_buffer_load_b64(rs_ws, OFF_GAM + q * 128u + (lane & 7u) * 8u, 0, SYS);
            a = gm[0];
            if (__builtin_amdgcn_ballot_w64(gm[1] == seq) == ~0ull) break;
            if (expired()) { a = 0xFFFFFFFFu; break; }
            __builtin_amdgcn_s_sleep(1);
        }
        a = wave_max_u32(a);
        LW[L_SCALE / 4 + 1] = a;
        LW[L_SCALE / 4] = 1u;
        STAMP(2);
    } else if (idx < 16u) {
        for (u32 f = wid - 9u; f < 128u; f += 3u) {
            while (LW[L_NRDY / 4] <= (f >> 1)) { if (expired()) break; __builtin_amdgcn_s_sleep(1); }
            CBAR();
            const u32 step = __builtin_amdgcn_readfirstlane(LW[L_STEPS / 4 + (f >> 1)]) & 63u;
            if (f >= 4u) {
                for (;;) {
                    if (__builtin_amdgcn_ballot_w64(LW[L_FREE / 4 + (lane & 7u)] >= f - 3u) == ~0ull) break;
                    if (expired()) break;
                    __builtin_amdgcn_s_sleep(1);
                }
            }
            CBAR();
            const u32 sbase = STAGE_OFF + ((step & 7u) * 8u + (step >> 3)) * STEP_BYTES + (f & 1u) * 16384u;
            LAS char* ldst = L + (f & 3u) * SLOT_BYTES;
#pragma unroll
            for (int i = 0; i < 8; ++i) {
                const u32 so = sbase + (i < 4 ? mb : 16u + nb) * PIECE + (i & 3) * 4096u;
                __builtin_amdgcn_raw_ptr_buffer_load_lds(rs_ws, ldst + i * 4096, 16, lane * 16u, so, 0, SYS);
                __builtin_amdgcn_raw_ptr_buffer_load_lds(rs_ws, ldst + i * 4096, 16, lane * 16u, so, 1024, SYS);
                __builtin_amdgcn_raw_ptr_buffer_load_lds(rs_ws, ldst + i * 4096, 16, lane * 16u, so, 2048, SYS);
                __builtin_amdgcn_raw_ptr_buffer_load_lds(rs_ws, ldst + i * 4096, 16, lane * 16u, so, 3072, SYS);
            }
            VMWAIT(0);
            LW[L_FULL / 4 + (f & 3u)] = (f >> 2) + 1u;
        }
        STAMP(1);
    } else {
        arrive(0u, 12u);
        const u32x4 sr_ws = mk_srd(wsl, WS_NEED);
        const u32 vone = lane == 0u ? 0u : OOB;
        const u32 sid = (idx - 16u) + 16u * wid;
        const u32 nc1 = q == 0u ? 32u : NCOMM1;
        if (sid < nc1) {
            const u32x4 sr_d = mk_srd(p.ws[dst], WS_NEED), sr_x = mk_srd(p.x, X_BYTES), sr_w = mk_srd(p.w, W_BYTES);
            const u32 vx = (lane >> 2) * 1024u + (((lane & 3u) ^ ((lane >> 4) & 2u)) << 4);
            const u32 vw = (lane >> 4) * 8u * 32768u + (lane & 15u) * 16u;
            const u32 vd0 = lane * 16u, vd1 = (lane ^ 8u) * 16u;
            const u32 vseq = seq;
            auto load_unit = [&](u32 u, u32x4 (&v)[8]) {
                u = min(u, 767u);
                const u32 ch = u / 96u, r = u % 96u;
                if (r < 32u) {
                    const u32 so = (ch * 64u + (r & 2u) * 16u) * 32768u + (dst * 2048u + (r >> 2) * 256u + (r & 1u) * 128u) * 2u;
                    LOAD8(v, sr_w, vw, so, 32768u, "");
                } else {
                    const u32 t = r - 32u, m = ((t >> 2) + 2u * q) & 15u;
                    const u32 so = (m * 256u + (t & 3u) * 64u) * 1024u + ch * 128u;
                    LOAD4X2(v, sr_x, vx, so, 16384u);
                }
            };
            auto store_unit = [&](u32 u, u32x4 (&v)[8]) -> u32 {
                const u32 ch = u / 96u, r = u % 96u, step = ch * 8u + q;
                const u32 sbase = STAGE_OFF + (q * 8u + ch) * STEP_BYTES;
                if (r < 32u) {
                    const u32 so = sbase + 16u * PIECE + r * 8192u;
                    WAIT8(v, THR1);
#pragma unroll
                    for (int e = 0; e < 8; ++e) {
                        const u32 sel = (e & 1) ? 0x07060302u : 0x05040100u;
                        u32x4 o;
#pragma unroll
                        for (int k = 0; k < 4; ++k) o[k] = __builtin_amdgcn_perm(v[2 * k + 1][e >> 1], v[2 * k][e >> 1], sel);
                        STORE1(o, sr_d, e >= 4 ? vd1 : vd0, so + e * 1024u, THR1);
                    }
                    return OFF_WF + (r >> 2) * 1024u + step * 16u + (r & 3u) * 4u;
                } else {
                    const u32 t = r - 32u, m = ((t >> 2) + 2u * q) & 15u;
                    const u32 so = sbase + m * PIECE + (t & 3u) * 4096u;
                    STORE4X2(v, sr_d, vd0, so, so + 16384u, THR1);
                    return OFF_XF + m * 1024u + step * 16u + (t & 3u) * 4u;
                }
            };
            const u32 cbase = (seq - 1u) * 768u - 2u * nc1, coff = OFF_CTR + q * 128u;
            u32x4 va[8], vb[8];
            u32 u0 = sid, u1 = sid + nc1, pflag = OOB, cl;
            load_unit(u0, va);
            VMWAIT(0);
            for (;;) {
                CLAIM(cl, sr_ws, vone, coff);
                load_unit(u1, vb);
                u32 f = store_unit(u0, va);
                FLAGST(vseq, sr_d, vone, pflag);
                pflag = f;
                CLAIMED(cl, 17);
                u0 = u1; u1 = __builtin_amdgcn_readfirstlane(cl) - cbase;
                if (u0 >= 768u) break;
                CLAIM(cl, sr_ws, vone, coff);
                load_unit(u1, va);
                f = store_unit(u0, vb);
                FLAGST(vseq, sr_d, vone, pflag);
                pflag = f;
                CLAIMED(cl, 17);
                u0 = u1; u1 = __builtin_amdgcn_readfirstlane(cl) - cbase;
                if (u0 >= 768u) break;
            }
            VMWAIT(0);
            FLAGST(vseq, sr_d, vone, pflag);
        }
        STAMP(1);
        if (blockIdx.x == AUX_WG && wid == 2u) {
            u32 a = 0;
            for (;;) {
                const u32x4 g0 = __builtin_amdgcn_raw_buffer_load_b128(rs_ws, OFF_AMX + lane * 32u, 0, AGT);
                const u32x4 g1 = __builtin_amdgcn_raw_buffer_load_b128(rs_ws, OFF_AMX + lane * 32u + 16u, 0, AGT);
                a = max(max(g0[0], g0[2]), max(g1[0], g1[2]));
                const bool ok = g0[1] == seq && g0[3] == seq && g1[1] == seq && g1[3] == seq;
                if (__builtin_amdgcn_ballot_w64(ok) == ~0ull) break;
                if (expired()) { a = 0xFFFFFFFFu; break; }
            }
            a = wave_max_u32(a);
            u64* gp = (u64*)(p.ws[lane & 7u] + OFF_GAM + (lane >> 3) * 128u + rank * 8u);
            __hip_atomic_store(gp, ((u64)seq << 32) | a, __ATOMIC_RELAXED, __HIP_MEMORY_SCOPE_SYSTEM);
            if (lane == 0) __builtin_amdgcn_raw_buffer_store_b32(seq, rs_ws, OFF_SEQ, 0, AGT);
            STAMP(3);
        }
        if (q != 0u && sid < NCOMM2) {
            auto wait_tiles = [&](u32 first, u32 count) {
                for (;;) {
                    u32 mn = 0xFFFFFFFFu;
                    for (u32 i = first; i < first + count; ++i) {
                        const u32x4 f = __builtin_amdgcn_raw_buffer_load_b128(rs_ws, OFF_TF + i * 1024u + lane * 16u, 0, AGT);
                        mn = min(min(mn, min(f[0], f[1])), min(f[2], f[3]));
                    }
                    if (__builtin_amdgcn_ballot_w64(mn >= seq) == ~0ull) break;
                    if (expired()) break;
                    __builtin_amdgcn_s_sleep(1);
                }
            };
            wait_tiles(NDIR, 4u - NDIR);
            STAMP(2);
            const u32x4 sr_ol = mk_srd(p.out[rank], OUT_BYTES), sr_or = mk_srd(p.out[dst], OUT_BYTES);
            const u32 vo = lane * 16u;
            constexpr u32 UA = (4u - NDIR) * 32u, UB = NDIR * 32u, NUA = 8u * UA, NU2 = NUA + 7u * UB;
            auto unit_off = [&](u32 u) -> u32 {
                u = min(u, NU2 - 1u);
                u32 r, row;
                if (u < NUA) { r = (q + u / UA) & 7u; row = NDIR * 256u + (u % UA) * 8u; }
                else { r = (q + 1u + (u - NUA) / UB) & 7u; row = ((u - NUA) % UB) * 8u; }
                return ((r >> 1) * 1024u + row) * NN + rank * 2048u + (r & 1u) * 1024u;
            };
            bool late_ok = false;
            auto load_unit = [&](u32 u, u32x4 (&v)[8]) {
                if (u >= NUA && !late_ok) { wait_tiles(0u, NDIR); late_ok = true; }
                const u32 so = unit_off(u);
                LOAD8(v, sr_ol, vo, so, NN, " sc1");
            };
            auto store_unit = [&](u32 u, u32x4 (&v)[8]) {
                const u32 so = unit_off(u);
                STORE8(v, sr_or, vo, so, NN, THR2);
            };
            const u32 cbase = (seq - 1u) * NU2 - 2u * NCOMM2, coff = OFF_CTR + 1024u + q * 128u;
            u32x4 va[8], vb[8];
            u32 u0 = sid, u1 = sid + NCOMM2, cl;
            load_unit(u0, va);
            VMWAIT(0);
            for (;;) {
                CLAIM(cl, sr_ws, vone, coff);
                load_unit(u1, vb);
                store_unit(u0, va);
                CLAIMED(cl, 16);
                u0 = u1; u1 = __builtin_amdgcn_readfirstlane(cl) - cbase;
                if (u0 >= NU2) break;
                CLAIM(cl, sr_ws, vone, coff);
                load_unit(u1, va);
                store_unit(u0, vb);
                CLAIMED(cl, 16);
                u0 = u1; u1 = __builtin_amdgcn_readfirstlane(cl) - cbase;
                if (u0 >= NU2) break;
            }
        }
    }
    VMWAIT(0);
    STAMP(6);
    __syncthreads();
    if (q != 0u && wid == 0u && lane == 0u)
        __builtin_amdgcn_raw_buffer_store_b32(seq, mk_rsrc(p.ws[dst], WS_NEED), OFF_DONE + rank * 128u + idx * 4u, 0, SYS);
    if (blockIdx.x == AUX_WG && wid == 0) {
        bool give_up = false;
        for (;;) {
            const u32x4 f = __builtin_amdgcn_raw_buffer_load_b128(rs_ws, OFF_DONE + lane * 16u, 0, SYS);
            const bool ok = (lane >> 3) == rank || (f[0] >= seq && f[1] >= seq && f[2] >= seq && f[3] >= seq);
            if (__builtin_amdgcn_ballot_w64(ok) == ~0ull) break;
            if (expired()) { give_up = true; break; }
            __builtin_amdgcn_s_sleep(1);
        }
        if (give_up || __builtin_amdgcn_raw_buffer_load_b32(rs_ws, OFF_TMO, 0, AGT) == seq)
            if (lane == 0) p.out[rank][rank * 2048u] = (char)-128;
        STAMP(7);
    }
}

extern "C" void kernel_launch(int rank, int nranks, ncclComm_t comm, hipStream_t stream,
                              void* const* d_in, const int* in_sizes, int n_in,
                              void* d_out, int out_size, const int* in_dtypes, int out_dtype,
                              void* const* d_out_all, void* const* d_ws_all, size_t ws_bytes) {
    if (nranks != NR || (unsigned)rank >= (unsigned)NR || n_in < 2 || ws_bytes < 3ull * WS_NEED ||
        in_sizes[0] != (int)(MM * KSH) || in_sizes[1] != (int)(KSH * NN) || out_size != (int)OUT_BYTES) {
        fprintf(stderr, "kernel_launch: unexpected problem shape\n");
        return;
    }
    static bool init = false;
    if (!init) {
        HIP_CHECK(hipFuncSetAttribute((const void*)gemm_ar_q8, hipFuncAttributeMaxDynamicSharedMemorySize, LDS_BYTES));
        init = true;
    }
    Params p;
    p.x = (const char*)d_in[0];
    p.w = (const char*)d_in[1];
    for (int r = 0; r < NR; ++r) { p.out[r] = (char*)d_out_all[r]; p.ws[r] = (char*)d_ws_all[r]; }
    p.rank = rank;
    hipLaunchKernelGGL(gemm_ar_q8, dim3(256), dim3(768), LDS_BYTES, stream, p);
    HIP_CHECK(hipGetLastError());
}
```

```cpp
#include <hip/hip_runtime.h>
#include <rccl/rccl.h>
#include <cstdio>
#include <cstdint>

#define HIP_CHECK(expr) do { hipError_t _e = (expr); if (_e != hipSuccess) { \
    fprintf(stderr, "HIP error %s at %s:%d\n", hipGetErrorString(_e), __FILE__, __LINE__); \
    return; } } while (0)

#define LAS __attribute__((address_space(3)))
typedef unsigned int u32;
typedef unsigned long long u64;
typedef u32 u32x4 __attribute__((ext_vector_type(4)));
typedef u32 u32x2 __attribute__((ext_vector_type(2)));
typedef unsigned short u16x2 __attribute__((ext_vector_type(2)));
typedef short bf16x8 __attribute__((ext_vector_type(8)));
typedef float f32x4 __attribute__((ext_vector_type(4)));

#define WIN1 4
#define WIN2 4
#define ND2 2
#define GCH 2
#define TDELAY 100
#define WDELAY1 200
#define WDELAY2 900
#define S2 16
#define DIAG_NDMA 8
#define CSLEEP 1
#define POLL_SLEEP 2
#define LEAD 8
#define FORCE_RAW1 0
#define FORCE_RAW2 0

constexpr int NR = 8;
constexpr u32 MM = 4096, NN = 16384, KSH = 512;
constexpr u32 X_BYTES = MM * KSH * 2, W_BYTES = KSH * NN * 2, OUT_BYTES = MM * NN;
constexpr u32 PIECE = 32768;
constexpr u32 STEP_BYTES = 24 * PIECE;
constexpr u32 STAGE_OFF = 1u << 20;
constexpr u32 E1_SLOT = 8192 + 128;
constexpr u32 E1X_OFF = STAGE_OFF + 64 * STEP_BYTES;
constexpr u32 E1W_OFF = E1X_OFF + 512 * E1_SLOT;
constexpr u32 E2_OFF = E1W_OFF + 2048 * E1_SLOT;
constexpr u32 WS_NEED = E2_OFF + 1024 * 8192;
constexpr u32 OFF_SEQ = 0, OFF_TMO = 256;
constexpr u32 OFF_XF = 4096;
constexpr u32 OFF_WF = OFF_XF + 16384;
constexpr u32 OFF_AMX = OFF_WF + 8192;
constexpr u32 OFF_GAM = OFF_AMX + 2048;
constexpr u32 OFF_DONE = OFF_GAM + 1024;
constexpr u32 OFF_F1 = OFF_DONE + 1024;
constexpr u32 OFF_CTR = OFF_F1 + 1024;
constexpr u32 OFF_F2 = 65536;
constexpr u32 OFF_STAMP = 131072;
static_assert(OFF_CTR + 80 * 128 <= OFF_F2 && OFF_F2 + 32768 <= OFF_STAMP && OFF_STAMP + 256 * 12 * 16 * 8 <= STAGE_OFF, "flags");

constexpr u32 SLOT_BYTES = 32768;
constexpr u32 CTRL = 4 * SLOT_BYTES;
constexpr u32 L_FULL = CTRL, L_FREE = CTRL + 32, L_NRDY = CTRL + 64, L_AMAX = CTRL + 68, L_ACNT = CTRL + 72,
              L_BAD = CTRL + 76, L_SCALE = CTRL + 80, L_GCNT = CTRL + 88, L_GSZ = CTRL + 96, L_STEPS = CTRL + 128;
constexpr u32 L_SEG = CTRL + 4, L_G2 = CTRL + 8, L_END = CTRL + 12, L_P1 = CTRL + 16, L_Q = CTRL + 128, L_SZ = CTRL + 256;
constexpr u32 LDS_BYTES = CTRL + 256 + 2048;
constexpr u64 DEADLINE = 100000000ull;
constexpr u32 OOB = WS_NEED;
constexpr int SYS = 17, AGT = 16;
constexpr u32 AUX_WG = 248;
constexpr u32 N2 = 64 - ND2;
constexpr u32 R_LEN = 5120, R_BITS = 5184;

struct Params {
    const char* x;
    const char* w;
    char* out[NR];
    char* ws[NR];
    int rank;
};

typedef __amdgpu_buffer_rsrc_t rsrc_t;
__device__ __forceinline__ rsrc_t mk_rsrc(const void* p, u32 bytes) {
    return __builtin_amdgcn_make_buffer_rsrc((void*)p, 0, bytes, 0x00020000);
}
__device__ __forceinline__ u64 now() { return __builtin_amdgcn_s_memrealtime(); }
#define VMWAIT(n) asm volatile("s_waitcnt vmcnt(%0)" :: "n"(n) : "memory")
#define CBAR() asm volatile("" ::: "memory")
#define ST128(v, rs, voff, soff, aux) do { const u32x4 d_ = (v); \
    __builtin_amdgcn_raw_buffer_store_b128(d_, rs, voff, soff, aux); \
    asm volatile("s_nop 1" :: "v"(d_) : "memory"); } while (0)
#define LD128(rs, voff, soff, aux) __builtin_amdgcn_raw_buffer_load_b128(rs, voff, soff, aux)
#define MFMA(a, b, c) __builtin_amdgcn_mfma_f32_16x16x32_bf16(a, b, c, 0, 0, 0)
#define PERM(a, b, s) __builtin_amdgcn_perm(a, b, s)
#define BALLOT(c) __builtin_amdgcn_ballot_w64(c)
#define RFL(v) ((u32)__builtin_amdgcn_readfirstlane(v))
__device__ __forceinline__ u32 bfi(u32 m, u32 a, u32 b) { return (a & m) | (b & ~m); }
__device__ __forceinline__ u32 rank_in(u64 m) {
    return __builtin_amdgcn_mbcnt_hi((u32)(m >> 32), __builtin_amdgcn_mbcnt_lo((u32)m, 0u));
}

#define STAMPV(i, v) do { } while (0)
#define STAMP(i) STAMPV(i, now())

#define DPP_STEPS(OP) \
    OP(0x111, 0xf, true) OP(0x112, 0xf, true) OP(0x114, 0xf, true) OP(0x118, 0xf, true) OP(0x142, 0xa, false) OP(0x143, 0xc, false)
__device__ __forceinline__ u32 wave_max_u32(u32 v) {
#define OP_(c, rm, bc) v = max(v, (u32)__builtin_amdgcn_update_dpp(0, (int)v, c, rm, 0xf, bc));
    DPP_STEPS(OP_)
#undef OP_
    return (u32)__builtin_amdgcn_readlane((int)v, 63);
}
__device__ __forceinline__ u32 wave_scan_u32(u32 v) {
#define OP_(c, rm, bc) v += (u32)__builtin_amdgcn_update_dpp(0, (int)v, c, rm, 0xf, bc);
    DPP_STEPS(OP_)
#undef OP_
    return v;
}
__device__ __forceinline__ u32 byte_signs(u32 a) { return PERM(a, a << 8, 0x0B090A08u); }

__global__ void __launch_bounds__(768, 3) gemm_ar_q8(Params p) {
    extern __shared__ __attribute__((aligned(16))) char smem[];
    LAS char* L = (LAS char*)smem;
    u32 lane = threadIdx.x & 63;
    const u32 wid = RFL(threadIdx.x >> 6);
    const u32 rank = (u32)p.rank & 7u;
    const u32 b = blockIdx.x;
    const u32 q = b & 7u, idx = b >> 3;
    const u32 mb = 4 * (q >> 1) + (idx & 3), nb = 4 * (q & 1) + ((idx >> 2) & 3);
    const u32 rel = 1u + (b - 128u) % 7u, pj = ((b - 128u) / 7u) & 15u;
    char* wsl = p.ws[rank];
    const rsrc_t rs_ws = mk_rsrc(wsl, WS_NEED);
    volatile LAS u32* LW = (volatile LAS u32*)L;
#define LDSV(a) (*(const LAS u32x4*)(L + (a)))

    STAMP(0);
    const u32 seqv = __builtin_amdgcn_raw_buffer_load_b32(rs_ws, OFF_SEQ, 0, AGT);
    if (threadIdx.x < 32) LW[CTRL / 4 + threadIdx.x] = 0u;
    asm volatile("s_waitcnt lgkmcnt(0)" ::: "memory");
    __builtin_amdgcn_s_barrier();
    CBAR();

    auto put_unit = [&](const u32x4 (&o)[8], rsrc_t rs, bool isw, u32 lk, u32 ch, u32 r) -> u32 {
        const u32 so = STAGE_OFF + (lk * 8u + ch) * STEP_BYTES + (isw ? 16u * PIECE + r * 8192u : (r >> 2) * PIECE + (r & 3u) * 4096u);
        const u32 v0 = lane * 16u, v1 = isw ? (lane ^ 8u) * 16u + 4096u : lane * 16u + 16384u;
#pragma unroll
        for (int e = 0; e < 8; ++e) ST128(o[e], rs, (e < 4 ? v0 : v1) + (e & 3) * 1024u, so, SYS);
        return (isw ? OFF_WF : OFF_XF) + (r >> 2) * 1024u + (ch * 8u + lk) * 16u + (r & 3u) * 4u;
    };

    {
        u32 t = 0, tend = 0, delay = 0;
        if (b < 128u) { t = wid * 128u + b; if (t < GCH * 96u) tend = t + 1u; delay = t / 96u * TDELAY; }
        else if (b < 240u) {
            if (wid < 8u) { t = GCH * 96u + (wid - 2u) * 112u + (b - 128u); if (wid >= 2u && t < 768u) tend = t + 1u; delay = t / 96u * TDELAY; }
            else { t = 768u + 224u + (wid - 8u) * 112u + (b - 128u); tend = t + 1u; delay = wid < 10u ? WDELAY1 : WDELAY2; }
        } else { t = 768u + wid * 16u + (b - 240u); tend = 768u + 8u * 224u; }
        if (t < tend) {
            const u64 ts = now();
            while (now() - ts < (u64)delay) __builtin_amdgcn_s_sleep(4);
        }
        for (; t < tend; t += t - (768u + 32u) < 192u ? 192u + 448u : 192u) {
            u32 kind, lk = 0, ch, r;
            if (t < 768u) { ch = t / 96u; r = t % 96u; kind = r >> 6; r &= 63u; }
            else { ch = (t - 768u) / 224u; r = (t - 768u) % 224u; lk = 1u + r % 7u; r /= 7u; kind = ch == 0u ? 1u : 2u; }
            const u32 to = (rank + lk) & 7u;
            u32x4 o[8];
            if (kind == 0u) {
                const rsrc_t rs_x = mk_rsrc(p.x, X_BYTES);
                const u32 vx = (lane >> 2) * 1024u + (((lane & 3u) ^ ((lane >> 4) & 2u)) << 4);
#pragma unroll
                for (int i = 0; i < 4; ++i) {
                    o[i] = LD128(rs_x, vx, r * 65536u + ch * 128u + i * 16384u, 0);
                    o[4 + i] = LD128(rs_x, vx + 64u, r * 65536u + ch * 128u + i * 16384u, 0);
                }
            } else {
                const rsrc_t rs_w = mk_rsrc(p.w, W_BYTES);
                const u32 vw = (lane >> 4) * 8u * 32768u + (lane & 15u) * 16u;
                const u32 so = (ch * 64u + (r & 2u) * 16u) * 32768u + (to * 2048u + (r >> 2) * 256u + (r & 1u) * 128u) * 2u;
                u32x4 v[8];
#pragma unroll
                for (int i = 0; i < 8; ++i) v[i] = LD128(rs_w, vw, so + i * 32768u, 0);
#pragma unroll
                for (int e = 0; e < 8; ++e)
#pragma unroll
                    for (int k = 0; k < 4; ++k)
                        o[e][k] = PERM(v[2 * k + 1][e >> 1], v[2 * k][e >> 1], (e & 1) ? 0x07060302u : 0x05040100u);
            }
            if (kind != 1u) {
                const u32 slot = kind == 0u ? E1X_OFF + (ch * 64u + r) * E1_SLOT : E1W_OFF + ((lk * 8u + ch) * 32u + r) * E1_SLOT;
                u32x4 mq[4];
                u32 ex[16], mx = 0;
#pragma unroll
                for (int j = 0; j < 16; ++j) {
                    const u32 d0 = o[j >> 1][(2 * j) & 3], d1 = o[j >> 1][(2 * j + 1) & 3];
                    const u32 r0 = __builtin_amdgcn_alignbit(d0, d0, 31), r1 = __builtin_amdgcn_alignbit(d1, d1, 31);
                    mq[j >> 2][j & 3] = PERM(r1, r0, 0x06040200u);
                    ex[j] = PERM(r1, r0, 0x07050301u);
                    mx = __builtin_bit_cast(u32, __builtin_elementwise_max(__builtin_bit_cast(u16x2, mx),
                             __builtin_elementwise_max(__builtin_bit_cast(u16x2, r0), __builtin_bit_cast(u16x2, r1))));
                }
                const u32 top = wave_max_u32(max(mx >> 24, (mx >> 8) & 255u));
                u32 any = 0, cnt = 0;
#pragma unroll
                for (int j = 0; j < 16; ++j) { ex[j] = top * 0x01010101u - ex[j]; any |= ex[j]; }
                if (BALLOT((any & 0xF0F0F0F0u) != 0u) != 0ull) {
#pragma unroll
                    for (int j = 0; j < 16; ++j) {
                        const bool far = (ex[j] & 0xF0F0F0F0u) != 0u;
                        const u64 m = BALLOT(far);
                        if (m != 0ull) {
                            const u32 k = cnt + rank_in(m);
                            if (far && k < 7u)
                                __builtin_amdgcn_raw_buffer_store_b64(u32x2{lane * 16u + j, ex[j]}, rs_ws, slot + 8u + k * 8u, 0, AGT);
                            cnt += (u32)__builtin_popcountll(m);
                            ex[j] &= 0x0F0F0F0Fu;
                        }
                    }
                }
                if (FORCE_RAW1 && (r & 1u)) cnt = 8u;
                if (lane == 0) __builtin_amdgcn_raw_buffer_store_b64(u32x2{top, cnt}, rs_ws, slot, 0, AGT);
                if (cnt > 7u) {
#pragma unroll
                    for (int e = 0; e < 8; ++e) ST128(o[e], rs_ws, lane * 16u + 128u, slot + e * 1024u, AGT);
                } else {
#pragma unroll
                    for (int e = 0; e < 4; ++e) ST128(mq[e], rs_ws, lane * 16u + 128u + e * 1024u, slot, AGT);
#pragma unroll
                    for (int e = 0; e < 2; ++e) {
                        u32x4 n;
#pragma unroll
                        for (int k = 0; k < 4; ++k) n[k] = ex[8 * e + 2 * k] | (ex[8 * e + 2 * k + 1] << 4);
                        ST128(n, rs_ws, lane * 16u + 128u + e * 1024u, slot + 4096u, AGT);
                    }
                }
            }
            const rsrc_t rs_to = mk_rsrc(p.ws[to], WS_NEED);
            u32 flag = OOB;
            if (kind != 2u) flag = put_unit(o, rs_to, kind == 1u, lk, ch, r);
            const u32 sq = RFL(seqv) + 1u;
            u32 old = 0;
            if (kind != 1u) {
                if (kind == 0u) VMWAIT(8); else VMWAIT(0);
                if (lane == 0)
                    old = __hip_atomic_fetch_add((u32*)(wsl + OFF_CTR + (lk * 8u + ch) * 128u), 1u, __ATOMIC_RELAXED, __HIP_MEMORY_SCOPE_AGENT);
            }
            VMWAIT(0);
            if (lane == 0) __builtin_amdgcn_raw_buffer_store_b32(sq, rs_to, flag, 0, SYS);
            if (kind != 1u && RFL(old) + 1u == sq * (kind == 0u ? 64u : 32u) && lane - 1u < 7u && (kind == 0u || lane == lk))
                __hip_atomic_store((u32*)(p.ws[(rank + lane) & 7u] + OFF_F1 + lane * 128u + (ch * 2u + (kind == 0u)) * 4u), sq,
                                   __ATOMIC_RELAXED, __HIP_MEMORY_SCOPE_SYSTEM);
        }
    }
    const u32 seq = RFL(seqv) + 1u;
    STAMP(10);

    u32 spins = 0;
    u64 t0 = 0;
    auto expired = [&]() -> bool {
        if ((++spins & 63u) != 0u) return false;
        if (LW[L_BAD / 4] != 0u) return true;
        const u64 t = now();
        if (t0 == 0) t0 = t;
        if (t - t0 > DEADLINE || __builtin_amdgcn_raw_buffer_load_b32(rs_ws, OFF_TMO, 0, AGT) == seq) {
            u32 sq = seq;
            asm volatile("" : "+s"(sq));
            __builtin_amdgcn_raw_buffer_store_b32(sq, rs_ws, OFF_TMO, 0, AGT);
            LW[L_BAD / 4] = 1u;
            return true;
        }
        return false;
    };

    auto arrive = [&](u32 wm, u32 n) {
        if (lane == 0) {
            __hip_atomic_fetch_max((LAS u32*)(L + L_AMAX), wm, __ATOMIC_RELAXED, __HIP_MEMORY_SCOPE_WORKGROUP);
            const u32 old = __hip_atomic_fetch_add((LAS u32*)(L + L_ACNT), 1u, __ATOMIC_RELAXED, __HIP_MEMORY_SCOPE_WORKGROUP);
            if (old == n - 1u) {
                u32 a = LW[L_AMAX / 4];
                if (LW[L_BAD / 4] != 0u) a = 0xFFFFFFFFu;
                __builtin_amdgcn_raw_buffer_store_b64(u32x2{a, seq}, rs_ws, OFF_AMX + b * 8u, 0, AGT);
            }
        }
    };

    if (b < 128u && wid < 8u) {
        u32 c = lane & 15u, g = lane >> 4;
        const u32 mw = wid & 3u, nh = wid >> 2;
        u32 xa = mw * 4096u + (c * 4u + (g ^ ((c >> 2) & 2u))) * 16u;
        u32 wa = 16384u + nh * 8192u + (c & 1u) * 4096u + g * 256u + ((8u * (c & 1u) + (c >> 1)) << 4);
        f32x4 acc[8][4];
#pragma unroll
        for (int j = 0; j < 8; ++j)
#pragma unroll
            for (int m = 0; m < 4; ++m) acc[j][m] = f32x4{0.f, 0.f, 0.f, 0.f};
        bf16x8 wl[4], wh[4], xf[4];
#define LDS16(a) (*(const LAS bf16x8*)(L + (a)))
        while (LW[L_FULL / 4] < 1u) { if (expired()) break; __builtin_amdgcn_s_sleep(CSLEEP); }
        CBAR();
        STAMP(1);
        STAMPV(8, __builtin_amdgcn_s_memtime());
#define SB() __builtin_amdgcn_sched_barrier(0)
        SB();
        wl[0] = LDS16(wa); SB(); wl[1] = LDS16(wa + 1024); SB();
        xf[0] = LDS16(xa); SB(); xf[1] = LDS16(xa + 1024); SB(); xf[2] = LDS16(xa + 2048); SB();
        wl[2] = LDS16(wa + 2048); SB(); xf[3] = LDS16(xa + 3072); SB(); wl[3] = LDS16(wa + 3072); SB();
        auto ctl = [&]() -> u32 {
            u32 l;
            asm volatile("v_mbcnt_lo_u32_b32 %0, -1, 0" : "=v"(l));
            return *(volatile LAS u32*)(L + L_FULL + (l & 15u) * 4u);
        };
#define MM_(W, j, m) acc[j][m] = MFMA(W[(j) & 3], xf[m], acc[j][m])
        for (u32 f = 0; f < 128; ++f) {
            SB();
            MM_(wl, 0, 0); MM_(wl, 1, 0); SB(); MM_(wl, 0, 1); MM_(wl, 1, 1); SB(); MM_(wl, 0, 2); MM_(wl, 1, 2); SB(); MM_(wl, 0, 3);
            wh[0] = LDS16(wa ^ 128u); SB();
            MM_(wl, 1, 3);
            wh[1] = LDS16((wa ^ 128u) + 1024); SB();
            MM_(wl, 2, 0); MM_(wl, 2, 1); MM_(wl, 2, 2); MM_(wl, 2, 3);
            wh[2] = LDS16((wa ^ 128u) + 2048);
            u32 cw = ctl(); SB();
            MM_(wl, 3, 0); MM_(wl, 3, 1); MM_(wl, 3, 2); MM_(wl, 3, 3);
            wh[3] = LDS16((wa ^ 128u) + 3072);
            CBAR();
            u32 fa = L_FREE + wid * 4u;
            asm volatile("" : "+s"(fa));
            *(volatile LAS u32*)(L + fa) = f + 1;
            SB();
            MM_(wh, 4, 0); MM_(wh, 4, 1); MM_(wh, 4, 2); MM_(wh, 4, 3);
            SB();
            auto go = [&](u32 w) -> bool {
                const u32 full = (u32)BALLOT(w > ((f + 1) >> 2));
                return ((full >> ((f + 1) & 3u)) & 1u) != 0u || f == 127;
            };
            if (!go(cw)) {
                while (!go(ctl())) { if (expired()) break; __builtin_amdgcn_s_sleep(CSLEEP); }
            }
            CBAR();
            xa = (xa + SLOT_BYTES) & (CTRL - 1u);
            wa = (wa + SLOT_BYTES) & (CTRL - 1u);
            wl[0] = LDS16(wa); SB();
            MM_(wh, 5, 0); MM_(wh, 5, 1); MM_(wh, 5, 2); MM_(wh, 5, 3);
            wl[1] = LDS16(wa + 1024); SB();
            MM_(wh, 6, 0); MM_(wh, 7, 0);
            xf[0] = LDS16(xa); SB();
            MM_(wh, 6, 1); MM_(wh, 7, 1);
            xf[1] = LDS16(xa + 1024); SB();
            MM_(wh, 6, 2); MM_(wh, 7, 2);
            xf[2] = LDS16(xa + 2048); SB();
            MM_(wh, 6, 3);
            wl[2] = LDS16(wa + 2048); SB();
            MM_(wh, 7, 3);
            xf[3] = LDS16(xa + 3072);
            wl[3] = LDS16(wa + 3072); SB();
        }
        asm volatile("v_mbcnt_lo_u32_b32 %0, -1, 0\n v_mbcnt_hi_u32_b32 %0, -1, %0" : "=v"(lane));
        c = lane & 15u; g = lane >> 4;
        STAMP(2);
        STAMPV(9, __builtin_amdgcn_s_memtime());
        float mx = 0.f;
#pragma unroll
        for (int j = 0; j < 8; ++j)
#pragma unroll
            for (int m = 0; m < 4; ++m) {
                mx = fmaxf(fmaxf(mx, fabsf(acc[j][m][0])), fabsf(acc[j][m][1]));
                mx = fmaxf(fmaxf(mx, fabsf(acc[j][m][2])), fabsf(acc[j][m][3]));
            }
        arrive(wave_max_u32(__float_as_uint(mx)), 8u);
        STAMP(3);
        const bool direct = q != 0u && wid >= 8u - ND2;
        if (direct) __builtin_amdgcn_s_setprio(3); else if (wid < 4u) __builtin_amdgcn_s_setprio(2);
        while (LW[L_SCALE / 4] != 1u) { if (expired()) break; __builtin_amdgcn_s_sleep(CSLEEP); }
        CBAR();
        const u32 gmax = LW[L_SCALE / 4 + 1];
        const bool bad = (gmax == 0xFFFFFFFFu) || (LW[L_BAD / 4] != 0u);
        const float amax = __uint_as_float(gmax);
        const float inv = amax > 0.f ? 127.0f / amax : 0.f;
        STAMP(4);
        const u32 reg = wid * 4096u;
#pragma unroll
        for (int k = 0; k < 3; ++k) *(LAS u32x4*)(L + reg + k * 1024u + lane * 16u) = u32x4{0u, 0u, 0u, 0u};
        const float MAGIC = 12582912.0f;
        const u32 ovoff = (c & 7u) * NN + ((c >> 3) * 4u + g) * 16u;
        const u32 osoff = (mb * 256u + mw * 64u) * NN + rank * 2048u + nb * 256u + nh * 128u;
        const rsrc_t rs_or = mk_rsrc(p.out[(rank + q) & 7u], OUT_BYTES);
        u32x4 qv[8];
#pragma unroll
        for (int m = 0; m < 4; ++m) {
#pragma unroll
            for (int r = 0; r < 4; ++r) {
                u32 o[2];
#pragma unroll
                for (int h = 0; h < 2; ++h) {
                    u32 t[4];
#pragma unroll
                    for (int jj = 0; jj < 4; ++jj)
                        t[jj] = __float_as_uint(__builtin_amdgcn_fmed3f(__builtin_fmaf(acc[4 * h + jj][m][r], inv, MAGIC),
                                                                        MAGIC - 127.0f, MAGIC + 127.0f));
                    o[h] = PERM(t[1], t[0], 0x0c0c0400u) | PERM(t[3], t[2], 0x04000c0cu);
                    if (bad) o[h] = 0x80808080u;
                }
                qv[2 * m][r] = (u32)__builtin_amdgcn_update_dpp((int)o[0], (int)o[1], 0x128, 0xf, 0xc, false);
                qv[2 * m + 1][r] = (u32)__builtin_amdgcn_update_dpp((int)o[1], (int)o[0], 0x128, 0xf, 0x3, false);
            }
            if (direct) {
                ST128(qv[2 * m], rs_or, ovoff, osoff + (2 * m) * 8u * NN, SYS);
                ST128(qv[2 * m + 1], rs_or, ovoff, osoff + (2 * m + 1) * 8u * NN, SYS);
            }
        }
        if (direct) __builtin_amdgcn_s_setprio(0);
        STAMP(7);
        u32 size = 8192u;
        const u32 slot = E2_OFF + (b * 8u + wid) * 8192u;
        if (!(q == 0u && wid == 7u) && !(FORCE_RAW2 && (wid & 1u))) {
            u32x4 nib[4], sg = u32x4{0u, 0u, 0u, 0u};
            u32 code[16], len[16], over = 0, tot = 0;
#pragma unroll
            for (int o = 0; o < 16; ++o) {
                u32 f[2], ps[2];
#pragma unroll
                for (int e = 0; e < 2; ++e) {
                    const int i = 2 * o + e;
                    const u32 v = qv[i >> 2][i & 3], s = byte_signs(v);
                    f[e] = v ^ s;
                    sg[i >> 3] = bfi(0x01010101u << (i & 7), s, sg[i >> 3]);
                    u32 h = (f[e] >> 4) & 0x07070707u;
                    h += h << 8;
                    ps[e] = h + (h << 16);
                }
                nib[o >> 2][o & 3] = bfi(0x0F0F0F0Fu, f[0], f[1] << 4);
                ps[1] += 0x07060504u + PERM(ps[0], ps[0], 0x03030303u);
                ps[0] += 0x03020100u;
                u32 cd = 0;
#pragma unroll
                for (int k = 0; k < 8; ++k) cd |= 1u << ((ps[k >> 2] >> (8 * (k & 3))) & 31u);
                code[o] = cd;
                len[o] = (ps[1] >> 24) + 1u;
                over |= ps[1];
                tot += len[o];
            }
            const u32 end = wave_scan_u32(tot);
            const u32 bits = (u32)__builtin_amdgcn_readlane((int)end, 63);
            if (BALLOT((over & 0x20000000u) != 0u || tot > 128u + 255u) == 0ull && bits <= (8192u - 64u - R_BITS) * 8u) {
                size = (R_BITS + ((bits + 127u) >> 7) * 16u);
#pragma unroll
                for (int e = 0; e < 4; ++e) ST128(nib[e], rs_ws, lane * 16u + e * 1024u, slot, AGT);
                ST128(sg, rs_ws, lane * 16u, slot + 4096u, AGT);
                *(LAS char*)(L + reg + lane) = (char)(tot - 128u);
                u32 pos = end - tot + (reg + 64u) * 8u;
#pragma unroll
                for (int o = 0; o < 16; ++o) {
                    const u64 w = (u64)code[o] << (pos & 31u);
                    LAS u32* a = (LAS u32*)(L + ((pos >> 5) << 2));
                    __hip_atomic_fetch_or(a, (u32)w, __ATOMIC_RELAXED, __HIP_MEMORY_SCOPE_WORKGROUP);
                    __hip_atomic_fetch_or(a + 1, (u32)(w >> 32), __ATOMIC_RELAXED, __HIP_MEMORY_SCOPE_WORKGROUP);
                    pos += len[o];
                }
                CBAR();
#pragma unroll
                for (int k = 0; k < 3; ++k) {
                    const u32 off = R_LEN + k * 1024u + lane * 16u;
                    if (R_LEN + k * 1024u < size) ST128(LDSV(reg + k * 1024u + lane * 16u), rs_ws, off < size ? off : OOB, slot, AGT);
                }
            }
        }
        if (size == 8192u) {
#pragma unroll
            for (int e = 0; e < 8; ++e) ST128(qv[e], rs_ws, lane * 16u, slot + e * 1024u, AGT);
        }
        STAMP(11);
        VMWAIT(0);
        {
            u32 old = 0;
            if (lane == 0) {
                *(volatile LAS unsigned short*)(L + L_GSZ + wid * 2u) = (unsigned short)(size >> 4);
                old = __hip_atomic_fetch_add((LAS u32*)(L + L_GCNT + nh * 4u), 1u, __ATOMIC_RELAXED, __HIP_MEMORY_SCOPE_WORKGROUP);
            }
            if (RFL(old) == 3u && lane - 1u < 7u) {
                const u32x4 fv = u32x4{seq, LW[L_GSZ / 4 + nh * 2u], LW[L_GSZ / 4 + nh * 2u + 1u], seq};
                char* fp = p.ws[(rank + lane) & 7u] + OFF_F2 + ((lane * 16u + idx) * 16u + q * 2u + nh) * 16u;
                asm volatile("global_store_dwordx4 %0, %1, off sc0 sc1\n s_nop 1" :: "v"(fp), "v"(fv) : "memory");
            }
        }
        STAMP(5);
        const rsrc_t rs_ol = mk_rsrc(p.out[rank], OUT_BYTES);
#pragma unroll
        for (int e = 0; e < 8; ++e) ST128(qv[e], rs_ol, ovoff, osoff + e * 8u * NN, AGT);
    } else if (b < 128u && wid == 8u) {
        u64 taken = 0;
        u32 n = 0;
        while (n < 64u) {
            const u64 want = ~taken & ((taken << 16) | 0xFFFFull);
            const u32 vo = ((want >> lane) & 1ull) ? lane * 16u : OOB;
            const u32x4 fx = LD128(rs_ws, vo, OFF_XF + mb * 1024u, SYS);
            const u32x4 fw = LD128(rs_ws, vo, OFF_WF + nb * 1024u, SYS);
            const u32 mn = min(min(min(fx[0], fx[1]), min(fx[2], fx[3])), min(min(fw[0], fw[1]), min(fw[2], fw[3])));
            u64 avail = BALLOT(mn >= seq);
            if (avail == 0) {
                if (!expired()) { __builtin_amdgcn_s_sleep(POLL_SLEEP); continue; }
                avail = ~taken;
            }
            if ((avail >> lane) & 1ull) LW[L_STEPS / 4 + n + rank_in(avail)] = lane;
            n += (u32)__builtin_popcountll(avail);
            taken |= avail;
            LW[L_NRDY / 4] = n;
        }
        STAMP(1);
        u32 a = 0;
        for (;;) {
            const u32x2 gm = __builtin_amdgcn_raw_buffer_load_b64(rs_ws, OFF_GAM + q * 128u + (lane & 7u) * 8u, 0, SYS);
            a = gm[0];
            if (BALLOT(gm[1] == seq) == ~0ull) break;
            if (expired()) { a = 0xFFFFFFFFu; break; }
            __builtin_amdgcn_s_sleep(1);
        }
        a = wave_max_u32(a);
        LW[L_SCALE / 4 + 1] = a;
        LW[L_SCALE / 4] = 1u;
        STAMP(2);
    } else if (b < 128u) {
        for (u32 f = wid - 9u; f < 128u; f += 3u) {
            while (LW[L_NRDY / 4] <= (f >> 1)) { if (expired()) break; __builtin_amdgcn_s_sleep(1); }
            CBAR();
            const u32 step = RFL(LW[L_STEPS / 4 + (f >> 1)]) & 63u;
            if (f >= 4u) {
                for (;;) {
                    if (BALLOT(LW[L_FREE / 4 + (lane & 7u)] >= f - 3u) == ~0ull) break;
                    if (expired()) break;
                    __builtin_amdgcn_s_sleep(1);
                }
            }
            CBAR();
            const u32 sbase = STAGE_OFF + ((step & 7u) * 8u + (step >> 3)) * STEP_BYTES + (f & 1u) * 16384u;
            LAS char* ldst = L + (f & 3u) * SLOT_BYTES;
#pragma unroll
            for (int i = 0; i < DIAG_NDMA; ++i) {
                const u32 so = sbase + (i < 4 ? mb : 16u + nb) * PIECE + (i & 3) * 4096u;
                __builtin_amdgcn_raw_ptr_buffer_load_lds(rs_ws, ldst + i * 4096, 16, lane * 16u, so, 0, SYS);
                __builtin_amdgcn_raw_ptr_buffer_load_lds(rs_ws, ldst + i * 4096, 16, lane * 16u, so, 1024, SYS);
                __builtin_amdgcn_raw_ptr_buffer_load_lds(rs_ws, ldst + i * 4096, 16, lane * 16u, so, 2048, SYS);
                __builtin_amdgcn_raw_ptr_buffer_load_lds(rs_ws, ldst + i * 4096, 16, lane * 16u, so, 3072, SYS);
            }
            VMWAIT(0);
            LW[L_FULL / 4 + (f & 3u)] = (f >> 2) + 1u;
        }
        STAMP(1);
    } else if (b < 240u) {
        arrive(0u, 12u);
        const u32 src = (rank - rel) & 7u;
        const rsrc_t rs_src = mk_rsrc(p.ws[src], WS_NEED);
        auto in_unit = [&](u32 u, u32& ch, u32& r, bool& isw) -> u32 {
            ch = u / 96u; r = u % 96u; isw = r < 32u;
            if (!isw) r = (r - 32u + 8u * rel) & 63u;
            return isw ? E1W_OFF + ((rel * 8u + ch) * 32u + r) * E1_SLOT : E1X_OFF + (ch * 64u + r) * E1_SLOT;
        };
        auto out_unit = [&](u32 o) -> u32 {
            const u32 k = o >> 4;
            u32 t = k + (k >= 7u);
            if (t >= rel * 8u + 8u - ND2) t += ND2;
            return (o & 15u) * 64u + (k == N2 - 1u ? 7u : t);
        };
        if (wid == 0u) {
            __builtin_amdgcn_s_setprio(3);
            auto ctl = [&](u32 a) -> u32 {
                u32 v;
                asm volatile("ds_read_b32 %0, %1\n s_waitcnt lgkmcnt(0)" : "=v"(v) : "v"(a) : "memory");
                return v;
            };
            auto put = [&](u32 a, u32 n) { asm volatile("ds_write_b32 %0, %1" :: "v"(a), "v"(n) : "memory"); };
            const u32 ca = L_FULL + (lane & 15u) * 4u;
            const u32 vone = lane == 0u ? 0u : OOB;
            u32 n = 0;
            auto wait = [&](int word, u32 bit) {
                auto ok = [&](u32 w) -> bool {
                    return (((u32)__builtin_amdgcn_readlane((int)w, word) >> bit) & 1u) != 0u &&
                           (u32)__builtin_amdgcn_readlane((int)w, 8 + (n & 7u)) + 15u >= n;
                };
                if (!ok(ctl(ca))) {
                    VMWAIT(0);
                    put(L_FULL, n);
                    while (!ok(ctl(ca))) { if (expired()) break; __builtin_amdgcn_s_sleep(1); }
                }
            };
#define CLAIM(cl, so) asm volatile("v_mov_b32 %0, 1\n s_nop 4\n buffer_atomic_add %0, %1, %2, %3 offen sc0 sc1" \
                                   : "=&v"(cl) : "v"(vone), "s"(rs_ws), "s"(so) : "memory")
#define CLAIMED(d, cl) asm volatile("v_readfirstlane_b32 %0, %1" : "=s"(d) : "v"(cl) : "memory")
            for (u32 u = 32u + pj; u < 768u; ++n) {
                u32 ch, r, cl; bool isw;
                const u32 so = in_unit(u, ch, r, isw);
                wait(1, ch * 2u + !isw);
                if (n == 0) STAMP(1);
                LAS char* ld = L + (n & 15u) * 8192u;
                put(L_Q + (n & 15u) * 4u, u);
                CLAIM(cl, OFF_CTR + (64u + rel) * 128u);
#pragma unroll
                for (int i = 0; i < 7; ++i) {
                    VMWAIT(WIN1 - (i < WIN1 - 1 ? 0 : 1));
                    if (i == WIN1 - 1) put(L_FULL, n);
                    if (i == 0) { if (lane < 4u) __builtin_amdgcn_raw_ptr_buffer_load_lds(rs_src, ld + 6144, 16, lane * 16u, so, 0, SYS); }
                    else __builtin_amdgcn_raw_ptr_buffer_load_lds(rs_src, ld + (i - 1) * 1024, 16, lane * 16u + 128u, so + (i - 1) * 1024u, 0, SYS);
                }
                CLAIMED(u, cl);
                u = u - (seq - 1u) * 736u + 48u;
            }
            VMWAIT(0);
            put(L_FULL, n);
            put(L_P1, 1u);
            STAMP(2);
            for (u32 o = pj; o < 16u * N2; ++n) {
                const u32 t = out_unit(o), so = E2_OFF + t * 8192u;
                const bool dyn = o >= 16u * (S2 - 1u);
                u32 cl;
                wait(2, o < 16u * S2 ? (t >> 2) & 15u : 16u);
                if (o == pj) STAMP(3);
                LAS char* ld = L + (n & 15u) * 8192u;
                const u32 sz = min(RFL(ctl(L_SZ + (t >> 1) * 4u)) >> ((t & 1u) * 16u) & 0xFFFFu, 512u);
                put(L_Q + (n & 15u) * 4u, 1024u + t + (sz << 16));
                CLAIM(cl, dyn ? OFF_CTR + (72u + rel) * 128u : OOB);
#pragma unroll
                for (int i = 0; i < 8; ++i) {
                    if (i >= 6 && i * 64u >= sz) break;
                    VMWAIT(WIN2 - (i < WIN2 - 1 ? 0 : 1));
                    if (i == WIN2 - 1) put(L_FULL, n);
                    if (i < 6 || i * 64u + lane < sz)
                        __builtin_amdgcn_raw_ptr_buffer_load_lds(rs_src, ld + i * 1024, 16, lane * 16u, so + i * 1024u, 0, SYS);
                }
                u32 c;
                CLAIMED(c, cl);
                o = dyn ? c - (seq - 1u) * (16u * (N2 - S2 + 1u)) + 16u * S2 : o + 16u;
            }
            VMWAIT(0);
            put(L_FULL, n);
            put(L_END, 1u);
            STAMP(4);
            if (lane == 0) __builtin_amdgcn_raw_buffer_store_b32(seq, rs_src, OFF_DONE + rank * 128u + pj * 4u, 0, SYS);
        } else if (wid == 1u) {
            for (u32 m = 0; m != 0xFFFFu;) {
                const u32 f = __builtin_amdgcn_raw_buffer_load_b32(rs_ws, OFF_F1 + rel * 128u + (lane & 15u) * 4u, 0, SYS);
                m = ((u32)BALLOT(f >= seq) & 0xFFFFu) | 1u;
                if (expired()) m = 0xFFFFu;
                LW[L_SEG / 4] = m;
                __builtin_amdgcn_s_sleep(4);
            }
            STAMP(1);
            while (LW[L_P1 / 4] == 0u) { if (expired()) break; __builtin_amdgcn_s_sleep(16); }
            u32 m = 0;
            while (m != 0xFFFFu) {
                const u32x4 f = LD128(rs_ws, lane < 16u ? lane * 16u : OOB, OFF_F2 + (rel * 16u + pj) * 256u, SYS);
                m = (u32)BALLOT(f[0] == seq && f[3] == seq) & 0xFFFFu;
                if (lane < 16u) *(LAS u32x2*)(L + L_SZ + pj * 128u + lane * 8u) = u32x2{f[1], f[2]};
                if (expired()) m = 0xFFFFu;
                CBAR();
                LW[L_G2 / 4] = m;
                __builtin_amdgcn_s_sleep(2);
            }
            STAMP(2);
            for (u32 i = 0; i < 4u;) {
                const u32x4 f = LD128(rs_ws, lane * 16u, OFF_F2 + rel * 4096u + i * 1024u, SYS);
                *(LAS u32x2*)(L + L_SZ + i * 512u + lane * 8u) = u32x2{f[1], f[2]};
                if (BALLOT(f[0] == seq && f[3] == seq) == ~0ull || expired()) ++i; else __builtin_amdgcn_s_sleep(16);
            }
            CBAR();
            LW[L_G2 / 4] = 0x1FFFFu;
            STAMP(3);
        } else if (wid < 10u) {
            const rsrc_t rs_ol = mk_rsrc(p.out[rank], OUT_BYTES);
            const u32 ovoff = (lane & 7u) * NN + (((lane >> 3) & 1u) * 4u + (lane >> 4)) * 16u;
            for (u32 n = wid - 2u;; n += 8u) {
                bool more = true;
                for (;;) {
                    const u32 end = LW[L_END / 4];
                    if (LW[L_FULL / 4] > n) break;
                    if (end != 0u) { more = false; break; }
                    expired();
                    __builtin_amdgcn_s_sleep(2);
                }
                if (!more) break;
                CBAR();
                const u32 sl = (n & 15u) * 8192u;
                const u32 id = RFL(LW[L_Q / 4 + (n & 15u)]);
                if (id < 1024u) {
                u32 ch, r; bool isw;
                const u32 so = in_unit(id, ch, r, isw);
                const u32 top = RFL(LW[(sl + 6144u) / 4]), cnt = RFL(LW[(sl + 6144u) / 4 + 1u]);
                u32x4 o[8];
                if (cnt > 7u) {
                    o[6] = LD128(rs_src, lane * 16u + 128u, so + 6144u, SYS);
                    o[7] = LD128(rs_src, lane * 16u + 128u, so + 7168u, SYS);
#pragma unroll
                    for (int e = 0; e < 6; ++e) o[e] = LDSV(sl + e * 1024u + lane * 16u);
                } else {
                    u32x4 mq[4];
                    u32 ex[16];
#pragma unroll
                    for (int e = 0; e < 4; ++e) mq[e] = LDSV(sl + e * 1024u + lane * 16u);
#pragma unroll
                    for (int e = 0; e < 2; ++e) {
                        const u32x4 v = LDSV(sl + 4096u + e * 1024u + lane * 16u);
#pragma unroll
                        for (int k = 0; k < 4; ++k) { ex[8 * e + 2 * k] = v[k] & 0x0F0F0F0Fu; ex[8 * e + 2 * k + 1] = (v[k] >> 4) & 0x0F0F0F0Fu; }
                    }
                    for (u32 k = 0; k < cnt; ++k) {
                        const u32 at = LW[(sl + 6144u) / 4 + 2u + 2u * k] - lane * 16u, val = LW[(sl + 6144u) / 4 + 3u + 2u * k];
#pragma unroll
                        for (int j = 0; j < 16; ++j) ex[j] = at == (u32)j ? val : ex[j];
                    }
#pragma unroll
                    for (int j = 0; j < 16; ++j) {
                        const u32 e = top * 0x01010101u - ex[j], m = mq[j >> 2][j & 3];
                        const u32 r0 = PERM(e, m, 0x05010400u), r1 = PERM(e, m, 0x07030602u);
                        o[j >> 1][(2 * j) & 3] = __builtin_amdgcn_alignbit(r0, r0, 1);
                        o[j >> 1][(2 * j + 1) & 3] = __builtin_amdgcn_alignbit(r1, r1, 1);
                    }
                }
                CBAR();
                LW[L_FREE / 4 + (n & 7u)] = n + 1u;
                const u32 flag = put_unit(o, rs_ws, isw, rel, ch, r);
                VMWAIT(0);
                if (lane == 0) __builtin_amdgcn_raw_buffer_store_b32(seq, rs_ws, flag, 0, SYS);
                STAMP(1);
                continue;
                }
                const u32 sz = id >> 16, x = (id >> 3) & 7u, ix = (id >> 6) & 15u;
                const u32 osoff = ((4u * (x >> 1) + (ix & 3u)) * 256u + (id & 3u) * 64u) * NN + src * 2048u +
                                  (4u * (x & 1u) + (ix >> 2)) * 256u + ((id >> 2) & 1u) * 128u;
                u32x4 qv[8];
                if (sz >= 512u) {
#pragma unroll
                    for (int e = 0; e < 8; ++e) qv[e] = LDSV(sl + e * 1024u + lane * 16u);
                } else {
                    const u32 tot = (u32)*(volatile LAS unsigned char*)(L + sl + R_LEN + lane) + 128u;
                    u32x4 nib[4];
#pragma unroll
                    for (int e = 0; e < 4; ++e) nib[e] = LDSV(sl + e * 1024u + lane * 16u);
                    const u32x4 sg = LDSV(sl + 4096u + lane * 16u);
                    u32 pos = wave_scan_u32(tot) - tot + (sl + R_BITS) * 8u;
#pragma unroll
                    for (int o = 0; o < 16; ++o) {
                        const LAS u32* a = (const LAS u32*)(L + ((pos >> 5) << 2));
                        u32 w = __builtin_amdgcn_alignbit(a[1], a[0], pos);
                        u32 ps[2] = {0u, 0u};
#pragma unroll
                        for (int k = 0; k < 8; ++k) {
                            ps[k >> 2] |= ((u32)__builtin_ctz(w) & 255u) << (8 * (k & 3));
                            w &= w - 1u;
                        }
                        pos += (ps[1] >> 24) + 1u;
                        const u32 h0 = ps[0] - ((ps[0] << 8) + 0x01010100u);
                        const u32 h1 = ps[1] - (__builtin_amdgcn_alignbit(ps[1], ps[0], 24) + 0x01010101u);
                        const u32 nv = nib[o >> 2][o & 3], s = sg[o >> 2] << (6 - 2 * (o & 3));
                        qv[o >> 1][(2 * o) & 3] = bfi(0x0F0F0F0Fu, nv, h0 << 4) ^ byte_signs(s << 1);
                        qv[o >> 1][(2 * o + 1) & 3] = bfi(0x0F0F0F0Fu, nv >> 4, h1 << 4) ^ byte_signs(s);
                    }
                }
                CBAR();
                LW[L_FREE / 4 + (n & 7u)] = n + 1u;
                if (LW[L_BAD / 4] != 0u) qv[0] = u32x4{0x80808080u, 0x80808080u, 0x80808080u, 0x80808080u};
#pragma unroll
                for (int e = 0; e < 8; ++e) ST128(qv[e], rs_ol, ovoff, osoff + e * 8u * NN, AGT);
            }
            STAMP(2);
        }
    } else {
        arrive(0u, 12u);
        if (b == AUX_WG && wid == 2u) {
            u32 a = 0;
            for (;;) {
                const u32x4 g0 = LD128(rs_ws, OFF_AMX + lane * 32u, 0, AGT);
                const u32x4 g1 = LD128(rs_ws, OFF_AMX + lane * 32u + 16u, 0, AGT);
                a = max(max(g0[0], g0[2]), max(g1[0], g1[2]));
                const bool ok = g0[1] == seq && g0[3] == seq && g1[1] == seq && g1[3] == seq;
                if (BALLOT(ok) == ~0ull) break;
                if (expired()) { a = 0xFFFFFFFFu; break; }
            }
            a = wave_max_u32(a);
            u64* gp = (u64*)(p.ws[lane & 7u] + OFF_GAM + (lane >> 3) * 128u + rank * 8u);
            __hip_atomic_store(gp, ((u64)seq << 32) | a, __ATOMIC_RELAXED, __HIP_MEMORY_SCOPE_SYSTEM);
            if (lane == 0) __builtin_amdgcn_raw_buffer_store_b32(seq, rs_ws, OFF_SEQ, 0, AGT);
            STAMP(3);
            bool give_up = false;
            for (;;) {
                const u32x4 f = LD128(rs_ws, OFF_DONE + lane * 16u, 0, SYS);
                const bool ok = (lane >> 3) == rank || (lane & 4u) != 0u || (f[0] >= seq && f[1] >= seq && f[2] >= seq && f[3] >= seq);
                if (BALLOT(ok) == ~0ull) break;
                if (expired()) { give_up = true; break; }
                __builtin_amdgcn_s_sleep(8);
            }
            if (give_up || __builtin_amdgcn_raw_buffer_load_b32(rs_ws, OFF_TMO, 0, AGT) == seq)
                if (lane == 0) p.out[rank][rank * 2048u] = (char)-128;
            STAMP(7);
        }
    }
}

extern "C" void kernel_launch(int rank, int nranks, ncclComm_t comm, hipStream_t stream,
                              void* const* d_in, const int* in_sizes, int n_in,
                              void* d_out, int out_size, const int* in_dtypes, int out_dtype,
                              void* const* d_out_all, void* const* d_ws_all, size_t ws_bytes) {
    if (nranks != NR || (unsigned)rank >= (unsigned)NR || n_in < 2 || ws_bytes < 3ull * WS_NEED ||
        in_sizes[0] != (int)(MM * KSH) || in_sizes[1] != (int)(KSH * NN) || out_size != (int)OUT_BYTES) {
        fprintf(stderr, "kernel_launch: unexpected problem shape\n");
        return;
    }
    static bool init = false;
    if (!init) {
        HIP_CHECK(hipFuncSetAttribute((const void*)gemm_ar_q8, hipFuncAttributeMaxDynamicSharedMemorySize, LDS_BYTES));
        init = true;
    }
    Params p;
    p.x = (const char*)d_in[0];
    p.w = (const char*)d_in[1];
    for (int r = 0; r < NR; ++r) { p.out[r] = (char*)d_out_all[r]; p.ws[r] = (char*)d_ws_all[r]; }
    p.rank = rank;
    hipLaunchKernelGGL(gemm_ar_q8, dim3(256), dim3(768), LDS_BYTES, stream, p);
    HIP_CHECK(hipGetLastError());
}
```

```cpp
#include <hip/hip_runtime.h>
#include <rccl/rccl.h>
#include <cstdio>
#include <cstdint>

#define HIP_CHECK(expr) do { hipError_t _e = (expr); if (_e != hipSuccess) { \
    fprintf(stderr, "HIP error %s at %s:%d\n", hipGetErrorString(_e), __FILE__, __LINE__); \
    return; } } while (0)

#define LAS __attribute__((address_space(3)))
typedef unsigned int u32;
typedef unsigned long long u64;
typedef u32 u32x4 __attribute__((ext_vector_type(4)));
typedef u32 u32x2 __attribute__((ext_vector_type(2)));
typedef unsigned short u16x2 __attribute__((ext_vector_type(2)));
typedef short bf16x8 __attribute__((ext_vector_type(8)));
typedef float f32x4 __attribute__((ext_vector_type(4)));

#define WIN1 4
#define WIN2 4
#define ND2 2
#define GCH 2
#define TDELAY 100
#define WDELAY1 200
#define WDELAY2 900
#define S2 16
#define DIAG_NDMA 8
#define CSLEEP 1
#define POLL_SLEEP 2
#define LEAD 8
#define FORCE_RAW1 0
#define FORCE_RAW2 0

constexpr int NR = 8;
constexpr u32 MM = 4096, NN = 16384, KSH = 512;
constexpr u32 X_BYTES = MM * KSH * 2, W_BYTES = KSH * NN * 2, OUT_BYTES = MM * NN;
constexpr u32 PIECE = 32768;
constexpr u32 STEP_BYTES = 28 * PIECE;
constexpr u32 STAGE_OFF = 1u << 20;
constexpr u32 E1_SLOT = 8192 + 128;
constexpr u32 E1X_OFF = STAGE_OFF + 64 * STEP_BYTES;
constexpr u32 E1W_OFF = E1X_OFF + 512 * E1_SLOT;
constexpr u32 E2_OFF = E1W_OFF + 2048 * E1_SLOT;
constexpr u32 WS_NEED = E2_OFF + 1024 * 8192;
constexpr u32 OFF_SEQ = 0, OFF_TMO = 256;
constexpr u32 OFF_XF = 4096;
constexpr u32 OFF_WF = OFF_XF + 16384;
constexpr u32 OFF_AMX = OFF_WF + 12288;
constexpr u32 OFF_GAM = OFF_AMX + 2048;
constexpr u32 OFF_DONE = OFF_GAM + 1024;
constexpr u32 OFF_F1 = OFF_DONE + 1024;
constexpr u32 OFF_CTR = OFF_F1 + 1024;
constexpr u32 OFF_F2 = 65536;
constexpr u32 OFF_STAMP = 131072;
static_assert(OFF_CTR + 81 * 128 <= OFF_F2 && OFF_F2 + 32768 <= OFF_STAMP && OFF_STAMP + 256 * 12 * 16 * 8 <= STAGE_OFF, "flags");

constexpr u32 SLOT_BYTES = 32768;
constexpr u32 CTRL = 4 * SLOT_BYTES;
constexpr u32 L_FULL = CTRL, L_FREE = CTRL + 32, L_NRDY = CTRL + 64, L_AMAX = CTRL + 68, L_ACNT = CTRL + 72,
              L_BAD = CTRL + 76, L_SCALE = CTRL + 80, L_GCNT = CTRL + 88, L_GSZ = CTRL + 96, L_STEPS = CTRL + 128;
constexpr u32 L_SEG = CTRL + 4, L_G2 = CTRL + 8, L_END = CTRL + 12, L_P1 = CTRL + 16, L_Q = CTRL + 128, L_SZ = CTRL + 256;
constexpr u32 L_PACK = CTRL + 1024;
constexpr u32 LDS_BYTES = L_PACK + 8 * 3072;
constexpr u64 DEADLINE = 100000000ull;
constexpr u32 OOB = WS_NEED;
constexpr int SYS = 17, AGT = 16;
constexpr u32 AUX_WG = 248;
constexpr u32 N1 = 768 + 112;
constexpr u32 N2 = 60 - ND2;
constexpr u32 R_LEN = 5120, R_BITS = 5184;

struct Params {
    const char* x;
    const char* w;
    char* out[NR];
    char* ws[NR];
    int rank;
};

typedef __amdgpu_buffer_rsrc_t rsrc_t;
__device__ __forceinline__ rsrc_t mk_rsrc(const void* p, u32 bytes) {
    return __builtin_amdgcn_make_buffer_rsrc((void*)p, 0, bytes, 0x00020000);
}
__device__ __forceinline__ u64 now() { return __builtin_amdgcn_s_memrealtime(); }
#define VMWAIT(n) asm volatile("s_waitcnt vmcnt(%0)" :: "n"(n) : "memory")
#define CBAR() asm volatile("" ::: "memory")
#define ST128(v, rs, voff, soff, aux) do { const u32x4 d_ = (v); \
    __builtin_amdgcn_raw_buffer_store_b128(d_, rs, voff, soff, aux); \
    asm volatile("s_nop 1" :: "v"(d_) : "memory"); } while (0)
#define LD128(rs, voff, soff, aux) __builtin_amdgcn_raw_buffer_load_b128(rs, voff, soff, aux)
#define MFMA(a, b, c) __builtin_amdgcn_mfma_f32_16x16x32_bf16(a, b, c, 0, 0, 0)
#define PERM(a, b, s) __builtin_amdgcn_perm(a, b, s)
#define BALLOT(c) __builtin_amdgcn_ballot_w64(c)
#define RFL(v) ((u32)__builtin_amdgcn_readfirstlane(v))
__device__ __forceinline__ u32 bfi(u32 m, u32 a, u32 b) { return (a & m) | (b & ~m); }
__device__ __forceinline__ u32 rank_in(u64 m) {
    return __builtin_amdgcn_mbcnt_hi((u32)(m >> 32), __builtin_amdgcn_mbcnt_lo((u32)m, 0u));
}

#define STAMPV(i, v) do { } while (0)
#define STAMP(i) STAMPV(i, now())

#define DPP_STEPS(OP) \
    OP(0x111, 0xf, true) OP(0x112, 0xf, true) OP(0x114, 0xf, true) OP(0x118, 0xf, true) OP(0x142, 0xa, false) OP(0x143, 0xc, false)
__device__ __forceinline__ u32 wave_max_u32(u32 v) {
#define OP_(c, rm, bc) v = max(v, (u32)__builtin_amdgcn_update_dpp(0, (int)v, c, rm, 0xf, bc));
    DPP_STEPS(OP_)
#undef OP_
    return (u32)__builtin_amdgcn_readlane((int)v, 63);
}
__device__ __forceinline__ u32 wave_scan_u32(u32 v) {
#define OP_(c, rm, bc) v += (u32)__builtin_amdgcn_update_dpp(0, (int)v, c, rm, 0xf, bc);
    DPP_STEPS(OP_)
#undef OP_
    return v;
}
__device__ __forceinline__ u32 byte_signs(u32 a) { return PERM(a, a << 8, 0x0B090A08u); }

__global__ void __launch_bounds__(768, 3) gemm_ar_q8(Params p) {
    extern __shared__ __attribute__((aligned(16))) char smem[];
    LAS char* L = (LAS char*)smem;
    u32 lane = threadIdx.x & 63;
    const u32 wid = RFL(threadIdx.x >> 6);
    const u32 rank = (u32)p.rank & 7u;
    const u32 b = blockIdx.x;
    const u32 q = b & 7u, idx = b >> 3;
    const u32 mb = 4 * (q >> 1) + (idx & 3), nb = 4 * (q & 1) + ((idx >> 2) & 3);
    const u32 nb2 = 8u + 2u * (q & 1u) + ((idx >> 2) & 1u), fills = idx < 8u ? 256u : 128u;
    const u32 rel = 1u + (b - 128u) % 7u, pj = ((b - 128u) / 7u) & 15u;
    char* wsl = p.ws[rank];
    const rsrc_t rs_ws = mk_rsrc(wsl, WS_NEED);
    volatile LAS u32* LW = (volatile LAS u32*)L;
#define LDSV(a) (*(const LAS u32x4*)(L + (a)))

    STAMP(0);
    const u32 seqv = __builtin_amdgcn_raw_buffer_load_b32(rs_ws, OFF_SEQ, 0, AGT);
    if (threadIdx.x < 32) LW[CTRL / 4 + threadIdx.x] = 0u;
    asm volatile("s_waitcnt lgkmcnt(0)" ::: "memory");
    __builtin_amdgcn_s_barrier();
    CBAR();

    auto put_unit = [&](const u32x4 (&o)[8], rsrc_t rs, bool isw, u32 lk, u32 ch, u32 r) -> u32 {
        const u32 so = STAGE_OFF + (lk * 8u + ch) * STEP_BYTES + (isw ? 16u * PIECE + r * 8192u : (r >> 2) * PIECE + (r & 3u) * 4096u);
        const u32 v0 = lane * 16u, v1 = isw ? (lane ^ 8u) * 16u + 4096u : lane * 16u + 16384u;
#pragma unroll
        for (int e = 0; e < 8; ++e) ST128(o[e], rs, (e < 4 ? v0 : v1) + (e & 3) * 1024u, so, SYS);
        return (isw ? OFF_WF : OFF_XF) + (r >> 2) * 1024u + (ch * 8u + lk) * 16u + (r & 3u) * 4u;
    };

    auto strip_at = [](u32 os, u32 r) -> u32 { return 32u + (os >> 1) * 4u + (r & 2u) + (os & 1u); };

    {
        u32 t = 0, tend = 0, delay = 0;
        if (b < 128u) { t = wid * 128u + b; if (t < GCH * 96u) tend = t + 1u; delay = t / 96u * TDELAY; }
        else if (b < 240u) {
            if (wid < 8u) { t = GCH * 96u + (wid - 2u) * 112u + (b - 128u); if (wid >= 2u && t < 768u) tend = t + 1u; delay = t / 96u * TDELAY; }
            else { t = 768u + 224u + (wid - 8u) * 112u + (b - 128u); tend = t + 1u; delay = wid < 10u ? WDELAY1 : WDELAY2; }
        } else { t = 768u + wid * 16u + (b - 240u); tend = 768u + 8u * 224u; }
        if (t < tend) {
            const u64 ts = now();
            while (now() - ts < (u64)delay) __builtin_amdgcn_s_sleep(4);
        }
        for (; t < tend; t += t - (768u + 32u) < 192u ? 192u + 448u : 192u) {
            u32 kind, lk = 0, ch, r;
            if (t < 768u) { ch = t / 96u; r = t % 96u; kind = r >> 6; r &= 63u; }
            else { ch = (t - 768u) / 224u; r = (t - 768u) % 224u; lk = 1u + r % 7u; r /= 7u; kind = ch == 0u ? 1u : 2u; }
            const u32 to = (rank + lk) & 7u;
            const bool strip = kind != 0u && (r | 2u) == 30u;
            u32x4 o[8];
            if (kind == 0u) {
                const rsrc_t rs_x = mk_rsrc(p.x, X_BYTES);
                const u32 vx = (lane >> 2) * 1024u + (((lane & 3u) ^ ((lane >> 4) & 2u)) << 4);
#pragma unroll
                for (int i = 0; i < 4; ++i) {
                    o[i] = LD128(rs_x, vx, r * 65536u + ch * 128u + i * 16384u, 0);
                    o[4 + i] = LD128(rs_x, vx + 64u, r * 65536u + ch * 128u + i * 16384u, 0);
                }
            } else {
                const rsrc_t rs_w = mk_rsrc(p.w, W_BYTES);
                const u32 vw = (lane >> 4) * 8u * 32768u + (lane & 15u) * 16u;
                const u32 so = (ch * 64u + (r & 2u) * 16u) * 32768u + (to * 2048u + (r >> 2) * 256u + (r & 1u) * 128u) * 2u;
                u32x4 v[8];
#pragma unroll
                for (int i = 0; i < 8; ++i) v[i] = LD128(rs_w, vw, so + i * 32768u, 0);
#pragma unroll
                for (int e = 0; e < 8; ++e)
#pragma unroll
                    for (int k = 0; k < 4; ++k)
                        o[e][k] = PERM(v[2 * k + 1][e >> 1], v[2 * k][e >> 1], (e & 1) ? 0x07060302u : 0x05040100u);
            }
            if (kind != 1u || strip) {
                const u32 slot = kind == 0u ? E1X_OFF + (ch * 64u + r) * E1_SLOT : E1W_OFF + ((lk * 8u + ch) * 32u + r) * E1_SLOT;
                u32x4 mq[4];
                u32 ex[16], mx = 0;
#pragma unroll
                for (int j = 0; j < 16; ++j) {
                    const u32 d0 = o[j >> 1][(2 * j) & 3], d1 = o[j >> 1][(2 * j + 1) & 3];
                    const u32 r0 = __builtin_amdgcn_alignbit(d0, d0, 31), r1 = __builtin_amdgcn_alignbit(d1, d1, 31);
                    mq[j >> 2][j & 3] = PERM(r1, r0, 0x06040200u);
                    ex[j] = PERM(r1, r0, 0x07050301u);
                    mx = __builtin_bit_cast(u32, __builtin_elementwise_max(__builtin_bit_cast(u16x2, mx),
                             __builtin_elementwise_max(__builtin_bit_cast(u16x2, r0), __builtin_bit_cast(u16x2, r1))));
                }
                const u32 top = wave_max_u32(max(mx >> 24, (mx >> 8) & 255u));
                u32 any = 0, cnt = 0;
#pragma unroll
                for (int j = 0; j < 16; ++j) { ex[j] = top * 0x01010101u - ex[j]; any |= ex[j]; }
                if (BALLOT((any & 0xF0F0F0F0u) != 0u) != 0ull) {
#pragma unroll
                    for (int j = 0; j < 16; ++j) {
                        const bool far = (ex[j] & 0xF0F0F0F0u) != 0u;
                        const u64 m = BALLOT(far);
                        if (m != 0ull) {
                            const u32 k = cnt + rank_in(m);
                            if (far && k < 7u)
                                __builtin_amdgcn_raw_buffer_store_b64(u32x2{lane * 16u + j, ex[j]}, rs_ws, slot + 8u + k * 8u, 0, AGT);
                            cnt += (u32)__builtin_popcountll(m);
                            ex[j] &= 0x0F0F0F0Fu;
                        }
                    }
                }
                if (FORCE_RAW1 && (r & 1u)) cnt = 8u;
                if (lane == 0) __builtin_amdgcn_raw_buffer_store_b64(u32x2{top, cnt}, rs_ws, slot, 0, AGT);
                if (cnt > 7u) {
#pragma unroll
                    for (int e = 0; e < 8; ++e) ST128(o[e], rs_ws, lane * 16u + 128u, slot + e * 1024u, AGT);
                } else {
#pragma unroll
                    for (int e = 0; e < 4; ++e) ST128(mq[e], rs_ws, lane * 16u + 128u + e * 1024u, slot, AGT);
#pragma unroll
                    for (int e = 0; e < 2; ++e) {
                        u32x4 n;
#pragma unroll
                        for (int k = 0; k < 4; ++k) n[k] = ex[8 * e + 2 * k] | (ex[8 * e + 2 * k + 1] << 4);
                        ST128(n, rs_ws, lane * 16u + 128u + e * 1024u, slot + 4096u, AGT);
                    }
                }
            }
            const rsrc_t rs_to = mk_rsrc(p.ws[to], WS_NEED);
            u32 flag = OOB, flag2 = OOB;
            if (kind != 2u) flag = put_unit(o, rs_to, kind == 1u, lk, ch, r);
            if (strip && lk != 0u) flag2 = put_unit(o, rs_ws, true, 0u, ch, strip_at(lk - 1u, r));
            const u32 sq = RFL(seqv) + 1u;
            u32 old = 0;
            if (kind == 0u) VMWAIT(8); else VMWAIT(0);
            if (lane == 0 ? kind != 1u : (lane == 1u && strip))
                old = 1u + __hip_atomic_fetch_add((u32*)(wsl + OFF_CTR + (lane == 0 ? lk * 8u + ch : 80u) * 128u), 1u, __ATOMIC_RELAXED,
                                                  __HIP_MEMORY_SCOPE_AGENT);
            VMWAIT(0);
            if (lane == 0) {
                __builtin_amdgcn_raw_buffer_store_b32(sq, rs_to, flag, 0, SYS);
                __builtin_amdgcn_raw_buffer_store_b32(sq, rs_ws, flag2, 0, SYS);
            }
            u32* f1 = (u32*)(p.ws[(rank + lane) & 7u] + OFF_F1 + lane * 128u);
            if (RFL(old) == sq * (kind == 0u ? 64u : 32u) && lane - 1u < 7u && (kind == 0u || lane == lk))
                __hip_atomic_store(f1 + ch * 2u + (kind == 0u), sq, __ATOMIC_RELAXED, __HIP_MEMORY_SCOPE_SYSTEM);
            if ((u32)__builtin_amdgcn_readlane((int)old, 1) == sq * 128u && lane - 1u < 7u)
                __hip_atomic_store(f1, sq, __ATOMIC_RELAXED, __HIP_MEMORY_SCOPE_SYSTEM);
        }
    }
    const u32 seq = RFL(seqv) + 1u;
    STAMP(10);

    u32 spins = 0;
    u64 t0 = 0;
    auto expired = [&]() -> bool {
        if ((++spins & 63u) != 0u) return false;
        if (LW[L_BAD / 4] != 0u) return true;
        const u64 t = now();
        if (t0 == 0) t0 = t;
        if (t - t0 > DEADLINE || __builtin_amdgcn_raw_buffer_load_b32(rs_ws, OFF_TMO, 0, AGT) == seq) {
            u32 sq = seq;
            asm volatile("" : "+s"(sq));
            __builtin_amdgcn_raw_buffer_store_b32(sq, rs_ws, OFF_TMO, 0, AGT);
            LW[L_BAD / 4] = 1u;
            return true;
        }
        return false;
    };

    auto arrive = [&](u32 wm, u32 n) {
        if (lane == 0) {
            __hip_atomic_fetch_max((LAS u32*)(L + L_AMAX), wm, __ATOMIC_RELAXED, __HIP_MEMORY_SCOPE_WORKGROUP);
            const u32 old = __hip_atomic_fetch_add((LAS u32*)(L + L_ACNT), 1u, __ATOMIC_RELAXED, __HIP_MEMORY_SCOPE_WORKGROUP);
            if (old == n - 1u) {
                u32 a = LW[L_AMAX / 4];
                if (LW[L_BAD / 4] != 0u) a = 0xFFFFFFFFu;
                __builtin_amdgcn_raw_buffer_store_b64(u32x2{a, seq}, rs_ws, OFF_AMX + b * 8u, 0, AGT);
            }
        }
    };

    if (b < 128u && wid < 8u) {
        u32 c = lane & 15u, g = lane >> 4;
        const u32 mw = wid & 3u, nh = wid >> 2;
        u32 xa = mw * 4096u + (c * 4u + (g ^ ((c >> 2) & 2u))) * 16u;
        u32 wa = 16384u + nh * 8192u + (c & 1u) * 4096u + g * 256u + ((8u * (c & 1u) + (c >> 1)) << 4);
        for (u32 f0 = 0; f0 < fills; f0 += 128u) {
        f32x4 acc[8][4];
#pragma unroll
        for (int j = 0; j < 8; ++j)
#pragma unroll
            for (int m = 0; m < 4; ++m) acc[j][m] = f32x4{0.f, 0.f, 0.f, 0.f};
        bf16x8 wl[4], wh[4], xf[4];
#define LDS16(a) (*(const LAS bf16x8*)(L + (a)))
        while (LW[L_FULL / 4] <= (f0 >> 2)) { if (expired()) break; __builtin_amdgcn_s_sleep(CSLEEP); }
        CBAR();
        if (f0 == 0u) { STAMP(1); STAMPV(8, __builtin_amdgcn_s_memtime()); } else STAMP(12);
#define SB() __builtin_amdgcn_sched_barrier(0)
        SB();
        wl[0] = LDS16(wa); SB(); wl[1] = LDS16(wa + 1024); SB();
        xf[0] = LDS16(xa); SB(); xf[1] = LDS16(xa + 1024); SB(); xf[2] = LDS16(xa + 2048); SB();
        wl[2] = LDS16(wa + 2048); SB(); xf[3] = LDS16(xa + 3072); SB(); wl[3] = LDS16(wa + 3072); SB();
        auto ctl = [&]() -> u32 {
            u32 l;
            asm volatile("v_mbcnt_lo_u32_b32 %0, -1, 0" : "=v"(l));
            return *(volatile LAS u32*)(L + L_FULL + (l & 15u) * 4u);
        };
#define MM_(W, j, m) acc[j][m] = MFMA(W[(j) & 3], xf[m], acc[j][m])
        for (u32 f = f0; f < f0 + 128u; ++f) {
            SB();
            MM_(wl, 0, 0); MM_(wl, 1, 0); SB(); MM_(wl, 0, 1); MM_(wl, 1, 1); SB(); MM_(wl, 0, 2); MM_(wl, 1, 2); SB(); MM_(wl, 0, 3);
            wh[0] = LDS16(wa ^ 128u); SB();
            MM_(wl, 1, 3);
            wh[1] = LDS16((wa ^ 128u) + 1024); SB();
            MM_(wl, 2, 0); MM_(wl, 2, 1); MM_(wl, 2, 2); MM_(wl, 2, 3);
            wh[2] = LDS16((wa ^ 128u) + 2048);
            u32 cw = ctl(); SB();
            MM_(wl, 3, 0); MM_(wl, 3, 1); MM_(wl, 3, 2); MM_(wl, 3, 3);
            wh[3] = LDS16((wa ^ 128u) + 3072);
            CBAR();
            u32 fa = L_FREE + wid * 4u;
            asm volatile("" : "+s"(fa));
            *(volatile LAS u32*)(L + fa) = f + 1;
            SB();
            MM_(wh, 4, 0); MM_(wh, 4, 1); MM_(wh, 4, 2); MM_(wh, 4, 3);
            SB();
            auto go = [&](u32 w) -> bool {
                const u32 full = (u32)BALLOT(w > ((f + 1) >> 2));
                return ((full >> ((f + 1) & 3u)) & 1u) != 0u || (f & 127u) == 127u;
            };
            if (!go(cw)) {
                while (!go(ctl())) { if (expired()) break; __builtin_amdgcn_s_sleep(CSLEEP); }
            }
            CBAR();
            xa = (xa + SLOT_BYTES) & (CTRL - 1u);
            wa = (wa + SLOT_BYTES) & (CTRL - 1u);
            wl[0] = LDS16(wa); SB();
            MM_(wh, 5, 0); MM_(wh, 5, 1); MM_(wh, 5, 2); MM_(wh, 5, 3);
            wl[1] = LDS16(wa + 1024); SB();
            MM_(wh, 6, 0); MM_(wh, 7, 0);
            xf[0] = LDS16(xa); SB();
            MM_(wh, 6, 1); MM_(wh, 7, 1);
            xf[1] = LDS16(xa + 1024); SB();
            MM_(wh, 6, 2); MM_(wh, 7, 2);
            xf[2] = LDS16(xa + 2048); SB();
            MM_(wh, 6, 3);
            wl[2] = LDS16(wa + 2048); SB();
            MM_(wh, 7, 3);
            xf[3] = LDS16(xa + 3072);
            wl[3] = LDS16(wa + 3072); SB();
        }
        asm volatile("v_mbcnt_lo_u32_b32 %0, -1, 0\n v_mbcnt_hi_u32_b32 %0, -1, %0" : "=v"(lane));
        c = lane & 15u; g = lane >> 4;
        const bool direct = f0 == 0u && q != 0u && wid >= 8u - ND2;
        const u32 reg = L_PACK + wid * 3072u;
        if (f0 == 0u) {
            STAMP(2);
            STAMPV(9, __builtin_amdgcn_s_memtime());
            float mx = 0.f;
#pragma unroll
            for (int j = 0; j < 8; ++j)
#pragma unroll
                for (int m = 0; m < 4; ++m) {
                    mx = fmaxf(fmaxf(mx, fabsf(acc[j][m][0])), fabsf(acc[j][m][1]));
                    mx = fmaxf(fmaxf(mx, fabsf(acc[j][m][2])), fabsf(acc[j][m][3]));
                }
            arrive(wave_max_u32(__float_as_uint(mx)), 8u);
            STAMP(3);
            if (direct) __builtin_amdgcn_s_setprio(3); else if (wid < 4u) __builtin_amdgcn_s_setprio(2);
#pragma unroll
            for (int k = 0; k < 3; ++k) *(LAS u32x4*)(L + reg + k * 1024u + lane * 16u) = u32x4{0u, 0u, 0u, 0u};
            while (LW[L_SCALE / 4] != 1u) { if (expired()) break; __builtin_amdgcn_s_sleep(CSLEEP); }
            CBAR();
        } else STAMP(13);
        const u32 gmax = LW[L_SCALE / 4 + 1];
        const bool bad = (gmax == 0xFFFFFFFFu) || (LW[L_BAD / 4] != 0u);
        const float amax = __uint_as_float(gmax);
        const float inv = amax > 0.f ? 127.0f / amax : 0.f;
        if (f0 == 0u) STAMP(4);
        const float MAGIC = 12582912.0f;
        const float qlo = bad ? MAGIC - 128.0f : MAGIC - 127.0f, qhi = bad ? MAGIC - 128.0f : MAGIC + 127.0f;
        const u32 ovoff = (c & 7u) * NN + ((c >> 3) * 4u + g) * 16u;
        const u32 os = (nb2 - 8u) * 2u + nh;
        const u32 osoff = (mb * 256u + mw * 64u) * NN + (f0 == 0u ? rank * 2048u + nb * 256u + nh * 128u : ((rank + 1u + os) & 7u) * 2048u + 1792u);
        const rsrc_t rs_or = mk_rsrc(p.out[(rank + q) & 7u], OUT_BYTES);
        u32x4 qv[8];
#pragma unroll
        for (int m = 0; m < 4; ++m) {
#pragma unroll
            for (int r = 0; r < 4; ++r) {
                u32 o[2];
#pragma unroll
                for (int h = 0; h < 2; ++h) {
                    u32 t[4];
#pragma unroll
                    for (int jj = 0; jj < 4; ++jj)
                        t[jj] = __float_as_uint(__builtin_amdgcn_fmed3f(__builtin_fmaf(acc[4 * h + jj][m][r], inv, MAGIC), qlo, qhi));
                    o[h] = PERM(t[1], t[0], 0x0c0c0400u) | PERM(t[3], t[2], 0x04000c0cu);
                }
                qv[2 * m][r] = (u32)__builtin_amdgcn_update_dpp((int)o[0], (int)o[1], 0x128, 0xf, 0xc, false);
                qv[2 * m + 1][r] = (u32)__builtin_amdgcn_update_dpp((int)o[1], (int)o[0], 0x128, 0xf, 0x3, false);
            }
            if (direct) {
                ST128(qv[2 * m], rs_or, ovoff, osoff + (2 * m) * 8u * NN, SYS);
                ST128(qv[2 * m + 1], rs_or, ovoff, osoff + (2 * m + 1) * 8u * NN, SYS);
            }
        }
        if (direct) __builtin_amdgcn_s_setprio(0);
        if (f0 == 0u) {
        STAMP(7);
        u32 size = nb == 7u && nh == 0u ? 0u : 8192u;
        const u32 slot = E2_OFF + (b * 8u + wid) * 8192u;
        if (size != 0u && !(q == 0u && wid == 7u) && !(FORCE_RAW2 && (wid & 1u))) {
            u32x4 nib[4], sg = u32x4{0u, 0u, 0u, 0u};
            u32 code[16], len[16], over = 0, tot = 0;
#pragma unroll
            for (int o = 0; o < 16; ++o) {
                u32 f[2], ps[2];
#pragma unroll
                for (int e = 0; e < 2; ++e) {
                    const int i = 2 * o + e;
                    const u32 v = qv[i >> 2][i & 3], s = byte_signs(v);
                    f[e] = v ^ s;
                    sg[i >> 3] = bfi(0x01010101u << (i & 7), s, sg[i >> 3]);
                    u32 h = (f[e] >> 4) & 0x07070707u;
                    h += h << 8;
                    ps[e] = h + (h << 16);
                }
                nib[o >> 2][o & 3] = bfi(0x0F0F0F0Fu, f[0], f[1] << 4);
                ps[1] += 0x07060504u + PERM(ps[0], ps[0], 0x03030303u);
                ps[0] += 0x03020100u;
                u32 cd = 0;
#pragma unroll
                for (int k = 0; k < 8; ++k) cd |= 1u << ((ps[k >> 2] >> (8 * (k & 3))) & 31u);
                code[o] = cd;
                len[o] = (ps[1] >> 24) + 1u;
                over |= ps[1];
                tot += len[o];
            }
            const u32 end = wave_scan_u32(tot);
            const u32 bits = (u32)__builtin_amdgcn_readlane((int)end, 63);
            if (BALLOT((over & 0x20000000u) != 0u || tot > 128u + 255u) == 0ull && bits <= (8192u - 64u - R_BITS) * 8u) {
                size = (R_BITS + ((bits + 127u) >> 7) * 16u);
#pragma unroll
                for (int e = 0; e < 4; ++e) ST128(nib[e], rs_ws, lane * 16u + e * 1024u, slot, AGT);
                ST128(sg, rs_ws, lane * 16u, slot + 4096u, AGT);
                *(LAS char*)(L + reg + lane) = (char)(tot - 128u);
                u32 pos = end - tot + (reg + 64u) * 8u;
#pragma unroll
                for (int o = 0; o < 16; ++o) {
                    LAS u32* a = (LAS u32*)(L + ((pos >> 5) << 2));
                    __hip_atomic_fetch_or(a, code[o] << (pos & 31u), __ATOMIC_RELAXED, __HIP_MEMORY_SCOPE_WORKGROUP);
                    __hip_atomic_fetch_or(a + 1, (code[o] >> 1) >> (~pos & 31u), __ATOMIC_RELAXED, __HIP_MEMORY_SCOPE_WORKGROUP);
                    pos += len[o];
                }
                CBAR();
#pragma unroll
                for (int k = 0; k < 3; ++k) {
                    const u32 off = R_LEN + k * 1024u + lane * 16u;
                    if (R_LEN + k * 1024u < size) ST128(LDSV(reg + k * 1024u + lane * 16u), rs_ws, off < size ? off : OOB, slot, AGT);
                }
            }
        }
        if (size == 8192u) {
#pragma unroll
            for (int e = 0; e < 8; ++e) ST128(qv[e], rs_ws, lane * 16u, slot + e * 1024u, AGT);
        }
        STAMP(11);
        VMWAIT(0);
        {
            u32 old = 0;
            if (lane == 0) {
                *(volatile LAS unsigned short*)(L + L_GSZ + wid * 2u) = (unsigned short)(size >> 4);
                old = __hip_atomic_fetch_add((LAS u32*)(L + L_GCNT + nh * 4u), 1u, __ATOMIC_RELAXED, __HIP_MEMORY_SCOPE_WORKGROUP);
            }
            if (RFL(old) == 3u && lane - 1u < 7u) {
                const u32x4 fv = u32x4{seq, LW[L_GSZ / 4 + nh * 2u], LW[L_GSZ / 4 + nh * 2u + 1u], seq};
                u32x2 fp = __builtin_bit_cast(u32x2, p.ws[(rank + lane) & 7u]);
                asm volatile("v_add_co_u32 %0, vcc, %0, %2\n v_addc_co_u32 %1, vcc, 0, %1, vcc" : "+v"(fp[0]), "+v"(fp[1])
                             : "v"(OFF_F2 + ((lane * 16u + idx) * 16u + q * 2u + nh) * 16u) : "vcc");
                asm volatile("global_store_dwordx4 %0, %1, off sc0 sc1\n s_nop 1" :: "v"(fp), "v"(fv) : "memory");
            }
        }
        STAMP(5);
        __builtin_amdgcn_s_setprio(0);
        }
        const rsrc_t rs_ol = mk_rsrc(p.out[rank], f0 == 0u || os < 7u ? OUT_BYTES : 0u);
#pragma unroll
        for (int e = 0; e < 8; ++e) ST128(qv[e], rs_ol, ovoff, osoff + e * 8u * NN, AGT);
        }
    } else if (b < 128u && wid == 8u) {
        for (u32 n = 0; n < fills / 2u;) {
            const u32 wn = n == 0u ? nb : nb2, hole = wn == 11u ? ~0u : 0u;
            u64 taken = 0;
            while (taken != ~0ull) {
                const u64 want = ~taken & ((taken << 16) | 0xFFFFull);
                const u32 vo = ((want >> lane) & 1ull) ? lane * 16u : OOB;
                const u32x4 fx = LD128(rs_ws, vo, OFF_XF + mb * 1024u, SYS);
                const u32x4 fw = LD128(rs_ws, vo, OFF_WF + wn * 1024u, SYS);
                const u32 mn = min(min(min(fx[0], fx[1]), min(fx[2], fx[3])), min(min(fw[0], fw[1] | hole), min(fw[2], fw[3] | hole)));
                u64 avail = BALLOT(mn >= seq);
                if (avail == 0) {
                    if (!expired()) { __builtin_amdgcn_s_sleep(POLL_SLEEP); continue; }
                    avail = ~taken;
                }
                if ((avail >> lane) & 1ull) LW[L_STEPS / 4 + n + rank_in(avail)] = lane;
                n += (u32)__builtin_popcountll(avail);
                taken |= avail;
                LW[L_NRDY / 4] = n;
            }
            if (n != 64u) break;
            STAMP(1);
            u32 a = 0;
            for (;;) {
                const u32x2 gm = __builtin_amdgcn_raw_buffer_load_b64(rs_ws, OFF_GAM + q * 128u + (lane & 7u) * 8u, 0, SYS);
                a = gm[0];
                if (BALLOT(gm[1] == seq) == ~0ull) break;
                if (expired()) { a = 0xFFFFFFFFu; break; }
                __builtin_amdgcn_s_sleep(1);
            }
            a = wave_max_u32(a);
            LW[L_SCALE / 4 + 1] = a;
            LW[L_SCALE / 4] = 1u;
            STAMP(2);
            while (LW[L_GCNT / 4] + LW[L_GCNT / 4 + 1] < 8u) { if (expired()) break; __builtin_amdgcn_s_sleep(8); }
        }
    } else if (b < 128u) {
        for (u32 f = wid - 9u; f < fills; f += 3u) {
            while (LW[L_NRDY / 4] <= (f >> 1)) { if (expired()) break; __builtin_amdgcn_s_sleep(1); }
            CBAR();
            const u32 step = RFL(LW[L_STEPS / 4 + (f >> 1)]) & 63u;
            if (f >= 4u) {
                for (;;) {
                    if (BALLOT(LW[L_FREE / 4 + (lane & 7u)] >= f - 3u) == ~0ull) break;
                    if (expired()) break;
                    __builtin_amdgcn_s_sleep(1);
                }
            }
            CBAR();
            const u32 sbase = STAGE_OFF + ((step & 7u) * 8u + (step >> 3)) * STEP_BYTES + (f & 1u) * 16384u;
            LAS char* ldst = L + (f & 3u) * SLOT_BYTES;
#pragma unroll
            for (int i = 0; i < DIAG_NDMA; ++i) {
                const u32 so = sbase + (i < 4 ? mb : 16u + (f < 128u ? nb : nb2)) * PIECE + (i & 3) * 4096u;
                __builtin_amdgcn_raw_ptr_buffer_load_lds(rs_ws, ldst + i * 4096, 16, lane * 16u, so, 0, SYS);
                __builtin_amdgcn_raw_ptr_buffer_load_lds(rs_ws, ldst + i * 4096, 16, lane * 16u, so, 1024, SYS);
                __builtin_amdgcn_raw_ptr_buffer_load_lds(rs_ws, ldst + i * 4096, 16, lane * 16u, so, 2048, SYS);
                __builtin_amdgcn_raw_ptr_buffer_load_lds(rs_ws, ldst + i * 4096, 16, lane * 16u, so, 3072, SYS);
            }
            VMWAIT(0);
            LW[L_FULL / 4 + (f & 3u)] = (f >> 2) + 1u;
            if (f - 125u < 3u) STAMP(1);
        }
    } else if (b < 240u) {
        arrive(0u, 12u);
        const u32 src = (rank - rel) & 7u;
        const rsrc_t rs_src = mk_rsrc(p.ws[src], WS_NEED);
        auto in_unit = [&](u32 u, u32& ch, u32& r, bool& isw) -> u32 {
            if (u >= 768u) {
                ch = (u - 768u) / 14u; r = (u - 768u) % 14u; isw = true;
                const u32 os = r >> 1, rs = 28u + (r & 1u) * 2u;
                r = strip_at(os, rs);
                return E1W_OFF + ((((rel + 1u + os) & 7u) * 8u + ch) * 32u + rs) * E1_SLOT;
            }
            ch = u / 96u; r = u % 96u; isw = r < 32u;
            if (!isw) r = (r - 32u + 8u * rel) & 63u;
            return isw ? E1W_OFF + ((rel * 8u + ch) * 32u + r) * E1_SLOT : E1X_OFF + (ch * 64u + r) * E1_SLOT;
        };
        auto out_unit = [&](u32 o) -> u32 {
            u32 k = o >> 4;
            if (k == N2 - 1u) return (o & 15u) * 64u + 7u;
            if (k >= N2 - 13u) {
                const u32 j = o - 16u * (N2 - 13u);
                return j % 12u * 64u + (j / 48u) * 16u + 8u + j / 12u % 4u;
            }
            k += k >= 7u;
            if (k >= (rel >> 1) * 12u + (rel & 1u) * 4u + 8u - ND2) k += ND2;
            return (o & 15u) * 64u + k / 12u * 16u + k % 12u + (k % 12u >= 8u ? 4u : 0u);
        };
        if (wid == 0u) {
            __builtin_amdgcn_s_setprio(3);
            auto ctl = [&](u32 a) -> u32 {
                u32 v;
                asm volatile("ds_read_b32 %0, %1\n s_waitcnt lgkmcnt(0)" : "=v"(v) : "v"(a) : "memory");
                return v;
            };
            auto put = [&](u32 a, u32 n) { asm volatile("ds_write_b32 %0, %1" :: "v"(a), "v"(n) : "memory"); };
            const u32 ca = L_FULL + (lane & 15u) * 4u;
            const u32 vone = lane == 0u ? 0u : OOB;
            u32 n = 0;
            auto wait = [&](int word, u32 bit) {
                auto ok = [&](u32 w) -> bool {
                    return (((u32)__builtin_amdgcn_readlane((int)w, word) >> bit) & 1u) != 0u &&
                           (u32)__builtin_amdgcn_readlane((int)w, 8 + (n & 7u)) + 15u >= n;
                };
                if (!ok(ctl(ca))) {
                    VMWAIT(0);
                    put(L_FULL, n);
                    while (!ok(ctl(ca))) { if (expired()) break; __builtin_amdgcn_s_sleep(1); }
                }
            };
#define CLAIM(cl, so) asm volatile("v_mov_b32 %0, 1\n s_nop 4\n buffer_atomic_add %0, %1, %2, %3 offen sc0 sc1" \
                                   : "=&v"(cl) : "v"(vone), "s"(rs_ws), "s"(so) : "memory")
#define CLAIMED(d, cl) asm volatile("v_readfirstlane_b32 %0, %1" : "=s"(d) : "v"(cl) : "memory")
            for (u32 u = 32u + pj; u < N1; ++n) {
                u32 ch, r, cl; bool isw;
                const u32 so = in_unit(u, ch, r, isw);
                if (u >= 768u) put(L_P1, 1u);
                wait(1, u >= 768u ? 0u : ch * 2u + !isw);
                if (n == 0) STAMP(1);
                LAS char* ld = L + (n & 15u) * 8192u;
                put(L_Q + (n & 15u) * 4u, u);
                CLAIM(cl, OFF_CTR + (64u + rel) * 128u);
#pragma unroll
                for (int i = 0; i < 7; ++i) {
                    VMWAIT(WIN1 - (i < WIN1 - 1 ? 0 : 1));
                    if (i == WIN1 - 1) put(L_FULL, n);
                    if (i == 0) { if (lane < 4u) __builtin_amdgcn_raw_ptr_buffer_load_lds(rs_src, ld + 6144, 16, lane * 16u, so, 0, SYS); }
                    else __builtin_amdgcn_raw_ptr_buffer_load_lds(rs_src, ld + (i - 1) * 1024, 16, lane * 16u + 128u, so + (i - 1) * 1024u, 0, SYS);
                }
                CLAIMED(u, cl);
                u = u - (seq - 1u) * (N1 - 32u) + 48u;
            }
            put(L_P1, 1u);
            STAMP(2);
            for (u32 o = pj; o < 16u * N2; ++n) {
                const u32 t = out_unit(o), so = E2_OFF + t * 8192u;
                const bool dyn = o >= 16u * (S2 - 1u);
                u32 cl;
                wait(2, o < 16u * S2 ? (t >> 2) & 15u : 16u);
                if (o == pj) STAMP(3);
                LAS char* ld = L + (n & 15u) * 8192u;
                const u32 sz = min(RFL(ctl(L_SZ + (t >> 1) * 4u)) >> ((t & 1u) * 16u) & 0xFFFFu, 512u);
                put(L_Q + (n & 15u) * 4u, 1024u + t + (sz << 16));
                CLAIM(cl, dyn ? OFF_CTR + (72u + rel) * 128u : OOB);
#pragma unroll
                for (int i = 0; i < 8; ++i) {
                    if (i >= 6 && i * 64u >= sz) break;
                    VMWAIT(WIN2 - (i < WIN2 - 1 ? 0 : 1));
                    if (i == WIN2 - 1) put(L_FULL, n);
                    if (i < 6 || i * 64u + lane < sz)
                        __builtin_amdgcn_raw_ptr_buffer_load_lds(rs_src, ld + i * 1024, 16, lane * 16u, so + i * 1024u, 0, SYS);
                }
                u32 c;
                CLAIMED(c, cl);
                o = dyn ? c - (seq - 1u) * (16u * (N2 - S2 + 1u)) + 16u * S2 : o + 16u;
            }
            VMWAIT(0);
            put(L_FULL, n);
            put(L_END, 1u);
            STAMP(4);
            if (lane == 0) __builtin_amdgcn_raw_buffer_store_b32(seq, rs_src, OFF_DONE + rank * 128u + pj * 4u, 0, SYS);
        } else if (wid == 1u) {
            for (u32 m = 0; m != 0xFFFFu;) {
                const u32 f = __builtin_amdgcn_raw_buffer_load_b32(rs_ws, OFF_F1 + rel * 128u + (lane & 15u) * 4u, 0, SYS);
                m = (u32)BALLOT(f >= seq) & 0xFFFFu;
                if (expired()) m = 0xFFFFu;
                LW[L_SEG / 4] = m;
                __builtin_amdgcn_s_sleep(4);
            }
            STAMP(1);
            while (LW[L_P1 / 4] == 0u) { if (expired()) break; __builtin_amdgcn_s_sleep(16); }
            u32 m = 0;
            while (m != 0xFFFFu) {
                const u32x4 f = LD128(rs_ws, lane < 16u ? lane * 16u : OOB, OFF_F2 + (rel * 16u + pj) * 256u, SYS);
                m = (u32)BALLOT(f[0] == seq && f[3] == seq) & 0xFFFFu;
                if (lane < 16u) *(LAS u32x2*)(L + L_SZ + pj * 128u + lane * 8u) = u32x2{f[1], f[2]};
                if (expired()) m = 0xFFFFu;
                CBAR();
                LW[L_G2 / 4] = m;
                __builtin_amdgcn_s_sleep(2);
            }
            STAMP(2);
            for (u32 i = 0; i < 4u;) {
                const u32x4 f = LD128(rs_ws, lane * 16u, OFF_F2 + rel * 4096u + i * 1024u, SYS);
                *(LAS u32x2*)(L + L_SZ + i * 512u + lane * 8u) = u32x2{f[1], f[2]};
                if (BALLOT(f[0] == seq && f[3] == seq) == ~0ull || expired()) ++i; else __builtin_amdgcn_s_sleep(16);
            }
            CBAR();
            LW[L_G2 / 4] = 0x1FFFFu;
            STAMP(3);
        } else if (wid < 10u) {
            const rsrc_t rs_ol = mk_rsrc(p.out[rank], OUT_BYTES);
            const u32 ovoff = (lane & 7u) * NN + (((lane >> 3) & 1u) * 4u + (lane >> 4)) * 16u;
            for (u32 n = wid - 2u;; n += 8u) {
                bool more = true;
                for (;;) {
                    const u32 end = LW[L_END / 4];
                    if (LW[L_FULL / 4] > n) break;
                    if (end != 0u) { more = false; break; }
                    expired();
                    __builtin_amdgcn_s_sleep(2);
                }
                if (!more) break;
                CBAR();
                const u32 sl = (n & 15u) * 8192u;
                const u32 id = RFL(LW[L_Q / 4 + (n & 15u)]);
                if (id < 1024u) {
                u32 ch, r; bool isw;
                const u32 so = in_unit(id, ch, r, isw);
                const u32 top = RFL(LW[(sl + 6144u) / 4]), cnt = RFL(LW[(sl + 6144u) / 4 + 1u]);
                u32x4 o[8];
                if (cnt > 7u) {
                    o[6] = LD128(rs_src, lane * 16u + 128u, so + 6144u, SYS);
                    o[7] = LD128(rs_src, lane * 16u + 128u, so + 7168u, SYS);
#pragma unroll
                    for (int e = 0; e < 6; ++e) o[e] = LDSV(sl + e * 1024u + lane * 16u);
                } else {
                    u32x4 mq[4];
                    u32 ex[16];
#pragma unroll
                    for (int e = 0; e < 4; ++e) mq[e] = LDSV(sl + e * 1024u + lane * 16u);
#pragma unroll
                    for (int e = 0; e < 2; ++e) {
                        const u32x4 v = LDSV(sl + 4096u + e * 1024u + lane * 16u);
#pragma unroll
                        for (int k = 0; k < 4; ++k) { ex[8 * e + 2 * k] = v[k] & 0x0F0F0F0Fu; ex[8 * e + 2 * k + 1] = (v[k] >> 4) & 0x0F0F0F0Fu; }
                    }
                    for (u32 k = 0; k < cnt; ++k) {
                        const u32 at = LW[(sl + 6144u) / 4 + 2u + 2u * k] - lane * 16u, val = LW[(sl + 6144u) / 4 + 3u + 2u * k];
#pragma unroll
                        for (int j = 0; j < 16; ++j) ex[j] = at == (u32)j ? val : ex[j];
                    }
#pragma unroll
                    for (int j = 0; j < 16; ++j) {
                        const u32 e = top * 0x01010101u - ex[j], m = mq[j >> 2][j & 3];
                        const u32 r0 = PERM(e, m, 0x05010400u), r1 = PERM(e, m, 0x07030602u);
                        o[j >> 1][(2 * j) & 3] = __builtin_amdgcn_alignbit(r0, r0, 1);
                        o[j >> 1][(2 * j + 1) & 3] = __builtin_amdgcn_alignbit(r1, r1, 1);
                    }
                }
                CBAR();
                LW[L_FREE / 4 + (n & 7u)] = n + 1u;
                const u32 flag = put_unit(o, rs_ws, isw, rel, ch, r);
                VMWAIT(0);
                if (lane == 0) __builtin_amdgcn_raw_buffer_store_b32(seq, rs_ws, flag, 0, SYS);
                STAMP(1);
                continue;
                }
                const u32 sz = id >> 16, x = (id >> 3) & 7u, ix = (id >> 6) & 15u;
                const u32 osoff = ((4u * (x >> 1) + (ix & 3u)) * 256u + (id & 3u) * 64u) * NN + src * 2048u +
                                  (4u * (x & 1u) + (ix >> 2)) * 256u + ((id >> 2) & 1u) * 128u;
                u32x4 qv[8];
                if (sz >= 512u) {
#pragma unroll
                    for (int e = 0; e < 8; ++e) qv[e] = LDSV(sl + e * 1024u + lane * 16u);
                } else {
                    const u32 tot = (u32)*(volatile LAS unsigned char*)(L + sl + R_LEN + lane) + 128u;
                    u32x4 nib[4];
#pragma unroll
                    for (int e = 0; e < 4; ++e) nib[e] = LDSV(sl + e * 1024u + lane * 16u);
                    const u32x4 sg = LDSV(sl + 4096u + lane * 16u);
                    u32 pos = wave_scan_u32(tot) - tot + (sl + R_BITS) * 8u;
#pragma unroll
                    for (int o = 0; o < 16; ++o) {
                        const LAS u32* a = (const LAS u32*)(L + ((pos >> 5) << 2));
                        u32 w = __builtin_amdgcn_alignbit(a[1], a[0], pos);
                        u32 ps[2] = {0u, 0u};
#pragma unroll
                        for (int k = 0; k < 8; ++k) {
                            ps[k >> 2] |= ((u32)__builtin_ctz(w) & 255u) << (8 * (k & 3));
                            w &= w - 1u;
                        }
                        pos += (ps[1] >> 24) + 1u;
                        const u32 h0 = ps[0] - ((ps[0] << 8) + 0x01010100u);
                        const u32 h1 = ps[1] - (__builtin_amdgcn_alignbit(ps[1], ps[0], 24) + 0x01010101u);
                        const u32 nv = nib[o >> 2][o & 3], s = sg[o >> 2] << (6 - 2 * (o & 3));
                        qv[o >> 1][(2 * o) & 3] = bfi(0x0F0F0F0Fu, nv, h0 << 4) ^ byte_signs(s << 1);
                        qv[o >> 1][(2 * o + 1) & 3] = bfi(0x0F0F0F0Fu, nv >> 4, h1 << 4) ^ byte_signs(s);
                    }
                }
                CBAR();
                LW[L_FREE / 4 + (n & 7u)] = n + 1u;
                if (LW[L_BAD / 4] != 0u) qv[0] = u32x4{0x80808080u, 0x80808080u, 0x80808080u, 0x80808080u};
#pragma unroll
                for (int e = 0; e < 8; ++e) ST128(qv[e], rs_ol, ovoff, osoff + e * 8u * NN, AGT);
            }
            STAMP(2);
        }
    } else {
        arrive(0u, 12u);
        if (b == AUX_WG && wid == 2u) {
            u32 a = 0;
            for (;;) {
                const u32x4 g0 = LD128(rs_ws, OFF_AMX + lane * 32u, 0, AGT);
                const u32x4 g1 = LD128(rs_ws, OFF_AMX + lane * 32u + 16u, 0, AGT);
                a = max(max(g0[0], g0[2]), max(g1[0], g1[2]));
                const bool ok = g0[1] == seq && g0[3] == seq && g1[1] == seq && g1[3] == seq;
                if (BALLOT(ok) == ~0ull) break;
                if (expired()) { a = 0xFFFFFFFFu; break; }
            }
            a = wave_max_u32(a);
            u64* gp = (u64*)(p.ws[lane & 7u] + OFF_GAM + (lane >> 3) * 128u + rank * 8u);
            __hip_atomic_store(gp, ((u64)seq << 32) | a, __ATOMIC_RELAXED, __HIP_MEMORY_SCOPE_SYSTEM);
            if (lane == 0) __builtin_amdgcn_raw_buffer_store_b32(seq, rs_ws, OFF_SEQ, 0, AGT);
            STAMP(3);
            bool give_up = false;
            for (;;) {
                const u32x4 f = LD128(rs_ws, OFF_DONE + lane * 16u, 0, SYS);
                const bool ok = (lane >> 3) == rank || (lane & 4u) != 0u || (f[0] >= seq && f[1] >= seq && f[2] >= seq && f[3] >= seq);
                if (BALLOT(ok) == ~0ull) break;
                if (expired()) { give_up = true; break; }
                __builtin_amdgcn_s_sleep(8);
            }
            if (give_up || __builtin_amdgcn_raw_buffer_load_b32(rs_ws, OFF_TMO, 0, AGT) == seq)
                if (lane == 0) p.out[rank][rank * 2048u] = (char)-128;
            STAMP(7);
        }
    }
}

extern "C" void kernel_launch(int rank, int nranks, ncclComm_t comm, hipStream_t stream,
                              void* const* d_in, const int* in_sizes, int n_in,
                              void* d_out, int out_size, const int* in_dtypes, int out_dtype,
                              void* const* d_out_all, void* const* d_ws_all, size_t ws_bytes) {
    if (nranks != NR || (unsigned)rank >= (unsigned)NR || n_in < 2 || ws_bytes < 3ull * WS_NEED ||
        in_sizes[0] != (int)(MM * KSH) || in_sizes[1] != (int)(KSH * NN) || out_size != (int)OUT_BYTES) {
        fprintf(stderr, "kernel_launch: unexpected problem shape\n");
        return;
    }
    static bool init = false;
    if (!init) {
        HIP_CHECK(hipFuncSetAttribute((const void*)gemm_ar_q8, hipFuncAttributeMaxDynamicSharedMemorySize, LDS_BYTES));
        init = true;
    }
    Params p;
    p.x = (const char*)d_in[0];
    p.w = (const char*)d_in[1];
    for (int r = 0; r < NR; ++r) { p.out[r] = (char*)d_out_all[r]; p.ws[r] = (char*)d_ws_all[r]; }
    p.rank = rank;
    hipLaunchKernelGGL(gemm_ar_q8, dim3(256), dim3(768), LDS_BYTES, stream, p);
    HIP_CHECK(hipGetLastError());
}
```

```cpp
#include <hip/hip_runtime.h>
#include <rccl/rccl.h>
#include <cstdio>
#include <cstdint>

#define HIP_CHECK(expr) do { hipError_t _e = (expr); if (_e != hipSuccess) { \
    fprintf(stderr, "HIP error %s at %s:%d\n", hipGetErrorString(_e), __FILE__, __LINE__); \
    return; } } while (0)

#define LAS __attribute__((address_space(3)))
typedef unsigned int u32;
typedef unsigned long long u64;
typedef u32 u32x4 __attribute__((ext_vector_type(4)));
typedef u32 u32x2 __attribute__((ext_vector_type(2)));
typedef unsigned short u16x2 __attribute__((ext_vector_type(2)));
typedef float f32x16 __attribute__((ext_vector_type(16)));

#define WIN1 4
#define WIN2 4
#define ND2 2
#define GCH 2
#define TDELAY 100
#define WDELAY1 200
#define WDELAY2 900
#define S2 16
#define DIAG_NDMA 8
#define CSLEEP 1
#define POLL_SLEEP 2
#define BALANCE 1
#define LSLEEP 1
#define KPAD "s_nop 0\n"
#define FLAGW 4
#define PAHEAD 4
#define LNOP 5
#define LADAPT 0
#define CPRIO 0
#define LPRIO 0
#define PRED_PAD 20
#define FORCE_RAW1 0
#define FORCE_RAW2 0

constexpr int NR = 8;
constexpr u32 MM = 4096, NN = 16384, KSH = 512;
constexpr u32 X_BYTES = MM * KSH * 2, W_BYTES = KSH * NN * 2, OUT_BYTES = MM * NN;
constexpr u32 PIECE = 32768;
constexpr u32 STEP_BYTES = 28 * PIECE;
constexpr u32 OFF_UF = 1u << 20;
constexpr u32 STAGE_OFF = 2u << 20;
constexpr u32 E1_SLOT = 8192 + 128;
constexpr u32 I_HDR = 4096, I_LEN = 16, I_BIT = 80, I_VAR = 592;
constexpr u32 E1X_OFF = STAGE_OFF + 64 * STEP_BYTES;
constexpr u32 E1W_OFF = E1X_OFF + 512 * E1_SLOT;
constexpr u32 E2_OFF = E1W_OFF + 2048 * E1_SLOT;
constexpr u32 WS_NEED = E2_OFF + 1024 * 8192;
constexpr u32 OFF_SEQ = 0, OFF_TMO = 256;
constexpr u32 OFF_AMX = 4096;
constexpr u32 OFF_GAM = OFF_AMX + 2048;
constexpr u32 OFF_DONE = OFF_GAM + 1024;
constexpr u32 OFF_F1 = OFF_DONE + 1024;
constexpr u32 OFF_CTR = OFF_F1 + 1024;
constexpr u32 OFF_F2 = 65536;
constexpr u32 OFF_STAMP = 131072;
static_assert(OFF_CTR + 81 * 128 <= OFF_F2 && OFF_F2 + 32768 <= OFF_STAMP && OFF_STAMP + 256 * 12 * 16 * 8 <= OFF_UF, "flags");

constexpr u32 SLOT_BYTES = 32768;
constexpr u32 CTRL = 16 * E1_SLOT;
constexpr u32 L_FULL = CTRL, L_FREE = CTRL + 32, L_NRDY = CTRL + 64, L_AMAX = CTRL + 68, L_ACNT = CTRL + 72,
              L_BAD = CTRL + 76, L_SCALE = CTRL + 80, L_GCNT = CTRL + 88, L_GSZ = CTRL + 96, L_STEPS = CTRL + 128;
constexpr u32 L_SEG = CTRL + 4, L_G2 = CTRL + 8, L_END = CTRL + 12, L_P1 = CTRL + 16, L_Q = CTRL + 128, L_PSZ = CTRL + 192,
              L_SZ = CTRL + 256;
constexpr u32 L_PACK = CTRL + 1024;
constexpr u32 LDS_BYTES = L_PACK + 8 * 3072;
constexpr u64 DEADLINE = 100000000ull;
constexpr u32 OOB = WS_NEED;
constexpr int SYS = 17, AGT = 16;
constexpr u32 AUX_WG = 248;
constexpr u32 N1 = 768 + 112;
constexpr u32 N2 = 60 - ND2;
constexpr u32 R_LEN = 5120, R_BITS = 5184;

struct Params {
    const char* x;
    const char* w;
    char* out[NR];
    char* ws[NR];
    int rank;
};

typedef __amdgpu_buffer_rsrc_t rsrc_t;
__device__ __forceinline__ rsrc_t mk_rsrc(const void* p, u32 bytes) {
    return __builtin_amdgcn_make_buffer_rsrc((void*)p, 0, bytes, 0x00020000);
}
__device__ __forceinline__ u64 now() { return __builtin_amdgcn_s_memrealtime(); }
#define VMWAIT(n) asm volatile("s_waitcnt vmcnt(%0)" :: "n"(n) : "memory")
#define CBAR() asm volatile("" ::: "memory")
#define ST128(v, rs, voff, soff, aux) do { const u32x4 d_ = (v); \
    __builtin_amdgcn_raw_buffer_store_b128(d_, rs, voff, soff, aux); \
    asm volatile("s_nop 1" :: "v"(d_) : "memory"); } while (0)
#define LD128(rs, voff, soff, aux) __builtin_amdgcn_raw_buffer_load_b128(rs, voff, soff, aux)
#define PERM(a, b, s) __builtin_amdgcn_perm(a, b, s)
#define BALLOT(c) __builtin_amdgcn_ballot_w64(c)
#define RFL(v) ((u32)__builtin_amdgcn_readfirstlane(v))
__device__ __forceinline__ u32 bfi(u32 m, u32 a, u32 b) { return (a & m) | (b & ~m); }
__device__ __forceinline__ u32 rank_in(u64 m) {
    return __builtin_amdgcn_mbcnt_hi((u32)(m >> 32), __builtin_amdgcn_mbcnt_lo((u32)m, 0u));
}

#define STAMPV(i, v) do { } while (0)
#define STAMP(i) STAMPV(i, now())

#define DPP_STEPS(OP) \
    OP(0x111, 0xf, true) OP(0x112, 0xf, true) OP(0x114, 0xf, true) OP(0x118, 0xf, true) OP(0x142, 0xa, false) OP(0x143, 0xc, false)
__device__ __forceinline__ u32 wave_max_u32(u32 v) {
#define OP_(c, rm, bc) v = max(v, (u32)__builtin_amdgcn_update_dpp(0, (int)v, c, rm, 0xf, bc));
    DPP_STEPS(OP_)
#undef OP_
    return (u32)__builtin_amdgcn_readlane((int)v, 63);
}
__device__ __forceinline__ u32 wave_scan_u32(u32 v) {
#define OP_(c, rm, bc) v += (u32)__builtin_amdgcn_update_dpp(0, (int)v, c, rm, 0xf, bc);
    DPP_STEPS(OP_)
#undef OP_
    return v;
}
__device__ __forceinline__ u32 byte_signs(u32 a) { return PERM(a, a << 8, 0x0B090A08u); }

constexpr u32 TFLIP0 = 0xFFFFFFFFu, TFLIP1 = 0xFFFFFF00u;
__device__ __forceinline__ u32 lut16(u32x4 t, u32 i) { return PERM(t[1], t[0], i) ^ PERM(t[3], t[2], i ^ 0x08080808u); }
__device__ __forceinline__ u32 unary8(u32 h0, u32 h1, u32& code) {
    h0 += h0 << 8; h0 += h0 << 16;
    h1 += h1 << 8; h1 += h1 << 16;
    h1 += 0x07060504u + PERM(h0, h0, 0x03030303u);
    h0 += 0x03020100u;
    code = 0;
#pragma unroll
    for (int k = 0; k < 4; ++k) code |= (1u << ((h0 >> (8 * k)) & 31u)) | (1u << ((h1 >> (8 * k)) & 31u));
    return h1 >> 24;
}
__device__ __forceinline__ void put_bits(LAS char* L, u32 pos, u32 code) {
    LAS u32* a = (LAS u32*)(L + ((pos >> 5) << 2));
    __hip_atomic_fetch_or(a, code << (pos & 31u), __ATOMIC_RELAXED, __HIP_MEMORY_SCOPE_WORKGROUP);
    __hip_atomic_fetch_or(a + 1, (code >> 1) >> (~pos & 31u), __ATOMIC_RELAXED, __HIP_MEMORY_SCOPE_WORKGROUP);
}
__device__ __forceinline__ void counts8(const LAS char* L, u32& pos, u32& h0, u32& h1) {
    const LAS u32* a = (const LAS u32*)(L + ((pos >> 5) << 2));
    u32 w = __builtin_amdgcn_alignbit(a[1], a[0], pos);
    u32 ps[2] = {0u, 0u};
#pragma unroll
    for (int k = 0; k < 8; ++k) {
        ps[k >> 2] |= ((u32)__builtin_ctz(w) & 255u) << (8 * (k & 3));
        w &= w - 1u;
    }
    pos += (ps[1] >> 24) + 1u;
    h0 = ps[0] - ((ps[0] << 8) + 0x01010100u);
    h1 = ps[1] - (__builtin_amdgcn_alignbit(ps[1], ps[0], 24) + 0x01010101u);
}

__global__ void __launch_bounds__(768, 3) gemm_ar_q8(Params p) {
    extern __shared__ __attribute__((aligned(16))) char smem[];
    LAS char* L = (LAS char*)smem;
    u32 lane = threadIdx.x & 63;
    const u32 wid = RFL(threadIdx.x >> 6);
    const u32 rank = (u32)p.rank & 7u;
    const u32 b = blockIdx.x;
    const u32 q = b & 7u, idx = b >> 3;
    const u32 mb = 4 * (q >> 1) + (idx & 3), nb = 4 * (q & 1) + ((idx >> 2) & 3);
    const u32 nb2 = 8u + 2u * (q & 1u) + ((idx >> 2) & 1u), fills = idx < 8u ? 256u : 128u;
    const u32 rel = 1u + (b - 128u) % 7u, pj = ((b - 128u) / 7u) & 15u;
    char* wsl = p.ws[rank];
    const rsrc_t rs_ws = mk_rsrc(wsl, WS_NEED);
    volatile LAS u32* LW = (volatile LAS u32*)L;
#define LDSV(a) (*(const LAS u32x4*)(L + (a)))

    STAMP(0);
    const u32 seqv = __builtin_amdgcn_raw_buffer_load_b32(rs_ws, OFF_SEQ, 0, AGT);
    if (threadIdx.x < 32) LW[CTRL / 4 + threadIdx.x] = 0u;
    asm volatile("s_waitcnt lgkmcnt(0)" ::: "memory");
    __builtin_amdgcn_s_barrier();
    CBAR();

    auto put_unit = [&](const u32x4 (&o)[8], rsrc_t rs, bool isw, u32 lk, u32 ch, u32 r) -> u32 {
        const u32 so = STAGE_OFF + (lk * 8u + ch) * STEP_BYTES + (isw ? 16u * PIECE + r * 8192u : (r >> 2) * PIECE + (r & 3u) * 4096u);
        const u32 v0 = isw ? ((lane & 48u) + (lane & 7u) * 2u + ((lane >> 3) & 1u)) * 16u : lane * 16u, v1 = v0 + (isw ? 4096u : 16384u);
#pragma unroll
        for (int e = 0; e < 8; ++e) ST128(o[e], rs, (e < 4 ? v0 : v1) + (e & 3) * 1024u, so, SYS);
        return OFF_UF + ((lk * 8u + ch) * 112u + (isw ? 64u : 0u) + r) * 128u;
    };
    auto put_flag = [&](rsrc_t rs, u32 flag, u32 sq) { if (lane < FLAGW) ST128((u32x4{sq, sq, sq, sq}), rs, flag + lane * 16u, 0, SYS); };

    auto strip_at = [](u32 os, u32 r) -> u32 { return 32u + (os >> 1) * 4u + (r & 2u) + (os & 1u); };

    {
        u32 t = 0, tend = 0, delay = 0;
        if (b < 128u) { t = wid * 128u + b; if (t < GCH * 96u) tend = t + 1u; delay = t / 96u * TDELAY; }
        else if (b < 240u) {
            if (wid < 8u) { t = GCH * 96u + (wid - 2u) * 112u + (b - 128u); if (wid >= 2u && t < 768u) tend = t + 1u; delay = t / 96u * TDELAY; }
            else { t = 768u + 224u + (wid - 8u) * 112u + (b - 128u); tend = t + 1u; delay = wid < 10u ? WDELAY1 : WDELAY2; }
        } else { t = 768u + wid * 16u + (b - 240u); tend = 768u + 8u * 224u; }
        if (t < tend) {
            const u64 ts = now();
            while (now() - ts < (u64)delay) __builtin_amdgcn_s_sleep(4);
        }
        for (; t < tend; t += t - (768u + 32u) < 192u ? 192u + 448u : 192u) {
            u32 kind, lk = 0, ch, r;
            if (t < 768u) { ch = t / 96u; r = t % 96u; kind = r >> 6; r &= 63u; }
            else { ch = (t - 768u) / 224u; r = (t - 768u) % 224u; lk = 1u + r % 7u; r /= 7u; kind = ch == 0u ? 1u : 2u; }
            const u32 to = (rank + lk) & 7u;
            const bool strip = kind != 0u && (r | 2u) == 30u;
            u32x4 o[8];
            if (kind == 0u) {
                const rsrc_t rs_x = mk_rsrc(p.x, X_BYTES);
                const u32 vx = (lane >> 2) * 1024u + (((lane & 3u) ^ ((lane >> 4) & 2u)) << 4);
#pragma unroll
                for (int i = 0; i < 4; ++i) {
                    o[i] = LD128(rs_x, vx, r * 65536u + ch * 128u + i * 16384u, 0);
                    o[4 + i] = LD128(rs_x, vx + 64u, r * 65536u + ch * 128u + i * 16384u, 0);
                }
            } else {
                const rsrc_t rs_w = mk_rsrc(p.w, W_BYTES);
                const u32 vw = (lane >> 4) * 8u * 32768u + (lane & 15u) * 16u;
                const u32 so = (ch * 64u + (r & 2u) * 16u) * 32768u + (to * 2048u + (r >> 2) * 256u + (r & 1u) * 128u) * 2u;
                u32x4 v[8];
#pragma unroll
                for (int i = 0; i < 8; ++i) v[i] = LD128(rs_w, vw, so + i * 32768u, 0);
#pragma unroll
                for (int e = 0; e < 4; ++e)
#pragma unroll
                    for (int k = 0; k < 4; ++k) {
                        const u32 lo = PERM(v[2 * k + 1][e >> 1], v[2 * k][e >> 1], (e & 1) ? 0x07060302u : 0x05040100u);
                        const u32 hi = PERM(v[2 * k + 1][2 + (e >> 1)], v[2 * k][2 + (e >> 1)], (e & 1) ? 0x07060302u : 0x05040100u);
                        o[e][k] = (u32)__builtin_amdgcn_update_dpp((int)lo, (int)hi, 0x128, 0xf, 0xc, false);
                        o[4 + e][k] = (u32)__builtin_amdgcn_update_dpp((int)hi, (int)lo, 0x128, 0xf, 0x3, false);
                    }
            }
            const rsrc_t rs_to = mk_rsrc(p.ws[to], WS_NEED);
            u32 flag = OOB, flag2 = OOB, size = E1_SLOT;
            if (kind != 2u) flag = put_unit(o, rs_to, kind == 1u, lk, ch, r);
            if (strip && lk != 0u) flag2 = put_unit(o, rs_ws, true, 0u, ch, strip_at(lk - 1u, r));
            const u32 sq = RFL(seqv) + 1u;
            auto staged = [&]() { put_flag(rs_to, flag, sq); put_flag(rs_ws, flag2, sq); };
            if (kind == 1u && !strip) { VMWAIT(0); staged(); continue; }
            const u32 slot = kind == 0u ? E1X_OFF + (ch * 64u + r) * E1_SLOT : E1W_OFF + ((lk * 8u + ch) * 32u + r) * E1_SLOT;
            const u32 reg = L_PACK + wid * 2048u;
            *(LAS u32x4*)(L + reg + lane * 16u) = u32x4{0u, 0u, 0u, 0u};
            *(LAS u32x4*)(L + reg + 1024u + lane * 16u) = u32x4{0u, 0u, 0u, 0u};
            u32 ex[16], mx = 0;
#pragma unroll
            for (int e = 0; e < 4; ++e) {
                u32x4 mq;
#pragma unroll
                for (int k = 0; k < 4; ++k) {
                    const int j = 4 * e + k;
                    const u32 d0 = o[j >> 1][(2 * j) & 3], d1 = o[j >> 1][(2 * j + 1) & 3];
                    const u32 r0 = __builtin_amdgcn_alignbit(d0, d0, 31), r1 = __builtin_amdgcn_alignbit(d1, d1, 31);
                    mq[k] = PERM(r1, r0, 0x06040200u);
                    ex[j] = PERM(r1, r0, 0x07050301u);
                    mx = __builtin_bit_cast(u32, __builtin_elementwise_max(__builtin_bit_cast(u16x2, mx),
                             __builtin_elementwise_max(__builtin_bit_cast(u16x2, r0), __builtin_bit_cast(u16x2, r1))));
                }
                ST128(mq, rs_ws, lane * 16u + e * 1024u, slot, AGT);
            }
            const u32 top = wave_max_u32(max(mx >> 24, (mx >> 8) & 255u));
            u32 any = 0, cnt = 0;
#pragma unroll
            for (int j = 0; j < 16; ++j) { ex[j] = top * 0x01010101u - ex[j]; any |= ex[j]; }
            CBAR();
#pragma unroll
            for (int j = 0; j < 16; j += 2)
                __hip_atomic_fetch_add((LAS u32*)(L + reg + 256u + (((ex[j] << 4) & 0xF0u) | ((lane & 3u) << 2))), 1u, __ATOMIC_RELAXED,
                                       __HIP_MEMORY_SCOPE_WORKGROUP);
            CBAR();
            const u32x4 hist = LDSV(reg + 256u + (lane & 15u) * 16u);
            CBAR();
            if (BALLOT((any & 0xF0F0F0F0u) != 0u) != 0ull) {
#pragma unroll
                for (int j = 0; j < 16; ++j) {
                    const bool far = (ex[j] & 0xF0F0F0F0u) != 0u;
                    const u64 m = BALLOT(far);
                    if (m != 0ull) {
                        const u32 k = cnt + rank_in(m);
                        if (far && k < 14u) *(LAS u32x2*)(L + reg + I_VAR + k * 8u) = u32x2{lane * 16u + j, ex[j]};
                        cnt += (u32)__builtin_popcountll(m);
                        ex[j] &= 0x0F0F0F0Fu;
                    }
                }
            }
            const u32 key = (((hist[0] + hist[1] + hist[2] + hist[3]) << 4) | (lane & 15u)) ^ 15u;
            u32 above = 0;
#define ROT_(n) above = __builtin_amdgcn_alignbit(above, key - (u32)__builtin_amdgcn_mov_dpp((int)key, 0x120 + n, 0xf, 0xf, false), 31);
            ROT_(1) ROT_(2) ROT_(3) ROT_(4) ROT_(5) ROT_(6) ROT_(7) ROT_(8) ROT_(9) ROT_(10) ROT_(11) ROT_(12) ROT_(13) ROT_(14) ROT_(15)
#undef ROT_
            above = (u32)__builtin_popcount(above);
            const u32 sym = (above >> 1) | ((above & 1u) << 3);
            u32 tb = sym << ((lane & 3u) * 8u);
            tb |= (u32)__builtin_amdgcn_mov_dpp((int)tb, 0xB1, 0xf, 0xf, false);
            tb |= (u32)__builtin_amdgcn_mov_dpp((int)tb, 0x4E, 0xf, 0xf, false);
            const u32x4 tab = u32x4{(u32)__builtin_amdgcn_readlane((int)tb, 0) ^ TFLIP0, (u32)__builtin_amdgcn_readlane((int)tb, 4) ^ TFLIP1,
                                    (u32)__builtin_amdgcn_readlane((int)tb, 8) ^ TFLIP0, (u32)__builtin_amdgcn_readlane((int)tb, 12) ^ TFLIP1};
            u32 code[8], len[8], over = 0, tot = 0;
            u32x2 pl = u32x2{0u, 0u};
#pragma unroll
            for (int g = 0; g < 8; ++g) {
                const u32 s0 = lut16(tab, ex[2 * g]), s1 = lut16(tab, ex[2 * g + 1]);
                pl[g >> 2] |= bfi(0x08080808u, s0, s1 << 4) >> (g & 3) & (0x88888888u >> (g & 3));
                const u32 last = unary8(s0 & 0x07070707u, s1 & 0x07070707u, code[g]);
                over |= last;
                len[g] = last + 1u;
                tot += len[g];
            }
            const u32 end = wave_scan_u32(tot);
            const u32 bytes = I_VAR + cnt * 8u + (((u32)__builtin_amdgcn_readlane((int)end, 63) + 7u) >> 3);
            VMWAIT(4);
            staged();
            if (BALLOT(over > 31u) == 0ull && cnt < 15u && bytes <= 2040u && !(FORCE_RAW1 && (r & 1u))) {
                size = I_HDR + bytes;
                *(LAS char*)(L + reg + sym) = (char)((lane & 15u) | (((top | (cnt << 8)) >> (sym * 4u & 31u)) << 4));
                *(LAS char*)(L + reg + I_LEN + lane) = (char)(tot - 64u);
                *(LAS u32x2*)(L + reg + I_BIT + lane * 8u) = pl;
                u32 pos = end - tot + (reg + I_VAR + cnt * 8u) * 8u;
#pragma unroll
                for (int g = 0; g < 8; ++g) { put_bits(L, pos, code[g]); pos += len[g]; }
                CBAR();
                ST128(LDSV(reg + lane * 16u), rs_ws, lane * 16u, slot + I_HDR, AGT);
                ST128(LDSV(reg + 1024u + lane * 16u), rs_ws, lane * 16u + 1024u < bytes ? lane * 16u + 1024u : OOB, slot + I_HDR, AGT);
                CBAR();
            } else {
                if (lane == 0) __builtin_amdgcn_raw_buffer_store_b32(~0u, rs_ws, slot + I_HDR, 0, AGT);
#pragma unroll
                for (int e = 0; e < 8; ++e) ST128(o[e], rs_ws, lane * 16u + (e & 3) * 1024u, slot + (e >> 2) * (I_HDR + 128u), AGT);
            }
            u32 old = 0;
            VMWAIT(0);
            if (lane == 0 ? kind != 1u : (lane == 1u && strip))
                old = 1u + __hip_atomic_fetch_add((u32*)(wsl + OFF_CTR + (lane == 0 ? lk * 8u + ch : 80u) * 128u), 1u, __ATOMIC_RELAXED,
                                                  __HIP_MEMORY_SCOPE_AGENT);
            u64* f1 = (u64*)(p.ws[(rank + lane) & 7u] + OFF_F1 + lane * 128u);
            const u64 fv = ((u64)min((size + PRED_PAD + 63u) & ~63u, E1_SLOT) << 28) | sq;
            if (RFL(old) == sq * (kind == 0u ? 64u : 32u) && lane - 1u < 7u && (kind == 0u || lane == lk))
                __hip_atomic_store(f1 + ch * 2u + (kind == 0u), fv, __ATOMIC_RELAXED, __HIP_MEMORY_SCOPE_SYSTEM);
            if ((u32)__builtin_amdgcn_readlane((int)old, 1) == sq * 128u && lane - 1u < 7u)
                __hip_atomic_store(f1, fv, __ATOMIC_RELAXED, __HIP_MEMORY_SCOPE_SYSTEM);
        }
    }
    const u32 seq = RFL(seqv) + 1u;
    STAMP(10);

    u32 spins = 0;
    u64 t0 = 0;
    auto expired = [&]() -> bool {
        if ((++spins & 63u) != 0u) return false;
        if (LW[L_BAD / 4] != 0u) return true;
        const u64 t = now();
        if (t0 == 0) t0 = t;
        if (t - t0 > DEADLINE || __builtin_amdgcn_raw_buffer_load_b32(rs_ws, OFF_TMO, 0, AGT) == seq) {
            u32 sq = seq;
            asm volatile("" : "+s"(sq));
            __builtin_amdgcn_raw_buffer_store_b32(sq, rs_ws, OFF_TMO, 0, AGT);
            LW[L_BAD / 4] = 1u;
            return true;
        }
        return false;
    };

    auto arrive = [&](u32 wm, u32 n) {
        if (lane == 0) {
            __hip_atomic_fetch_max((LAS u32*)(L + L_AMAX), wm, __ATOMIC_RELAXED, __HIP_MEMORY_SCOPE_WORKGROUP);
            const u32 old = __hip_atomic_fetch_add((LAS u32*)(L + L_ACNT), 1u, __ATOMIC_RELAXED, __HIP_MEMORY_SCOPE_WORKGROUP);
            if (old == n - 1u) {
                u32 a = LW[L_AMAX / 4];
                if (LW[L_BAD / 4] != 0u) a = 0xFFFFFFFFu;
                __builtin_amdgcn_raw_buffer_store_b64(u32x2{a, seq}, rs_ws, OFF_AMX + b * 8u, 0, AGT);
            }
        }
    };

    if (b < 128u && wid < 8u) {
        u32 c = lane & 15u, g = lane >> 4;
        const u32 mw = wid & 3u, nh = wid >> 2;
        for (u32 f0 = 0; f0 < fills; f0 += 128u) {
        f32x16 acc[8];
        u32 xa = mw * 4096u + (c * 4u + (g ^ ((c >> 2) & 2u))) * 16u, wa = 16384u + nh * 8192u + lane * 16u;
        const u32 ca = lane < 32u ? L_FULL + ((lane - 1u) & 3u) * 4u : lane < 48u ? L_FREE + (wid ^ 4u) * 4u : L_FREE + wid * 4u - 768u;
        u32 f2 = f0 + 1u, spin = 0, tv;
#define S0_(x) #x
#define S_(x) S0_(x)
#define R4(r) "v[" S_(r) ":" S_(r) "+3]"
#define MM_(j, m, w, x) "v_mfma_f32_16x16x32_bf16 " R4(16*j+4*m) ", " R4(w) ", " R4(x) ", " R4(16*j+4*m) "\n"
#define MM4(j, w, x0) MM_(j, 0, w, x0) MM_(j, 1, w, 132) MM_(j, 2, w, 136) MM_(j, 3, w, 140)
#define RW(r, o) "ds_read_b128 " R4(r) ", %[wa] offset:" S_(o) "\n"
#define RX(r, o) "ds_read_b128 " R4(r) ", %[xa] offset:" S_(o) "\n"
#define LG(n) "s_waitcnt lgkmcnt(" S_(n) ")\n" KPAD
#define KWAIT(l) \
    "8" l ":\n s_cmp_eq_u32 %[f2], %[last]\n s_cbranch_scc1 9" l "b\n ds_read_b32 %[tv], %[ca]\n s_waitcnt lgkmcnt(0)\n" \
    "v_cmp_le_u32 vcc, %[f2], %[tv]\n s_nop 4\n s_bitcmp1_b32 vcc_lo, %[f2]\n s_cbranch_scc1 9" l "b\n s_sleep 1\n" \
    "s_add_u32 %[spin], %[spin], 1\n s_cmp_lt_u32 %[spin], 0x1000000\n s_cbranch_scc1 8" l "b\n s_branch 9" l "b\n"
#define KFILL(l, x0, y0, cur, nxt, TOGGLE) \
    LG(5) MM_(0, 0, 144, x0) MM_(1, 0, 148, x0) LG(4) MM_(0, 1, 144, 132) MM_(1, 1, 148, 132) \
    LG(3) MM_(0, 2, 144, 136) MM_(1, 2, 148, 136) LG(2) MM_(0, 3, 144, 140) RW(144, cur+4096) MM_(1, 3, 148, 140) RW(148, cur+5120) \
    "ds_read_b32 %[tv], %[ca]\n" \
    "s_waitcnt lgkmcnt(4)\n s_add_u32 %[f2], %[f2], 1\n" MM4(2, 152, x0) RW(152, cur+6144) \
    LG(4) MM4(3, 156, x0) RW(156, cur+7168) \
    "s_waitcnt lgkmcnt(2)\n v_cmp_le_u32 vcc, %[f2], %[tv]\n" MM_(4, 0, 144, x0) \
    "v_mov_b32_e64 %[tv], %[f2]\n" MM_(4, 1, 144, 132) \
    "ds_write_b32 %[ca], %[tv] offset:768\n"         \
    MM_(4, 2, 144, 136) TOGGLE MM_(4, 3, 144, 140) \
      \
    "s_setprio " S_(CPRIO) "\n s_bitcmp0_b32 vcc_hi, 0\n s_cbranch_scc1 7" l "f\n s_setprio " S_(CPRIO+BALANCE) "\n7" l ":\n" \
    "s_bitcmp0_b32 vcc_lo, %[f2]\n s_cbranch_scc1 8" l "f\n9" l ":\n" \
    RW(144, nxt) RX(y0, nxt) MM4(5, 148, x0) RW(148, nxt+1024) \
    LG(4) MM_(6, 1, 152, 132) MM_(7, 1, 156, 132) RX(132, nxt+1024) MM_(6, 2, 152, 136) MM_(7, 2, 156, 136) RX(136, nxt+2048) \
    MM_(6, 3, 152, 140) MM_(7, 3, 156, 140) RX(140, nxt+3072) MM_(6, 0, 152, x0) RW(152, nxt+2048) MM_(7, 0, 156, x0) RW(156, nxt+3072)
        asm volatile(
            ".set i_, 0\n .rept 64\n v_mov_b64 v[i_:i_+1], 0\n .set i_, i_+2\n .endr\n"
            "s_branch 80f\n90:\n"
            RW(144, 0) RX(128, 0) RW(148, 1024) RX(132, 1024) RX(136, 2048) RX(140, 3072) RW(152, 2048) RW(156, 3072)
            ".p2align 3\n1:\n"
            KFILL("1", 128, 160, 0, 32768, "")
            KFILL("2", 160, 128, 32768, 0, "v_xor_b32 %[wa], 0x10000, %[wa]\n v_xor_b32 %[xa], 0x10000, %[xa]\n")
            "s_cmp_lt_u32 %[f2], %[last]\n s_cbranch_scc1 1b\n s_branch 2f\n"
            KWAIT("0") KWAIT("1") KWAIT("2")
            "2:\n s_waitcnt lgkmcnt(0)\n s_nop 7\n s_nop 7\n"
            : "={v[0:15]}"(acc[0]), "={v[16:31]}"(acc[1]), "={v[32:47]}"(acc[2]), "={v[48:63]}"(acc[3]), "={v[64:79]}"(acc[4]),
              "={v[80:95]}"(acc[5]), "={v[96:111]}"(acc[6]), "={v[112:127]}"(acc[7]),
              [xa] "+{v164}"(xa), [wa] "+{v165}"(wa), [tv] "={v166}"(tv), [f2] "+s"(f2), [spin] "+s"(spin)
            : [ca] "{v167}"(ca), [last] "s"(f0 + 129u)
            : "v128", "v129", "v130", "v131", "v132", "v133", "v134", "v135", "v136", "v137", "v138", "v139", "v140", "v141", "v142",
              "v143", "v144", "v145", "v146", "v147", "v148", "v149", "v150", "v151", "v152", "v153", "v154", "v155", "v156", "v157",
              "v158", "v159", "v160", "v161", "v162", "v163", "vcc", "scc", "memory");
        if (spin >= 0x1000000u) LW[L_BAD / 4] = 1u;
        asm volatile("v_mbcnt_lo_u32_b32 %0, -1, 0\n v_mbcnt_hi_u32_b32 %0, -1, %0" : "=v"(lane));
        c = lane & 15u; g = lane >> 4;
        STAMPV(f0 == 0u ? 14 : 15, (u64)spin << 24);
        const bool direct = f0 == 0u && q != 0u && wid >= 8u - ND2;
        const u32 reg = L_PACK + wid * 3072u;
        if (f0 == 0u) {
            STAMP(2);
            STAMPV(9, __builtin_amdgcn_s_memtime());
            float mx = 0.f;
#pragma unroll
            for (int j = 0; j < 8; ++j)
#pragma unroll
                for (int m = 0; m < 4; ++m) {
                    mx = fmaxf(fmaxf(mx, fabsf(acc[j][4 * m])), fabsf(acc[j][4 * m + 1]));
                    mx = fmaxf(fmaxf(mx, fabsf(acc[j][4 * m + 2])), fabsf(acc[j][4 * m + 3]));
                }
            arrive(wave_max_u32(__float_as_uint(mx)), 8u);
            STAMP(3);
            if (direct) __builtin_amdgcn_s_setprio(3); else if (wid < 4u) __builtin_amdgcn_s_setprio(2); else __builtin_amdgcn_s_setprio(0);
#pragma unroll
            for (int k = 0; k < 3; ++k) *(LAS u32x4*)(L + reg + k * 1024u + lane * 16u) = u32x4{0u, 0u, 0u, 0u};
            while (LW[L_SCALE / 4] != 1u) { if (expired()) break; __builtin_amdgcn_s_sleep(CSLEEP); }
            CBAR();
        } else STAMP(13);
        const u32 gmax = LW[L_SCALE / 4 + 1];
        const bool bad = (gmax == 0xFFFFFFFFu) || (LW[L_BAD / 4] != 0u);
        const float amax = __uint_as_float(gmax);
        const float inv = amax > 0.f ? 127.0f / amax : 0.f;
        if (f0 == 0u) STAMP(4);
        const float MAGIC = 12582912.0f;
        const float qlo = bad ? MAGIC - 128.0f : MAGIC - 127.0f, qhi = bad ? MAGIC - 128.0f : MAGIC + 127.0f;
        const u32 ovoff = (c & 7u) * NN + ((c >> 3) * 4u + g) * 16u;
        const u32 os = (nb2 - 8u) * 2u + nh;
        const u32 osoff = (mb * 256u + mw * 64u) * NN + (f0 == 0u ? rank * 2048u + nb * 256u + nh * 128u : ((rank + 1u + os) & 7u) * 2048u + 1792u);
        const rsrc_t rs_or = mk_rsrc(p.out[(rank + q) & 7u], OUT_BYTES);
        u32x4 qv[8];
#pragma unroll
        for (int m = 0; m < 4; ++m) {
#pragma unroll
            for (int r = 0; r < 4; ++r) {
                u32 o[2];
#pragma unroll
                for (int h = 0; h < 2; ++h) {
                    u32 t[4];
#pragma unroll
                    for (int jj = 0; jj < 4; ++jj)
                        t[jj] = __float_as_uint(__builtin_amdgcn_fmed3f(__builtin_fmaf(acc[4 * h + jj][4 * m + r], inv, MAGIC), qlo, qhi));
                    o[h] = PERM(t[1], t[0], 0x0c0c0400u) | PERM(t[3], t[2], 0x04000c0cu);
                }
                qv[2 * m][r] = (u32)__builtin_amdgcn_update_dpp((int)o[0], (int)o[1], 0x128, 0xf, 0xc, false);
                qv[2 * m + 1][r] = (u32)__builtin_amdgcn_update_dpp((int)o[1], (int)o[0], 0x128, 0xf, 0x3, false);
            }
            if (direct) {
                ST128(qv[2 * m], rs_or, ovoff, osoff + (2 * m) * 8u * NN, SYS);
                ST128(qv[2 * m + 1], rs_or, ovoff, osoff + (2 * m + 1) * 8u * NN, SYS);
            }
        }
        if (direct) __builtin_amdgcn_s_setprio(0);
        if (f0 == 0u) {
        STAMP(7);
        u32 size = nb == 7u && nh == 0u ? 0u : 8192u;
        const u32 slot = E2_OFF + (b * 8u + wid) * 8192u;
        if (size != 0u && !(q == 0u && wid == 7u) && !(FORCE_RAW2 && (wid & 1u))) {
            u32x4 nib[4], sg = u32x4{0u, 0u, 0u, 0u};
            u32 code[16], len[16], over = 0, tot = 0;
#pragma unroll
            for (int o = 0; o < 16; ++o) {
                u32 f[2];
#pragma unroll
                for (int e = 0; e < 2; ++e) {
                    const int i = 2 * o + e;
                    const u32 v = qv[i >> 2][i & 3], s = byte_signs(v);
                    f[e] = v ^ s;
                    sg[i >> 3] = bfi(0x01010101u << (i & 7), s, sg[i >> 3]);
                }
                nib[o >> 2][o & 3] = bfi(0x0F0F0F0Fu, f[0], f[1] << 4);
                const u32 last = unary8((f[0] >> 4) & 0x07070707u, (f[1] >> 4) & 0x07070707u, code[o]);
                over |= last;
                len[o] = last + 1u;
                tot += len[o];
            }
            const u32 end = wave_scan_u32(tot);
            const u32 bits = (u32)__builtin_amdgcn_readlane((int)end, 63);
            if (BALLOT(over > 31u || tot > 128u + 255u) == 0ull && bits <= (8192u - 64u - R_BITS) * 8u) {
                size = (R_BITS + ((bits + 127u) >> 7) * 16u);
#pragma unroll
                for (int e = 0; e < 4; ++e) ST128(nib[e], rs_ws, lane * 16u + e * 1024u, slot, AGT);
                ST128(sg, rs_ws, lane * 16u, slot + 4096u, AGT);
                *(LAS char*)(L + reg + lane) = (char)(tot - 128u);
                u32 pos = end - tot + (reg + 64u) * 8u;
#pragma unroll
                for (int o = 0; o < 16; ++o) { put_bits(L, pos, code[o]); pos += len[o]; }
                CBAR();
#pragma unroll
                for (int k = 0; k < 3; ++k) {
                    const u32 off = R_LEN + k * 1024u + lane * 16u;
                    if (R_LEN + k * 1024u < size) ST128(LDSV(reg + k * 1024u + lane * 16u), rs_ws, off < size ? off : OOB, slot, AGT);
                }
            }
        }
        if (size == 8192u) {
#pragma unroll
            for (int e = 0; e < 8; ++e) ST128(qv[e], rs_ws, lane * 16u, slot + e * 1024u, AGT);
        }
        STAMP(11);
        VMWAIT(0);
        {
            u32 old = 0;
            if (lane == 0) {
                *(volatile LAS unsigned short*)(L + L_GSZ + wid * 2u) = (unsigned short)(size >> 4);
                old = __hip_atomic_fetch_add((LAS u32*)(L + L_GCNT + nh * 4u), 1u, __ATOMIC_RELAXED, __HIP_MEMORY_SCOPE_WORKGROUP);
            }
            if (RFL(old) == 3u && lane - 1u < 7u) {
                const u32x4 fv = u32x4{seq, LW[L_GSZ / 4 + nh * 2u], LW[L_GSZ / 4 + nh * 2u + 1u], seq};
                u32x2 fp = __builtin_bit_cast(u32x2, p.ws[(rank + lane) & 7u]);
                asm volatile("v_add_co_u32 %0, vcc, %0, %2\n v_addc_co_u32 %1, vcc, 0, %1, vcc" : "+v"(fp[0]), "+v"(fp[1])
                             : "v"(OFF_F2 + ((lane * 16u + idx) * 16u + q * 2u + nh) * 16u) : "vcc");
                asm volatile("global_store_dwordx4 %0, %1, off sc0 sc1\n s_nop 1" :: "v"(fp), "v"(fv) : "memory");
            }
        }
        STAMP(5);
        __builtin_amdgcn_s_setprio(0);
        }
        const rsrc_t rs_ol = mk_rsrc(p.out[rank], f0 == 0u || os < 7u ? OUT_BYTES : 0u);
#pragma unroll
        for (int e = 0; e < 8; ++e) ST128(qv[e], rs_ol, ovoff, osoff + e * 8u * NN, AGT);
        }
    } else if (b < 128u && wid == 8u) {
        __builtin_amdgcn_s_setprio(LPRIO);
        for (u32 n = 0; n < fills / 2u;) {
            const u32 wn = n == 0u ? nb : nb2, end = n + 64u;
            const bool hole = wn == 11u && (lane & 5u) == 5u;
            u32 step = lane & 56u, va = OFF_UF + (step * 112u + (lane & 4u ? 64u + wn * 4u : mb * 4u) + (lane & 3u)) * 128u;
            bool seen = hole;
            while (n != end) {
                while (2u * n > LW[L_FREE / 4] + PAHEAD) { if (expired()) break; __builtin_amdgcn_s_sleep(8); }
                seen |= __builtin_amdgcn_raw_buffer_load_b32(rs_ws, seen ? OOB : va, 0, SYS) >= seq;
                u64 avail = BALLOT(seen);
                avail &= avail >> 1; avail &= avail >> 2; avail &= avail >> 4;
                avail &= 0x0101010101010101ull;
                if (avail == 0) {
                    if (!expired()) { __builtin_amdgcn_s_sleep(POLL_SLEEP); continue; }
                    avail = BALLOT(va != OOB) & 0x0101010101010101ull;
                }
                if ((avail >> lane) & 1ull) LW[L_STEPS / 4 + n + rank_in(avail)] = step;
                n += (u32)__builtin_popcountll(avail);
                LW[L_NRDY / 4] = n;
                if ((avail >> (lane & 56u)) & 1ull) {
                    seen = hole;
                    va = (++step & 7u) ? va + 112u * 128u : OOB;
                }
            }
            if (n != 64u) break;
            STAMP(1);
            while (LW[L_FREE / 4] < 128u) { if (expired()) break; __builtin_amdgcn_s_sleep(8); }
            u32 a = 0;
            for (;;) {
                const u32x2 gm = __builtin_amdgcn_raw_buffer_load_b64(rs_ws, OFF_GAM + q * 128u + (lane & 7u) * 8u, 0, SYS);
                a = gm[0];
                if (BALLOT(gm[1] == seq) == ~0ull) break;
                if (expired()) { a = 0xFFFFFFFFu; break; }
                __builtin_amdgcn_s_sleep(1);
            }
            a = wave_max_u32(a);
            LW[L_SCALE / 4 + 1] = a;
            LW[L_SCALE / 4] = 1u;
            STAMP(2);
            while (LW[L_GCNT / 4] + LW[L_GCNT / 4 + 1] < 8u) { if (expired()) break; __builtin_amdgcn_s_sleep(8); }
        }
    } else if (b < 128u) {
#define CLK(c)
        __builtin_amdgcn_s_setprio(LPRIO);
        for (u32 f = wid - 9u; f < fills; f += 3u) {
            CLK(c0);
            while (LW[L_NRDY / 4] <= (f >> 1)) { if (expired()) break; __builtin_amdgcn_s_sleep(1); }
            CBAR();
            const u32 step = RFL(LW[L_STEPS / 4 + (f >> 1)]) & 63u;
            CLK(c1);
            bool ahead = false;
            while (BALLOT(LW[L_FREE / 4 + (lane & 7u)] + 2u >= f) != ~0ull && f >= 4u) {
                ahead = true;
                if (expired()) break;
                __builtin_amdgcn_s_sleep(LSLEEP);
            }
            CBAR();
            const u32 sbase = STAGE_OFF + step * STEP_BYTES + (f & 1u) * 16384u;
            LAS char* ldst = L + (f & 3u) * SLOT_BYTES;
            CLK(c2);
            auto fill = [&](auto pace) {
#pragma unroll
                for (int i = 0; i < DIAG_NDMA; ++i) {
                    const u32 so = sbase + (i < 4 ? mb : 16u + (f < 128u ? nb : nb2)) * PIECE + (i & 3) * 4096u;
                    __builtin_amdgcn_raw_ptr_buffer_load_lds(rs_ws, ldst + i * 4096, 16, lane * 16u, so, 0, SYS); pace();
                    __builtin_amdgcn_raw_ptr_buffer_load_lds(rs_ws, ldst + i * 4096, 16, lane * 16u, so, 1024, SYS); pace();
                    __builtin_amdgcn_raw_ptr_buffer_load_lds(rs_ws, ldst + i * 4096, 16, lane * 16u, so, 2048, SYS); pace();
                    __builtin_amdgcn_raw_ptr_buffer_load_lds(rs_ws, ldst + i * 4096, 16, lane * 16u, so, 3072, SYS); pace();
                }
            };
            if (ahead || !LADAPT) fill([] { asm volatile("s_nop %0" :: "n"(LNOP) : "memory"); });
            else fill([] { });
            CBAR();
            CLK(c3);
            VMWAIT(0);
            CLK(c4);
            LW[L_FULL / 4 + (f & 3u)] = f + 1u;
            if (f - 125u < 3u) STAMP(1);
        }
    } else if (b < 240u) {
        arrive(0u, 12u);
        const u32 src = (rank - rel) & 7u;
        const rsrc_t rs_src = mk_rsrc(p.ws[src], WS_NEED);
        auto in_unit = [&](u32 u, u32& ch, u32& r, bool& isw) -> u32 {
            if (u >= 768u) {
                ch = (u - 768u) / 14u; r = (u - 768u) % 14u; isw = true;
                const u32 os = r >> 1, rs = 28u + (r & 1u) * 2u;
                r = strip_at(os, rs);
                return E1W_OFF + ((((rel + 1u + os) & 7u) * 8u + ch) * 32u + rs) * E1_SLOT;
            }
            ch = u / 96u; r = u % 96u; isw = r < 32u;
            if (!isw) r = (r - 32u + 8u * rel) & 63u;
            return isw ? E1W_OFF + ((rel * 8u + ch) * 32u + r) * E1_SLOT : E1X_OFF + (ch * 64u + r) * E1_SLOT;
        };
        auto out_unit = [&](u32 o) -> u32 {
            u32 k = o >> 4;
            if (k == N2 - 1u) return (o & 15u) * 64u + 7u;
            if (k >= N2 - 13u) {
                const u32 j = o - 16u * (N2 - 13u);
                return j % 12u * 64u + (j / 48u) * 16u + 8u + j / 12u % 4u;
            }
            k += k >= 7u;
            if (k >= (rel >> 1) * 12u + (rel & 1u) * 4u + 8u - ND2) k += ND2;
            return (o & 15u) * 64u + k / 12u * 16u + k % 12u + (k % 12u >= 8u ? 4u : 0u);
        };
        if (wid == 0u) {
            __builtin_amdgcn_s_setprio(3);
            auto ctl = [&](u32 a) -> u32 {
                u32 v;
                asm volatile("ds_read_b32 %0, %1\n s_waitcnt lgkmcnt(0)" : "=v"(v) : "v"(a) : "memory");
                return v;
            };
            auto put = [&](u32 a, u32 n) { asm volatile("ds_write_b32 %0, %1" :: "v"(a), "v"(n) : "memory"); };
            const u32 ca = L_FULL + (lane & 15u) * 4u;
            const u32 vone = lane == 0u ? 0u : OOB;
            u32 n = 0;
            auto wait = [&](int word, u32 bit) {
                auto ok = [&](u32 w) -> bool {
                    return (((u32)__builtin_amdgcn_readlane((int)w, word) >> bit) & 1u) != 0u &&
                           (u32)__builtin_amdgcn_readlane((int)w, 8 + (n & 7u)) + 15u >= n;
                };
                if (!ok(ctl(ca))) {
                    VMWAIT(0);
                    put(L_FULL, n);
                    while (!ok(ctl(ca))) { if (expired()) break; __builtin_amdgcn_s_sleep(1); }
                }
            };
#define CLAIM(cl, so) asm volatile("v_mov_b32 %0, 1\n s_nop 4\n buffer_atomic_add %0, %1, %2, %3 offen sc0 sc1" \
                                   : "=&v"(cl) : "v"(vone), "s"(rs_ws), "s"(so) : "memory")
#define CLAIMED(d, cl) asm volatile("v_readfirstlane_b32 %0, %1" : "=s"(d) : "v"(cl) : "memory")
            for (u32 u = 32u + pj; u < N1; ++n) {
                u32 ch, r, cl; bool isw;
                const u32 so = in_unit(u, ch, r, isw);
                if (u >= 768u) put(L_P1, 1u);
                const u32 seg = u >= 768u ? 0u : ch * 2u + !isw;
                wait(1, seg);
                if (n == 0) STAMP(1);
                LAS char* ld = L + (n & 15u) * E1_SLOT;
                const u32 sz = min(max(RFL(ctl(L_PSZ + seg * 4u)), 321u), E1_SLOT / 16u);
                put(L_Q + (n & 15u) * 4u, u + (sz << 16));
                CLAIM(cl, OFF_CTR + (64u + rel) * 128u);
#pragma unroll
                for (int i = 0; i < 9; ++i) {
                    if (i >= 6 && i * 64u >= sz) break;
                    VMWAIT(WIN1 - (i < WIN1 - 1 ? 0 : 1));
                    if (i == WIN1 - 1) put(L_FULL, n);
                    if (i < 5 || i * 64u + lane < sz)
                        __builtin_amdgcn_raw_ptr_buffer_load_lds(rs_src, ld + i * 1024, 16, lane * 16u, so + i * 1024u, 0, SYS);
                }
                CLAIMED(u, cl);
                u = u - (seq - 1u) * (N1 - 32u) + 48u;
            }
            put(L_P1, 1u);
            STAMP(2);
            for (u32 o = pj; o < 16u * N2; ++n) {
                const u32 t = out_unit(o), so = E2_OFF + t * 8192u;
                const bool dyn = o >= 16u * (S2 - 1u);
                u32 cl;
                wait(2, o < 16u * S2 ? (t >> 2) & 15u : 16u);
                if (o == pj) STAMP(3);
                LAS char* ld = L + (n & 15u) * E1_SLOT;
                const u32 sz = min(RFL(ctl(L_SZ + (t >> 1) * 4u)) >> ((t & 1u) * 16u) & 0xFFFFu, 512u);
                put(L_Q + (n & 15u) * 4u, 1024u + t + (sz << 16));
                CLAIM(cl, dyn ? OFF_CTR + (72u + rel) * 128u : OOB);
#pragma unroll
                for (int i = 0; i < 8; ++i) {
                    if (i >= 6 && i * 64u >= sz) break;
                    VMWAIT(WIN2 - (i < WIN2 - 1 ? 0 : 1));
                    if (i == WIN2 - 1) put(L_FULL, n);
                    if (i < 6 || i * 64u + lane < sz)
                        __builtin_amdgcn_raw_ptr_buffer_load_lds(rs_src, ld + i * 1024, 16, lane * 16u, so + i * 1024u, 0, SYS);
                }
                u32 c;
                CLAIMED(c, cl);
                o = dyn ? c - (seq - 1u) * (16u * (N2 - S2 + 1u)) + 16u * S2 : o + 16u;
            }
            VMWAIT(0);
            put(L_FULL, n);
            put(L_END, 1u);
            STAMP(4);
            if (lane == 0) __builtin_amdgcn_raw_buffer_store_b32(seq, rs_src, OFF_DONE + rank * 128u + pj * 4u, 0, SYS);
        } else if (wid == 1u) {
            for (u32 m = 0; m != 0xFFFFu;) {
                const u32x2 f = __builtin_amdgcn_raw_buffer_load_b64(rs_ws, OFF_F1 + rel * 128u + (lane & 15u) * 8u, 0, SYS);
                m = (u32)BALLOT(f[0] >= seq) & 0xFFFFu;
                LW[L_PSZ / 4 + (lane & 15u)] = f[1];
                if (expired()) m = 0xFFFFu;
                LW[L_SEG / 4] = m;
                __builtin_amdgcn_s_sleep(4);
            }
            STAMP(1);
            while (LW[L_P1 / 4] == 0u) { if (expired()) break; __builtin_amdgcn_s_sleep(16); }
            u32 m = 0;
            while (m != 0xFFFFu) {
                const u32x4 f = LD128(rs_ws, lane < 16u ? lane * 16u : OOB, OFF_F2 + (rel * 16u + pj) * 256u, SYS);
                m = (u32)BALLOT(f[0] == seq && f[3] == seq) & 0xFFFFu;
                if (lane < 16u) *(LAS u32x2*)(L + L_SZ + pj * 128u + lane * 8u) = u32x2{f[1], f[2]};
                if (expired()) m = 0xFFFFu;
                CBAR();
                LW[L_G2 / 4] = m;
                __builtin_amdgcn_s_sleep(2);
            }
            STAMP(2);
            for (u32 i = 0; i < 4u;) {
                const u32x4 f = LD128(rs_ws, lane * 16u, OFF_F2 + rel * 4096u + i * 1024u, SYS);
                *(LAS u32x2*)(L + L_SZ + i * 512u + lane * 8u) = u32x2{f[1], f[2]};
                if (BALLOT(f[0] == seq && f[3] == seq) == ~0ull || expired()) ++i; else __builtin_amdgcn_s_sleep(16);
            }
            CBAR();
            LW[L_G2 / 4] = 0x1FFFFu;
            STAMP(3);
        } else if (wid < 10u) {
            const rsrc_t rs_ol = mk_rsrc(p.out[rank], OUT_BYTES);
            const u32 ovoff = (lane & 7u) * NN + (((lane >> 3) & 1u) * 4u + (lane >> 4)) * 16u;
            for (u32 n = wid - 2u;; n += 8u) {
                bool more = true;
                for (;;) {
                    const u32 end = LW[L_END / 4];
                    if (LW[L_FULL / 4] > n) break;
                    if (end != 0u) { more = false; break; }
                    expired();
                    __builtin_amdgcn_s_sleep(2);
                }
                if (!more) break;
                CBAR();
                const u32 sl = (n & 15u) * E1_SLOT;
                const u32 id = RFL(LW[L_Q / 4 + (n & 15u)]);
                if ((id & 1024u) == 0u) {
                u32 ch, r; bool isw;
                const u32 so = in_unit(id & 1023u, ch, r, isw);
                const u32 hd = sl + I_HDR;
                u32x4 tab = LDSV(hd);
                const u32 tot = (u32)*(volatile LAS unsigned char*)(L + hd + I_LEN + lane) + 64u;
                const u32 h0 = RFL(tab[0]), top = ((h0 >> 4) & 15u) | ((h0 >> 8) & 0xF0u), cnt = (h0 >> 20) & 15u;
                const u32 end = wave_scan_u32(tot);
                const u32 size = cnt == 15u ? E1_SLOT : I_HDR + I_VAR + cnt * 8u + (((u32)__builtin_amdgcn_readlane((int)end, 63) + 7u) >> 3);
                if (size > (id >> 16) * 16u) {
                    for (u32 at = (id >> 16) * 16u; at < size; at += 1024u)
                        if (at + lane * 16u < size)
                            __builtin_amdgcn_raw_ptr_buffer_load_lds(rs_src, L + sl + at, 16, lane * 16u, so + at, 0, SYS);
                    VMWAIT(0);
                }
                u32x4 o[8];
                if (cnt == 15u) {
#pragma unroll
                    for (int e = 0; e < 8; ++e) o[e] = LDSV(sl + (e >> 2) * (I_HDR + 128u) + (e & 3) * 1024u + lane * 16u);
                } else {
                    u32x4 mq[4];
                    u32 ex[16];
#pragma unroll
                    for (int e = 0; e < 4; ++e) mq[e] = LDSV(sl + e * 1024u + lane * 16u);
                    const u32x2 pl = *(const LAS u32x2*)(L + hd + I_BIT + lane * 8u);
                    tab = (tab & 0x0F0F0F0Fu) ^ u32x4{TFLIP0, TFLIP1, TFLIP0, TFLIP1};
                    u32 pos = end - tot + (hd + I_VAR + cnt * 8u) * 8u;
#pragma unroll
                    for (int g = 0; g < 8; ++g) {
                        u32 s0, s1;
                        counts8(L, pos, s0, s1);
                        ex[2 * g] = lut16(tab, bfi(0x08080808u, pl[g >> 2] << (g & 3), s0));
                        ex[2 * g + 1] = lut16(tab, bfi(0x08080808u, pl[g >> 2] >> (4 - (g & 3)), s1));
                    }
                    for (u32 k = 0; k < cnt; ++k) {
                        const u32 at = LW[(hd + I_VAR) / 4 + 2u * k] - lane * 16u, val = LW[(hd + I_VAR) / 4 + 1u + 2u * k];
#pragma unroll
                        for (int j = 0; j < 16; ++j) ex[j] = at == (u32)j ? val : ex[j];
                    }
#pragma unroll
                    for (int j = 0; j < 16; ++j) {
                        const u32 e = top * 0x01010101u - ex[j], m = mq[j >> 2][j & 3];
                        const u32 r0 = PERM(e, m, 0x05010400u), r1 = PERM(e, m, 0x07030602u);
                        o[j >> 1][(2 * j) & 3] = __builtin_amdgcn_alignbit(r0, r0, 1);
                        o[j >> 1][(2 * j + 1) & 3] = __builtin_amdgcn_alignbit(r1, r1, 1);
                    }
                }
                CBAR();
                LW[L_FREE / 4 + (n & 7u)] = n + 1u;
                const u32 flag = put_unit(o, rs_ws, isw, rel, ch, r);
                VMWAIT(0);
                put_flag(rs_ws, flag, seq);
                STAMP(1);
                continue;
                }
                const u32 sz = id >> 16, x = (id >> 3) & 7u, ix = (id >> 6) & 15u;
                const u32 osoff = ((4u * (x >> 1) + (ix & 3u)) * 256u + (id & 3u) * 64u) * NN + src * 2048u +
                                  (4u * (x & 1u) + (ix >> 2)) * 256u + ((id >> 2) & 1u) * 128u;
                u32x4 qv[8];
                if (sz >= 512u) {
#pragma unroll
                    for (int e = 0; e < 8; ++e) qv[e] = LDSV(sl + e * 1024u + lane * 16u);
                } else {
                    const u32 tot = (u32)*(volatile LAS unsigned char*)(L + sl + R_LEN + lane) + 128u;
                    u32x4 nib[4];
#pragma unroll
                    for (int e = 0; e < 4; ++e) nib[e] = LDSV(sl + e * 1024u + lane * 16u);
                    const u32x4 sg = LDSV(sl + 4096u + lane * 16u);
                    u32 pos = wave_scan_u32(tot) - tot + (sl + R_BITS) * 8u;
#pragma unroll
                    for (int o = 0; o < 16; ++o) {
                        u32 h0, h1;
                        counts8(L, pos, h0, h1);
                        const u32 nv = nib[o >> 2][o & 3], s = sg[o >> 2] << (6 - 2 * (o & 3));
                        qv[o >> 1][(2 * o) & 3] = bfi(0x0F0F0F0Fu, nv, h0 << 4) ^ byte_signs(s << 1);
                        qv[o >> 1][(2 * o + 1) & 3] = bfi(0x0F0F0F0Fu, nv >> 4, h1 << 4) ^ byte_signs(s);
                    }
                }
                CBAR();
                LW[L_FREE / 4 + (n & 7u)] = n + 1u;
                if (LW[L_BAD / 4] != 0u) qv[0] = u32x4{0x80808080u, 0x80808080u, 0x80808080u, 0x80808080u};
#pragma unroll
                for (int e = 0; e < 8; ++e) ST128(qv[e], rs_ol, ovoff, osoff + e * 8u * NN, AGT);
            }
            STAMP(2);
        }
    } else {
        arrive(0u, 12u);
        if (b == AUX_WG && wid == 2u) {
            u32 a = 0;
            for (;;) {
                const u32x4 g0 = LD128(rs_ws, OFF_AMX + lane * 32u, 0, AGT);
                const u32x4 g1 = LD128(rs_ws, OFF_AMX + lane * 32u + 16u, 0, AGT);
                a = max(max(g0[0], g0[2]), max(g1[0], g1[2]));
                const bool ok = g0[1] == seq && g0[3] == seq && g1[1] == seq && g1[3] == seq;
                if (BALLOT(ok) == ~0ull) break;
                if (expired()) { a = 0xFFFFFFFFu; break; }
            }
            a = wave_max_u32(a);
            u64* gp = (u64*)(p.ws[lane & 7u] + OFF_GAM + (lane >> 3) * 128u + rank * 8u);
            __hip_atomic_store(gp, ((u64)seq << 32) | a, __ATOMIC_RELAXED, __HIP_MEMORY_SCOPE_SYSTEM);
            if (lane == 0) __builtin_amdgcn_raw_buffer_store_b32(seq, rs_ws, OFF_SEQ, 0, AGT);
            STAMP(3);
            bool give_up = false;
            for (;;) {
                const u32x4 f = LD128(rs_ws, OFF_DONE + lane * 16u, 0, SYS);
                const bool ok = (lane >> 3) == rank || (lane & 4u) != 0u || (f[0] >= seq && f[1] >= seq && f[2] >= seq && f[3] >= seq);
                if (BALLOT(ok) == ~0ull) break;
                if (expired()) { give_up = true; break; }
                __builtin_amdgcn_s_sleep(8);
            }
            if (give_up || __builtin_amdgcn_raw_buffer_load_b32(rs_ws, OFF_TMO, 0, AGT) == seq)
                if (lane == 0) p.out[rank][rank * 2048u] = (char)-128;
            STAMP(7);
        }
    }
}

extern "C" void kernel_launch(int rank, int nranks, ncclComm_t comm, hipStream_t stream,
                              void* const* d_in, const int* in_sizes, int n_in,
                              void* d_out, int out_size, const int* in_dtypes, int out_dtype,
                              void* const* d_out_all, void* const* d_ws_all, size_t ws_bytes) {
    if (nranks != NR || (unsigned)rank >= (unsigned)NR || n_in < 2 || ws_bytes < 3ull * WS_NEED ||
        in_sizes[0] != (int)(MM * KSH) || in_sizes[1] != (int)(KSH * NN) || out_size != (int)OUT_BYTES) {
        fprintf(stderr, "kernel_launch: unexpected problem shape\n");
        return;
    }
    static bool init = false;
    if (!init) {
        HIP_CHECK(hipFuncSetAttribute((const void*)gemm_ar_q8, hipFuncAttributeMaxDynamicSharedMemorySize, LDS_BYTES));
        init = true;
    }
    Params p;
    p.x = (const char*)d_in[0];
    p.w = (const char*)d_in[1];
    for (int r = 0; r < NR; ++r) { p.out[r] = (char*)d_out_all[r]; p.ws[r] = (char*)d_ws_all[r]; }
    p.rank = rank;
    hipLaunchKernelGGL(gemm_ar_q8, dim3(256), dim3(768), LDS_BYTES, stream, p);
    HIP_CHECK(hipGetLastError());
}
```

```cpp
#include <hip/hip_runtime.h>
#include <rccl/rccl.h>
#include <cstdio>
#include <cstdint>

#define HIP_CHECK(expr) do { hipError_t _e = (expr); if (_e != hipSuccess) { \
    fprintf(stderr, "HIP error %s at %s:%d\n", hipGetErrorString(_e), __FILE__, __LINE__); \
    return; } } while (0)

#define LAS __attribute__((address_space(3)))
#define GAS __attribute__((address_space(1)))
typedef unsigned int u32;
typedef unsigned long long u64;
typedef u32 u32x4 __attribute__((ext_vector_type(4)));
typedef u32 u32x2 __attribute__((ext_vector_type(2)));
typedef unsigned short u16x2 __attribute__((ext_vector_type(2)));
typedef float f32x16 __attribute__((ext_vector_type(16)));

#define WIN1 4
#define WIN2 4
#define SGM 0xBBBBu
#define PCH 2
#define PSTEP 7u
#define PSTEP2 11u
#define PBACK 20u
#define PLEAD 8u
#define WARM 1
#define TDELAY 100
#define WDELAY1 50
#define WDELAY2 200
#define S2 (SGM == 0xAAAAu ? 32u : 48u)
#define DIAG_NDMA 8
#define CSLEEP 1
#define POLL_SLEEP 2
#define BALANCE 1
#define LSLEEP 1
#define KPAD "s_nop 0\n"
#define FLAGW 4
#define PAHEAD 4
#define LNOP 5
#define LADAPT 0
#define CPRIO 0
#define LPRIO 0
#define PRED_PAD 20
#define FORCE_RAW1 0
#define FORCE_RAW2 0

constexpr int NR = 8;
constexpr u32 MM = 4096, NN = 16384, KSH = 512;
constexpr u32 X_BYTES = MM * KSH * 2, W_BYTES = KSH * NN * 2, OUT_BYTES = MM * NN;
constexpr u32 PIECE = 32768;
constexpr u32 STEP_BYTES = 28 * PIECE;
constexpr u32 OFF_UF = 1u << 20;
constexpr u32 OFF_F3 = OFF_UF + 64 * 112 * 128;
constexpr u32 STAGE_OFF = 2u << 20;
static_assert(OFF_F3 + 2048 * 64 <= STAGE_OFF, "flags");
constexpr u32 E1_SLOT = 8192 + 128;
constexpr u32 I_HDR = 4096, I_LEN = 16, I_BIT = 80, I_VAR = 592;
constexpr u32 E1X_OFF = STAGE_OFF + 64 * STEP_BYTES;
constexpr u32 E1W_OFF = E1X_OFF + 512 * E1_SLOT;
constexpr u32 E1P_OFF = E1W_OFF + 2048 * E1_SLOT;
constexpr u32 E2_OFF = E1P_OFF + 512 * E1_SLOT;
constexpr u32 WS_NEED = E2_OFF + 1024 * 8192;
constexpr u32 OFF_SEQ = 0, OFF_TMO = 256;
constexpr u32 OFF_AMX = 4096;
constexpr u32 OFF_GAM = OFF_AMX + 2048;
constexpr u32 OFF_DONE = OFF_GAM + 1024;
constexpr u32 OFF_F1 = OFF_DONE + 1024;
constexpr u32 OFF_CTR = OFF_F1 + 2048;
constexpr u32 OFF_F2 = 65536;
constexpr u32 OFF_FP = OFF_F2 + 32768;
constexpr u32 OFF_STAMP = 131072;
static_assert(OFF_CTR + 81 * 128 <= OFF_F2 && OFF_FP + 16384 <= OFF_STAMP && OFF_STAMP + 256 * 12 * 32 * 8 <= OFF_UF, "flags");

constexpr u32 SLOT_BYTES = 32768;
constexpr u32 CTRL = 16 * E1_SLOT;
constexpr u32 L_FULL = CTRL, L_FREE = CTRL + 32, L_NRDY = CTRL + 64, L_AMAX = CTRL + 68, L_ACNT = CTRL + 72,
              L_BAD = CTRL + 76, L_SCALE = CTRL + 80, L_GCNT = CTRL + 88, L_GSZ = CTRL + 96, L_SCNT = CTRL + 112, L_STEPS = CTRL + 128, L_PTR = CTRL + 640;
constexpr u32 L_SEG = CTRL + 4, L_G2 = CTRL + 8, L_END = CTRL + 12, L_P1 = CTRL + 16, L_G3 = CTRL + 20, L_Q = CTRL + 128,
              L_PSZ = CTRL + 192, L_SG = 8 * E1_SLOT;
constexpr u32 L_PACK = CTRL + 1024;
constexpr u32 L_SZ = L_PACK + 8 * 3072;
constexpr u32 LDS_BYTES = L_SZ + 2048;
constexpr u64 DEADLINE = 100000000ull;
constexpr u32 OOB = WS_NEED;
constexpr int SYS = 17, AGT = 16;
constexpr u32 CODE_BYTES = 40960;
constexpr u32 AUX_WG = 248;
constexpr u32 N1 = 768 + 112;
constexpr u32 N2 = 60;
constexpr u32 R_BIT = 1024, R_NIB = 2048, R_VAR = 6144, R_MAX = 3064;

struct Params {
    char* ws[NR];
    const char* x;
    const char* w;
    char* out;
    int rank;
};

typedef __amdgpu_buffer_rsrc_t rsrc_t;
__device__ __forceinline__ rsrc_t mk_rsrc(const void* p, u32 bytes) {
    return __builtin_amdgcn_make_buffer_rsrc((void*)p, 0, bytes, 0x00020000);
}
__device__ __forceinline__ u64 now() { return __builtin_amdgcn_s_memrealtime(); }
#define VMWAIT(n) asm volatile("s_waitcnt vmcnt(%0)" :: "n"(n) : "memory")
#define CBAR() asm volatile("" ::: "memory")
#define ST128(v, rs, voff, soff, aux) do { const u32x4 d_ = (v); \
    __builtin_amdgcn_raw_buffer_store_b128(d_, rs, voff, soff, aux); \
    asm volatile("s_nop 1" :: "v"(d_) : "memory"); } while (0)
#define LD128(rs, voff, soff, aux) __builtin_amdgcn_raw_buffer_load_b128(rs, voff, soff, aux)
#define PERM(a, b, s) __builtin_amdgcn_perm(a, b, s)
#define BALLOT(c) __builtin_amdgcn_ballot_w64(c)
#define RFL(v) ((u32)__builtin_amdgcn_readfirstlane(v))
__device__ __forceinline__ u32 bfi(u32 m, u32 a, u32 b) { return (a & m) | (b & ~m); }
__device__ __forceinline__ u32 rank_in(u64 m) {
    return __builtin_amdgcn_mbcnt_hi((u32)(m >> 32), __builtin_amdgcn_mbcnt_lo((u32)m, 0u));
}

#define STAMPV(i, v) do { } while (0)
#define STAMP(i) STAMPV(i, now())
#define TK(i)
#define TKOUT()

#define DPP_STEPS(OP) \
    OP(0x111, 0xf, true) OP(0x112, 0xf, true) OP(0x114, 0xf, true) OP(0x118, 0xf, true) OP(0x142, 0xa, false) OP(0x143, 0xc, false)
__device__ __forceinline__ u32 wave_max_u32(u32 v) {
#define OP_(c, rm, bc) v = max(v, (u32)__builtin_amdgcn_update_dpp(0, (int)v, c, rm, 0xf, bc));
    DPP_STEPS(OP_)
#undef OP_
    return (u32)__builtin_amdgcn_readlane((int)v, 63);
}
__device__ __forceinline__ u32 wave_scan_u32(u32 v) {
#define OP_(c, rm, bc) v += (u32)__builtin_amdgcn_update_dpp(0, (int)v, c, rm, 0xf, bc);
    DPP_STEPS(OP_)
#undef OP_
    return v;
}
__device__ __forceinline__ u32 byte_signs(u32 a) { return PERM(a, a << 8, 0x0B090A08u); }

constexpr u32 TFLIP0 = 0xFFFFFFFFu, TFLIP1 = 0xFFFFFF00u;
__device__ __forceinline__ u32 lut16(u32x4 t, u32 i) { return PERM(t[1], t[0], i) ^ PERM(t[3], t[2], i ^ 0x08080808u); }
__device__ __forceinline__ u32 unary8(u32 h0, u32 h1, u32& code) {
    h0 += h0 << 8; h0 += h0 << 16;
    h1 += h1 << 8; h1 += h1 << 16;
    h1 += 0x07060504u + PERM(h0, h0, 0x03030303u);
    h0 += 0x03020100u;
    code = 0;
#pragma unroll
    for (int k = 0; k < 4; ++k) code |= (1u << ((h0 >> (8 * k)) & 31u)) | (1u << ((h1 >> (8 * k)) & 31u));
    return h1 >> 24;
}
__device__ __forceinline__ void put_bits(LAS char* L, u32 pos, u32 code) {
    LAS u32* a = (LAS u32*)(L + ((pos >> 5) << 2));
    __hip_atomic_fetch_or(a, code << (pos & 31u), __ATOMIC_RELAXED, __HIP_MEMORY_SCOPE_WORKGROUP);
    __hip_atomic_fetch_or(a + 1, (code >> 1) >> (~pos & 31u), __ATOMIC_RELAXED, __HIP_MEMORY_SCOPE_WORKGROUP);
}
__device__ __forceinline__ u32 counts8(u32 w, u32& h0, u32& h1) {
    u32 p0, p1, t;
#define ONE_(p, k, keep) "v_ffbl_b32_sdwa " p ", %3 dst_sel:BYTE_" #k " dst_unused:" keep " src0_sel:DWORD\n"
#define CLR_ "v_add_u32 %2, -1, %3\n v_and_b32 %3, %2, %3\n"
    asm(ONE_("%0", 0, "UNUSED_PAD") CLR_ ONE_("%0", 1, "UNUSED_PRESERVE") CLR_ ONE_("%0", 2, "UNUSED_PRESERVE") CLR_ ONE_("%0", 3, "UNUSED_PRESERVE") CLR_
        ONE_("%1", 0, "UNUSED_PAD") CLR_ ONE_("%1", 1, "UNUSED_PRESERVE") CLR_ ONE_("%1", 2, "UNUSED_PRESERVE") CLR_ ONE_("%1", 3, "UNUSED_PRESERVE") "s_nop 0"
        : "=&v"(p0), "=&v"(p1), "=&v"(t), "+v"(w));
    h0 = p0 - ((p0 << 8) + 0x01010100u);
    h1 = p1 - (__builtin_amdgcn_alignbit(p1, p0, 24) + 0x01010101u);
    return p1 >> 24;
}
__device__ __forceinline__ u32 get_bits(const LAS char* L, u32 pos) {
    const LAS u32* a = (const LAS u32*)(L + ((pos >> 5) << 2));
    return __builtin_amdgcn_alignbit(a[1], a[0], pos);
}

#define LP(a) ((const LAS u32*)(u32)(a))
__device__ __forceinline__ u32 unary_list(u32 lane, u32 from, u32 sb, u32 mine, u32 area, u32 cap) {
    u32 pos = from + lane * sb;
    const u32 stop = pos + sb;
    const LAS u32* a = LP(((pos - 32u) >> 5) << 2);
    u32 cnt = 0, prev = __builtin_amdgcn_alignbit(a[1], a[0], pos) | 1u;
#pragma unroll
    for (u32 i = 0; i < 3u; ++i)
        cnt += (u32)__builtin_popcount(__builtin_amdgcn_alignbit(a[i + 2u], a[i + 1u], pos) & (u32)((1ull << (min(max(sb, 32u * i), 32u * i + 32u) - 32u * i)) - 1ull));
    asm volatile("" : "+v"(prev));
    for (u32 i = 3u; 32u * i < sb; ++i)
        cnt += (u32)__builtin_popcount(__builtin_amdgcn_alignbit(a[i + 2u], a[i + 1u], pos) & (u32)((1ull << min(sb - 32u * i, 32u)) - 1ull));
    pos -= (u32)__builtin_clz(lane == 0u ? ~0u : prev);
    u32 at = (pos >> 5) << 2, d0 = LP(at)[0], d1 = LP(at)[1];
    cnt = (cnt << 16) + mine;
    const u32 end = wave_scan_u32(cnt);
    const u32 last = area + min((u32)__builtin_amdgcn_readlane((int)end, 63) & 0xFFFFu, cap);
    u32 k = area + ((end - cnt) >> 16);
    asm volatile("" : "+v"(d0), "+v"(d1));
    u64 go = BALLOT(k < last);
    for (u32 i = 0; i < 64u && go != 0ull; ++i) {
        const u32 d2 = *(const volatile LAS u32*)LP(at + 8u);
        const u32 w = __builtin_amdgcn_alignbit(d1, d0, pos);
        u32 h0, h1;
        const u32 p8 = counts8(w, h0, h1), nv = min((u32)__builtin_popcount(w), 8u);
        const u32 np = pos + min(p8 + 1u, 32u - (u32)__builtin_clz(w | 1u)), cross = (np ^ pos) & 32u;
        pos = np;
        at += cross >> 3;
        d0 = cross ? d1 : d0;
        d1 = cross ? d2 : d1;
        const u64 more = BALLOT(k + nv < last), part = go & more & BALLOT(nv < 8u);
        asm volatile("" : "+v"(d0), "+v"(d1));
        if (__builtin_amdgcn_inverse_ballot_w64(go & ~part)) asm volatile("ds_write_b64 %0, %1" :: "v"(k), "v"(u32x2{h0, h1}) : "memory");
        if (part != 0ull) {
#pragma unroll 1
            for (u32 e = 0; e < 7u; ++e)
                if (__builtin_amdgcn_inverse_ballot_w64(part) && e < nv) *(volatile LAS char*)(k + e) = (char)((e < 4u ? h0 : h1) >> (e * 8u & 31u));
        }
        k += nv;
        go &= more & BALLOT(pos < stop);
    }
    return area + ((end - cnt) & 0xFFFFu);
}
#define LIST8(d, ad) do { const LAS u32* a_ = LP((ad) & ~3u); d[0] = a_[0]; d[1] = a_[1]; d[2] = a_[2]; } while (0)
__device__ __forceinline__ void deal8(const u32 (&d)[3], u32 ad, u32 es, u32& c0, u32& c1) {
    const u32 lo = __builtin_amdgcn_alignbyte(d[1], d[0], ad), hi = __builtin_amdgcn_alignbyte(d[2], d[1], ad);
    u32 pre = es << 1;
    asm("" : "+v"(pre));
    const u32 mk = pre - (es >> 3);
    pre += pre << 4;
    asm("" : "+v"(pre));
    pre += pre << 8;
    asm("" : "+v"(pre));
    pre += pre << 16;
    const u32 sel = __builtin_amdgcn_bitop3_b32(mk, pre, 0xCCCCCCCCu, 0xCA);
    c0 = PERM(hi, lo, sel & 0x0F0F0F0Fu);
    c1 = PERM(hi, lo, (sel >> 4) & 0x0F0F0F0Fu);
}

__global__ void __launch_bounds__(768, 3) gemm_ar_q8(Params p) {
    extern __shared__ __attribute__((aligned(16))) char smem[];
    LAS char* L = (LAS char*)smem;
    u32 lane = threadIdx.x & 63;
    const u32 wid = RFL(threadIdx.x >> 6);
    const u32 rank = (u32)p.rank & 7u;
    const u32 b = blockIdx.x;
    const u32 q = b & 7u, idx = b >> 3;
    const u32 mb = 4 * (q >> 1) + (idx & 3), nb = 4 * (q & 1) + ((idx >> 2) & 3);
    const u32 nb2 = 8u + 2u * (q & 1u) + ((idx >> 2) & 1u), fills = idx < 8u ? 256u : 128u;
    const u32 rel = 1u + (b - 128u) % 7u, pj = ((b - 128u) / 7u) & 15u;
    char* wsl = p.ws[0];
    asm volatile("" :: "s"(p.x), "s"(p.w), "s"(wsl), "s"(rank));
    const rsrc_t rs_ws = mk_rsrc(wsl, WS_NEED);
    volatile LAS u32* LW = (volatile LAS u32*)L;
#define LDSV(a) (*(const LAS u32x4*)(L + (a)))

    STAMPV(0, tstart);
    STAMP(22);
    const u32 seqv = __builtin_amdgcn_raw_buffer_load_b32(rs_ws, OFF_SEQ, 0, AGT);
    if (threadIdx.x < 32) LW[CTRL / 4 + threadIdx.x] = b >= 128u && threadIdx.x - (16u - 2u * PCH) < 2u * PCH ? ~7u : 0u;
    auto barrier = [] {
        asm volatile("s_waitcnt lgkmcnt(0)" ::: "memory");
        __builtin_amdgcn_s_barrier();
        CBAR();
    };
#define PEERS(d) asm volatile("global_load_dwordx2 %0, %1, %2" : "=v"(d) : "v"((lane & 7u) * 8u), "s"(__builtin_amdgcn_kernarg_segment_ptr()) : "memory")

    auto put_unit = [&](const u32x4 (&o)[8], bool isw, u32 lk, u32 ch, u32 r) -> u32 {
        const u32 so = STAGE_OFF + (lk * 8u + ch) * STEP_BYTES + (isw ? 16u * PIECE + r * 8192u : (r >> 2) * PIECE + (r & 3u) * 4096u);
        const u32 v0 = isw ? ((lane & 48u) + (lane & 7u) * 2u + ((lane >> 3) & 1u)) * 16u : lane * 16u, v1 = v0 + (isw ? 4096u : 16384u);
#pragma unroll
        for (int e = 0; e < 8; ++e) ST128(o[e], rs_ws, (e < 4 ? v0 : v1) + (e & 3) * 1024u, so, SYS);
        return OFF_UF + ((lk * 8u + ch) * 112u + (isw ? 64u : 0u) + r) * 128u;
    };
    auto put_flag = [&](u32 flag, u32 sq) { if (lane < FLAGW) ST128((u32x4{sq, sq, sq, sq}), rs_ws, flag + lane * 16u, 0, SYS); };

    auto strip_at = [](u32 os, u32 r) -> u32 { return 32u + (os >> 1) * 4u + (r & 2u) + (os & 1u); };

    {
        u32 t = 0, tend = 0, delay = 0, step = 1536u;
        if (b < 128u) { t = wid * 128u + b; if (t < 464u) tend = t + 1u; delay = wid < 3u ? 0u : WDELAY1; }
        else if (b < 240u) {
            t = (wid < 6u ? wid - 2u : wid - 6u) * 112u + (b - 128u);
            delay = (t < 256u ? t >> 6 : 4u + (t - 256u) / 96u) * TDELAY;
            if (wid - 2u < 4u) tend = t + 1025u;
            if (wid >= 10u) tend = t + 1025u + step;
            t += t < 640u ? 1024u : 1024u + step;
        } else {
            t = 1776u + wid * 16u + (b - 240u); tend = 3008u; step = 192u;
            if (wid < 4u) __builtin_amdgcn_s_setprio(2); else if (wid < 7u) __builtin_amdgcn_s_setprio(1); else delay = WDELAY2;
        }
        bool bar = t < tend && delay == 0u;
        auto pause = [](u32 d) {
            const u32 ts = (u32)now();
            while ((u32)now() - ts < d) __builtin_amdgcn_s_sleep(2);
        };
        if (!bar) {
            barrier();
            if (t < tend) pause(delay);
        }
        if ((WARM & 1) && b < 128u && wid >= 8u) {
            u64 pc;
            asm volatile("s_getpc_b64 %0" : "=s"(pc));
            const u32 v = __builtin_amdgcn_raw_buffer_load_b32(mk_rsrc((void*)(pc & ~255ull), CODE_BYTES), lane * 128u, ((idx * 4u + wid) % 6u) * 8192u, 0);
            asm volatile("" :: "v"(v));
        }
        for (; t < tend; t += step) {
            TK(0);
            u32 kind, lk = 0, ch, r;
            if (t < 384u) { ch = t >> 8; r = (t >> 3) & 31u; lk = (t + r) & 7u; kind = lk == 0u ? 1u : 2u; }
            else if (t < 464u) { ch = (t - 336u) >> 5; r = (t - 336u) & 31u; kind = 1u; }
            else if (t < 1280u) { ch = (t >> 6) & 3u; r = t & 63u; kind = 0u; }
            else if (t < 1664u) { ch = 4u + (t - 1280u) / 96u; r = (t - 1280u) % 96u; kind = r >> 6; r &= 63u; }
            else { ch = (0x3765421u >> ((t - 1664u) / 224u * 4u)) & 7u; r = (t - 1664u) % 224u; lk = 1u + r % 7u; r /= 7u; kind = 2u; }
            const u32 to = (rank + lk) & 7u;
            const bool strip = kind != 0u && (r | 2u) == 30u;
            const bool push = kind == 2u && ch < PCH, both = push && strip;
            char* pto = wsl;
            u32x4 o[8];
            if (kind == 0u) {
                const rsrc_t rs_x = mk_rsrc(p.x, X_BYTES);
                const u32 vx = (lane >> 2) * 1024u + (((lane & 3u) ^ ((lane >> 4) & 2u)) << 4);
#pragma unroll
                for (int i = 0; i < 4; ++i) {
                    o[i] = LD128(rs_x, vx, r * 65536u + ch * 128u + i * 16384u, 0);
                    o[4 + i] = LD128(rs_x, vx + 64u, r * 65536u + ch * 128u + i * 16384u, 0);
                }
                if (bar) barrier();
                asm volatile("" : "+v"(o[0]), "+v"(o[1]), "+v"(o[2]), "+v"(o[3]), "+v"(o[4]), "+v"(o[5]), "+v"(o[6]), "+v"(o[7]));
            } else {
                const rsrc_t rs_w = mk_rsrc(p.w, W_BYTES);
                const u32 vw = (lane >> 4) * 8u * 32768u + (lane & 15u) * 16u;
                const u32 so = (ch * 64u + (r & 2u) * 16u) * 32768u + (to * 2048u + (r >> 2) * 256u + (r & 1u) * 128u) * 2u;
                u32x4 v[8];
#pragma unroll
                for (int i = 0; i < 8; ++i) v[i] = LD128(rs_w, vw, so + i * 32768u, 0);
                pto = p.ws[push ? lk : 0u];
                asm volatile("" : "+s"(pto));
                if (bar) barrier();
#pragma unroll
                for (int e = 0; e < 4; ++e)
#pragma unroll
                    for (int k = 0; k < 4; ++k) {
                        const u32 lo = PERM(v[2 * k + 1][e >> 1], v[2 * k][e >> 1], (e & 1) ? 0x07060302u : 0x05040100u);
                        const u32 hi = PERM(v[2 * k + 1][2 + (e >> 1)], v[2 * k][2 + (e >> 1)], (e & 1) ? 0x07060302u : 0x05040100u);
                        o[e][k] = (u32)__builtin_amdgcn_update_dpp((int)lo, (int)hi, 0x128, 0xf, 0xc, false);
                        o[4 + e][k] = (u32)__builtin_amdgcn_update_dpp((int)hi, (int)lo, 0x128, 0xf, 0x3, false);
                    }
            }
            bar = false;
            u32 flag = OOB, size = E1_SLOT, sq = RFL(seqv) + 1u;
            u32 reg = L_PACK + wid * 2048u;
            asm volatile("" : "+s"(sq), "+s"(reg), "+v"(lane));
            if (kind != 2u || strip) flag = put_unit(o, kind != 0u, 0u, ch, kind == 2u ? strip_at(lk - 1u, r) : r);
            if (kind == 1u && !strip) { TK(2); VMWAIT(0); TK(3); put_flag(flag, sq); TK(4); TKOUT(); continue; }
            u64 f1;
            PEERS(f1);
            const u32 kept = kind == 0u ? E1X_OFF + (ch * 64u + r) * E1_SLOT : E1W_OFF + ((lk * 8u + ch) * 32u + r) * E1_SLOT;
            const u32 slot = push ? E1P_OFF + ((lk * 2u + ch) * 32u + r) * E1_SLOT : kept;
            const rsrc_t rs_to = mk_rsrc(pto, WS_NEED);
#define PST(v, voff, at) do { const u32x4 p_ = (v); ST128(p_, rs_to, voff, slot + (at), SYS); if (both) ST128(p_, rs_ws, voff, kept + (at), SYS); } while (0)
            *(LAS u32x4*)(L + reg + lane * 16u) = u32x4{0u, 0u, 0u, 0u};
            *(LAS u32x4*)(L + reg + 1024u + lane * 16u) = u32x4{0u, 0u, 0u, 0u};
            u32 ex[16], mx = 0;
            const bool late = push && ch != 0u;
            if (push && !late && r > PLEAD) pause((r - PLEAD) * PSTEP);
#pragma unroll
            for (int e = 0; e < 4; ++e) {
                u32x4 mq;
#pragma unroll
                for (int k = 0; k < 4; ++k) {
                    const int j = 4 * e + k;
                    const u32 d0 = o[j >> 1][(2 * j) & 3], d1 = o[j >> 1][(2 * j + 1) & 3];
                    const u32 r0 = __builtin_amdgcn_alignbit(d0, d0, 31), r1 = __builtin_amdgcn_alignbit(d1, d1, 31);
                    mq[k] = PERM(r1, r0, 0x06040200u);
                    ex[j] = PERM(r1, r0, 0x07050301u);
                    mx = __builtin_bit_cast(u32, __builtin_elementwise_max(__builtin_bit_cast(u16x2, mx),
                             __builtin_elementwise_max(__builtin_bit_cast(u16x2, r0), __builtin_bit_cast(u16x2, r1))));
                }
                if (!late) PST(mq, lane * 16u + e * 1024u, 0u);
            }
            const u32 top = wave_max_u32(max(mx >> 24, (mx >> 8) & 255u));
            u32 any = 0, cnt = 0;
#pragma unroll
            for (int j = 0; j < 16; ++j) { ex[j] = top * 0x01010101u - ex[j]; any |= ex[j]; }
            CBAR();
#pragma unroll
            for (int j = 0; j < 16; j += 2)
                __hip_atomic_fetch_add((LAS u32*)(L + reg + 256u + (((ex[j] << 4) & 0xF0u) | ((lane & 3u) << 2))), 1u, __ATOMIC_RELAXED,
                                       __HIP_MEMORY_SCOPE_WORKGROUP);
            CBAR();
            const u32x4 hist = LDSV(reg + 256u + (lane & 15u) * 16u);
            CBAR();
            if (BALLOT((any & 0xF0F0F0F0u) != 0u) != 0ull) {
#pragma unroll
                for (int j = 0; j < 16; ++j) {
                    const bool far = (ex[j] & 0xF0F0F0F0u) != 0u;
                    const u64 m = BALLOT(far);
                    if (m != 0ull) {
                        const u32 k = cnt + rank_in(m);
                        if (far && k < 14u) *(LAS u32x2*)(L + reg + I_VAR + k * 8u) = u32x2{lane * 16u + j, ex[j]};
                        cnt += (u32)__builtin_popcountll(m);
                        ex[j] &= 0x0F0F0F0Fu;
                    }
                }
            }
            const u32 key = (((hist[0] + hist[1] + hist[2] + hist[3]) << 4) | (lane & 15u)) ^ 15u;
            u32 above = 0;
#define ROT_(n) above = __builtin_amdgcn_alignbit(above, key - (u32)__builtin_amdgcn_mov_dpp((int)key, 0x120 + n, 0xf, 0xf, false), 31);
            ROT_(1) ROT_(2) ROT_(3) ROT_(4) ROT_(5) ROT_(6) ROT_(7) ROT_(8) ROT_(9) ROT_(10) ROT_(11) ROT_(12) ROT_(13) ROT_(14) ROT_(15)
#undef ROT_
            above = (u32)__builtin_popcount(above);
            const u32 sym = (above >> 1) | ((above & 1u) << 3);
            u32 tb = sym << ((lane & 3u) * 8u);
            tb |= (u32)__builtin_amdgcn_mov_dpp((int)tb, 0xB1, 0xf, 0xf, false);
            tb |= (u32)__builtin_amdgcn_mov_dpp((int)tb, 0x4E, 0xf, 0xf, false);
            const u32x4 tab = u32x4{(u32)__builtin_amdgcn_readlane((int)tb, 0) ^ TFLIP0, (u32)__builtin_amdgcn_readlane((int)tb, 4) ^ TFLIP1,
                                    (u32)__builtin_amdgcn_readlane((int)tb, 8) ^ TFLIP0, (u32)__builtin_amdgcn_readlane((int)tb, 12) ^ TFLIP1};
            u32 code[8], len[8], over = 0, tot = 0;
            u32x2 pl = u32x2{0u, 0u};
#pragma unroll
            for (int g = 0; g < 8; ++g) {
                const u32 s0 = lut16(tab, ex[2 * g]), s1 = lut16(tab, ex[2 * g + 1]);
                pl[g >> 2] |= bfi(0x08080808u, s0, s1 << 4) >> (g & 3) & (0x88888888u >> (g & 3));
                const u32 last = unary8(s0 & 0x07070707u, s1 & 0x07070707u, code[g]);
                over |= last;
                len[g] = last + 1u;
                tot += len[g];
            }
            const u32 end = wave_scan_u32(tot);
            const u32 bytes = I_VAR + cnt * 8u + (((u32)__builtin_amdgcn_readlane((int)end, 63) + 7u) >> 3);
            if (kind != 2u) { VMWAIT(4); put_flag(flag, sq); }
            if (BALLOT(over > 31u) == 0ull && cnt < 15u && bytes <= 2040u && !(FORCE_RAW1 && (r & 1u))) {
                size = I_HDR + bytes;
                *(LAS char*)(L + reg + sym) = (char)((lane & 15u) | (((top | (cnt << 8)) >> (sym * 4u & 31u)) << 4));
                *(LAS char*)(L + reg + I_LEN + lane) = (char)(tot - 64u);
                *(LAS u32x2*)(L + reg + I_BIT + lane * 8u) = pl;
                u32 pos = end - tot + (reg + I_VAR + cnt * 8u) * 8u;
#pragma unroll
                for (int g = 0; g < 8; ++g) { put_bits(L, pos, code[g]); pos += len[g]; }
                CBAR();
                if (late) {
                    pause((r < 25u ? r : r - PBACK) * PSTEP2);
#pragma unroll
                    for (int e = 0; e < 4; ++e) {
                        u32x4 mq;
#pragma unroll
                        for (int k = 0; k < 4; ++k) {
                            const u32 d0 = o[2 * e + (k >> 1)][(2 * k) & 3], d1 = o[2 * e + (k >> 1)][(2 * k + 1) & 3];
                            mq[k] = PERM(__builtin_amdgcn_alignbit(d1, d1, 31), __builtin_amdgcn_alignbit(d0, d0, 31), 0x06040200u);
                        }
                        PST(mq, lane * 16u + e * 1024u, 0u);
                    }
                }
                PST(LDSV(reg + lane * 16u), lane * 16u, I_HDR);
                PST(LDSV(reg + 1024u + lane * 16u), (lane & 60u) * 16u + 1024u < bytes ? lane * 16u + 1024u : OOB, I_HDR);
                CBAR();
            } else {
                PST((u32x4{~0u, ~0u, ~0u, ~0u}), lane == 0u ? 0u : OOB, I_HDR);
#pragma unroll
                for (int e = 0; e < 8; ++e) PST(o[e], lane * 16u + (e & 3) * 1024u, (e >> 2) * (I_HDR + 128u));
            }
            u32 old = 0;
            TK(2);
            VMWAIT(0);
            TK(3);
            asm volatile("" : "+v"(f1));
            if (kind == 2u) put_flag(flag, sq);
            if (push) {
                if (lane == 0) __builtin_amdgcn_raw_buffer_store_b64(u32x2{sq, (size + 15u) >> 4}, rs_to, 0u, OFF_FP + (lk * 16u + (r >> 1)) * 128u + (ch * 2u + (r & 1u)) * 8u, SYS);
                __builtin_amdgcn_s_setprio(0);
                if (!strip) { TK(4); TKOUT(); continue; }
            }
            if (lane == 0 ? kind != 1u && !push : (lane == 1u && strip))
                old = 1u + __hip_atomic_fetch_add((u32*)(wsl + OFF_CTR + (lane == 0 ? lk * 8u + ch : 80u) * 128u), 1u, __ATOMIC_RELAXED,
                                                  __HIP_MEMORY_SCOPE_AGENT);
            f1 += OFF_F1 + lane * 256u;
            const u64 fv = ((u64)min((size + PRED_PAD + 63u) & ~63u, E1_SLOT) << 28) | sq;
            if (RFL(old) == sq * (kind == 0u ? 64u : 32u) && lane - 1u < 7u && (kind == 0u || lane == lk))
                __hip_atomic_store((GAS u64*)f1 + ch * 2u + (kind == 0u), fv, __ATOMIC_RELAXED, __HIP_MEMORY_SCOPE_SYSTEM);
            if ((u32)__builtin_amdgcn_readlane((int)old, 1) == sq * 128u && lane - 1u < 7u)
                __hip_atomic_store((GAS u64*)f1 + 16, fv, __ATOMIC_RELAXED, __HIP_MEMORY_SCOPE_SYSTEM);
            TKOUT();
        }
    }
    const u32 seq = RFL(seqv) + 1u;
    lane = threadIdx.x & 63;
    STAMP(10);

    u32 spins = 0;
    u64 t0 = 0;
    auto expired = [&]() -> bool {
        if ((++spins & 63u) != 0u) return false;
        if (LW[L_BAD / 4] != 0u) return true;
        const u64 t = now();
        if (t0 == 0) t0 = t;
        if (t - t0 > DEADLINE || __builtin_amdgcn_raw_buffer_load_b32(rs_ws, OFF_TMO, 0, AGT) == seq) {
            u32 sq = seq;
            asm volatile("" : "+s"(sq));
            __builtin_amdgcn_raw_buffer_store_b32(sq, rs_ws, OFF_TMO, 0, AGT);
            LW[L_BAD / 4] = 1u;
            return true;
        }
        return false;
    };

    auto arrive = [&](u32 wm, u32 n) {
        if (lane == 0) {
            __hip_atomic_fetch_max((LAS u32*)(L + L_AMAX), wm, __ATOMIC_RELAXED, __HIP_MEMORY_SCOPE_WORKGROUP);
            const u32 old = __hip_atomic_fetch_add((LAS u32*)(L + L_ACNT), 1u, __ATOMIC_RELAXED, __HIP_MEMORY_SCOPE_WORKGROUP);
            if (old == n - 1u) {
                u32 a = LW[L_AMAX / 4];
                if (LW[L_BAD / 4] != 0u) a = 0xFFFFFFFFu;
                __builtin_amdgcn_raw_buffer_store_b64(u32x2{a, seq}, rs_ws, OFF_AMX + b * 8u, 0, AGT);
            }
        }
    };

    if (b < 128u && wid < 8u) {
        u32 c = lane & 15u, g = lane >> 4;
        const u32 mw = wid & 3u, nh = wid >> 2;
        for (u32 f0 = 0; f0 < fills; f0 += 128u) {
        f32x16 acc[8];
        u32 xa = mw * 4096u + (c * 4u + (g ^ ((c >> 2) & 2u))) * 16u, wa = 16384u + nh * 8192u + lane * 16u;
        const u32 ca = lane < 32u ? L_FULL + ((lane - 1u) & 3u) * 4u : lane < 48u ? L_FREE + (wid ^ 4u) * 4u : L_FREE + wid * 4u - 768u;
        u32 f2 = f0 + 1u, spin = 0, tv;
#define S0_(x) #x
#define S_(x) S0_(x)
#define R4(r) "v[" S_(r) ":" S_(r) "+3]"
#define MM_(j, m, w, x) "v_mfma_f32_16x16x32_bf16 " R4(16*j+4*m) ", " R4(w) ", " R4(x) ", " R4(16*j+4*m) "\n"
#define MM4(j, w, x0) MM_(j, 0, w, x0) MM_(j, 1, w, 132) MM_(j, 2, w, 136) MM_(j, 3, w, 140)
#define RW(r, o) "ds_read_b128 " R4(r) ", %[wa] offset:" S_(o) "\n"
#define RX(r, o) "ds_read_b128 " R4(r) ", %[xa] offset:" S_(o) "\n"
#define LG(n) "s_waitcnt lgkmcnt(" S_(n) ")\n" KPAD
#define KWAIT(l) \
    "8" l ":\n s_cmp_eq_u32 %[f2], %[last]\n s_cbranch_scc1 9" l "b\n ds_read_b32 %[tv], %[ca]\n s_waitcnt lgkmcnt(0)\n" \
    "v_cmp_le_u32 vcc, %[f2], %[tv]\n s_nop 4\n s_bitcmp1_b32 vcc_lo, %[f2]\n s_cbranch_scc1 9" l "b\n s_sleep 1\n" \
    "s_add_u32 %[spin], %[spin], 1\n s_cmp_lt_u32 %[spin], 0x1000000\n s_cbranch_scc1 8" l "b\n s_branch 9" l "b\n"
#define KFILL(l, x0, y0, cur, nxt, TOGGLE) \
    LG(5) MM_(0, 0, 144, x0) MM_(1, 0, 148, x0) LG(4) MM_(0, 1, 144, 132) MM_(1, 1, 148, 132) \
    LG(3) MM_(0, 2, 144, 136) MM_(1, 2, 148, 136) LG(2) MM_(0, 3, 144, 140) RW(144, cur+4096) MM_(1, 3, 148, 140) RW(148, cur+5120) \
    "ds_read_b32 %[tv], %[ca]\n" \
    "s_waitcnt lgkmcnt(4)\n s_add_u32 %[f2], %[f2], 1\n" MM4(2, 152, x0) RW(152, cur+6144) \
    LG(4) MM4(3, 156, x0) RW(156, cur+7168) \
    "s_waitcnt lgkmcnt(2)\n v_cmp_le_u32 vcc, %[f2], %[tv]\n" MM_(4, 0, 144, x0) \
    "v_mov_b32_e64 %[tv], %[f2]\n" MM_(4, 1, 144, 132) \
    "ds_write_b32 %[ca], %[tv] offset:768\n"         \
    MM_(4, 2, 144, 136) TOGGLE MM_(4, 3, 144, 140) \
      \
    "s_setprio " S_(CPRIO) "\n s_bitcmp0_b32 vcc_hi, 0\n s_cbranch_scc1 7" l "f\n s_setprio " S_(CPRIO+BALANCE) "\n7" l ":\n" \
    "s_bitcmp0_b32 vcc_lo, %[f2]\n s_cbranch_scc1 8" l "f\n9" l ":\n" \
    RW(144, nxt) RX(y0, nxt) MM4(5, 148, x0) RW(148, nxt+1024) \
    LG(4) MM_(6, 1, 152, 132) MM_(7, 1, 156, 132) RX(132, nxt+1024) MM_(6, 2, 152, 136) MM_(7, 2, 156, 136) RX(136, nxt+2048) \
    MM_(6, 3, 152, 140) MM_(7, 3, 156, 140) RX(140, nxt+3072) MM_(6, 0, 152, x0) RW(152, nxt+2048) MM_(7, 0, 156, x0) RW(156, nxt+3072)
        asm volatile(
            ".set i_, 0\n .rept 64\n v_mov_b64 v[i_:i_+1], 0\n .set i_, i_+2\n .endr\n"
            "s_branch 80f\n90:\n"
            RW(144, 0) RX(128, 0) RW(148, 1024) RX(132, 1024) RX(136, 2048) RX(140, 3072) RW(152, 2048) RW(156, 3072)
            ".p2align 3\n1:\n"
            KFILL("1", 128, 160, 0, 32768, "")
            KFILL("2", 160, 128, 32768, 0, "v_xor_b32 %[wa], 0x10000, %[wa]\n v_xor_b32 %[xa], 0x10000, %[xa]\n")
            "s_cmp_lt_u32 %[f2], %[last]\n s_cbranch_scc1 1b\n s_branch 2f\n"
            KWAIT("0") KWAIT("1") KWAIT("2")
            "2:\n s_waitcnt lgkmcnt(0)\n s_nop 7\n s_nop 7\n"
            : "={v[0:15]}"(acc[0]), "={v[16:31]}"(acc[1]), "={v[32:47]}"(acc[2]), "={v[48:63]}"(acc[3]), "={v[64:79]}"(acc[4]),
              "={v[80:95]}"(acc[5]), "={v[96:111]}"(acc[6]), "={v[112:127]}"(acc[7]),
              [xa] "+{v164}"(xa), [wa] "+{v165}"(wa), [tv] "={v166}"(tv), [f2] "+s"(f2), [spin] "+s"(spin)
            : [ca] "{v167}"(ca), [last] "s"(f0 + 129u)
            : "v128", "v129", "v130", "v131", "v132", "v133", "v134", "v135", "v136", "v137", "v138", "v139", "v140", "v141", "v142",
              "v143", "v144", "v145", "v146", "v147", "v148", "v149", "v150", "v151", "v152", "v153", "v154", "v155", "v156", "v157",
              "v158", "v159", "v160", "v161", "v162", "v163", "vcc", "scc", "memory");
        if (spin >= 0x1000000u) LW[L_BAD / 4] = 1u;
        asm volatile("v_mbcnt_lo_u32_b32 %0, -1, 0\n v_mbcnt_hi_u32_b32 %0, -1, %0" : "=v"(lane));
        c = lane & 15u; g = lane >> 4;
        STAMPV(f0 == 0u ? 14 : 15, (u64)spin << 24);
        const u32 reg = L_PACK + wid * 2048u, slot = E2_OFF + (b * 8u + wid) * 8192u;
        auto post = [&](u32 cnt, u32 at) {
            u32 old = 0;
            VMWAIT(0);
            if (lane == 0) old = __hip_atomic_fetch_add((LAS u32*)(L + cnt + nh * 4u), 1u, __ATOMIC_RELAXED, __HIP_MEMORY_SCOPE_WORKGROUP);
            if (RFL(old) == 3u && lane - 1u < 7u) {
                const u32x4 fv = u32x4{seq, LW[L_GSZ / 4 + nh * 2u], LW[L_GSZ / 4 + nh * 2u + 1u], seq};
                u32x2 fp = *(volatile LAS u32x2*)(L + L_PTR + lane * 8u);
                asm volatile("v_add_co_u32 %0, vcc, %0, %2\n v_addc_co_u32 %1, vcc, 0, %1, vcc" : "+v"(fp[0]), "+v"(fp[1]) : "v"(at) : "vcc");
                asm volatile("global_store_dwordx4 %0, %1, off sc0 sc1\n s_nop 1" :: "v"(fp), "v"(fv) : "memory");
            }
        };
        if (f0 == 0u) {
            STAMP(2);
            STAMPV(9, __builtin_amdgcn_s_memtime());
            u32x4 sg;
#pragma unroll
            for (int m = 0; m < 4; ++m) {
                u32 d = 0;
#pragma unroll
                for (int i = 31; i >= 0; --i) d = __builtin_amdgcn_alignbit(d, __float_as_uint(acc[(i & 4) + (i >> 3)][4 * m + (i & 3)]), 31);
                const u32 o = (u32)__builtin_amdgcn_mov_dpp((int)d, 0x128, 0xf, 0xf, false);
                sg[m] = bfi(c < 8u ? 0x0F0F0F0Fu : 0xF0F0F0F0u, d, __builtin_amdgcn_alignbit(o, o, c < 8u ? 28u : 4u));
            }
            ST128(sg, rs_ws, lane * 16u, slot, AGT);
            float mx = 0.f;
#pragma unroll
            for (int j = 0; j < 8; ++j)
#pragma unroll
                for (int m = 0; m < 4; ++m) {
                    mx = fmaxf(fmaxf(mx, fabsf(acc[j][4 * m])), fabsf(acc[j][4 * m + 1]));
                    mx = fmaxf(fmaxf(mx, fabsf(acc[j][4 * m + 2])), fabsf(acc[j][4 * m + 3]));
                }
            arrive(wave_max_u32(__float_as_uint(mx)), 8u);
            STAMP(3);
            if (wid >= 4u) __builtin_amdgcn_s_setprio(2); else __builtin_amdgcn_s_setprio(0);
#pragma unroll
            for (int k = 0; k < 2; ++k) *(LAS u32x4*)(L + reg + k * 1024u + lane * 16u) = u32x4{0u, 0u, 0u, 0u};
            if ((SGM >> (q * 2u + nh)) & 1u) post(L_SCNT, OFF_F3 + ((lane * 16u + idx) * 16u + q * 2u + nh) * 64u);
            STAMP(23);
            while (LW[L_SCALE / 4] != 1u) { if (expired()) break; __builtin_amdgcn_s_sleep(CSLEEP); }
            CBAR();
        } else STAMP(13);
        const u32 gmax = LW[L_SCALE / 4 + 1];
        const bool bad = (gmax == 0xFFFFFFFFu) || (LW[L_BAD / 4] != 0u);
        const float amax = __uint_as_float(gmax);
        const float inv = amax > 0.f ? 127.0f / amax : 0.f;
        if (f0 == 0u) STAMP(4);
        const float MAGIC = 12582912.0f;
        const float qlo = bad ? MAGIC - 128.0f : MAGIC - 127.0f, qhi = bad ? MAGIC - 128.0f : MAGIC + 127.0f;
        const u32 ovoff = (c & 7u) * NN + ((c >> 3) * 4u + g) * 16u;
        const u32 os = (nb2 - 8u) * 2u + nh;
        const u32 osoff = (mb * 256u + mw * 64u) * NN + (f0 == 0u ? rank * 2048u + nb * 256u + nh * 128u : ((rank + 1u + os) & 7u) * 2048u + 1792u);
        u32x4 qv[8];
#pragma unroll
        for (int m = 0; m < 4; ++m) {
#pragma unroll
            for (int r = 0; r < 4; ++r) {
                u32 o[2];
#pragma unroll
                for (int h = 0; h < 2; ++h) {
                    u32 t[4];
#pragma unroll
                    for (int jj = 0; jj < 4; ++jj)
                        t[jj] = __float_as_uint(__builtin_amdgcn_fmed3f(__builtin_fmaf(acc[4 * h + jj][4 * m + r], inv, MAGIC), qlo, qhi));
                    o[h] = PERM(t[1], t[0], 0x0c0c0400u) | PERM(t[3], t[2], 0x04000c0cu);
                }
                qv[2 * m][r] = (u32)__builtin_amdgcn_update_dpp((int)o[0], (int)o[1], 0x128, 0xf, 0xc, false);
                qv[2 * m + 1][r] = (u32)__builtin_amdgcn_update_dpp((int)o[1], (int)o[0], 0x128, 0xf, 0x3, false);
            }
        }
        if (f0 == 0u) {
        STAMP(7);
        u32 size = nb == 7u && nh == 0u ? 0u : 8192u;
        if (size != 0u && !bad && !(q == 7u && wid == 3u) && !(FORCE_RAW2 && (wid & 1u))) {
            u32x4 nib[4], pl = u32x4{0u, 0u, 0u, 0u};
            u32 code[16], len[16], over = 0, tot = 0;
#pragma unroll
            for (int o = 0; o < 16; ++o) {
                u32 y[2], l[2];
#pragma unroll
                for (int e = 0; e < 2; ++e) {
                    const u32 v = qv[o >> 1][(2 * o + e) & 3], s = byte_signs(v);
                    y[e] = (v ^ s) + ((s & 0x01010101u) | 0x70707070u);
                    l[e] = (y[e] & byte_signs(y[e]) & 0x78787878u) >> 3;
                }
                const u32 h = bfi(0x0F0F0F0Fu, y[0] >> 4, y[1]);
                nib[o >> 2][o & 3] = bfi(0x0F0F0F0Fu, y[0], y[1] << 4) | (((h & 0x77777777u) + 0x77777777u) & h & 0x88888888u);
                pl[o >> 2] = bfi(0x88888888u >> (o & 3), h >> (o & 3), pl[o >> 2]);
                u32 p1 = l[0] + l[1];
                p1 += p1 << 8; p1 += p1 << 16;
                const u32 p0 = p1 - l[1];
                u32 cd = 0;
#pragma unroll
                for (int k = 0; k < 4; ++k) cd |= (1u << ((p0 >> (8 * k)) & 31u)) | (1u << ((p1 >> (8 * k)) & 31u));
                code[o] = cd >> 1;
                len[o] = p1 >> 24;
                over |= len[o];
                tot += len[o] + ((u32)__builtin_popcount(code[o]) << 16);
            }
            const u32 end = wave_scan_u32(tot);
            const u32 all = (u32)__builtin_amdgcn_readlane((int)end, 63), bits = all & 0xFFFFu;
            if (BALLOT(over > 31u) == 0ull && bits <= (8192u - 64u - R_VAR) * 8u && (all >> 16) <= R_MAX) {
                size = (R_VAR + ((bits + 127u) >> 7) * 16u);
                ST128(pl, rs_ws, lane * 16u, slot + R_BIT, AGT);
#pragma unroll
                for (int e = 0; e < 4; ++e) ST128(nib[e], rs_ws, lane * 16u + e * 1024u, slot + R_NIB, AGT);
                u32 pos = ((end - tot) & 0xFFFFu) + reg * 8u;
#pragma unroll
                for (int o = 0; o < 16; ++o) { put_bits(L, pos, code[o]); pos += len[o]; }
                CBAR();
#pragma unroll
                for (int k = 0; k < 2; ++k) {
                    const u32 off = R_VAR + k * 1024u + lane * 16u;
                    if (R_VAR + k * 1024u < size) ST128(LDSV(reg + k * 1024u + lane * 16u), rs_ws, off < size ? off : OOB, slot, AGT);
                }
            }
        }
        if (size == 8192u) {
#pragma unroll
            for (int e = 0; e < 8; ++e) ST128(qv[e], rs_ws, lane * 16u, slot + e * 1024u, AGT);
        }
        STAMP(11);
        if (lane == 0) *(volatile LAS unsigned short*)(L + L_GSZ + wid * 2u) = (unsigned short)(size >> 4);
        post(L_GCNT, OFF_F2 + ((lane * 16u + idx) * 16u + q * 2u + nh) * 16u);
        STAMP(5);
        __builtin_amdgcn_s_setprio(0);
        }
        const rsrc_t rs_ol = mk_rsrc(p.out, f0 == 0u || os < 7u ? OUT_BYTES : 0u);
#pragma unroll
        for (int e = 0; e < 8; ++e) ST128(qv[e], rs_ol, ovoff, osoff + e * 8u * NN, AGT);
        }
    } else if (b < 128u && wid == 8u) {
        __builtin_amdgcn_s_setprio(LPRIO);
        *(volatile LAS u64*)(L + L_PTR + (lane & 7u) * 8u) = (u64)p.ws[lane & 7u];
        for (u32 n = 0; n < fills / 2u;) {
            const u32 wn = n == 0u ? nb : nb2, end = n + 64u;
            const bool hole = wn == 11u && (lane & 5u) == 5u;
            u32 step = lane & 56u, va = OFF_UF + (step * 112u + (lane & 4u ? 64u + wn * 4u : mb * 4u) + (lane & 3u)) * 128u;
            bool seen = hole;
            while (n != end) {
                while (2u * n > LW[L_FREE / 4] + PAHEAD) { if (expired()) break; __builtin_amdgcn_s_sleep(8); }
                seen |= __builtin_amdgcn_raw_buffer_load_b32(rs_ws, seen ? OOB : va, 0, SYS) >= seq;
                u64 avail = BALLOT(seen);
                avail &= avail >> 1; avail &= avail >> 2; avail &= avail >> 4;
                avail &= 0x0101010101010101ull;
                if (avail == 0) {
                    if (!expired()) { __builtin_amdgcn_s_sleep(POLL_SLEEP); continue; }
                    avail = BALLOT(va != OOB) & 0x0101010101010101ull;
                }
                if ((avail >> lane) & 1ull) LW[L_STEPS / 4 + n + rank_in(avail)] = step;
                n += (u32)__builtin_popcountll(avail);
                LW[L_NRDY / 4] = n;
                if ((avail >> (lane & 56u)) & 1ull) {
                    seen = hole;
                    va = (++step & 7u) ? va + 112u * 128u : OOB;
                }
            }
            if (n != 64u) break;
            STAMP(1);
            while (LW[L_FREE / 4] < 128u) { if (expired()) break; __builtin_amdgcn_s_sleep(8); }
            u32 a = 0;
            for (;;) {
                const u32x2 gm = __builtin_amdgcn_raw_buffer_load_b64(rs_ws, OFF_GAM + q * 128u + (lane & 7u) * 8u, 0, SYS);
                a = gm[0];
                if (BALLOT(gm[1] == seq) == ~0ull) break;
                if (expired()) { a = 0xFFFFFFFFu; break; }
                __builtin_amdgcn_s_sleep(1);
            }
            a = wave_max_u32(a);
            LW[L_SCALE / 4 + 1] = a;
            LW[L_SCALE / 4] = 1u;
            STAMP(2);
            while (LW[L_GCNT / 4] + LW[L_GCNT / 4 + 1] < 8u) { if (expired()) break; __builtin_amdgcn_s_sleep(8); }
        }
    } else if (b < 128u) {
#define CLK(c)
        __builtin_amdgcn_s_setprio(LPRIO);
        for (u32 f = wid - 9u; f < fills; f += 3u) {
            CLK(c0);
            while (LW[L_NRDY / 4] <= (f >> 1)) { if (expired()) break; __builtin_amdgcn_s_sleep(1); }
            CBAR();
            const u32 step = RFL(LW[L_STEPS / 4 + (f >> 1)]) & 63u;
            CLK(c1);
            bool ahead = false;
            while (BALLOT(LW[L_FREE / 4 + (lane & 7u)] + 2u >= f) != ~0ull && f >= 4u) {
                ahead = true;
                if (expired()) break;
                __builtin_amdgcn_s_sleep(LSLEEP);
            }
            CBAR();
            const u32 sbase = STAGE_OFF + step * STEP_BYTES + (f & 1u) * 16384u;
            LAS char* ldst = L + (f & 3u) * SLOT_BYTES;
            CLK(c2);
            auto fill = [&](auto pace) {
#pragma unroll
                for (int i = 0; i < DIAG_NDMA; ++i) {
                    const u32 so = sbase + (i < 4 ? mb : 16u + (f < 128u ? nb : nb2)) * PIECE + (i & 3) * 4096u;
                    __builtin_amdgcn_raw_ptr_buffer_load_lds(rs_ws, ldst + i * 4096, 16, lane * 16u, so, 0, SYS); pace();
                    __builtin_amdgcn_raw_ptr_buffer_load_lds(rs_ws, ldst + i * 4096, 16, lane * 16u, so, 1024, SYS); pace();
                    __builtin_amdgcn_raw_ptr_buffer_load_lds(rs_ws, ldst + i * 4096, 16, lane * 16u, so, 2048, SYS); pace();
                    __builtin_amdgcn_raw_ptr_buffer_load_lds(rs_ws, ldst + i * 4096, 16, lane * 16u, so, 3072, SYS); pace();
                }
            };
            if (ahead || !LADAPT) fill([] { asm volatile("s_nop %0" :: "n"(LNOP) : "memory"); });
            else fill([] { });
            CBAR();
            CLK(c3);
            VMWAIT(0);
            CLK(c4);
            LW[L_FULL / 4 + (f & 3u)] = f + 1u;
            if (f - 125u < 3u) STAMP(1);
        }
    } else if (b < 240u) {
        arrive(0u, 12u);
        const u32 src = (rank - rel) & 7u;
        const rsrc_t rs_src = mk_rsrc(p.ws[(8u - rel) & 7u], WS_NEED);
        auto in_unit = [&](u32 u, u32& ch, u32& r, bool& isw) -> u32 {
            if (u >= 768u) {
                ch = (u - 768u) / 14u; r = (u - 768u) % 14u; isw = true;
                const u32 os = r >> 1, rs = 28u + (r & 1u) * 2u;
                r = strip_at(os, rs);
                return E1W_OFF + ((((rel + 1u + os) & 7u) * 8u + ch) * 32u + rs) * E1_SLOT;
            }
            ch = u / 96u; r = u % 96u; isw = r < 32u;
            if (!isw) r = (r - 32u + 8u * rel) & 63u;
            return !isw ? E1X_OFF + (ch * 64u + r) * E1_SLOT : ch < PCH ? E1P_OFF + ((rel * 2u + ch) * 32u + r) * E1_SLOT
                                                                         : E1W_OFF + ((rel * 8u + ch) * 32u + r) * E1_SLOT;
        };
        auto out_unit = [&](u32 o) -> u32 {
            const u32 k = o >> 4, j = o - 768u;
            if (k >= 48u) return j % 12u * 64u + (j / 48u) * 16u + 8u + j / 12u % 4u;
            return (o & 15u) * 64u + (k < 32u ? (k >> 2) * 8u + 4u : ((k >> 2) & 3u) * 16u) + (k & 3u);
        };
        if (wid == 0u) {
            __builtin_amdgcn_s_setprio(3);
            auto ctl = [&](u32 a) -> u32 {
                u32 v;
                asm volatile("ds_read_b32 %0, %1\n s_waitcnt lgkmcnt(0)" : "=v"(v) : "v"(a) : "memory");
                return v;
            };
            auto put = [&](u32 a, u32 n) { asm volatile("ds_write_b32 %0, %1" :: "v"(a), "v"(n) : "memory"); };
            const u32 ca = L_FULL + (lane & 15u) * 4u;
            const u32 vone = lane == 0u ? 0u : OOB;
            u32 n = 0;
            auto wait = [&](int word, u32 bit) {
                auto ok = [&](u32 w) -> bool {
                    return (((u32)__builtin_amdgcn_readlane((int)w, word) >> bit) & 1u) != 0u &&
                           (u32)__builtin_amdgcn_readlane((int)w, 8 + (n & 7u)) + (n < (word == 1 ? 16u : 8u) ? 15u : 7u) >= n;
                };
                if (!ok(ctl(ca))) {
                    VMWAIT(0);
                    put(L_FULL, n);
                    while (!ok(ctl(ca))) { if (expired()) break; __builtin_amdgcn_s_sleep(1); }
                }
            };
#define CLAIM(cl, so) asm volatile("v_mov_b32 %0, 1\n s_nop 4\n buffer_atomic_add %0, %1, %2, %3 offen sc0 sc1" \
                                   : "=&v"(cl) : "v"(vone), "s"(rs_ws), "s"(so) : "memory")
#define CLAIMED(d, cl) asm volatile("v_readfirstlane_b32 %0, %1" : "=s"(d) : "v"(cl) : "memory")
            for (u32 u = 32u + pj; u < N1; ++n) {
                u32 ch, r, cl; bool isw;
                const u32 so = in_unit(u, ch, r, isw);
                if (u >= 768u) put(L_P1, 1u);
                const u32 seg = u >= 768u ? 16u : ch * 2u + !isw;
                wait(1, seg);
                if (n == 0) STAMP(1);
                LAS char* ld = L + (n < 16u ? n : n & 7u) * E1_SLOT;
                const u32 sz = min(max(RFL(ctl(L_PSZ + seg * 4u)), 321u), E1_SLOT / 16u);
                put(L_Q + (n & 15u) * 4u, u + (sz << 16));
                CLAIM(cl, OFF_CTR + (64u + rel) * 128u);
#pragma unroll
                for (int i = 0; i < 9; ++i) {
                    if (i >= 6 && i * 64u >= sz) break;
                    VMWAIT(WIN1 - (i < WIN1 - 1 ? 0 : 1));
                    if (i == WIN1 - 1) put(L_FULL, n);
                    if (i < 5 || i * 64u + lane < sz)
                        __builtin_amdgcn_raw_ptr_buffer_load_lds(rs_src, ld + i * 1024, 16, lane * 16u, so + i * 1024u, 0, SYS);
                }
                CLAIMED(u, cl);
                u = u - (seq - 1u) * (N1 - 32u * PCH) + 16u;
                u += u < 64u ? 32u : 32u * PCH;
            }
            put(L_P1, 1u);
            STAMP(2);
            for (u32 got = 0, i = 0; got != SGM;) {
                const u32 w = ctl(ca);
                const u32 m = (u32)__builtin_amdgcn_readlane((int)w, 5) & ~got;
                if (m == 0u || (n < 24u && (BALLOT((int)w + 8 > (int)n) >> 8 & 255u) != 255u)) {
                    VMWAIT(0);
                    put(L_FULL, n);
                    if (expired()) break;
                    __builtin_amdgcn_s_sleep(1);
                    continue;
                }
                if (got == 0u) STAMP(23);
                const u32 t = (u32)__builtin_ctz(m);
                got |= 1u << t;
#pragma unroll
                for (int e = 0; e < 4; ++e) {
                    VMWAIT(WIN2 - 1);
                    if (++i == WIN2) put(L_FULL, n);
                    __builtin_amdgcn_raw_ptr_buffer_load_lds(rs_src, L + L_SG + (t * 4u + e) * 1024u, 16, lane * 16u,
                                                             E2_OFF + (pj * 64u + t * 4u + e) * 8192u, 0, SYS);
                }
            }
            STAMP(24);
            for (u32 o = pj; o < 16u * N2; ++n) {
                const u32 t = out_unit(o);
                const bool dyn = o >= 16u * (S2 - 1u);
                u32 cl;
                wait(2, o < 16u * S2 ? (t >> 2) & 15u : 16u);
                if (o == pj) STAMP(3);
                u32 sz = min(RFL(ctl(L_SZ + (t >> 1) * 4u)) >> ((t & 1u) * 16u) & 0xFFFFu, 512u);
                const u32 skip = sz < 512u && ((SGM >> ((t >> 2) & 15u)) & 1u) ? 1024u : 0u;
                put(L_Q + (n & 15u) * 4u, 1024u + t + skip * 2u + (sz << 16));
                LAS char* ld = L + (n & 7u) * E1_SLOT + skip;
                const u32 so = E2_OFF + t * 8192u + skip;
                sz -= skip >> 4;
                CLAIM(cl, dyn ? OFF_CTR + (72u + rel) * 128u : OOB);
#pragma unroll
                for (int i = 0; i < 8; ++i) {
                    if (i >= 5 && (int)(i * 64u) >= (int)sz) break;
                    VMWAIT(WIN2 - (i < WIN2 - 1 ? 0 : 1));
                    if (i == WIN2 - 1) put(L_FULL, n);
                    if (i < 5 || i * 64u + lane < sz)
                        __builtin_amdgcn_raw_ptr_buffer_load_lds(rs_src, ld + i * 1024, 16, lane * 16u, so + i * 1024u, 0, SYS);
                }
                u32 c;
                CLAIMED(c, cl);
                o = dyn ? c - (seq - 1u) * (16u * (N2 - S2 + 1u)) + 16u * S2 : o + 16u;
            }
            VMWAIT(0);
            put(L_FULL, n);
            put(L_END, 1u);
            STAMP(4);
            if (lane == 0) __builtin_amdgcn_raw_buffer_store_b32(seq, rs_src, OFF_DONE + rank * 128u + pj * 4u, 0, SYS);
        } else if (wid == 1u) {
            const u32 fa = (lane & 31u) < 20u ? OFF_F1 + rel * 256u + (lane & 31u) * 8u : OFF_FP + (rel * 16u + pj) * 128u + (lane & 3u) * 8u;
            for (u32 m = 0; m != 0xF1FFFAu;) {
                const u32x2 f = __builtin_amdgcn_raw_buffer_load_b64(rs_ws, fa, 0, SYS);
                m = (u32)BALLOT(f[0] >= seq) & 0xF1FFFAu;
                LW[L_PSZ / 4 + (lane & 31u)] = f[1];
                if (expired()) m = 0xF1FFFAu;
                LW[L_SEG / 4] = m;
                __builtin_amdgcn_s_sleep(4);
            }
            STAMP(1);
            while (LW[L_P1 / 4] == 0u) { if (expired()) break; __builtin_amdgcn_s_sleep(16); }
            u32 m = 0;
            while (m != SGM) {
                m |= (u32)BALLOT(__builtin_amdgcn_raw_buffer_load_b32(rs_ws, lane < 16u && ((SGM & ~m) >> lane & 1u) ? lane * 64u : OOB,
                                                                      OFF_F3 + (rel * 16u + pj) * 1024u, SYS) == seq);
                if (expired()) m = SGM;
                LW[L_G3 / 4] = m;
                __builtin_amdgcn_s_sleep(2);
            }
            m = 0;
            while (m != 0xFFFFu) {
                const u32x4 f = LD128(rs_ws, lane < 16u ? lane * 16u : OOB, OFF_F2 + (rel * 16u + pj) * 256u, SYS);
                m = (u32)BALLOT(f[0] == seq && f[3] == seq) & 0xFFFFu;
                if (lane < 16u) *(LAS u32x2*)(L + L_SZ + pj * 128u + lane * 8u) = u32x2{f[1], f[2]};
                if (expired()) m = 0xFFFFu;
                CBAR();
                LW[L_G2 / 4] = m;
                __builtin_amdgcn_s_sleep(2);
            }
            STAMP(2);
            for (u32 i = 0; i < 4u;) {
                const u32x4 f = LD128(rs_ws, lane * 16u, OFF_F2 + rel * 4096u + i * 1024u, SYS);
                *(LAS u32x2*)(L + L_SZ + i * 512u + lane * 8u) = u32x2{f[1], f[2]};
                if (BALLOT(f[0] == seq && f[3] == seq) == ~0ull || expired()) ++i; else __builtin_amdgcn_s_sleep(16);
            }
            CBAR();
            LW[L_G2 / 4] = 0x1FFFFu;
            STAMP(3);
        } else if (wid < 10u) {
            const rsrc_t rs_ol = mk_rsrc(p.out, OUT_BYTES);
            const u32 ovoff = (lane & 7u) * NN + (((lane >> 3) & 1u) * 4u + (lane >> 4)) * 16u;
            for (u32 n = wid - (wid >= 10u - 2u * PCH ? 10u : 2u);; n += 8u) {
                const bool pre = (int)n < 0;
                bool more = true;
                const u32 k = n + 2u * PCH;
                while (pre && ((LW[L_SEG / 4] >> (20u + k)) & 1u) == 0u) __builtin_amdgcn_s_sleep(2);
                while (!pre) {
                    const u32 end = LW[L_END / 4];
                    if (LW[L_FULL / 4] > n) break;
                    if (end != 0u) { more = false; break; }
                    expired();
                    __builtin_amdgcn_s_sleep(2);
                }
                if (!more) break;
                CBAR();
                const u32 id = pre ? (k >> 1) * 96u + pj * 2u + (k & 1u) + (min(max(RFL(LW[L_PSZ / 4 + 20u + k]), 321u), E1_SLOT / 16u) << 16)
                                   : RFL(LW[L_Q / 4 + (n & 15u)]);
                const u32 sl = (n & ((int)n < 16 && (id & 1024u) == 0u ? 15u : 7u)) * E1_SLOT;
                if ((id & 1024u) == 0u) {
                u32 ch, r; bool isw;
                const u32 so = in_unit(id & 1023u, ch, r, isw);
                const rsrc_t rs_in = mk_rsrc(p.ws[pre ? 0u : (8u - rel) & 7u], WS_NEED);
                auto fetch = [&](u32 at, u32 size) {
                    for (; at < size; at += 1024u)
                        if (at + lane * 16u < size)
                            __builtin_amdgcn_raw_ptr_buffer_load_lds(rs_in, L + sl + at, 16, lane * 16u, so + at, 0, SYS);
                    VMWAIT(0);
                };
                if (pre) fetch(0u, (id >> 16) * 16u);
                const u32 hd = sl + I_HDR;
                u32x4 tab = LDSV(hd);
                const u32 tot = (u32)*(volatile LAS unsigned char*)(L + hd + I_LEN + lane) + 64u;
                const u32 h0 = RFL(tab[0]), top = ((h0 >> 4) & 15u) | ((h0 >> 8) & 0xF0u), cnt = (h0 >> 20) & 15u;
                const u32 end = wave_scan_u32(tot);
                const u32 size = cnt == 15u ? E1_SLOT : I_HDR + I_VAR + cnt * 8u + (((u32)__builtin_amdgcn_readlane((int)end, 63) + 7u) >> 3);
                if (size > (id >> 16) * 16u) fetch((id >> 16) * 16u, size);
                u32x4 o[8];
                if (cnt == 15u) {
#pragma unroll
                    for (int e = 0; e < 8; ++e) o[e] = LDSV(sl + (e >> 2) * (I_HDR + 128u) + (e & 3) * 1024u + lane * 16u);
                } else {
                    u32x4 mq[4];
                    u32 ex[16];
#pragma unroll
                    for (int e = 0; e < 4; ++e) mq[e] = LDSV(sl + e * 1024u + lane * 16u);
                    const u32x2 pl = *(const LAS u32x2*)(L + hd + I_BIT + lane * 8u);
                    tab = (tab & 0x0F0F0F0Fu) ^ u32x4{TFLIP0, TFLIP1, TFLIP0, TFLIP1};
                    u32 pos = end - tot + (hd + I_VAR + cnt * 8u) * 8u;
#pragma unroll
                    for (int g = 0; g < 8; ++g) {
                        u32 s0, s1;
                        pos += counts8(get_bits(L, pos), s0, s1) + 1u;
                        ex[2 * g] = lut16(tab, bfi(0x08080808u, pl[g >> 2] << (g & 3), s0));
                        ex[2 * g + 1] = lut16(tab, bfi(0x08080808u, pl[g >> 2] >> (4 - (g & 3)), s1));
                    }
#pragma unroll 1
                    for (u32 k = 0; k < cnt; ++k) {
                        const u32 at = LW[(hd + I_VAR) / 4 + 2u * k] - lane * 16u, val = LW[(hd + I_VAR) / 4 + 1u + 2u * k];
#pragma unroll
                        for (int j = 0; j < 16; ++j) { ex[j] = at == (u32)j ? val : ex[j]; if ((j & 3) == 3) __builtin_amdgcn_sched_barrier(0); }
                    }
#pragma unroll
                    for (int j = 0; j < 16; ++j) {
                        const u32 e = top * 0x01010101u - ex[j], m = mq[j >> 2][j & 3];
                        const u32 r0 = PERM(e, m, 0x05010400u), r1 = PERM(e, m, 0x07030602u);
                        o[j >> 1][(2 * j) & 3] = __builtin_amdgcn_alignbit(r0, r0, 1);
                        o[j >> 1][(2 * j + 1) & 3] = __builtin_amdgcn_alignbit(r1, r1, 1);
                    }
                }
                CBAR();
                LW[L_FREE / 4 + (n & 7u)] = n + 1u;
                const u32 flag = put_unit(o, isw, rel, ch, r);
                VMWAIT(0);
                put_flag(flag, seq);
                STAMP(pre ? 3 : 1);
                continue;
                }
                const u32 sz = id >> 16, x = (id >> 3) & 7u, ix = (id >> 6) & 15u;
                const u32 osoff = ((4u * (x >> 1) + (ix & 3u)) * 256u + (id & 3u) * 64u) * NN + src * 2048u +
                                  (4u * (x & 1u) + (ix >> 2)) * 256u + ((id >> 2) & 1u) * 128u;
                u32x4 qv[8];
                if (sz >= 512u) {
#pragma unroll
                    for (int e = 0; e < 8; ++e) qv[e] = LDSV(sl + e * 1024u + lane * 16u);
                } else {
                    u32x4 nib[4];
#pragma unroll
                    for (int e = 0; e < 4; ++e) nib[e] = LDSV(sl + R_NIB + e * 1024u + lane * 16u);
                    const u32x4 pl = LDSV(sl + R_BIT + lane * 16u);
                    const u32x4 sg = LDSV((id & 2048u ? L_SG + (id & 63u) * 1024u : sl) + lane * 16u);
                    u32 cnt = 0, at[16], d[16][3];
#pragma unroll
                    for (int o = 0; o < 16; ++o) {
                        at[o] = cnt;
                        cnt += (u32)__builtin_popcount(nib[o >> 2][o & 3] & (pl[o >> 2] << (o & 3)) & 0x88888888u);
                    }
                    const u32 a0 = unary_list(lane, (sl + R_VAR) * 8u, (max(sz, R_VAR / 16u) - R_VAR / 16u) * 2u, cnt,
                                              L_PACK + (wid - 2u) * 3072u, R_MAX);
                    CBAR();
#pragma unroll
                    for (int o = 0; o < 16; ++o) { at[o] += a0; LIST8(d[o], at[o]); }
                    __builtin_amdgcn_sched_barrier(0);
#pragma unroll
                    for (int o = 0; o < 16; ++o) {
                        const u32 nv = nib[o >> 2][o & 3], pb = pl[o >> 2] << (o & 3), sv = sg[o >> 2] << (6 - 2 * (o & 3));
                        u32 c0, c1;
                        deal8(d[o], at[o], nv & pb & 0x88888888u, c0, c1);
                        const u32 m0 = (c0 << 3) + (bfi(0x0F0F0F0Fu, nv, pb << 1) & 0x1F1F1F1Fu);
                        const u32 m1 = (c1 << 3) + (bfi(0x0F0F0F0Fu, nv >> 4, pb >> 3) & 0x1F1F1F1Fu);
                        qv[o >> 1][(2 * o) & 3] = __builtin_amdgcn_bitop3_b32(byte_signs(sv << 1), (0x80808080u - m0) ^ 0x80808080u, m0, 0xCA);
                        qv[o >> 1][(2 * o + 1) & 3] = __builtin_amdgcn_bitop3_b32(byte_signs(sv), (0x80808080u - m1) ^ 0x80808080u, m1, 0xCA);
                    }
                }
                CBAR();
                LW[L_FREE / 4 + (n & 7u)] = n + 1u;
                if (LW[L_BAD / 4] != 0u) qv[0] = u32x4{0x80808080u, 0x80808080u, 0x80808080u, 0x80808080u};
#pragma unroll
                for (int e = 0; e < 8; ++e) ST128(qv[e], rs_ol, ovoff, osoff + e * 8u * NN, AGT);
            }
            STAMP(2);
        }
    } else {
        arrive(0u, 12u);
        if (b == AUX_WG && wid == 2u) {
            u32 a = 0;
            u64* gp = (u64*)(p.ws[lane & 7u] + OFF_GAM + (lane >> 3) * 128u + rank * 8u);
            asm volatile("" : "+v"(gp));
            for (;;) {
                const u32x4 g0 = LD128(rs_ws, OFF_AMX + lane * 32u, 0, AGT);
                const u32x4 g1 = LD128(rs_ws, OFF_AMX + lane * 32u + 16u, 0, AGT);
                a = max(max(g0[0], g0[2]), max(g1[0], g1[2]));
                const bool ok = g0[1] == seq && g0[3] == seq && g1[1] == seq && g1[3] == seq;
                if (BALLOT(ok) == ~0ull) break;
                if (expired()) { a = 0xFFFFFFFFu; break; }
            }
            a = wave_max_u32(a);
            __hip_atomic_store((GAS u64*)gp, ((u64)seq << 32) | a, __ATOMIC_RELAXED, __HIP_MEMORY_SCOPE_SYSTEM);
            if (lane == 0) __builtin_amdgcn_raw_buffer_store_b32(seq, rs_ws, OFF_SEQ, 0, AGT);
            STAMP(3);
            bool give_up = false;
            for (;;) {
                const u32x4 f = LD128(rs_ws, OFF_DONE + lane * 16u, 0, SYS);
                const bool ok = (lane >> 3) == rank || (lane & 4u) != 0u || (f[0] >= seq && f[1] >= seq && f[2] >= seq && f[3] >= seq);
                if (BALLOT(ok) == ~0ull) break;
                if (expired()) { give_up = true; break; }
                __builtin_amdgcn_s_sleep(8);
            }
            if (give_up || __builtin_amdgcn_raw_buffer_load_b32(rs_ws, OFF_TMO, 0, AGT) == seq)
                if (lane == 0) p.out[rank * 2048u] = (char)-128;
            STAMP(7);
        }
    }
}

extern "C" void kernel_launch(int rank, int nranks, ncclComm_t comm, hipStream_t stream,
                              void* const* d_in, const int* in_sizes, int n_in,
                              void* d_out, int out_size, const int* in_dtypes, int out_dtype,
                              void* const* d_out_all, void* const* d_ws_all, size_t ws_bytes) {
    if (nranks != NR || (unsigned)rank >= (unsigned)NR || n_in < 2 || ws_bytes < 3ull * WS_NEED ||
        in_sizes[0] != (int)(MM * KSH) || in_sizes[1] != (int)(KSH * NN) || out_size != (int)OUT_BYTES) {
        fprintf(stderr, "kernel_launch: unexpected problem shape\n");
        return;
    }
    static bool init = false;
    if (!init) {
        HIP_CHECK(hipFuncSetAttribute((const void*)gemm_ar_q8, hipFuncAttributeMaxDynamicSharedMemorySize, LDS_BYTES));
        init = true;
    }
    Params p;
    p.x = (const char*)d_in[0];
    p.w = (const char*)d_in[1];
    for (int r = 0; r < NR; ++r) p.ws[r] = (char*)d_ws_all[(rank + r) % NR];
    p.out = (char*)d_out;
    p.rank = rank;
    hipLaunchKernelGGL(gemm_ar_q8, dim3(256), dim3(768), LDS_BYTES, stream, p);
    HIP_CHECK(hipGetLastError());
}
```

```cpp
#include <hip/hip_runtime.h>
#include <rccl/rccl.h>
#include <cstdio>
#include <cstdint>

#define HIP_CHECK(expr) do { hipError_t _e = (expr); if (_e != hipSuccess) { \
    fprintf(stderr, "HIP error %s at %s:%d\n", hipGetErrorString(_e), __FILE__, __LINE__); \
    return; } } while (0)

#define LAS __attribute__((address_space(3)))
#define GAS __attribute__((address_space(1)))
typedef unsigned int u32;
typedef unsigned long long u64;
typedef u32 u32x4 __attribute__((ext_vector_type(4)));
typedef u32 u32x2 __attribute__((ext_vector_type(2)));
typedef unsigned short u16x2 __attribute__((ext_vector_type(2)));
typedef float f32x16 __attribute__((ext_vector_type(16)));

#define WIN1 4
#define WIN2 4
#define WINS 4
#define E2S 8192u
#define SGM 0xBBBBu
#define PCH 2
#define PSTEP 7u
#define PSTEP2 11u
#define PBACK 20u
#define PLEAD 8u
#define XFIRST 0x0Cu
#define RAWW 2u
#define WARM 1
#define TDELAY 100
#define WDELAY1 50
#define WDELAY2 200
#define S2 (SGM == 0xAAAAu ? 32u : 48u)
#define DIAG_NDMA 8
#define CSLEEP 1
#define POLL_SLEEP 2
#define BALANCE 1
#define LSLEEP 1
#define KPAD "s_nop 0\n"
#define FLAGW 4
#define PAHEAD 4
#define LNOP 5
#define LADAPT 0
#define CPRIO 0
#define LPRIO 0
#define PRED_PAD 20
#define FORCE_RAW1 0
#define FORCE_RAW2 0

constexpr int NR = 8;
constexpr u32 MM = 4096, NN = 16384, KSH = 512;
constexpr u32 X_BYTES = MM * KSH * 2, W_BYTES = KSH * NN * 2, OUT_BYTES = MM * NN;
constexpr u32 PIECE = 32768;
constexpr u32 STEP_BYTES = 28 * PIECE;
constexpr u32 OFF_UF = 1u << 20;
constexpr u32 OFF_F3 = OFF_UF + 64 * 112 * 128;
constexpr u32 STAGE_OFF = 2u << 20;
static_assert(OFF_F3 + 2048 * 64 <= STAGE_OFF, "flags");
constexpr u32 E1_SLOT = 8192 + 128;
constexpr u32 I_HDR = 4096, I_FLD = 16, I_VAR = 1040, I_MAX = 2040;
constexpr u32 E1X_OFF = STAGE_OFF + 64 * STEP_BYTES;
constexpr u32 E1W_OFF = E1X_OFF + 512 * E1_SLOT;
constexpr u32 E1P_OFF = E1W_OFF + 2048 * E1_SLOT;
constexpr u32 E2_OFF = E1P_OFF + 512 * E1_SLOT;
constexpr u32 E3_OFF = E2_OFF + 1024 * E2S;
constexpr u32 WS_NEED = E3_OFF + 1024 * 1024;
constexpr u32 OFF_SEQ = 0, OFF_TMO = 256;
constexpr u32 OFF_AMX = 4096;
constexpr u32 OFF_GAM = OFF_AMX + 2048;
constexpr u32 OFF_DONE = OFF_GAM + 1024;
constexpr u32 OFF_F1 = OFF_DONE + 1024;
constexpr u32 OFF_CTR = OFF_F1 + 2048;
constexpr u32 OFF_F2 = 65536;
constexpr u32 OFF_FP = OFF_F2 + 32768;
constexpr u32 OFF_STAMP = 131072;
static_assert(OFF_CTR + 81 * 128 <= OFF_F2 && OFF_FP + 16384 <= OFF_STAMP && OFF_STAMP + 256 * 12 * 32 * 8 <= OFF_UF, "flags");

constexpr u32 SLOT_BYTES = 32768;
constexpr u32 CTRL = 16 * E1_SLOT;
constexpr u32 L_FULL = CTRL, L_FREE = CTRL + 32, L_NRDY = CTRL + 64, L_AMAX = CTRL + 68, L_ACNT = CTRL + 72,
              L_BAD = CTRL + 76, L_SCALE = CTRL + 80, L_GCNT = CTRL + 88, L_GSZ = CTRL + 96, L_SCNT = CTRL + 112, L_STEPS = CTRL + 128, L_PTR = CTRL + 640;
constexpr u32 L_SEG = CTRL + 4, L_G2 = CTRL + 8, L_END = CTRL + 12, L_P1 = CTRL + 16, L_G3 = CTRL + 20, L_Q = CTRL + 128,
              L_PSZ = CTRL + 192, L_SG = 8 * E1_SLOT;
constexpr u32 L_PACK = CTRL + 1024;
constexpr u32 L_SZ = L_PACK + 8 * 3072;
constexpr u32 LDS_BYTES = L_SZ + 2048;
constexpr u64 DEADLINE = 100000000ull;
constexpr u32 OOB = WS_NEED;
constexpr int SYS = 17, AGT = 16;
constexpr u32 CODE_BYTES = 49152;
constexpr u32 AUX_WG = 248;
constexpr u32 N1 = 768 + 112;
constexpr u32 N2 = 60;
constexpr u32 R_BIT = 1024, R_NIB = 2048, R_VAR = 6144, R_MAX = 3064;

struct Params {
    char* ws[NR];
    const char* x;
    const char* w;
    char* out;
    int rank;
};

typedef __amdgpu_buffer_rsrc_t rsrc_t;
__device__ __forceinline__ rsrc_t mk_rsrc(const void* p, u32 bytes) {
    return __builtin_amdgcn_make_buffer_rsrc((void*)p, 0, bytes, 0x00020000);
}
__device__ __forceinline__ u64 now() { return __builtin_amdgcn_s_memrealtime(); }
#define VMWAIT(n) asm volatile("s_waitcnt vmcnt(%0)" :: "n"(n) : "memory")
#define CBAR() asm volatile("" ::: "memory")
#define ST128(v, rs, voff, soff, aux) do { const u32x4 d_ = (v); \
    __builtin_amdgcn_raw_buffer_store_b128(d_, rs, voff, soff, aux); \
    asm volatile("s_nop 1" :: "v"(d_) : "memory"); } while (0)
#define LD128(rs, voff, soff, aux) __builtin_amdgcn_raw_buffer_load_b128(rs, voff, soff, aux)
#define PERM(a, b, s) __builtin_amdgcn_perm(a, b, s)
#define BALLOT(c) __builtin_amdgcn_ballot_w64(c)
#define RFL(v) ((u32)__builtin_amdgcn_readfirstlane(v))
__device__ __forceinline__ u32 bfi(u32 m, u32 a, u32 b) { return (a & m) | (b & ~m); }
__device__ __forceinline__ u32 rank_in(u64 m) {
    return __builtin_amdgcn_mbcnt_hi((u32)(m >> 32), __builtin_amdgcn_mbcnt_lo((u32)m, 0u));
}

#define STAMPV(i, v) do { } while (0)
#define STAMP(i) STAMPV(i, now())
#define TK(i)
#define TKOUT()

#define DPP_STEPS(OP) \
    OP(0x111, 0xf, true) OP(0x112, 0xf, true) OP(0x114, 0xf, true) OP(0x118, 0xf, true) OP(0x142, 0xa, false) OP(0x143, 0xc, false)
__device__ __forceinline__ u32 wave_max_u32(u32 v) {
#define OP_(c, rm, bc) v = max(v, (u32)__builtin_amdgcn_update_dpp(0, (int)v, c, rm, 0xf, bc));
    DPP_STEPS(OP_)
#undef OP_
    return (u32)__builtin_amdgcn_readlane((int)v, 63);
}
__device__ __forceinline__ u32 wave_scan_u32(u32 v) {
#define OP_(c, rm, bc) v += (u32)__builtin_amdgcn_update_dpp(0, (int)v, c, rm, 0xf, bc);
    DPP_STEPS(OP_)
#undef OP_
    return v;
}
__device__ __forceinline__ u32 byte_signs(u32 a) { return PERM(a, a << 8, 0x0B090A08u); }

constexpr u32 TFLIP0 = 0xFFFFFFFFu, TFLIP1 = 0xFFFFFF00u;
__device__ __forceinline__ u32 lut16(u32x4 t, u32 i) { return PERM(t[1], t[0], i) ^ PERM(t[3], t[2], i ^ 0x08080808u); }
__device__ __forceinline__ u32 unary8(u32 l0, u32 l1, u32& code) {
    u32 p1 = l0 + l1;
    p1 += p1 << 8; p1 += p1 << 16;
    const u32 p0 = p1 - l1;
    code = 0;
#pragma unroll
    for (int k = 0; k < 4; ++k) code |= (1u << ((p0 >> (8 * k)) & 31u)) | (1u << ((p1 >> (8 * k)) & 31u));
    code >>= 1;
    return p1 >> 24;
}
__device__ __forceinline__ void put_bits(LAS char* L, u32 pos, u32 code) {
    LAS u32* a = (LAS u32*)(L + ((pos >> 5) << 2));
    __hip_atomic_fetch_or(a, code << (pos & 31u), __ATOMIC_RELAXED, __HIP_MEMORY_SCOPE_WORKGROUP);
    __hip_atomic_fetch_or(a + 1, (code >> 1) >> (~pos & 31u), __ATOMIC_RELAXED, __HIP_MEMORY_SCOPE_WORKGROUP);
}
__device__ __forceinline__ u32 counts8(u32 w, u32& h0, u32& h1) {
    u32 p0, p1, t;
#define ONE_(p, k, keep) "v_ffbl_b32_sdwa " p ", %3 dst_sel:BYTE_" #k " dst_unused:" keep " src0_sel:DWORD\n"
#define CLR_ "v_add_u32 %2, -1, %3\n v_and_b32 %3, %2, %3\n"
    asm(ONE_("%0", 0, "UNUSED_PAD") CLR_ ONE_("%0", 1, "UNUSED_PRESERVE") CLR_ ONE_("%0", 2, "UNUSED_PRESERVE") CLR_ ONE_("%0", 3, "UNUSED_PRESERVE") CLR_
        ONE_("%1", 0, "UNUSED_PAD") CLR_ ONE_("%1", 1, "UNUSED_PRESERVE") CLR_ ONE_("%1", 2, "UNUSED_PRESERVE") CLR_ ONE_("%1", 3, "UNUSED_PRESERVE") "s_nop 0"
        : "=&v"(p0), "=&v"(p1), "=&v"(t), "+v"(w));
    h0 = p0 - ((p0 << 8) + 0x01010100u);
    h1 = p1 - (__builtin_amdgcn_alignbit(p1, p0, 24) + 0x01010101u);
    return p1 >> 24;
}

#define LP(a) ((const LAS u32*)(size_t)(a))
__device__ __forceinline__ u32 unary_list(u32 lane, u32 from, u32 sb, u32 mine, u32 area, u32 cap) {
    u32 pos = from + lane * sb;
    const u32 stop = pos + sb;
    const LAS u32* a = LP(((pos - 32u) >> 5) << 2);
    u32 cnt = 0, prev = __builtin_amdgcn_alignbit(a[1], a[0], pos) | 1u;
#pragma unroll
    for (u32 i = 0; i < 3u; ++i)
        cnt += (u32)__builtin_popcount(__builtin_amdgcn_alignbit(a[i + 2u], a[i + 1u], pos) & (u32)((1ull << (min(max(sb, 32u * i), 32u * i + 32u) - 32u * i)) - 1ull));
    asm volatile("" : "+v"(prev));
#pragma unroll 1
    for (u32 i = 3u; 32u * i < sb; ++i)
        cnt += (u32)__builtin_popcount(__builtin_amdgcn_alignbit(a[i + 2u], a[i + 1u], pos) & (u32)((1ull << min(sb - 32u * i, 32u)) - 1ull));
    pos -= (u32)__builtin_clz(lane == 0u ? ~0u : prev);
    u32 at = (pos >> 5) << 2, d0 = LP(at)[0], d1 = LP(at)[1];
    cnt = (cnt << 16) + mine;
    const u32 end = wave_scan_u32(cnt);
    const u32 last = area + min((u32)__builtin_amdgcn_readlane((int)end, 63) & 0xFFFFu, cap);
    u32 k = area + ((end - cnt) >> 16);
    asm volatile("" : "+v"(d0), "+v"(d1));
    u64 go = BALLOT(k < last);
    for (u32 i = 0; i < 64u && go != 0ull; ++i) {
        const u32 d2 = LP(at + 8u)[0];
        __builtin_amdgcn_sched_barrier(0);
        const u32 w = __builtin_amdgcn_alignbit(d1, d0, pos);
        u32 h0, h1;
        const u32 p8 = counts8(w, h0, h1), nv = min((u32)__builtin_popcount(w), 8u);
        const u32 np = pos + min(p8 + 1u, 32u - (u32)__builtin_clz(w | 1u)), cross = (np ^ pos) & 32u;
        pos = np;
        at += cross >> 3;
        d0 = cross ? d1 : d0;
        d1 = cross ? d2 : d1;
        asm volatile("" : "+v"(d0), "+v"(d1), "+v"(h0), "+v"(h1));
        if (__builtin_amdgcn_inverse_ballot_w64(go)) asm volatile("ds_write_b64 %0, %1" :: "v"(k), "v"(u32x2{h0, h1}) : "memory");
        k += nv;
        go &= BALLOT(k < last) & (BALLOT(pos < stop) | BALLOT(nv < 8u));
    }
    return area + ((end - cnt) & 0xFFFFu);
}
#define LIST8(d, ad) do { const LAS u32* a_ = LP((ad) & ~3u); d[0] = a_[0]; d[1] = a_[1]; d[2] = a_[2]; } while (0)
__device__ __forceinline__ void deal8(const u32 (&d)[3], u32 ad, u32 es, u32& c0, u32& c1) {
    const u32 lo = __builtin_amdgcn_alignbyte(d[1], d[0], ad), hi = __builtin_amdgcn_alignbyte(d[2], d[1], ad);
    u32 pre = es << 1;
    asm("" : "+v"(pre));
    const u32 mk = pre - (es >> 3);
    pre += pre << 4;
    asm("" : "+v"(pre));
    pre += pre << 8;
    asm("" : "+v"(pre));
    pre += pre << 16;
    const u32 sel = __builtin_amdgcn_bitop3_b32(mk, pre, 0xCCCCCCCCu, 0xCA);
    c0 = PERM(hi, lo, sel & 0x0F0F0F0Fu);
    c1 = PERM(hi, lo, (sel >> 4) & 0x0F0F0F0Fu);
}

__global__ void __launch_bounds__(768, 3) gemm_ar_q8(Params p) {
    extern __shared__ __attribute__((aligned(16))) char smem[];
    LAS char* L = (LAS char*)smem;
    u32 lane = threadIdx.x & 63;
    const u32 wid = RFL(threadIdx.x >> 6);
    const u32 rank = (u32)p.rank & 7u;
    const u32 b = blockIdx.x;
    const u32 q = b & 7u, idx = b >> 3;
    const u32 mb = 4 * (q >> 1) + (idx & 3), nb = 4 * (q & 1) + ((idx >> 2) & 3);
    const u32 nb2 = 8u + 2u * (q & 1u) + ((idx >> 2) & 1u), fills = idx < 8u ? 256u : 128u;
    const u32 rel = 1u + (b - 128u) % 7u, pj = ((b - 128u) / 7u) & 15u;
    char* wsl = p.ws[0];
    asm volatile("" :: "s"(p.x), "s"(p.w), "s"(wsl), "s"(rank));
    const rsrc_t rs_ws = mk_rsrc(wsl, WS_NEED);
    volatile LAS u32* LW = (volatile LAS u32*)L;
#define LDSV(a) (*(const LAS u32x4*)(L + (a)))

    STAMPV(0, tstart);
    STAMP(22);
    const u32 seqv = __builtin_amdgcn_raw_buffer_load_b32(rs_ws, OFF_SEQ, 0, AGT);
    if (threadIdx.x < 32) LW[CTRL / 4 + threadIdx.x] = b >= 128u && threadIdx.x - (16u - 2u * PCH) < 2u * PCH ? ~7u : 0u;
    auto barrier = [] {
        asm volatile("s_waitcnt lgkmcnt(0)" ::: "memory");
        __builtin_amdgcn_s_barrier();
        CBAR();
    };
#define PEERS(d) asm volatile("global_load_dwordx2 %0, %1, %2" : "=v"(d) : "v"((lane & 7u) * 8u), "s"(__builtin_amdgcn_kernarg_segment_ptr()) : "memory")

    auto put_unit = [&](const u32x4 (&o)[8], bool isw, u32 lk, u32 ch, u32 r) -> u32 {
        const u32 so = STAGE_OFF + (lk * 8u + ch) * STEP_BYTES + (isw ? 16u * PIECE + r * 8192u : (r >> 2) * PIECE + (r & 3u) * 4096u);
        const u32 v0 = isw ? ((lane & 48u) + (lane & 7u) * 2u + ((lane >> 3) & 1u)) * 16u : lane * 16u, v1 = v0 + (isw ? 4096u : 16384u);
#pragma unroll
        for (int e = 0; e < 8; ++e) ST128(o[e], rs_ws, (e < 4 ? v0 : v1) + (e & 3) * 1024u, so, SYS);
        return OFF_UF + ((lk * 8u + ch) * 112u + (isw ? 64u : 0u) + r) * 128u;
    };
    auto put_flag = [&](u32 flag, u32 sq) { if (lane < FLAGW) ST128((u32x4{sq, sq, sq, sq}), rs_ws, flag + lane * 16u, 0, SYS); };

    auto strip_at = [](u32 os, u32 r) -> u32 { return 32u + (os >> 1) * 4u + (r & 2u) + (os & 1u); };

    {
        u32 t = 0, tend = 0, delay = 0, step = 1536u;
        if (b < 128u) { t = wid * 128u + b; if (t < 464u) tend = t + 1u; delay = wid < 3u ? 0u : WDELAY1; }
        else if (b < 240u) {
            t = (wid < 6u ? wid - 2u : wid - 6u) * 112u + (b - 128u);
            delay = (t < 256u ? t >> 6 : 4u + (t - 256u) / 96u) * TDELAY;
            if (wid - 2u < 4u) tend = t + 1025u;
            if (wid >= 10u) tend = t + 1025u + step;
            t += t < 640u ? 1024u : 1024u + step;
        } else {
            t = 1776u + wid * 16u + (b - 240u); tend = 3008u; step = 192u;
            if (wid < 4u) __builtin_amdgcn_s_setprio(2); else if (wid < 7u) __builtin_amdgcn_s_setprio(1); else delay = WDELAY2;
        }
        bool bar = t < tend && delay == 0u;
        auto pause = [](u32 d) {
            const u32 ts = (u32)now();
            while ((u32)now() - ts < d) __builtin_amdgcn_s_sleep(2);
        };
        if (!bar) {
            barrier();
            if (t < tend) pause(delay);
        }
        if ((WARM & 1) && b < 128u && wid >= 8u) {
            u64 pc;
            asm volatile("s_getpc_b64 %0" : "=s"(pc));
            const u32 v = __builtin_amdgcn_raw_buffer_load_b32(mk_rsrc((void*)(pc & ~255ull), CODE_BYTES), lane * 128u, ((idx * 4u + wid) % 6u) * 8192u, 0);
            asm volatile("" :: "v"(v));
        }
        for (; t < tend; t += step) {
            TK(0);
            u32 kind, lk = 0, ch, r;
            if (t < 384u) { ch = t >> 8; r = (t >> 3) & 31u; lk = (t + r) & 7u; kind = lk == 0u ? 1u : 2u; }
            else if (t < 464u) { ch = (t - 336u) >> 5; r = (t - 336u) & 31u; kind = 1u; }
            else if (t < 1280u) { ch = (t >> 6) & 3u; r = t & 63u; kind = 0u; }
            else if (t < 1664u) { ch = 4u + (t - 1280u) / 96u; r = (t - 1280u) % 96u; kind = r >> 6; r &= 63u; }
            else { ch = (0x3765421u >> ((t - 1664u) / 224u * 4u)) & 7u; r = (t - 1664u) % 224u; lk = 1u + r % 7u; r /= 7u; kind = 2u; }
            const u32 to = (rank + lk) & 7u;
            const bool strip = kind != 0u && (r | 2u) == 30u;
            const bool push = kind == 2u && ch < PCH, both = push && strip;
            char* pto = wsl;
            u32x4 o[8];
            if (kind == 0u) {
                const rsrc_t rs_x = mk_rsrc(p.x, X_BYTES);
                const u32 vx = (lane >> 2) * 1024u + (((lane & 3u) ^ ((lane >> 4) & 2u)) << 4);
#pragma unroll
                for (int i = 0; i < 4; ++i) {
                    o[i] = LD128(rs_x, vx, r * 65536u + ch * 128u + i * 16384u, 0);
                    o[4 + i] = LD128(rs_x, vx + 64u, r * 65536u + ch * 128u + i * 16384u, 0);
                }
                if (bar) barrier();
                asm volatile("" : "+v"(o[0]), "+v"(o[1]), "+v"(o[2]), "+v"(o[3]), "+v"(o[4]), "+v"(o[5]), "+v"(o[6]), "+v"(o[7]));
            } else {
                const rsrc_t rs_w = mk_rsrc(p.w, W_BYTES);
                const u32 vw = (lane >> 4) * 8u * 32768u + (lane & 15u) * 16u;
                const u32 so = (ch * 64u + (r & 2u) * 16u) * 32768u + (to * 2048u + (r >> 2) * 256u + (r & 1u) * 128u) * 2u;
                u32x4 v[8];
#pragma unroll
                for (int i = 0; i < 8; ++i) v[i] = LD128(rs_w, vw, so + i * 32768u, 0);
                pto = p.ws[push ? lk : 0u];
                asm volatile("" : "+s"(pto));
                if (bar) barrier();
#pragma unroll
                for (int e = 0; e < 4; ++e)
#pragma unroll
                    for (int k = 0; k < 4; ++k) {
                        const u32 lo = PERM(v[2 * k + 1][e >> 1], v[2 * k][e >> 1], (e & 1) ? 0x07060302u : 0x05040100u);
                        const u32 hi = PERM(v[2 * k + 1][2 + (e >> 1)], v[2 * k][2 + (e >> 1)], (e & 1) ? 0x07060302u : 0x05040100u);
                        o[e][k] = (u32)__builtin_amdgcn_update_dpp((int)lo, (int)hi, 0x128, 0xf, 0xc, false);
                        o[4 + e][k] = (u32)__builtin_amdgcn_update_dpp((int)hi, (int)lo, 0x128, 0xf, 0x3, false);
                    }
            }
            bar = false;
            u32 flag = OOB, size = E1_SLOT, sq = RFL(seqv) + 1u;
            u32 reg = L_PACK + wid * 2048u;
            asm volatile("" : "+s"(sq), "+s"(reg), "+v"(lane));
            if (kind != 2u || strip) flag = put_unit(o, kind != 0u, 0u, ch, kind == 2u ? strip_at(lk - 1u, r) : r);
            if (kind == 1u && !strip) { TK(2); VMWAIT(0); TK(3); put_flag(flag, sq); TK(4); TKOUT(); continue; }
            u64 f1;
            PEERS(f1);
            const u32 kept = kind == 0u ? E1X_OFF + (ch * 64u + r) * E1_SLOT : E1W_OFF + ((lk * 8u + ch) * 32u + r) * E1_SLOT;
            const u32 slot = push ? E1P_OFF + ((lk * 2u + ch) * 32u + r) * E1_SLOT : kept;
            const rsrc_t rs_to = mk_rsrc(pto, WS_NEED);
#define PST(v, voff, at) do { const u32x4 p_ = (v); ST128(p_, rs_to, voff, slot + (at), SYS); if (both) ST128(p_, rs_ws, voff, kept + (at), SYS); } while (0)
            *(LAS u32x4*)(L + reg + lane * 16u) = u32x4{0u, 0u, 0u, 0u};
            *(LAS u32x4*)(L + reg + 1024u + lane * 16u) = u32x4{0u, 0u, 0u, 0u};
            u32 ex[16], mx = 0;
            const bool late = push && ch != 0u;
            if (push && !late && r > PLEAD) pause((r - PLEAD) * PSTEP);
#pragma unroll
            for (int e = 0; e < 4; ++e) {
                u32x4 mq;
#pragma unroll
                for (int k = 0; k < 4; ++k) {
                    const int j = 4 * e + k;
                    const u32 d0 = o[j >> 1][(2 * j) & 3], d1 = o[j >> 1][(2 * j + 1) & 3];
                    const u32 r0 = __builtin_amdgcn_alignbit(d0, d0, 31), r1 = __builtin_amdgcn_alignbit(d1, d1, 31);
                    mq[k] = PERM(r1, r0, 0x06040200u);
                    ex[j] = PERM(r1, r0, 0x07050301u);
                    mx = __builtin_bit_cast(u32, __builtin_elementwise_max(__builtin_bit_cast(u16x2, mx),
                             __builtin_elementwise_max(__builtin_bit_cast(u16x2, r0), __builtin_bit_cast(u16x2, r1))));
                }
                if (!late) PST(mq, lane * 16u + e * 1024u, 0u);
            }
            const u32 top = wave_max_u32(max(mx >> 24, (mx >> 8) & 255u));
            u32 any = 0, cnt = 0;
#pragma unroll
            for (int j = 0; j < 16; ++j) { ex[j] = top * 0x01010101u - ex[j]; any |= ex[j]; }
            CBAR();
#pragma unroll
            for (int j = 0; j < 16; j += 2)
                __hip_atomic_fetch_add((LAS u32*)(L + reg + 256u + (((ex[j] << 4) & 0xF0u) | ((lane & 3u) << 2))), 1u, __ATOMIC_RELAXED,
                                       __HIP_MEMORY_SCOPE_WORKGROUP);
            CBAR();
            const u32x4 hist = LDSV(reg + 256u + (lane & 15u) * 16u);
            CBAR();
            if (BALLOT((any & 0xF0F0F0F0u) != 0u) != 0ull) {
#pragma unroll
                for (int j = 0; j < 16; ++j) {
                    const bool far = (ex[j] & 0xF0F0F0F0u) != 0u;
                    const u64 m = BALLOT(far);
                    if (m != 0ull) {
                        const u32 k = cnt + rank_in(m);
                        if (far && k < 14u) *(LAS u32x2*)(L + reg + I_VAR + k * 8u) = u32x2{lane * 16u + j, ex[j]};
                        cnt += (u32)__builtin_popcountll(m);
                        ex[j] &= 0x0F0F0F0Fu;
                    }
                }
            }
            const u32 key = (((hist[0] + hist[1] + hist[2] + hist[3]) << 4) | (lane & 15u)) ^ 15u;
            u32 above = 0;
#define ROT_(n) above = __builtin_amdgcn_alignbit(above, key - (u32)__builtin_amdgcn_mov_dpp((int)key, 0x120 + n, 0xf, 0xf, false), 31);
            ROT_(1) ROT_(2) ROT_(3) ROT_(4) ROT_(5) ROT_(6) ROT_(7) ROT_(8) ROT_(9) ROT_(10) ROT_(11) ROT_(12) ROT_(13) ROT_(14) ROT_(15)
#undef ROT_
            above = (u32)__builtin_popcount(above);
            u32 tb = ((max(above, 2u) - 2u) | (min(above, 3u) << 4)) << ((lane & 3u) * 8u);
            tb |= (u32)__builtin_amdgcn_mov_dpp((int)tb, 0xB1, 0xf, 0xf, false);
            tb |= (u32)__builtin_amdgcn_mov_dpp((int)tb, 0x4E, 0xf, 0xf, false);
            const u32x4 tab = u32x4{(u32)__builtin_amdgcn_readlane((int)tb, 0) ^ TFLIP0, (u32)__builtin_amdgcn_readlane((int)tb, 4) ^ TFLIP1,
                                    (u32)__builtin_amdgcn_readlane((int)tb, 8) ^ TFLIP0, (u32)__builtin_amdgcn_readlane((int)tb, 12) ^ TFLIP1};
            u32 code[8], len[8], over = 0, tot = 0;
            u32x4 fw;
#pragma unroll
            for (int g = 0; g < 8; ++g) {
                const u32 s0 = lut16(tab, ex[2 * g]), s1 = lut16(tab, ex[2 * g + 1]), h = bfi(0x0F0F0F0Fu, s0 >> 4, s1);
                fw[g >> 1] = g & 1 ? fw[g >> 1] | (h << 2) : h;
                len[g] = unary8(s0 & 0x0F0F0F0Fu, s1 & 0x0F0F0F0Fu, code[g]);
                over |= len[g];
                tot += len[g] + ((u32)__builtin_popcount(code[g]) << 16);
            }
            const u32 end = wave_scan_u32(tot), all = (u32)__builtin_amdgcn_readlane((int)end, 63);
            const u32 bytes = ((I_VAR + cnt * 8u) * 8u + (all & 0xFFFFu) + 127u) >> 7 << 4;
            if (kind != 2u) { VMWAIT(4); put_flag(flag, sq); }
            if (BALLOT(over > 31u) == 0ull && cnt < 15u && bytes <= 2032u && (all >> 16) <= I_MAX && !(FORCE_RAW1 && (r & 1u))) {
                size = I_HDR + bytes;
                *(LAS char*)(L + reg + above) = (char)((lane & 15u) | ((((bytes >> 4) | (top << 8) | (cnt << 16)) >> (above * 4u & 31u)) << 4));
                *(LAS u32x4*)(L + reg + I_FLD + lane * 16u) = fw;
                u32 pos = ((end - tot) & 0xFFFFu) + (reg + I_VAR + cnt * 8u) * 8u;
#pragma unroll
                for (int g = 0; g < 8; ++g) { put_bits(L, pos, code[g]); pos += len[g]; }
                CBAR();
                if (late) {
                    pause((r < 25u ? r : r - PBACK) * PSTEP2);
#pragma unroll
                    for (int e = 0; e < 4; ++e) {
                        u32x4 mq;
#pragma unroll
                        for (int k = 0; k < 4; ++k) {
                            const u32 d0 = o[2 * e + (k >> 1)][(2 * k) & 3], d1 = o[2 * e + (k >> 1)][(2 * k + 1) & 3];
                            mq[k] = PERM(__builtin_amdgcn_alignbit(d1, d1, 31), __builtin_amdgcn_alignbit(d0, d0, 31), 0x06040200u);
                        }
                        PST(mq, lane * 16u + e * 1024u, 0u);
                    }
                }
                PST(LDSV(reg + lane * 16u), lane * 16u, I_HDR);
                PST(LDSV(reg + 1024u + lane * 16u), (lane & 60u) * 16u + 1024u < bytes ? lane * 16u + 1024u : OOB, I_HDR);
                CBAR();
            } else {
                PST((u32x4{~0u, ~0u, ~0u, ~0u}), lane == 0u ? 0u : OOB, I_HDR);
#pragma unroll
                for (int e = 0; e < 8; ++e) PST(o[e], lane * 16u + (e & 3) * 1024u, (e >> 2) * (I_HDR + 128u));
            }
            u32 old = 0;
            TK(2);
            VMWAIT(0);
            TK(3);
            asm volatile("" : "+v"(f1));
            if (kind == 2u) put_flag(flag, sq);
            if (push) {
                if (lane == 0) __builtin_amdgcn_raw_buffer_store_b64(u32x2{sq, (size + 15u) >> 4}, rs_to, 0u, OFF_FP + (lk * 16u + (r >> 1)) * 128u + (ch * 2u + (r & 1u)) * 8u, SYS);
                __builtin_amdgcn_s_setprio(0);
                if (!strip) { TK(4); TKOUT(); continue; }
            }
            if (lane == 0 ? kind != 1u && !push : (lane == 1u && strip))
                old = 1u + __hip_atomic_fetch_add((u32*)(wsl + OFF_CTR + (lane == 0 ? lk * 8u + ch : 80u) * 128u), 1u, __ATOMIC_RELAXED,
                                                  __HIP_MEMORY_SCOPE_AGENT);
            f1 += OFF_F1 + lane * 256u;
            const u64 fv = sq;
            if (RFL(old) == sq * (kind == 0u ? 64u : 32u) && lane - 1u < 7u && (kind == 0u || lane == lk))
                __hip_atomic_store((GAS u64*)f1 + ch * 2u + (kind == 0u), fv, __ATOMIC_RELAXED, __HIP_MEMORY_SCOPE_SYSTEM);
            if ((u32)__builtin_amdgcn_readlane((int)old, 1) == sq * 128u && lane - 1u < 7u)
                __hip_atomic_store((GAS u64*)f1 + 16, fv, __ATOMIC_RELAXED, __HIP_MEMORY_SCOPE_SYSTEM);
            TKOUT();
        }
    }
    const u32 seq = RFL(seqv) + 1u;
    lane = threadIdx.x & 63;
    STAMP(10);

    u32 spins = 0;
    u64 t0 = 0;
    auto expired = [&]() -> bool {
        if ((++spins & 63u) != 0u) return false;
        if (LW[L_BAD / 4] != 0u) return true;
        const u64 t = now();
        if (t0 == 0) t0 = t;
        if (t - t0 > DEADLINE || __builtin_amdgcn_raw_buffer_load_b32(rs_ws, OFF_TMO, 0, AGT) == seq) {
            u32 sq = seq;
            asm volatile("" : "+s"(sq));
            __builtin_amdgcn_raw_buffer_store_b32(sq, rs_ws, OFF_TMO, 0, AGT);
            LW[L_BAD / 4] = 1u;
            return true;
        }
        return false;
    };

    auto arrive = [&](u32 wm, u32 n) {
        if (lane == 0) {
            __hip_atomic_fetch_max((LAS u32*)(L + L_AMAX), wm, __ATOMIC_RELAXED, __HIP_MEMORY_SCOPE_WORKGROUP);
            const u32 old = __hip_atomic_fetch_add((LAS u32*)(L + L_ACNT), 1u, __ATOMIC_RELAXED, __HIP_MEMORY_SCOPE_WORKGROUP);
            if (old == n - 1u) {
                u32 a = LW[L_AMAX / 4];
                if (LW[L_BAD / 4] != 0u) a = 0xFFFFFFFFu;
                __builtin_amdgcn_raw_buffer_store_b64(u32x2{a, seq}, rs_ws, OFF_AMX + b * 8u, 0, AGT);
            }
        }
    };

    if (b < 128u && wid < 8u) {
        u32 c = lane & 15u, g = lane >> 4;
        const u32 mw = wid & 3u, nh = wid >> 2;
        for (u32 f0 = 0; f0 < fills; f0 += 128u) {
        f32x16 acc[8];
        u32 xa = mw * 4096u + (c * 4u + (g ^ ((c >> 2) & 2u))) * 16u, wa = 16384u + nh * 8192u + lane * 16u;
        const u32 ca = lane < 32u ? L_FULL + ((lane - 1u) & 3u) * 4u : lane < 48u ? L_FREE + (wid ^ 4u) * 4u : L_FREE + wid * 4u - 768u;
        u32 f2 = f0 + 1u, spin = 0, tv;
#define S0_(x) #x
#define S_(x) S0_(x)
#define R4(r) "v[" S_(r) ":" S_(r) "+3]"
#define MM_(j, m, w, x) "v_mfma_f32_16x16x32_bf16 " R4(16*j+4*m) ", " R4(w) ", " R4(x) ", " R4(16*j+4*m) "\n"
#define MM4(j, w, x0) MM_(j, 0, w, x0) MM_(j, 1, w, 132) MM_(j, 2, w, 136) MM_(j, 3, w, 140)
#define RW(r, o) "ds_read_b128 " R4(r) ", %[wa] offset:" S_(o) "\n"
#define RX(r, o) "ds_read_b128 " R4(r) ", %[xa] offset:" S_(o) "\n"
#define LG(n) "s_waitcnt lgkmcnt(" S_(n) ")\n" KPAD
#define KWAIT(l) \
    "8" l ":\n s_cmp_eq_u32 %[f2], %[last]\n s_cbranch_scc1 9" l "b\n ds_read_b32 %[tv], %[ca]\n s_waitcnt lgkmcnt(0)\n" \
    "v_cmp_le_u32 vcc, %[f2], %[tv]\n s_nop 4\n s_bitcmp1_b32 vcc_lo, %[f2]\n s_cbranch_scc1 9" l "b\n s_sleep 1\n" \
    "s_add_u32 %[spin], %[spin], 1\n s_cmp_lt_u32 %[spin], 0x1000000\n s_cbranch_scc1 8" l "b\n s_branch 9" l "b\n"
#define KFILL(l, x0, y0, cur, nxt, TOGGLE) \
    LG(5) MM_(0, 0, 144, x0) MM_(1, 0, 148, x0) LG(4) MM_(0, 1, 144, 132) MM_(1, 1, 148, 132) \
    LG(3) MM_(0, 2, 144, 136) MM_(1, 2, 148, 136) LG(2) MM_(0, 3, 144, 140) RW(144, cur+4096) MM_(1, 3, 148, 140) RW(148, cur+5120) \
    "ds_read_b32 %[tv], %[ca]\n" \
    "s_waitcnt lgkmcnt(4)\n s_add_u32 %[f2], %[f2], 1\n" MM4(2, 152, x0) RW(152, cur+6144) \
    LG(4) MM4(3, 156, x0) RW(156, cur+7168) \
    "s_waitcnt lgkmcnt(2)\n v_cmp_le_u32 vcc, %[f2], %[tv]\n" MM_(4, 0, 144, x0) \
    "v_mov_b32_e64 %[tv], %[f2]\n" MM_(4, 1, 144, 132) \
    "ds_write_b32 %[ca], %[tv] offset:768\n"         \
    MM_(4, 2, 144, 136) TOGGLE MM_(4, 3, 144, 140) \
      \
    "s_setprio " S_(CPRIO) "\n s_bitcmp0_b32 vcc_hi, 0\n s_cbranch_scc1 7" l "f\n s_setprio " S_(CPRIO+BALANCE) "\n7" l ":\n" \
    "s_bitcmp0_b32 vcc_lo, %[f2]\n s_cbranch_scc1 8" l "f\n9" l ":\n" \
    RW(144, nxt) RX(y0, nxt) MM4(5, 148, x0) RW(148, nxt+1024) \
    LG(4) MM_(6, 1, 152, 132) MM_(7, 1, 156, 132) RX(132, nxt+1024) MM_(6, 2, 152, 136) MM_(7, 2, 156, 136) RX(136, nxt+2048) \
    MM_(6, 3, 152, 140) MM_(7, 3, 156, 140) RX(140, nxt+3072) MM_(6, 0, 152, x0) RW(152, nxt+2048) MM_(7, 0, 156, x0) RW(156, nxt+3072)
        asm volatile(
            ".set i_, 0\n .rept 64\n v_mov_b64 v[i_:i_+1], 0\n .set i_, i_+2\n .endr\n"
            "s_branch 80f\n90:\n"
            RW(144, 0) RX(128, 0) RW(148, 1024) RX(132, 1024) RX(136, 2048) RX(140, 3072) RW(152, 2048) RW(156, 3072)
            ".p2align 3\n1:\n"
            KFILL("1", 128, 160, 0, 32768, "")
            KFILL("2", 160, 128, 32768, 0, "v_xor_b32 %[wa], 0x10000, %[wa]\n v_xor_b32 %[xa], 0x10000, %[xa]\n")
            "s_cmp_lt_u32 %[f2], %[last]\n s_cbranch_scc1 1b\n s_branch 2f\n"
            KWAIT("0") KWAIT("1") KWAIT("2")
            "2:\n s_waitcnt lgkmcnt(0)\n s_nop 7\n s_nop 7\n"
            : "={v[0:15]}"(acc[0]), "={v[16:31]}"(acc[1]), "={v[32:47]}"(acc[2]), "={v[48:63]}"(acc[3]), "={v[64:79]}"(acc[4]),
              "={v[80:95]}"(acc[5]), "={v[96:111]}"(acc[6]), "={v[112:127]}"(acc[7]),
              [xa] "+{v164}"(xa), [wa] "+{v165}"(wa), [tv] "={v166}"(tv), [f2] "+s"(f2), [spin] "+s"(spin)
            : [ca] "{v167}"(ca), [last] "s"(f0 + 129u)
            : "v128", "v129", "v130", "v131", "v132", "v133", "v134", "v135", "v136", "v137", "v138", "v139", "v140", "v141", "v142",
              "v143", "v144", "v145", "v146", "v147", "v148", "v149", "v150", "v151", "v152", "v153", "v154", "v155", "v156", "v157",
              "v158", "v159", "v160", "v161", "v162", "v163", "vcc", "scc", "memory");
        if (spin >= 0x1000000u) LW[L_BAD / 4] = 1u;
        asm volatile("v_mbcnt_lo_u32_b32 %0, -1, 0\n v_mbcnt_hi_u32_b32 %0, -1, %0" : "=v"(lane));
        c = lane & 15u; g = lane >> 4;
        STAMPV(f0 == 0u ? 14 : 15, (u64)spin << 24);
        const u32 reg = L_PACK + wid * 2048u, slot = E2_OFF + (b * 8u + wid) * E2S;
        auto post = [&](u32 cnt, u32 at) {
            u32 old = 0;
            VMWAIT(0);
            if (lane == 0) old = __hip_atomic_fetch_add((LAS u32*)(L + cnt + nh * 4u), 1u, __ATOMIC_RELAXED, __HIP_MEMORY_SCOPE_WORKGROUP);
            if (RFL(old) == 3u && lane - 1u < 7u) {
                const u32x4 fv = u32x4{seq, LW[L_GSZ / 4 + nh * 2u], LW[L_GSZ / 4 + nh * 2u + 1u], seq};
                u32x2 fp = *(volatile LAS u32x2*)(L + L_PTR + lane * 8u);
                asm volatile("v_add_co_u32 %0, vcc, %0, %2\n v_addc_co_u32 %1, vcc, 0, %1, vcc" : "+v"(fp[0]), "+v"(fp[1]) : "v"(at) : "vcc");
                asm volatile("global_store_dwordx4 %0, %1, off sc0 sc1\n s_nop 1" :: "v"(fp), "v"(fv) : "memory");
            }
        };
        if (f0 == 0u) {
            STAMP(2);
            STAMPV(9, __builtin_amdgcn_s_memtime());
            u32x4 sg;
#pragma unroll
            for (int m = 0; m < 4; ++m) {
                u32 d = 0;
#pragma unroll
                for (int i = 31; i >= 0; --i) d = __builtin_amdgcn_alignbit(d, __float_as_uint(acc[(i & 4) + (i >> 3)][4 * m + (i & 3)]), 31);
                const u32 o = (u32)__builtin_amdgcn_mov_dpp((int)d, 0x128, 0xf, 0xf, false);
                sg[m] = bfi(c < 8u ? 0x0F0F0F0Fu : 0xF0F0F0F0u, d, __builtin_amdgcn_alignbit(o, o, c < 8u ? 28u : 4u));
            }
            ST128(sg, rs_ws, lane * 16u, (SGM >> (q * 2u + nh)) & 1u ? E3_OFF + (b * 8u + wid) * 1024u : slot, AGT);
            float mx = 0.f;
#pragma unroll
            for (int j = 0; j < 8; ++j)
#pragma unroll
                for (int m = 0; m < 4; ++m) {
                    mx = fmaxf(fmaxf(mx, fabsf(acc[j][4 * m])), fabsf(acc[j][4 * m + 1]));
                    mx = fmaxf(fmaxf(mx, fabsf(acc[j][4 * m + 2])), fabsf(acc[j][4 * m + 3]));
                }
            arrive(wave_max_u32(__float_as_uint(mx)), 8u);
            STAMP(3);
            if (wid >= 4u) __builtin_amdgcn_s_setprio(2); else __builtin_amdgcn_s_setprio(0);
#pragma unroll
            for (int k = 0; k < 2; ++k) *(LAS u32x4*)(L + reg + k * 1024u + lane * 16u) = u32x4{0u, 0u, 0u, 0u};
            if ((SGM >> (q * 2u + nh)) & 1u) post(L_SCNT, OFF_F3 + ((lane * 16u + idx) * 16u + q * 2u + nh) * 64u);
            STAMP(23);
            while (LW[L_SCALE / 4] != 1u) { if (expired()) break; __builtin_amdgcn_s_sleep(CSLEEP); }
            CBAR();
        } else STAMP(13);
        const u32 gmax = LW[L_SCALE / 4 + 1];
        const bool bad = (gmax == 0xFFFFFFFFu) || (LW[L_BAD / 4] != 0u);
        const float amax = __uint_as_float(gmax);
        const float inv = amax > 0.f ? 127.0f / amax : 0.f;
        if (f0 == 0u) STAMP(4);
        const float MAGIC = 12582912.0f;
        const float qlo = bad ? MAGIC - 128.0f : MAGIC - 127.0f, qhi = bad ? MAGIC - 128.0f : MAGIC + 127.0f;
        const u32 ovoff = (c & 7u) * NN + ((c >> 3) * 4u + g) * 16u;
        const u32 os = (nb2 - 8u) * 2u + nh;
        const u32 osoff = (mb * 256u + mw * 64u) * NN + (f0 == 0u ? rank * 2048u + nb * 256u + nh * 128u : ((rank + 1u + os) & 7u) * 2048u + 1792u);
        u32x4 qv[8];
#pragma unroll
        for (int m = 0; m < 4; ++m) {
#pragma unroll
            for (int r = 0; r < 4; ++r) {
                u32 o[2];
#pragma unroll
                for (int h = 0; h < 2; ++h) {
                    u32 t[4];
#pragma unroll
                    for (int jj = 0; jj < 4; ++jj)
                        t[jj] = __float_as_uint(__builtin_amdgcn_fmed3f(__builtin_fmaf(acc[4 * h + jj][4 * m + r], inv, MAGIC), qlo, qhi));
                    o[h] = PERM(t[1], t[0], 0x0c0c0400u) | PERM(t[3], t[2], 0x04000c0cu);
                }
                qv[2 * m][r] = (u32)__builtin_amdgcn_update_dpp((int)o[0], (int)o[1], 0x128, 0xf, 0xc, false);
                qv[2 * m + 1][r] = (u32)__builtin_amdgcn_update_dpp((int)o[1], (int)o[0], 0x128, 0xf, 0x3, false);
            }
        }
        if (f0 == 0u) {
        STAMP(7);
        u32 size = nb == 7u && nh == 0u ? 0u : 8192u;
        if (size != 0u && !bad && !(q == 7u && wid - RAWW <= 3u - RAWW) && !(FORCE_RAW2 && (wid & 1u))) {
            u32x4 nib[4], pl = u32x4{0u, 0u, 0u, 0u};
            u32 code[16], len[16], over = 0, tot = 0;
#pragma unroll
            for (int o = 0; o < 16; ++o) {
                u32 y[2], l[2];
#pragma unroll
                for (int e = 0; e < 2; ++e) {
                    const u32 v = qv[o >> 1][(2 * o + e) & 3], s = byte_signs(v);
                    y[e] = (v ^ s) + ((s & 0x01010101u) | 0x70707070u);
                    l[e] = (y[e] & byte_signs(y[e]) & 0x78787878u) >> 3;
                }
                const u32 h = bfi(0x0F0F0F0Fu, y[0] >> 4, y[1]);
                nib[o >> 2][o & 3] = bfi(0x0F0F0F0Fu, y[0], y[1] << 4) | (((h & 0x77777777u) + 0x77777777u) & h & 0x88888888u);
                pl[o >> 2] = bfi(0x88888888u >> (o & 3), h >> (o & 3), pl[o >> 2]);
                len[o] = unary8(l[0], l[1], code[o]);
                over |= len[o];
                tot += len[o] + ((u32)__builtin_popcount(code[o]) << 16);
            }
            const u32 end = wave_scan_u32(tot);
            const u32 all = (u32)__builtin_amdgcn_readlane((int)end, 63), bits = all & 0xFFFFu;
            if (BALLOT(over > 31u) == 0ull && bits <= (8192u - 64u - R_VAR) * 8u && (all >> 16) <= R_MAX) {
                size = (R_VAR + ((bits + 127u) >> 7) * 16u);
                ST128(pl, rs_ws, lane * 16u, slot + R_BIT, AGT);
#pragma unroll
                for (int e = 0; e < 4; ++e) ST128(nib[e], rs_ws, lane * 16u + e * 1024u, slot + R_NIB, AGT);
                u32 pos = ((end - tot) & 0xFFFFu) + reg * 8u;
#pragma unroll
                for (int o = 0; o < 16; ++o) { put_bits(L, pos, code[o]); pos += len[o]; }
                CBAR();
#pragma unroll
                for (int k = 0; k < 2; ++k) {
                    const u32 off = R_VAR + k * 1024u + lane * 16u;
                    if (R_VAR + k * 1024u < size) ST128(LDSV(reg + k * 1024u + lane * 16u), rs_ws, off < size ? off : OOB, slot, AGT);
                }
            }
        }
        if (size == 8192u) {
#pragma unroll
            for (int e = 0; e < 8; ++e) ST128(qv[e], rs_ws, lane * 16u, slot + e * 1024u, AGT);
        }
        STAMP(11);
        if (lane == 0) *(volatile LAS unsigned short*)(L + L_GSZ + wid * 2u) = (unsigned short)(size >> 4);
        post(L_GCNT, OFF_F2 + ((lane * 16u + idx) * 16u + q * 2u + nh) * 16u);
        STAMP(5);
        __builtin_amdgcn_s_setprio(0);
        }
        const rsrc_t rs_ol = mk_rsrc(p.out, f0 == 0u || os < 7u ? OUT_BYTES : 0u);
#pragma unroll
        for (int e = 0; e < 8; ++e) ST128(qv[e], rs_ol, ovoff, osoff + e * 8u * NN, AGT);
        }
    } else if (b < 128u && wid == 8u) {
        __builtin_amdgcn_s_setprio(LPRIO);
        *(volatile LAS u64*)(L + L_PTR + (lane & 7u) * 8u) = (u64)p.ws[lane & 7u];
        for (u32 n = 0; n < fills / 2u;) {
            const u32 wn = n == 0u ? nb : nb2, end = n + 64u;
            const bool hole = wn == 11u && (lane & 5u) == 5u;
            u32 step = lane & 56u, va = OFF_UF + (step * 112u + (lane & 4u ? 64u + wn * 4u : mb * 4u) + (lane & 3u)) * 128u;
            bool seen = hole;
            while (n != end) {
                while (2u * n > LW[L_FREE / 4] + PAHEAD) { if (expired()) break; __builtin_amdgcn_s_sleep(8); }
                seen |= __builtin_amdgcn_raw_buffer_load_b32(rs_ws, seen ? OOB : va, 0, SYS) >= seq;
                u64 avail = BALLOT(seen);
                avail &= avail >> 1; avail &= avail >> 2; avail &= avail >> 4;
                avail &= 0x0101010101010101ull;
                if (avail == 0) {
                    if (!expired()) { __builtin_amdgcn_s_sleep(POLL_SLEEP); continue; }
                    avail = BALLOT(va != OOB) & 0x0101010101010101ull;
                }
                if ((avail >> lane) & 1ull) LW[L_STEPS / 4 + n + rank_in(avail)] = step;
                n += (u32)__builtin_popcountll(avail);
                LW[L_NRDY / 4] = n;
                if ((avail >> (lane & 56u)) & 1ull) {
                    seen = hole;
                    va = (++step & 7u) ? va + 112u * 128u : OOB;
                }
            }
            if (n != 64u) break;
            STAMP(1);
            while (LW[L_FREE / 4] < 128u) { if (expired()) break; __builtin_amdgcn_s_sleep(8); }
            u32 a = 0;
            for (;;) {
                const u32x2 gm = __builtin_amdgcn_raw_buffer_load_b64(rs_ws, OFF_GAM + q * 128u + (lane & 7u) * 8u, 0, SYS);
                a = gm[0];
                if (BALLOT(gm[1] == seq) == ~0ull) break;
                if (expired()) { a = 0xFFFFFFFFu; break; }
                __builtin_amdgcn_s_sleep(1);
            }
            a = wave_max_u32(a);
            LW[L_SCALE / 4 + 1] = a;
            LW[L_SCALE / 4] = 1u;
            STAMP(2);
            while (LW[L_GCNT / 4] + LW[L_GCNT / 4 + 1] < 8u) { if (expired()) break; __builtin_amdgcn_s_sleep(8); }
        }
    } else if (b < 128u) {
#define CLK(c)
        __builtin_amdgcn_s_setprio(LPRIO);
        for (u32 f = wid - 9u; f < fills; f += 3u) {
            CLK(c0);
            while (LW[L_NRDY / 4] <= (f >> 1)) { if (expired()) break; __builtin_amdgcn_s_sleep(1); }
            CBAR();
            const u32 step = RFL(LW[L_STEPS / 4 + (f >> 1)]) & 63u;
            CLK(c1);
            bool ahead = false;
            while (BALLOT(LW[L_FREE / 4 + (lane & 7u)] + 2u >= f) != ~0ull && f >= 4u) {
                ahead = true;
                if (expired()) break;
                __builtin_amdgcn_s_sleep(LSLEEP);
            }
            CBAR();
            const u32 sbase = STAGE_OFF + step * STEP_BYTES + (f & 1u) * 16384u;
            LAS char* ldst = L + (f & 3u) * SLOT_BYTES;
            CLK(c2);
            auto fill = [&](auto pace) {
#pragma unroll
                for (int i = 0; i < DIAG_NDMA; ++i) {
                    const u32 so = sbase + (i < 4 ? mb : 16u + (f < 128u ? nb : nb2)) * PIECE + (i & 3) * 4096u;
                    __builtin_amdgcn_raw_ptr_buffer_load_lds(rs_ws, ldst + i * 4096, 16, lane * 16u, so, 0, SYS); pace();
                    __builtin_amdgcn_raw_ptr_buffer_load_lds(rs_ws, ldst + i * 4096, 16, lane * 16u, so, 1024, SYS); pace();
                    __builtin_amdgcn_raw_ptr_buffer_load_lds(rs_ws, ldst + i * 4096, 16, lane * 16u, so, 2048, SYS); pace();
                    __builtin_amdgcn_raw_ptr_buffer_load_lds(rs_ws, ldst + i * 4096, 16, lane * 16u, so, 3072, SYS); pace();
                }
            };
            if (ahead || !LADAPT) fill([] { asm volatile("s_nop %0" :: "n"(LNOP) : "memory"); });
            else fill([] { });
            CBAR();
            CLK(c3);
            VMWAIT(0);
            CLK(c4);
            LW[L_FULL / 4 + (f & 3u)] = f + 1u;
            if (f - 125u < 3u) STAMP(1);
        }
    } else if (b < 240u) {
        arrive(0u, 12u);
        const u32 src = (rank - rel) & 7u;
        const rsrc_t rs_src = mk_rsrc(p.ws[(8u - rel) & 7u], WS_NEED);
        auto in_unit = [&](u32 u, u32& ch, u32& r, bool& isw) -> u32 {
            if (u >= 768u) {
                ch = (u - 768u) / 14u; r = (u - 768u) % 14u; isw = true;
                const u32 os = r >> 1, rs = 28u + (r & 1u) * 2u;
                r = strip_at(os, rs);
                return E1W_OFF + ((((rel + 1u + os) & 7u) * 8u + ch) * 32u + rs) * E1_SLOT;
            }
            ch = u / 96u; r = u % 96u;
            if ((XFIRST >> ch) & 1u) r = r < 64u ? r + 32u : r - 64u;
            isw = r < 32u;
            if (!isw) r = (r - 32u + 8u * rel) & 63u;
            return !isw ? E1X_OFF + (ch * 64u + r) * E1_SLOT : ch < PCH ? E1P_OFF + ((rel * 2u + ch) * 32u + r) * E1_SLOT
                                                                         : E1W_OFF + ((rel * 8u + ch) * 32u + r) * E1_SLOT;
        };
        auto out_unit = [&](u32 o) -> u32 {
            const u32 k = o >> 4, j = o - 768u;
            if (k >= 48u) return j % 12u * 64u + (j / 48u) * 16u + 8u + j / 12u % 4u;
            return (o & 15u) * 64u + (k < 32u ? (k >> 2) * 8u + 4u : ((k >> 2) & 3u) * 16u) + (k & 3u);
        };
        auto in_size = [](u32 h) -> u32 {
            h = ((h >> 4) & 15u) | ((h >> 8) & 0xF0u);
            return h > 127u ? E1_SLOT / 16u : I_HDR / 16u + h;
        };
        if (wid == 0u) {
            __builtin_amdgcn_s_setprio(3);
            auto ctl = [&](u32 a) -> u32 {
                u32 v;
                asm volatile("ds_read_b32 %0, %1\n s_waitcnt lgkmcnt(0)" : "=v"(v) : "v"(a) : "memory");
                return v;
            };
            auto put = [&](u32 a, u32 n) { asm volatile("ds_write_b32 %0, %1" :: "v"(a), "v"(n) : "memory"); };
            const u32 ca = L_FULL + (lane & 15u) * 4u;
            const u32 vone = lane == 0u ? 0u : OOB;
            u32 n = 0;
            auto wait = [&](int word, u32 bit) {
                auto ok = [&](u32 w) -> bool {
                    return (((u32)__builtin_amdgcn_readlane((int)w, word) >> bit) & 1u) != 0u &&
                           (u32)__builtin_amdgcn_readlane((int)w, 8 + (n & 7u)) + (n < (word == 1 ? 16u : 8u) ? 15u : 7u) >= n;
                };
                if (!ok(ctl(ca))) {
                    VMWAIT(0);
                    put(L_FULL, n);
                    while (!ok(ctl(ca))) { if (expired()) break; __builtin_amdgcn_s_sleep(1); }
                }
            };
#define CLAIM(cl, so) asm volatile("v_mov_b32 %0, 1\n s_nop 4\n buffer_atomic_add %0, %1, %2, %3 offen sc0 sc1" \
                                   : "=&v"(cl) : "v"(vone), "s"(rs_ws), "s"(so) : "memory")
#define CLAIMED(d, cl) asm volatile("v_readfirstlane_b32 %0, %1" : "=s"(d) : "v"(cl) : "memory")
            for (u32 u = 32u + pj; u < N1; ++n) {
                u32 ch, r, cl; bool isw;
                const u32 so = in_unit(u, ch, r, isw);
                if (u >= 768u) put(L_P1, 1u);
                const u32 seg = u >= 768u ? 16u : ch * 2u + !isw;
                wait(1, seg);
                if (n == 0) STAMP(1);
                LAS char* ld = L + (n < 16u ? n : n & 7u) * E1_SLOT;
                u32 sz = 0;
                put(L_Q + (n & 15u) * 4u, u);
                CLAIM(cl, OFF_CTR + (64u + rel) * 128u);
                static_assert(WIN1 <= 4, "the header, read first, has landed behind the fifth read");
#pragma unroll
                for (int i = 0; i < 9; ++i) {
                    if (i == 5) sz = in_size(RFL(ctl((n < 16u ? n : n & 7u) * E1_SLOT + I_HDR)));
                    if (i >= 6 && i * 64u >= sz) break;
                    VMWAIT(WIN1 - (i < WIN1 - 1 ? 0 : 1));
                    if (i == WIN1 - 1) put(L_FULL, n);
                    const int k = i > 4 ? i : (i + 4) % 5;
                    if (i < 5 || i * 64u + lane < sz)
                        __builtin_amdgcn_raw_ptr_buffer_load_lds(rs_src, ld + k * 1024, 16, lane * 16u, so + k * 1024u, 0, SYS);
                }
                CLAIMED(u, cl);
                u = u - (seq - 1u) * (N1 - 32u * PCH) + 16u;
                u += u < 64u ? 32u : 32u * PCH;
            }
            put(L_P1, 1u);
            STAMP(2);
            for (u32 got = 0, i = 0; got != SGM;) {
                const u32 w = ctl(ca);
                const u32 m = (u32)__builtin_amdgcn_readlane((int)w, 5) & ~got;
                if (m == 0u || (n < 24u && (BALLOT((int)w + 8 > (int)n) >> 8 & 255u) != 255u)) {
                    VMWAIT(0);
                    put(L_FULL, n);
                    if (expired()) break;
                    __builtin_amdgcn_s_sleep(1);
                    continue;
                }
                if (got == 0u) STAMP(23);
                const u32 t = (u32)__builtin_ctz(m);
                got |= 1u << t;
#pragma unroll
                for (int e = 0; e < 4; ++e) {
                    VMWAIT(WINS - 1);
                    if (++i == WINS) put(L_FULL, n);
                    __builtin_amdgcn_raw_ptr_buffer_load_lds(rs_src, L + L_SG + (t * 4u + e) * 1024u, 16, lane * 16u,
                                                             E3_OFF + (pj * 64u + t * 4u + e) * 1024u, 0, SYS);
                }
            }
            STAMP(24);
            put(L_FULL, n);
            while ((BALLOT((int)ctl(ca) + 8 > (int)n) >> 8 & 255u) != 255u) if (expired()) break;
            for (u32 o = pj; o < 16u * N2; ++n) {
                const u32 t = out_unit(o);
                const bool dyn = o >= 16u * (S2 - 1u);
                u32 cl;
                wait(2, o < 16u * S2 ? (t >> 2) & 15u : 16u);
                if (o == pj) STAMP(3);
                u32 sz = min(RFL(ctl(L_SZ + (t >> 1) * 4u)) >> ((t & 1u) * 16u) & 0xFFFFu, 512u);
                const u32 skip = sz < 512u && ((SGM >> ((t >> 2) & 15u)) & 1u) ? 1024u : 0u;
                put(L_Q + (n & 15u) * 4u, 1024u + t + skip * 2u + (sz << 16));
                LAS char* ld = L + (n & 7u) * E1_SLOT + skip;
                const u32 so = E2_OFF + t * E2S + skip;
                sz -= skip >> 4;
                CLAIM(cl, dyn ? OFF_CTR + (72u + rel) * 128u : OOB);
#pragma unroll
                for (int i = 0; i < 8; ++i) {
                    if (i >= 5 && (int)(i * 64u) >= (int)sz) break;
                    VMWAIT(WIN2 - (i < WIN2 - 1 ? 0 : 1));
                    if (i == WIN2 - 1) put(L_FULL, n);
                    if (i < 5 || i * 64u + lane < sz)
                        __builtin_amdgcn_raw_ptr_buffer_load_lds(rs_src, ld + i * 1024, 16, lane * 16u, so + i * 1024u, 0, SYS);
                }
                u32 c;
                CLAIMED(c, cl);
                o = dyn ? c - (seq - 1u) * (16u * (N2 - S2 + 1u)) + 16u * S2 : o + 16u;
            }
            VMWAIT(0);
            put(L_FULL, n);
            put(L_END, 1u);
            STAMP(4);
            if (lane == 0) __builtin_amdgcn_raw_buffer_store_b32(seq, rs_src, OFF_DONE + rank * 128u + pj * 4u, 0, SYS);
        } else if (wid == 1u) {
            const u32 fa = (lane & 31u) < 20u ? OFF_F1 + rel * 256u + (lane & 31u) * 8u : OFF_FP + (rel * 16u + pj) * 128u + (lane & 3u) * 8u;
            for (u32 m = 0; m != 0xF1FFFAu;) {
                const u32x2 f = __builtin_amdgcn_raw_buffer_load_b64(rs_ws, fa, 0, SYS);
                m = (u32)BALLOT(f[0] >= seq) & 0xF1FFFAu;
                LW[L_PSZ / 4 + (lane & 31u)] = f[1];
                if (expired()) m = 0xF1FFFAu;
                LW[L_SEG / 4] = m;
                __builtin_amdgcn_s_sleep(4);
            }
            STAMP(1);
            while (LW[L_P1 / 4] == 0u) { if (expired()) break; __builtin_amdgcn_s_sleep(16); }
            u32 m = 0;
            while (m != SGM) {
                m |= (u32)BALLOT(__builtin_amdgcn_raw_buffer_load_b32(rs_ws, lane < 16u && ((SGM & ~m) >> lane & 1u) ? lane * 64u : OOB,
                                                                      OFF_F3 + (rel * 16u + pj) * 1024u, SYS) == seq);
                if (expired()) m = SGM;
                LW[L_G3 / 4] = m;
                __builtin_amdgcn_s_sleep(2);
            }
            m = 0;
            while (m != 0xFFFFu) {
                const u32x4 f = LD128(rs_ws, lane < 16u ? lane * 16u : OOB, OFF_F2 + (rel * 16u + pj) * 256u, SYS);
                m = (u32)BALLOT(f[0] == seq && f[3] == seq) & 0xFFFFu;
                if (lane < 16u) *(LAS u32x2*)(L + L_SZ + pj * 128u + lane * 8u) = u32x2{f[1], f[2]};
                if (expired()) m = 0xFFFFu;
                CBAR();
                LW[L_G2 / 4] = m;
                __builtin_amdgcn_s_sleep(2);
            }
            STAMP(2);
            for (u32 i = 0; i < 4u;) {
                const u32x4 f = LD128(rs_ws, lane * 16u, OFF_F2 + rel * 4096u + i * 1024u, SYS);
                *(LAS u32x2*)(L + L_SZ + i * 512u + lane * 8u) = u32x2{f[1], f[2]};
                if (BALLOT(f[0] == seq && f[3] == seq) == ~0ull || expired()) ++i; else __builtin_amdgcn_s_sleep(16);
            }
            CBAR();
            LW[L_G2 / 4] = 0x1FFFFu;
            STAMP(3);
        } else if (wid < 10u) {
            const rsrc_t rs_ol = mk_rsrc(p.out, OUT_BYTES);
            const u32 ovoff = (lane & 7u) * NN + (((lane >> 3) & 1u) * 4u + (lane >> 4)) * 16u;
            for (u32 n = wid - (wid >= 10u - 2u * PCH ? 10u : 2u);; n += 8u) {
                const bool pre = (int)n < 0;
                bool more = true;
                const u32 k = n + 2u * PCH;
                while (pre && ((LW[L_SEG / 4] >> (20u + k)) & 1u) == 0u) __builtin_amdgcn_s_sleep(2);
                while (!pre) {
                    const u32 end = LW[L_END / 4];
                    if (LW[L_FULL / 4] > n) break;
                    if (end != 0u) { more = false; break; }
                    expired();
                    __builtin_amdgcn_s_sleep(2);
                }
                if (!more) break;
                CBAR();
                const u32 id = pre ? (k >> 1) * 96u + pj * 2u + (k & 1u) + (min(max(RFL(LW[L_PSZ / 4 + 20u + k]), 321u), E1_SLOT / 16u) << 16)
                                   : RFL(LW[L_Q / 4 + (n & 15u)]);
                const u32 sl = (n & ((int)n < 16 && (id & 1024u) == 0u ? 15u : 7u)) * E1_SLOT;
                if ((id & 1024u) == 0u) {
                u32 ch, r; bool isw;
                const u32 so = in_unit(id & 1023u, ch, r, isw);
                if (pre) {
#pragma unroll 1
                    for (u32 at = 0; at < (id >> 16) * 16u; at += 1024u)
                        if (at + lane * 16u < (id >> 16) * 16u)
                            __builtin_amdgcn_raw_ptr_buffer_load_lds(rs_ws, L + sl + at, 16, lane * 16u, so + at, 0, SYS);
                    VMWAIT(0);
                }
                const u32 hd = sl + I_HDR;
                u32x4 tab = LDSV(hd);
                const u32x4 fw = LDSV(hd + I_FLD + lane * 16u);
                const u32 h0 = RFL(tab[0]), sz = in_size(h0), top = ((h0 >> 20) & 15u) | ((h0 >> 24) & 0xF0u), cnt = (RFL(tab[1]) >> 4) & 15u;
                u32x4 o[8];
                if (sz == E1_SLOT / 16u) {
#pragma unroll
                    for (int e = 0; e < 8; ++e) o[e] = LDSV(sl + (e >> 2) * (I_HDR + 128u) + (e & 3) * 1024u + lane * 16u);
                } else {
                    u32x4 mq[4];
                    u32 ex[16], es[8], at[8], d[8][3], ne = 0;
#pragma unroll
                    for (int g = 0; g < 8; ++g) {
                        es[g] = ((fw[g >> 1] & (fw[g >> 1] << 1)) << (g & 1 ? 0 : 2)) & 0x88888888u;
                        at[g] = ne;
                        ne += (u32)__builtin_popcount(es[g]);
                    }
                    const u32 v0 = (I_HDR + I_VAR) / 8u + cnt;
                    const u32 a0 = unary_list(lane, (sl + v0 * 8u) * 8u, max(sz * 2u, v0) - v0, ne, L_PACK + wid * 2048u, I_MAX);
                    CBAR();
#pragma unroll
                    for (int g = 0; g < 8; ++g) { at[g] += a0; LIST8(d[g], at[g]); }
#pragma unroll
                    for (int e = 0; e < 4; ++e) mq[e] = LDSV(sl + e * 1024u + lane * 16u);
                    __builtin_amdgcn_sched_barrier(0);
                    tab = (tab & 0x0F0F0F0Fu) ^ u32x4{TFLIP0, TFLIP1, TFLIP0, TFLIP1};
#pragma unroll
                    for (int g = 0; g < 8; ++g) {
                        u32 c0, c1;
                        deal8(d[g], at[g], es[g], c0, c1);
                        ex[2 * g] = lut16(tab, ((fw[g >> 1] >> (2 * (g & 1))) & 0x03030303u) + c0);
                        ex[2 * g + 1] = lut16(tab, ((fw[g >> 1] >> (2 * (g & 1) + 4)) & 0x03030303u) + c1);
                    }
#pragma unroll 1
                    for (u32 k = 0; k < cnt; ++k) {
                        const u32 at = LW[(hd + I_VAR) / 4 + 2u * k] - lane * 16u, val = LW[(hd + I_VAR) / 4 + 1u + 2u * k];
#pragma unroll
                        for (int j = 0; j < 16; ++j) { ex[j] = at == (u32)j ? val : ex[j]; if ((j & 3) == 3) __builtin_amdgcn_sched_barrier(0); }
                    }
#pragma unroll
                    for (int j = 0; j < 16; ++j) {
                        const u32 e = top * 0x01010101u - ex[j], m = mq[j >> 2][j & 3];
                        const u32 r0 = PERM(e, m, 0x05010400u), r1 = PERM(e, m, 0x07030602u);
                        o[j >> 1][(2 * j) & 3] = __builtin_amdgcn_alignbit(r0, r0, 1);
                        o[j >> 1][(2 * j + 1) & 3] = __builtin_amdgcn_alignbit(r1, r1, 1);
                    }
                }
                CBAR();
                LW[L_FREE / 4 + (n & 7u)] = n + 1u;
                const u32 flag = put_unit(o, isw, rel, ch, r);
                VMWAIT(0);
                put_flag(flag, seq);
                STAMPV(26, (u64)(c1 - c0) | ((u64)((u32)__builtin_amdgcn_s_memtime() - c1) << 20) | ((u64)sz << 40));
                STAMP(pre ? 3 : 1);
                continue;
                }
                const u32 sz = id >> 16, x = (id >> 3) & 7u, ix = (id >> 6) & 15u;
                const u32 osoff = ((4u * (x >> 1) + (ix & 3u)) * 256u + (id & 3u) * 64u) * NN + src * 2048u +
                                  (4u * (x & 1u) + (ix >> 2)) * 256u + ((id >> 2) & 1u) * 128u;
                u32x4 qv[8];
                if (sz >= 512u) {
#pragma unroll
                    for (int e = 0; e < 8; ++e) qv[e] = LDSV(sl + e * 1024u + lane * 16u);
                } else {
                    u32x4 nib[4];
#pragma unroll
                    for (int e = 0; e < 4; ++e) nib[e] = LDSV(sl + R_NIB + e * 1024u + lane * 16u);
                    const u32x4 pl = LDSV(sl + R_BIT + lane * 16u);
                    const u32x4 sg = LDSV((id & 2048u ? L_SG + (id & 63u) * 1024u : sl) + lane * 16u);
                    u32 cnt = 0, at[16], d[16][3];
#pragma unroll
                    for (int o = 0; o < 16; ++o) {
                        at[o] = cnt;
                        cnt += (u32)__builtin_popcount(nib[o >> 2][o & 3] & (pl[o >> 2] << (o & 3)) & 0x88888888u);
                    }
                    const u32 a0 = unary_list(lane, (sl + R_VAR) * 8u, (max(sz, R_VAR / 16u) - R_VAR / 16u) * 2u, cnt,
                                              L_PACK + (wid - 2u) * 3072u, R_MAX);
                    CBAR();
#pragma unroll
                    for (int o = 0; o < 16; ++o) { at[o] += a0; LIST8(d[o], at[o]); }
                    __builtin_amdgcn_sched_barrier(0);
#pragma unroll
                    for (int o = 0; o < 16; ++o) {
                        const u32 nv = nib[o >> 2][o & 3], pb = pl[o >> 2] << (o & 3), sv = sg[o >> 2] << (6 - 2 * (o & 3));
                        u32 c0, c1;
                        deal8(d[o], at[o], nv & pb & 0x88888888u, c0, c1);
                        const u32 m0 = (c0 << 3) + (bfi(0x0F0F0F0Fu, nv, pb << 1) & 0x1F1F1F1Fu);
                        const u32 m1 = (c1 << 3) + (bfi(0x0F0F0F0Fu, nv >> 4, pb >> 3) & 0x1F1F1F1Fu);
                        qv[o >> 1][(2 * o) & 3] = __builtin_amdgcn_bitop3_b32(byte_signs(sv << 1), (0x80808080u - m0) ^ 0x80808080u, m0, 0xCA);
                        qv[o >> 1][(2 * o + 1) & 3] = __builtin_amdgcn_bitop3_b32(byte_signs(sv), (0x80808080u - m1) ^ 0x80808080u, m1, 0xCA);
                    }
                }
                CBAR();
                LW[L_FREE / 4 + (n & 7u)] = n + 1u;
                if (LW[L_BAD / 4] != 0u) qv[0] = u32x4{0x80808080u, 0x80808080u, 0x80808080u, 0x80808080u};
#pragma unroll
                for (int e = 0; e < 8; ++e) ST128(qv[e], rs_ol, ovoff, osoff + e * 8u * NN, AGT);
            }
            STAMP(2);
        }
    } else {
        arrive(0u, 12u);
        if (b == AUX_WG && wid == 2u) {
            u32 a = 0;
            u64* gp = (u64*)(p.ws[lane & 7u] + OFF_GAM + (lane >> 3) * 128u + rank * 8u);
            asm volatile("" : "+v"(gp));
            for (;;) {
                const u32x4 g0 = LD128(rs_ws, OFF_AMX + lane * 32u, 0, AGT);
                const u32x4 g1 = LD128(rs_ws, OFF_AMX + lane * 32u + 16u, 0, AGT);
                a = max(max(g0[0], g0[2]), max(g1[0], g1[2]));
                const bool ok = g0[1] == seq && g0[3] == seq && g1[1] == seq && g1[3] == seq;
                if (BALLOT(ok) == ~0ull) break;
                if (expired()) { a = 0xFFFFFFFFu; break; }
            }
            a = wave_max_u32(a);
            __hip_atomic_store((GAS u64*)gp, ((u64)seq << 32) | a, __ATOMIC_RELAXED, __HIP_MEMORY_SCOPE_SYSTEM);
            if (lane == 0) __builtin_amdgcn_raw_buffer_store_b32(seq, rs_ws, OFF_SEQ, 0, AGT);
            STAMP(3);
            bool give_up = false;
            for (;;) {
                const u32x4 f = LD128(rs_ws, OFF_DONE + lane * 16u, 0, SYS);
                const bool ok = (lane >> 3) == rank || (lane & 4u) != 0u || (f[0] >= seq && f[1] >= seq && f[2] >= seq && f[3] >= seq);
                if (BALLOT(ok) == ~0ull) break;
                if (expired()) { give_up = true; break; }
                __builtin_amdgcn_s_sleep(8);
            }
            if (give_up || __builtin_amdgcn_raw_buffer_load_b32(rs_ws, OFF_TMO, 0, AGT) == seq)
                if (lane == 0) p.out[rank * 2048u] = (char)-128;
            STAMP(7);
        }
    }
}

extern "C" void kernel_launch(int rank, int nranks, ncclComm_t comm, hipStream_t stream,
                              void* const* d_in, const int* in_sizes, int n_in,
                              void* d_out, int out_size, const int* in_dtypes, int out_dtype,
                              void* const* d_out_all, void* const* d_ws_all, size_t ws_bytes) {
    if (nranks != NR || (unsigned)rank >= (unsigned)NR || n_in < 2 || ws_bytes < 3ull * WS_NEED ||
        in_sizes[0] != (int)(MM * KSH) || in_sizes[1] != (int)(KSH * NN) || out_size != (int)OUT_BYTES) {
        fprintf(stderr, "kernel_launch: unexpected problem shape\n");
        return;
    }
    static bool init = false;
    if (!init) {
        HIP_CHECK(hipFuncSetAttribute((const void*)gemm_ar_q8, hipFuncAttributeMaxDynamicSharedMemorySize, LDS_BYTES));
        init = true;
    }
    Params p;
    p.x = (const char*)d_in[0];
    p.w = (const char*)d_in[1];
    for (int r = 0; r < NR; ++r) p.ws[r] = (char*)d_ws_all[(rank + r) % NR];
    p.out = (char*)d_out;
    p.rank = rank;
    hipLaunchKernelGGL(gemm_ar_q8, dim3(256), dim3(768), LDS_BYTES, stream, p);
    HIP_CHECK(hipGetLastError());
}
```

```cpp
#include <hip/hip_runtime.h>
#include <rccl/rccl.h>
#include <cstdio>
#include <cstdint>

#define HIP_CHECK(expr) do { hipError_t _e = (expr); if (_e != hipSuccess) { \
    fprintf(stderr, "HIP error %s at %s:%d\n", hipGetErrorString(_e), __FILE__, __LINE__); \
    return; } } while (0)

#define LAS __attribute__((address_space(3)))
#define GAS __attribute__((address_space(1)))
typedef unsigned int u32;
typedef unsigned long long u64;
typedef u32 u32x4 __attribute__((ext_vector_type(4)));
typedef u32 u32x2 __attribute__((ext_vector_type(2)));
typedef unsigned short u16x2 __attribute__((ext_vector_type(2)));
typedef float f32x16 __attribute__((ext_vector_type(16)));

#define WIN1 4
#define WIN2 4
#define WINS 4
#define E2S 8192u
#define SGM 0xBBBBu
#define PCH 2
#define PUSHED(ch, r) ((ch) == 0u || ((ch) == 1u && (r) >= 16u))
#define PB1 12u
#define PSTEP 7u
#define PSTEP2 11u
#define PBACK 20u
#define PLEAD 8u
#define XFIRST 0x0Cu
#define NRAW 24u
#define STEAL 1
#define VSLEEP 8
#define SMARK 64u
#define LDIV 256
#define LMIN 12
#define JIT 15u
#define SFROM 120u
#define TDELAY 100
#define WDELAY1 50
#define WDELAY2 200
#define S2 (SGM == 0xAAAAu ? 32u : 48u)
#define DIAG_NDMA 8
#define CSLEEP 1
#define POLL_SLEEP 2
#define BALANCE 1
#define LSLEEP 1
#define KPAD "s_nop 0\n"
#define FLAGW 4
#define PAHEAD 4
#define LNOP 5
#define LADAPT 0
#define CPRIO 0
#define LPRIO 0
#define PRED_PAD 20
#define FORCE_RAW1 0
#define FORCE_RAW2 0

constexpr int NR = 8;
constexpr u32 MM = 4096, NN = 16384, KSH = 512;
constexpr u32 X_BYTES = MM * KSH * 2, W_BYTES = KSH * NN * 2, OUT_BYTES = MM * NN;
constexpr u32 PIECE = 32768;
constexpr u32 STEP_BYTES = 28 * PIECE;
constexpr u32 OFF_UF = 1u << 20;
constexpr u32 OFF_F3 = OFF_UF + 64 * 112 * 128;
constexpr u32 STAGE_OFF = 2u << 20;
static_assert(OFF_F3 + 2048 * 64 <= STAGE_OFF, "flags");
constexpr u32 E1_SLOT = 8192 + 128;
constexpr u32 I_HDR = 4096, I_FLD = 16, I_VAR = 1040, I_MAX = 1900;
constexpr u32 E1X_OFF = STAGE_OFF + 64 * STEP_BYTES;
constexpr u32 E1W_OFF = E1X_OFF + 512 * E1_SLOT;
constexpr u32 E1P_OFF = E1W_OFF + 2048 * E1_SLOT;
constexpr u32 E2_OFF = E1P_OFF + 512 * E1_SLOT;
constexpr u32 E3_OFF = E2_OFF + 1024 * E2S;
constexpr u32 WS_NEED = E3_OFF + 1024 * 1024;
constexpr u32 OFF_SEQ = 0, OFF_TMO = 256;
constexpr u32 OFF_AMX = 4096;
constexpr u32 OFF_GAM = OFF_AMX + 2048;
constexpr u32 OFF_DONE = OFF_GAM + 1024;
constexpr u32 OFF_F1 = OFF_DONE + 1024;
constexpr u32 OFF_CTR = OFF_F1 + 2048;
constexpr u32 OFF_RF = OFF_CTR + 160 * 128;
constexpr u32 OFF_F2 = 65536;
constexpr u32 OFF_FP = OFF_F2 + 32768;
constexpr u32 OFF_STAMP = 131072;
static_assert(OFF_RF + 1024 <= OFF_F2 && OFF_FP + 16384 <= OFF_STAMP && OFF_STAMP + 256 * 12 * 32 * 8 <= OFF_UF, "flags");

constexpr u32 SLOT_BYTES = 32768;
constexpr u32 RING = 2048;
constexpr u32 CTRL = RING + 16 * E1_SLOT;
constexpr u32 L_FULL = CTRL, L_FREE = CTRL + 32, L_NRDY = CTRL + 64, L_AMAX = CTRL + 68, L_ACNT = CTRL + 72,
              L_BAD = CTRL + 76, L_SCALE = CTRL + 80, L_GCNT = CTRL + 88, L_GSZ = CTRL + 96, L_SCNT = CTRL + 112, L_STEPS = CTRL + 128, L_PTR = CTRL + 640;
constexpr u32 L_SEG = CTRL + 4, L_G2 = CTRL + 8, L_END = CTRL + 12, L_P1 = CTRL + 16, L_G3 = CTRL + 20, L_VICT = CTRL + 24, L_Q = CTRL + 128,
              L_PSZ = CTRL + 192, L_SG = RING + 8 * E1_SLOT;
constexpr u32 L_PACK = CTRL + 1024;
constexpr u32 L_SZ = L_PACK + 8 * 3072;
constexpr u32 LDS_BYTES = L_SZ + 2048;
constexpr u64 DEADLINE = 100000000ull;
constexpr u32 OOB = WS_NEED;
constexpr int SYS = 17, AGT = 16;
constexpr u32 CODE_BYTES = 49152;
constexpr u32 AUX_WG = 248;
constexpr u32 N1 = 768 + 112;
constexpr u32 N2 = 60;
constexpr u32 R_BIT = 1024, R_NIB = 2048, R_VAR = 6144, R_MAX = 2800;

struct Params {
    char* ws[NR];
    const char* x;
    const char* w;
    int rank;
    char* out[NR];
};

typedef __amdgpu_buffer_rsrc_t rsrc_t;
__device__ __forceinline__ rsrc_t mk_rsrc(const void* p, u32 bytes) {
    return __builtin_amdgcn_make_buffer_rsrc((void*)p, 0, bytes, 0x00020000);
}
__device__ __forceinline__ u64 now() { return __builtin_amdgcn_s_memrealtime(); }
#define VMWAIT(n) asm volatile("s_waitcnt vmcnt(%0)" :: "n"(n) : "memory")
#define CBAR() asm volatile("" ::: "memory")
#define ST128(v, rs, voff, soff, aux) do { const u32x4 d_ = (v); \
    __builtin_amdgcn_raw_buffer_store_b128(d_, rs, voff, soff, aux); \
    asm volatile("s_nop 1" :: "v"(d_) : "memory"); } while (0)
#define LD128(rs, voff, soff, aux) __builtin_amdgcn_raw_buffer_load_b128(rs, voff, soff, aux)
#define PERM(a, b, s) __builtin_amdgcn_perm(a, b, s)
#define BALLOT(c) __builtin_amdgcn_ballot_w64(c)
#define RFL(v) ((u32)__builtin_amdgcn_readfirstlane(v))
__device__ __forceinline__ u32 bfi(u32 m, u32 a, u32 b) { return (a & m) | (b & ~m); }
__device__ __forceinline__ u32 rank_in(u64 m) {
    return __builtin_amdgcn_mbcnt_hi((u32)(m >> 32), __builtin_amdgcn_mbcnt_lo((u32)m, 0u));
}

#define STAMPV(i, v) do { } while (0)
#define STAMP(i) STAMPV(i, now())
#define TK(i)
#define TKOUT()

#define DPP_STEPS(OP) \
    OP(0x111, 0xf, true) OP(0x112, 0xf, true) OP(0x114, 0xf, true) OP(0x118, 0xf, true) OP(0x142, 0xa, false) OP(0x143, 0xc, false)
__device__ __forceinline__ u32 wave_max_u32(u32 v) {
#define OP_(c, rm, bc) v = max(v, (u32)__builtin_amdgcn_update_dpp(0, (int)v, c, rm, 0xf, bc));
    DPP_STEPS(OP_)
#undef OP_
    return (u32)__builtin_amdgcn_readlane((int)v, 63);
}
__device__ __forceinline__ u32 wave_scan_u32(u32 v) {
#define OP_(c, rm, bc) v += (u32)__builtin_amdgcn_update_dpp(0, (int)v, c, rm, 0xf, bc);
    DPP_STEPS(OP_)
#undef OP_
    return v;
}
__device__ __forceinline__ u32 byte_signs(u32 a) { return PERM(a, a << 8, 0x0B090A08u); }

constexpr u32 TFLIP0 = 0xFFFFFFFFu, TFLIP1 = 0xFFFFFF00u;
__device__ __forceinline__ u32 lut16(u32x4 t, u32 i) { return PERM(t[1], t[0], i) ^ PERM(t[3], t[2], i ^ 0x08080808u); }
__device__ __forceinline__ u32 unary8(u32 l0, u32 l1, u32& code) {
    u32 p1 = l0 + l1;
    p1 += p1 << 8; p1 += p1 << 16;
    const u32 p0 = p1 - l1;
    code = 0;
#pragma unroll
    for (int k = 0; k < 4; ++k) code |= (1u << ((p0 >> (8 * k)) & 31u)) | (1u << ((p1 >> (8 * k)) & 31u));
    code >>= 1;
    return p1 >> 24;
}
__device__ __forceinline__ void put_bits(LAS char* L, u32 pos, u32 code) {
    LAS u32* a = (LAS u32*)(L + ((pos >> 5) << 2));
    __hip_atomic_fetch_or(a, code << (pos & 31u), __ATOMIC_RELAXED, __HIP_MEMORY_SCOPE_WORKGROUP);
    __hip_atomic_fetch_or(a + 1, (code >> 1) >> (~pos & 31u), __ATOMIC_RELAXED, __HIP_MEMORY_SCOPE_WORKGROUP);
}
__device__ __forceinline__ u32x2 table_of(u32 v) {
    u64 t = 0;
#pragma unroll 1
    for (u32 i = 0, n = 0; i < 8u; ++i) {
        if ((v >> i) & 1u) n += 8u; else t += 1ull << n;
    }
    return __builtin_bit_cast(u32x2, t);
}
#define LP(a) ((const LAS u32*)(size_t)(a))
template <u32 KB>
__device__ __forceinline__ u32 unary_list(u32 lane, u32 from, u32 bytes, u32 mine, u32 area) {
    const u32 sb = RFL((bytes + 63u) >> 6), at = from + lane * sb;
    const LAS u32* a = LP(at & ~3u);
    auto word = [&](u32 i) -> u32 { return __builtin_amdgcn_alignbyte(a[i + 1u], a[i], at); };
    auto ones = [&](u32 w, u32 i) -> u32 { return (u32)__builtin_popcount(w & RFL(sb >= 4u * i + 4u ? ~0u : sb > 4u * i ? ~(~0u << (sb & 3u) * 8u) : 0u)); };
    u32 w[KB], ix[4 * KB], cnt = 0;
    u32x2 t[4 * KB];
#pragma unroll
    for (u32 i = 0; i < KB; ++i) w[i] = word(i);
#pragma unroll
    for (u32 i = 0; i < KB; ++i) *(LAS u32x4*)(size_t)(area + i * 1024u + lane * 16u) = u32x4{0u, 0u, 0u, 0u};
#pragma unroll
    for (u32 e = 0; e < 4u * KB; ++e) { ix[e] = ((w[e >> 2] >> (8 * (e & 3))) & 255u) << 3; t[e] = *(const LAS u32x2*)(size_t)ix[e]; }
#pragma unroll
    for (u32 i = 0; i < KB; ++i) cnt += ones(w[i], i);
#pragma unroll 1
    for (u32 i = KB; 4u * i < sb; ++i) cnt += ones(word(i), i);
    const bool on = lane * sb < bytes;
    cnt = (on ? cnt << 16 : 0u) + mine;
    const u32 end = wave_scan_u32(cnt);
    u32 k = area - 1u + ((end - cnt) >> 16);
    auto add = [&](u32x2 v, u32 x) {
        LAS u32* d = (LAS u32*)(size_t)(k & ~3u);
        const u32 m = ~k;
        __hip_atomic_fetch_add(d, __builtin_amdgcn_alignbyte(v[0], 0u, m), __ATOMIC_RELAXED, __HIP_MEMORY_SCOPE_WORKGROUP);
        __hip_atomic_fetch_add(d + 1, __builtin_amdgcn_alignbyte(v[1], v[0], m), __ATOMIC_RELAXED, __HIP_MEMORY_SCOPE_WORKGROUP);
        __hip_atomic_fetch_add(d + 2, __builtin_amdgcn_alignbyte(0u, v[1], m), __ATOMIC_RELAXED, __HIP_MEMORY_SCOPE_WORKGROUP);
        k += (u32)__builtin_popcount(x);
    };
    asm volatile("" : "+v"(t[4 * KB - 1]), "+v"(k));
    if (on && (u32)__builtin_amdgcn_readlane((int)end, 63) < (KB * 1024u - 12u) << 16) do {
#define STEP_(e) if (e < 4u * KB) { add(t[e % (4u * KB)], ix[e % (4u * KB)]); if (sb == e + 1u) break; }
        STEP_(0u) STEP_(1u) STEP_(2u) STEP_(3u) STEP_(4u) STEP_(5u) STEP_(6u) STEP_(7u) STEP_(8u) STEP_(9u) STEP_(10u) STEP_(11u)
#undef STEP_
#pragma unroll 1
        for (u32 e = 4u * KB; e < sb; ++e) {
            const u32 x = (u32)*(const LAS unsigned char*)(size_t)(at + e) << 3;
            add(*(const LAS u32x2*)(size_t)x, x);
        }
    } while (0);
    return area + ((end - cnt) & 0xFFFFu);
}
#define LIST8(d, ad) do { const LAS u32* a_ = LP((ad) & ~3u); d[0] = a_[0]; d[1] = a_[1]; d[2] = a_[2]; } while (0)
__device__ __forceinline__ void deal8(const u32 (&d)[3], u32 ad, u32 es, u32& c0, u32& c1) {
    const u32 lo = __builtin_amdgcn_alignbyte(d[1], d[0], ad), hi = __builtin_amdgcn_alignbyte(d[2], d[1], ad);
    u32 pre = es << 1;
    asm("" : "+v"(pre));
    const u32 mk = pre - (es >> 3);
    pre += pre << 4;
    asm("" : "+v"(pre));
    pre += pre << 8;
    asm("" : "+v"(pre));
    pre += pre << 16;
    const u32 sel = __builtin_amdgcn_bitop3_b32(mk, pre, 0xCCCCCCCCu, 0xCA);
    c0 = PERM(hi, lo, sel & 0x0F0F0F0Fu);
    c1 = PERM(hi, lo, (sel >> 4) & 0x0F0F0F0Fu);
}

__global__ void __launch_bounds__(768, 3) gemm_ar_q8(Params p) {
    extern __shared__ __attribute__((aligned(16))) char smem[];
    LAS char* L = (LAS char*)smem;
    u32 lane = threadIdx.x & 63;
    const u32 wid = RFL(threadIdx.x >> 6);
    const u32 rank = (u32)p.rank & 7u;
    const u32 b = blockIdx.x;
    const u32 q = b & 7u, idx = b >> 3;
    const u32 mb = 4 * (q >> 1) + (idx & 3), nb = 4 * (q & 1) + ((idx >> 2) & 3);
    const u32 nb2 = 8u + 2u * (q & 1u) + ((idx >> 2) & 1u), fills = idx < 8u ? 256u : 128u;
    const u32 rel = 1u + (b - 128u) % 7u, pj = ((b - 128u) / 7u) & 15u;
    char* wsl = p.ws[0];
    asm volatile("" :: "s"(p.x), "s"(p.w), "s"(wsl), "s"(rank));
    const rsrc_t rs_ws = mk_rsrc(wsl, WS_NEED);
    volatile LAS u32* LW = (volatile LAS u32*)L;
#define LDSV(a) (*(const LAS u32x4*)(L + (a)))

    STAMPV(0, tstart);
    STAMP(22);
    const u32 seqv = __builtin_amdgcn_raw_buffer_load_b32(rs_ws, OFF_SEQ, 0, AGT);
    if (threadIdx.x < 32) LW[CTRL / 4 + threadIdx.x] = b >= 128u && threadIdx.x - 13u < 3u ? ~7u : 0u;
    auto barrier = [] {
        asm volatile("s_waitcnt lgkmcnt(0)" ::: "memory");
        __builtin_amdgcn_s_barrier();
        CBAR();
    };
#define PEERS(d) asm volatile("global_load_dwordx2 %0, %1, %2" : "=v"(d) : "v"((lane & 7u) * 8u), "s"(__builtin_amdgcn_kernarg_segment_ptr()) : "memory")

    auto put_unit = [&](const u32x4 (&o)[8], bool isw, u32 lk, u32 ch, u32 r) -> u32 {
        const u32 so = STAGE_OFF + (lk * 8u + ch) * STEP_BYTES + (isw ? 16u * PIECE + r * 8192u : (r >> 2) * PIECE + (r & 3u) * 4096u);
        const u32 v0 = isw ? ((lane & 48u) + (lane & 7u) * 2u + ((lane >> 3) & 1u)) * 16u : lane * 16u, v1 = v0 + (isw ? 4096u : 16384u);
#pragma unroll
        for (int e = 0; e < 8; ++e) ST128(o[e], rs_ws, (e < 4 ? v0 : v1) + (e & 3) * 1024u, so, SYS);
        return OFF_UF + ((lk * 8u + ch) * 112u + (isw ? 64u : 0u) + r) * 128u;
    };
    auto put_flag = [&](u32 flag, u32 sq) { if (lane < FLAGW) ST128((u32x4{sq, sq, sq, sq}), rs_ws, flag + lane * 16u, 0, SYS); };

    auto strip_at = [](u32 os, u32 r) -> u32 { return 32u + (os >> 1) * 4u + (r & 2u) + (os & 1u); };

    {
        u32 t = 0, tend = 0, delay = 0, step = 1536u;
        if (b < 128u) { t = wid * 128u + b; if (t < 464u) tend = t + 1u; delay = wid < 3u ? 0u : WDELAY1; }
        else if (b < 240u) {
            t = (wid < 6u ? wid - 2u : wid - 6u) * 112u + (b - 128u);
            delay = (t < 256u ? t >> 6 : 4u + (t - 256u) / 96u) * TDELAY;
            if (wid - 2u < 4u) tend = t + 1025u;
            if (wid >= 10u) tend = t + 1025u + step;
            t += t < 640u ? 1024u : 1024u + step;
        } else {
            t = 1776u + wid * 16u + (b - 240u); tend = 3008u; step = 192u;
            if (wid < 4u) __builtin_amdgcn_s_setprio(2); else if (wid < 7u) __builtin_amdgcn_s_setprio(1); else delay = WDELAY2;
        }
        bool bar = t < tend && delay == 0u;
        auto pause = [](u32 d) {
            const u32 ts = (u32)now();
            while ((u32)now() - ts < d) __builtin_amdgcn_s_sleep(2);
        };
        if (!bar) {
            if (b - 128u < 112u && wid - 6u < 4u) *(LAS u32x2*)(L + (threadIdx.x - 384u) * 8u) = table_of(threadIdx.x - 384u);
            barrier();
            if (t < tend) pause(delay);
        }
        if (b < 128u && wid >= 8u) {
            u64 pc;
            asm volatile("s_getpc_b64 %0" : "=s"(pc));
            const u32 v = __builtin_amdgcn_raw_buffer_load_b32(mk_rsrc((void*)(pc & ~255ull), CODE_BYTES), lane * 128u, ((idx * 4u + wid) % 6u) * 8192u, 0);
            asm volatile("" :: "v"(v));
        }
        for (; t < tend; t += step) {
            TK(0);
            u32 kind, lk = 0, ch, r;
            if (t < 384u) { ch = t >> 8; r = (t >> 3) & 31u; lk = (t + r) & 7u; kind = lk == 0u ? 1u : 2u; }
            else if (t < 464u) { ch = (t - 336u) >> 5; r = (t - 336u) & 31u; kind = 1u; }
            else if (t < 1280u) { ch = (t >> 6) & 3u; r = t & 63u; kind = 0u; }
            else if (t < 1664u) { ch = 4u + (t - 1280u) / 96u; r = (t - 1280u) % 96u; kind = r >> 6; r &= 63u; }
            else { ch = (0x3765421u >> ((t - 1664u) / 224u * 4u)) & 7u; r = (t - 1664u) % 224u; lk = 1u + r % 7u; r /= 7u; kind = 2u; }
            const u32 to = (rank + lk) & 7u;
            const bool strip = kind != 0u && (r | 2u) == 30u;
            const bool push = kind == 2u && PUSHED(ch, r), both = push && strip;
            char* pto = wsl;
            u32x4 o[8];
            if (kind == 0u) {
                const rsrc_t rs_x = mk_rsrc(p.x, X_BYTES);
                const u32 vx = (lane >> 2) * 1024u + (((lane & 3u) ^ ((lane >> 4) & 2u)) << 4);
#pragma unroll
                for (int i = 0; i < 4; ++i) {
                    o[i] = LD128(rs_x, vx, r * 65536u + ch * 128u + i * 16384u, 0);
                    o[4 + i] = LD128(rs_x, vx + 64u, r * 65536u + ch * 128u + i * 16384u, 0);
                }
                if (bar) barrier();
                asm volatile("" : "+v"(o[0]), "+v"(o[1]), "+v"(o[2]), "+v"(o[3]), "+v"(o[4]), "+v"(o[5]), "+v"(o[6]), "+v"(o[7]));
            } else {
                const rsrc_t rs_w = mk_rsrc(p.w, W_BYTES);
                const u32 vw = (lane >> 4) * 8u * 32768u + (lane & 15u) * 16u;
                const u32 so = (ch * 64u + (r & 2u) * 16u) * 32768u + (to * 2048u + (r >> 2) * 256u + (r & 1u) * 128u) * 2u;
                u32x4 v[8];
#pragma unroll
                for (int i = 0; i < 8; ++i) v[i] = LD128(rs_w, vw, so + i * 32768u, 0);
                pto = p.ws[push ? lk : 0u];
                asm volatile("" : "+s"(pto));
                if (bar) barrier();
#pragma unroll
                for (int e = 0; e < 4; ++e)
#pragma unroll
                    for (int k = 0; k < 4; ++k) {
                        const u32 lo = PERM(v[2 * k + 1][e >> 1], v[2 * k][e >> 1], (e & 1) ? 0x07060302u : 0x05040100u);
                        const u32 hi = PERM(v[2 * k + 1][2 + (e >> 1)], v[2 * k][2 + (e >> 1)], (e & 1) ? 0x07060302u : 0x05040100u);
                        o[e][k] = (u32)__builtin_amdgcn_update_dpp((int)lo, (int)hi, 0x128, 0xf, 0xc, false);
                        o[4 + e][k] = (u32)__builtin_amdgcn_update_dpp((int)hi, (int)lo, 0x128, 0xf, 0x3, false);
                    }
            }
            bar = false;
            u32 flag = OOB, size = E1_SLOT, sq = RFL(seqv) + 1u;
            u32 reg = L_PACK + wid * 2048u;
            asm volatile("" : "+s"(sq), "+s"(reg), "+v"(lane));
            if (kind != 2u || strip) flag = put_unit(o, kind != 0u, 0u, ch, kind == 2u ? strip_at(lk - 1u, r) : r);
            if (kind == 1u && !strip) { TK(2); VMWAIT(0); TK(3); put_flag(flag, sq); TK(4); TKOUT(); continue; }
            u64 f1;
            PEERS(f1);
            const u32 kept = kind == 0u ? E1X_OFF + (ch * 64u + r) * E1_SLOT : E1W_OFF + ((lk * 8u + ch) * 32u + r) * E1_SLOT;
            const u32 slot = push ? E1P_OFF + ((lk * 2u + ch) * 32u + r) * E1_SLOT : kept;
            const rsrc_t rs_to = mk_rsrc(pto, WS_NEED);
#define PST(v, voff, at) do { const u32x4 p_ = (v); ST128(p_, rs_to, voff, slot + (at), SYS); if (both) ST128(p_, rs_ws, voff, kept + (at), SYS); } while (0)
            *(LAS u32x4*)(L + reg + lane * 16u) = u32x4{0u, 0u, 0u, 0u};
            *(LAS u32x4*)(L + reg + 1024u + lane * 16u) = u32x4{0u, 0u, 0u, 0u};
            u32 ex[16], mx = 0;
            const bool late = push && ch != 0u;
            if (push && !late && r > PLEAD) pause((r - PLEAD) * PSTEP);
#pragma unroll
            for (int e = 0; e < 4; ++e) {
                u32x4 mq;
#pragma unroll
                for (int k = 0; k < 4; ++k) {
                    const int j = 4 * e + k;
                    const u32 d0 = o[j >> 1][(2 * j) & 3], d1 = o[j >> 1][(2 * j + 1) & 3];
                    const u32 r0 = __builtin_amdgcn_alignbit(d0, d0, 31), r1 = __builtin_amdgcn_alignbit(d1, d1, 31);
                    mq[k] = PERM(r1, r0, 0x06040200u);
                    ex[j] = PERM(r1, r0, 0x07050301u);
                    mx = __builtin_bit_cast(u32, __builtin_elementwise_max(__builtin_bit_cast(u16x2, mx),
                             __builtin_elementwise_max(__builtin_bit_cast(u16x2, r0), __builtin_bit_cast(u16x2, r1))));
                }
                if (!late) PST(mq, lane * 16u + e * 1024u, 0u);
            }
            const u32 top = wave_max_u32(max(mx >> 24, (mx >> 8) & 255u));
            u32 any = 0, cnt = 0;
#pragma unroll
            for (int j = 0; j < 16; ++j) { ex[j] = top * 0x01010101u - ex[j]; any |= ex[j]; }
            CBAR();
#pragma unroll
            for (int j = 0; j < 16; j += 2)
                __hip_atomic_fetch_add((LAS u32*)(L + reg + 256u + (((ex[j] << 4) & 0xF0u) | ((lane & 3u) << 2))), 1u, __ATOMIC_RELAXED,
                                       __HIP_MEMORY_SCOPE_WORKGROUP);
            CBAR();
            const u32x4 hist = LDSV(reg + 256u + (lane & 15u) * 16u);
            CBAR();
            if (BALLOT((any & 0xF0F0F0F0u) != 0u) != 0ull) {
#pragma unroll
                for (int j = 0; j < 16; ++j) {
                    const bool far = (ex[j] & 0xF0F0F0F0u) != 0u;
                    const u64 m = BALLOT(far);
                    if (m != 0ull) {
                        const u32 k = cnt + rank_in(m);
                        if (far && k < 14u) *(LAS u32x2*)(L + reg + I_VAR + k * 8u) = u32x2{lane * 16u + j, ex[j]};
                        cnt += (u32)__builtin_popcountll(m);
                        ex[j] &= 0x0F0F0F0Fu;
                    }
                }
            }
            const u32 key = (((hist[0] + hist[1] + hist[2] + hist[3]) << 4) | (lane & 15u)) ^ 15u;
            u32 above = 0;
#define ROT_(n) above = __builtin_amdgcn_alignbit(above, key - (u32)__builtin_amdgcn_mov_dpp((int)key, 0x120 + n, 0xf, 0xf, false), 31);
            ROT_(1) ROT_(2) ROT_(3) ROT_(4) ROT_(5) ROT_(6) ROT_(7) ROT_(8) ROT_(9) ROT_(10) ROT_(11) ROT_(12) ROT_(13) ROT_(14) ROT_(15)
#undef ROT_
            above = (u32)__builtin_popcount(above);
            u32 tb = ((max(above, 2u) - 2u) | (min(above, 3u) << 4)) << ((lane & 3u) * 8u);
            tb |= (u32)__builtin_amdgcn_mov_dpp((int)tb, 0xB1, 0xf, 0xf, false);
            tb |= (u32)__builtin_amdgcn_mov_dpp((int)tb, 0x4E, 0xf, 0xf, false);
            const u32x4 tab = u32x4{(u32)__builtin_amdgcn_readlane((int)tb, 0) ^ TFLIP0, (u32)__builtin_amdgcn_readlane((int)tb, 4) ^ TFLIP1,
                                    (u32)__builtin_amdgcn_readlane((int)tb, 8) ^ TFLIP0, (u32)__builtin_amdgcn_readlane((int)tb, 12) ^ TFLIP1};
            u32 code[8], len[8], over = 0, tot = 0;
            u32x4 fw;
#pragma unroll
            for (int g = 0; g < 8; ++g) {
                const u32 s0 = lut16(tab, ex[2 * g]), s1 = lut16(tab, ex[2 * g + 1]), h = bfi(0x0F0F0F0Fu, s0 >> 4, s1);
                fw[g >> 1] = g & 1 ? fw[g >> 1] | (h << 2) : h;
                len[g] = unary8(s0 & 0x0F0F0F0Fu, s1 & 0x0F0F0F0Fu, code[g]);
                over |= len[g];
                tot += len[g] + ((u32)__builtin_popcount(code[g]) << 16);
            }
            const u32 end = wave_scan_u32(tot), all = (u32)__builtin_amdgcn_readlane((int)end, 63);
            const u32 bytes = ((I_VAR + cnt * 8u) * 8u + (all & 0xFFFFu) + 127u) >> 7 << 4;
            if (kind != 2u) { VMWAIT(4); put_flag(flag, sq); }
            if (BALLOT(over > 31u) == 0ull && cnt < 15u && bytes <= 2032u && (all >> 16) <= I_MAX && !(FORCE_RAW1 && (r & 1u))) {
                size = I_HDR + bytes;
                *(LAS char*)(L + reg + above) = (char)((lane & 15u) | ((((bytes >> 4) | (top << 8) | (cnt << 16)) >> (above * 4u & 31u)) << 4));
                *(LAS u32x4*)(L + reg + I_FLD + lane * 16u) = fw;
                u32 pos = ((end - tot) & 0xFFFFu) + (reg + I_VAR + cnt * 8u) * 8u;
#pragma unroll
                for (int g = 0; g < 8; ++g) { put_bits(L, pos, code[g]); pos += len[g]; }
                CBAR();
                if (late) {
                    pause((r - (r < 25u ? PB1 : PBACK)) * PSTEP2);
#pragma unroll
                    for (int e = 0; e < 4; ++e) {
                        u32x4 mq;
#pragma unroll
                        for (int k = 0; k < 4; ++k) {
                            const u32 d0 = o[2 * e + (k >> 1)][(2 * k) & 3], d1 = o[2 * e + (k >> 1)][(2 * k + 1) & 3];
                            mq[k] = PERM(__builtin_amdgcn_alignbit(d1, d1, 31), __builtin_amdgcn_alignbit(d0, d0, 31), 0x06040200u);
                        }
                        PST(mq, lane * 16u + e * 1024u, 0u);
                    }
                }
                PST(LDSV(reg + lane * 16u), lane * 16u, I_HDR);
                PST(LDSV(reg + 1024u + lane * 16u), (lane & 60u) * 16u + 1024u < bytes ? lane * 16u + 1024u : OOB, I_HDR);
                CBAR();
            } else {
                PST((u32x4{~0u, ~0u, ~0u, ~0u}), lane == 0u ? 0u : OOB, I_HDR);
#pragma unroll
                for (int e = 0; e < 8; ++e) PST(o[e], lane * 16u + (e & 3) * 1024u, (e >> 2) * (I_HDR + 128u));
            }
            u32 old = 0;
            TK(2);
            VMWAIT(0);
            TK(3);
            asm volatile("" : "+v"(f1));
            if (kind == 2u) put_flag(flag, sq);
            if (push) {
                if (lane == 0) __builtin_amdgcn_raw_buffer_store_b64(u32x2{sq, (size + 15u) >> 4}, rs_to, 0u, OFF_FP + (lk * 16u + (r & 15u)) * 128u + (ch + (r >> 4)) * 8u, SYS);
                __builtin_amdgcn_s_setprio(0);
                if (!strip) { TK(4); TKOUT(); continue; }
            }
            if (kind == 0u && ch == 0u) {
                const u32 j = (r - 8u * lane) & 63u;
                if (lane - 1u < 7u)
                    __hip_atomic_store((GAS u64*)(f1 + OFF_FP + (lane * 16u + (j & 15u)) * 128u + 32u + (j >> 4) * 8u), (u64)sq, __ATOMIC_RELAXED, __HIP_MEMORY_SCOPE_SYSTEM);
                TK(4); TKOUT();
                continue;
            }
            if (lane == 0 ? kind != 1u && !push : (lane == 1u && strip))
                old = 1u + __hip_atomic_fetch_add((u32*)(wsl + OFF_CTR + (lane == 0 ? lk * 8u + ch : 80u) * 128u), 1u, __ATOMIC_RELAXED,
                                                  __HIP_MEMORY_SCOPE_AGENT);
            f1 += OFF_F1 + lane * 256u;
            const u64 fv = sq;
            if (RFL(old) == sq * (kind == 0u ? 64u : ch == 1u ? 16u : 32u) && lane - 1u < 7u && (kind == 0u || lane == lk))
                __hip_atomic_store((GAS u64*)f1 + ch * 2u + (kind == 0u), fv, __ATOMIC_RELAXED, __HIP_MEMORY_SCOPE_SYSTEM);
            if ((u32)__builtin_amdgcn_readlane((int)old, 1) == sq * 128u && lane - 1u < 7u)
                __hip_atomic_store((GAS u64*)f1 + 16, fv, __ATOMIC_RELAXED, __HIP_MEMORY_SCOPE_SYSTEM);
            TKOUT();
        }
    }
    const u32 seq = RFL(seqv) + 1u;
    lane = threadIdx.x & 63;
    STAMP(10);

    u32 spins = 0;
    u64 t0 = 0;
    auto expired = [&]() -> bool {
        if ((++spins & 63u) != 0u) return false;
        if (LW[L_BAD / 4] != 0u) return true;
        const u64 t = now();
        if (t0 == 0) t0 = t;
        if (t - t0 > DEADLINE || __builtin_amdgcn_raw_buffer_load_b32(rs_ws, OFF_TMO, 0, AGT) == seq) {
            u32 sq = seq;
            asm volatile("" : "+s"(sq));
            __builtin_amdgcn_raw_buffer_store_b32(sq, rs_ws, OFF_TMO, 0, AGT);
            LW[L_BAD / 4] = 1u;
            return true;
        }
        return false;
    };

    auto arrive = [&](u32 wm, u32 n) {
        if (lane == 0) {
            __hip_atomic_fetch_max((LAS u32*)(L + L_AMAX), wm, __ATOMIC_RELAXED, __HIP_MEMORY_SCOPE_WORKGROUP);
            const u32 old = __hip_atomic_fetch_add((LAS u32*)(L + L_ACNT), 1u, __ATOMIC_RELAXED, __HIP_MEMORY_SCOPE_WORKGROUP);
            if (old == n - 1u) {
                u32 a = LW[L_AMAX / 4];
                if (LW[L_BAD / 4] != 0u) a = 0xFFFFFFFFu;
                __builtin_amdgcn_raw_buffer_store_b64(u32x2{a, seq}, rs_ws, OFF_AMX + b * 8u, 0, AGT);
            }
        }
    };

    if (b < 128u && wid < 8u) {
        u32 c = lane & 15u, g = lane >> 4;
        const u32 mw = wid & 3u, nh = wid >> 2;
        for (u32 f0 = 0; f0 < fills; f0 += 128u) {
        f32x16 acc[8];
        u32 xa = mw * 4096u + (c * 4u + (g ^ ((c >> 2) & 2u))) * 16u, wa = 16384u + nh * 8192u + lane * 16u;
        const u32 ca = lane < 32u ? L_FULL + ((lane - 1u) & 3u) * 4u : lane < 48u ? L_FREE + (wid ^ 4u) * 4u : L_FREE + wid * 4u - 768u;
        u32 f2 = f0 + 1u, spin = 0, tv;
#define S0_(x) #x
#define S_(x) S0_(x)
#define R4(r) "v[" S_(r) ":" S_(r) "+3]"
#define MM_(j, m, w, x) "v_mfma_f32_16x16x32_bf16 " R4(16*j+4*m) ", " R4(w) ", " R4(x) ", " R4(16*j+4*m) "\n"
#define MM4(j, w, x0) MM_(j, 0, w, x0) MM_(j, 1, w, 132) MM_(j, 2, w, 136) MM_(j, 3, w, 140)
#define RW(r, o) "ds_read_b128 " R4(r) ", %[wa] offset:" S_(o) "\n"
#define RX(r, o) "ds_read_b128 " R4(r) ", %[xa] offset:" S_(o) "\n"
#define LG(n) "s_waitcnt lgkmcnt(" S_(n) ")\n" KPAD
#define KWAIT(l) \
    "8" l ":\n s_cmp_eq_u32 %[f2], %[last]\n s_cbranch_scc1 9" l "b\n ds_read_b32 %[tv], %[ca]\n s_waitcnt lgkmcnt(0)\n" \
    "v_cmp_le_u32 vcc, %[f2], %[tv]\n s_nop 4\n s_bitcmp1_b32 vcc_lo, %[f2]\n s_cbranch_scc1 9" l "b\n s_sleep 1\n" \
    "s_add_u32 %[spin], %[spin], 1\n s_cmp_lt_u32 %[spin], 0x1000000\n s_cbranch_scc1 8" l "b\n s_branch 9" l "b\n"
#define KFILL(l, x0, y0, cur, nxt, TOGGLE) \
    LG(5) MM_(0, 0, 144, x0) MM_(1, 0, 148, x0) LG(4) MM_(0, 1, 144, 132) MM_(1, 1, 148, 132) \
    LG(3) MM_(0, 2, 144, 136) MM_(1, 2, 148, 136) LG(2) MM_(0, 3, 144, 140) RW(144, cur+4096) MM_(1, 3, 148, 140) RW(148, cur+5120) \
    "ds_read_b32 %[tv], %[ca]\n" \
    "s_waitcnt lgkmcnt(4)\n s_add_u32 %[f2], %[f2], 1\n" MM4(2, 152, x0) RW(152, cur+6144) \
    LG(4) MM4(3, 156, x0) RW(156, cur+7168) \
    "s_waitcnt lgkmcnt(2)\n v_cmp_le_u32 vcc, %[f2], %[tv]\n" MM_(4, 0, 144, x0) \
    "v_mov_b32_e64 %[tv], %[f2]\n" MM_(4, 1, 144, 132) \
    "ds_write_b32 %[ca], %[tv] offset:768\n"         \
    MM_(4, 2, 144, 136) TOGGLE MM_(4, 3, 144, 140) \
      \
    "s_setprio " S_(CPRIO) "\n s_bitcmp0_b32 vcc_hi, 0\n s_cbranch_scc1 7" l "f\n s_setprio " S_(CPRIO+BALANCE) "\n7" l ":\n" \
    "s_bitcmp0_b32 vcc_lo, %[f2]\n s_cbranch_scc1 8" l "f\n9" l ":\n" \
    RW(144, nxt) RX(y0, nxt) MM4(5, 148, x0) RW(148, nxt+1024) \
    LG(4) MM_(6, 1, 152, 132) MM_(7, 1, 156, 132) RX(132, nxt+1024) MM_(6, 2, 152, 136) MM_(7, 2, 156, 136) RX(136, nxt+2048) \
    MM_(6, 3, 152, 140) MM_(7, 3, 156, 140) RX(140, nxt+3072) MM_(6, 0, 152, x0) RW(152, nxt+2048) MM_(7, 0, 156, x0) RW(156, nxt+3072)
        asm volatile(
            ".set i_, 0\n .rept 64\n v_mov_b64 v[i_:i_+1], 0\n .set i_, i_+2\n .endr\n"
            "s_branch 80f\n90:\n"
            RW(144, 0) RX(128, 0) RW(148, 1024) RX(132, 1024) RX(136, 2048) RX(140, 3072) RW(152, 2048) RW(156, 3072)
            ".p2align 3\n1:\n"
            KFILL("1", 128, 160, 0, 32768, "")
            KFILL("2", 160, 128, 32768, 0, "v_xor_b32 %[wa], 0x10000, %[wa]\n v_xor_b32 %[xa], 0x10000, %[xa]\n")
            "s_cmp_lt_u32 %[f2], %[last]\n s_cbranch_scc1 1b\n s_branch 2f\n"
            KWAIT("0") KWAIT("1") KWAIT("2")
            "2:\n s_waitcnt lgkmcnt(0)\n s_nop 7\n s_nop 7\n"
            : "={v[0:15]}"(acc[0]), "={v[16:31]}"(acc[1]), "={v[32:47]}"(acc[2]), "={v[48:63]}"(acc[3]), "={v[64:79]}"(acc[4]),
              "={v[80:95]}"(acc[5]), "={v[96:111]}"(acc[6]), "={v[112:127]}"(acc[7]),
              [xa] "+{v164}"(xa), [wa] "+{v165}"(wa), [tv] "={v166}"(tv), [f2] "+s"(f2), [spin] "+s"(spin)
            : [ca] "{v167}"(ca), [last] "s"(f0 + 129u)
            : "v128", "v129", "v130", "v131", "v132", "v133", "v134", "v135", "v136", "v137", "v138", "v139", "v140", "v141", "v142",
              "v143", "v144", "v145", "v146", "v147", "v148", "v149", "v150", "v151", "v152", "v153", "v154", "v155", "v156", "v157",
              "v158", "v159", "v160", "v161", "v162", "v163", "vcc", "scc", "memory");
        if (spin >= 0x1000000u) LW[L_BAD / 4] = 1u;
        asm volatile("v_mbcnt_lo_u32_b32 %0, -1, 0\n v_mbcnt_hi_u32_b32 %0, -1, %0" : "=v"(lane));
        c = lane & 15u; g = lane >> 4;
        STAMPV(f0 == 0u ? 14 : 15, (u64)spin << 24);
        const u32 reg = L_PACK + wid * 2048u, slot = E2_OFF + (b * 8u + wid) * E2S;
        auto post = [&](u32 cnt, u32 at) {
            u32 old = 0;
            VMWAIT(0);
            if (lane == 0) old = __hip_atomic_fetch_add((LAS u32*)(L + cnt + nh * 4u), 1u, __ATOMIC_RELAXED, __HIP_MEMORY_SCOPE_WORKGROUP);
            if (RFL(old) == 3u && lane - 1u < 7u) {
                const u32x4 fv = u32x4{seq, LW[L_GSZ / 4 + nh * 2u], LW[L_GSZ / 4 + nh * 2u + 1u], seq};
                u32x2 fp = *(volatile LAS u32x2*)(L + L_PTR + lane * 8u);
                asm volatile("v_add_co_u32 %0, vcc, %0, %2\n v_addc_co_u32 %1, vcc, 0, %1, vcc" : "+v"(fp[0]), "+v"(fp[1]) : "v"(at) : "vcc");
                asm volatile("global_store_dwordx4 %0, %1, off sc0 sc1\n s_nop 1" :: "v"(fp), "v"(fv) : "memory");
            }
        };
        if (f0 == 0u) {
            STAMP(2);
            STAMPV(9, __builtin_amdgcn_s_memtime());
            u32x4 sg;
#pragma unroll
            for (int m = 0; m < 4; ++m) {
                u32 d = 0;
#pragma unroll
                for (int i = 31; i >= 0; --i) d = __builtin_amdgcn_alignbit(d, __float_as_uint(acc[(i & 4) + (i >> 3)][4 * m + (i & 3)]), 31);
                const u32 o = (u32)__builtin_amdgcn_mov_dpp((int)d, 0x128, 0xf, 0xf, false);
                sg[m] = bfi(c < 8u ? 0x0F0F0F0Fu : 0xF0F0F0F0u, d, __builtin_amdgcn_alignbit(o, o, c < 8u ? 28u : 4u));
            }
            ST128(sg, rs_ws, lane * 16u, (SGM >> (q * 2u + nh)) & 1u ? E3_OFF + (b * 8u + wid) * 1024u : slot, AGT);
            float mx = 0.f;
#pragma unroll
            for (int j = 0; j < 8; ++j)
#pragma unroll
                for (int m = 0; m < 4; ++m) {
                    mx = fmaxf(fmaxf(mx, fabsf(acc[j][4 * m])), fabsf(acc[j][4 * m + 1]));
                    mx = fmaxf(fmaxf(mx, fabsf(acc[j][4 * m + 2])), fabsf(acc[j][4 * m + 3]));
                }
            arrive(wave_max_u32(__float_as_uint(mx)), 8u);
            STAMP(3);
            if (wid >= 4u) __builtin_amdgcn_s_setprio(2); else __builtin_amdgcn_s_setprio(0);
#pragma unroll
            for (int k = 0; k < 2; ++k) *(LAS u32x4*)(L + reg + k * 1024u + lane * 16u) = u32x4{0u, 0u, 0u, 0u};
            if ((SGM >> (q * 2u + nh)) & 1u) post(L_SCNT, OFF_F3 + ((lane * 16u + idx) * 16u + q * 2u + nh) * 64u);
            STAMP(23);
            while (LW[L_SCALE / 4] != 1u) { if (expired()) break; __builtin_amdgcn_s_sleep(CSLEEP); }
            CBAR();
        } else STAMP(13);
        const u32 gmax = LW[L_SCALE / 4 + 1];
        const bool bad = (gmax == 0xFFFFFFFFu) || (LW[L_BAD / 4] != 0u);
        const float amax = __uint_as_float(gmax);
        const float inv = amax > 0.f ? 127.0f / amax : 0.f;
        if (f0 == 0u) STAMP(4);
        const float MAGIC = 12582912.0f;
        const float qlo = bad ? MAGIC - 128.0f : MAGIC - 127.0f, qhi = bad ? MAGIC - 128.0f : MAGIC + 127.0f;
        const u32 ovoff = (c & 7u) * NN + ((c >> 3) * 4u + g) * 16u;
        const u32 os = (nb2 - 8u) * 2u + nh;
        const u32 osoff = (mb * 256u + mw * 64u) * NN + (f0 == 0u ? rank * 2048u + nb * 256u + nh * 128u : ((rank + 1u + os) & 7u) * 2048u + 1792u);
        u32x4 qv[8];
#pragma unroll
        for (int m = 0; m < 4; ++m) {
#pragma unroll
            for (int r = 0; r < 4; ++r) {
                u32 o[2];
#pragma unroll
                for (int h = 0; h < 2; ++h) {
                    u32 t[4];
#pragma unroll
                    for (int jj = 0; jj < 4; ++jj)
                        t[jj] = __float_as_uint(__builtin_amdgcn_fmed3f(__builtin_fmaf(acc[4 * h + jj][4 * m + r], inv, MAGIC), qlo, qhi));
                    o[h] = PERM(t[1], t[0], 0x0c0c0400u) | PERM(t[3], t[2], 0x04000c0cu);
                }
                qv[2 * m][r] = (u32)__builtin_amdgcn_update_dpp((int)o[0], (int)o[1], 0x128, 0xf, 0xc, false);
                qv[2 * m + 1][r] = (u32)__builtin_amdgcn_update_dpp((int)o[1], (int)o[0], 0x128, 0xf, 0x3, false);
            }
        }
        if (f0 == 0u) {
        STAMP(7);
        u32 size = nb == 7u && nh == 0u ? 0u : 8192u;
        if (size != 0u && !bad && !(FORCE_RAW2 && (wid & 1u))) {
            u32x4 nib[4], pl = u32x4{0u, 0u, 0u, 0u};
            u32 code[16], len[16], over = 0, tot = 0;
#pragma unroll
            for (int o = 0; o < 16; ++o) {
                u32 y[2], l[2];
#pragma unroll
                for (int e = 0; e < 2; ++e) {
                    const u32 v = qv[o >> 1][(2 * o + e) & 3], s = byte_signs(v);
                    y[e] = (v ^ s) + ((s & 0x01010101u) | 0x70707070u);
                    l[e] = (y[e] & byte_signs(y[e]) & 0x78787878u) >> 3;
                }
                const u32 h = bfi(0x0F0F0F0Fu, y[0] >> 4, y[1]);
                nib[o >> 2][o & 3] = bfi(0x0F0F0F0Fu, y[0], y[1] << 4) | (((h & 0x77777777u) + 0x77777777u) & h & 0x88888888u);
                pl[o >> 2] = bfi(0x88888888u >> (o & 3), h >> (o & 3), pl[o >> 2]);
                len[o] = unary8(l[0], l[1], code[o]);
                over |= len[o];
                tot += len[o] + ((u32)__builtin_popcount(code[o]) << 16);
            }
            const u32 end = wave_scan_u32(tot);
            const u32 all = (u32)__builtin_amdgcn_readlane((int)end, 63), bits = all & 0xFFFFu;
            if (BALLOT(over > 31u) == 0ull && bits <= (8192u - 64u - R_VAR) * 8u && (all >> 16) <= R_MAX) {
                size = (R_VAR + ((bits + 127u) >> 7) * 16u);
                ST128(pl, rs_ws, lane * 16u, slot + R_BIT, AGT);
#pragma unroll
                for (int e = 0; e < 4; ++e) ST128(nib[e], rs_ws, lane * 16u + e * 1024u, slot + R_NIB, AGT);
                u32 pos = ((end - tot) & 0xFFFFu) + reg * 8u;
#pragma unroll
                for (int o = 0; o < 16; ++o) { put_bits(L, pos, code[o]); pos += len[o]; }
                CBAR();
#pragma unroll
                for (int k = 0; k < 2; ++k) {
                    const u32 off = R_VAR + k * 1024u + lane * 16u;
                    if (R_VAR + k * 1024u < size) ST128(LDSV(reg + k * 1024u + lane * 16u), rs_ws, off < size ? off : OOB, slot, AGT);
                }
            }
        }
        if (size == 8192u) {
#pragma unroll
            for (int e = 0; e < 8; ++e) ST128(qv[e], rs_ws, lane * 16u, slot + e * 1024u, AGT);
        }
        STAMP(11);
        if (lane == 0) *(volatile LAS unsigned short*)(L + L_GSZ + wid * 2u) = (unsigned short)(size >> 4);
        }
        const bool first = f0 == 0u && (nh != 0u || (q & 1u) == 0u);
        if (first) { post(L_GCNT, OFF_F2 + ((lane * 16u + idx) * 16u + q * 2u + nh) * 16u); STAMP(5); __builtin_amdgcn_s_setprio(0); }
        const rsrc_t rs_ol = mk_rsrc(p.out[0], f0 == 0u || os < 7u ? OUT_BYTES : 0u);
#pragma unroll
        for (int e = 0; e < 8; ++e) ST128(qv[e], rs_ol, ovoff, osoff + e * 8u * NN, AGT);
        if (f0 == 0u && !first) { post(L_GCNT, OFF_F2 + ((lane * 16u + idx) * 16u + q * 2u + nh) * 16u); STAMP(5); }
        }
    } else if (b < 128u && wid == 8u) {
        __builtin_amdgcn_s_setprio(LPRIO);
        *(volatile LAS u64*)(L + L_PTR + (lane & 7u) * 8u) = (u64)p.ws[lane & 7u];
        for (u32 n = 0; n < fills / 2u;) {
            const u32 wn = n == 0u ? nb : nb2, end = n + 64u;
            const bool hole = wn == 11u && (lane & 5u) == 5u;
            u32 step = lane & 56u, va = OFF_UF + (step * 112u + (lane & 4u ? 64u + wn * 4u : mb * 4u) + (lane & 3u)) * 128u;
            bool seen = hole;
            while (n != end) {
                while (2u * n > LW[L_FREE / 4] + PAHEAD) { if (expired()) break; __builtin_amdgcn_s_sleep(8); }
                seen |= __builtin_amdgcn_raw_buffer_load_b32(rs_ws, seen ? OOB : va, 0, SYS) >= seq;
                u64 avail = BALLOT(seen);
                avail &= avail >> 1; avail &= avail >> 2; avail &= avail >> 4;
                avail &= 0x0101010101010101ull;
                if (avail == 0) {
                    if (!expired()) { __builtin_amdgcn_s_sleep(POLL_SLEEP); continue; }
                    avail = BALLOT(va != OOB) & 0x0101010101010101ull;
                }
                if ((avail >> lane) & 1ull) LW[L_STEPS / 4 + n + rank_in(avail)] = step;
                n += (u32)__builtin_popcountll(avail);
                LW[L_NRDY / 4] = n;
                if ((avail >> (lane & 56u)) & 1ull) {
                    seen = hole;
                    va = (++step & 7u) ? va + 112u * 128u : OOB;
                }
            }
            if (n != 64u) break;
            STAMP(1);
            while (LW[L_FREE / 4] < 128u) { if (expired()) break; __builtin_amdgcn_s_sleep(8); }
            u32 a = 0;
            for (;;) {
                const u32x2 gm = __builtin_amdgcn_raw_buffer_load_b64(rs_ws, OFF_GAM + q * 128u + (lane & 7u) * 8u, 0, SYS);
                a = gm[0];
                if (BALLOT(gm[1] == seq) == ~0ull) break;
                if (expired()) { a = 0xFFFFFFFFu; break; }
                __builtin_amdgcn_s_sleep(1);
            }
            a = wave_max_u32(a);
            LW[L_SCALE / 4 + 1] = a;
            LW[L_SCALE / 4] = 1u;
            STAMP(2);
            while (LW[L_GCNT / 4] + LW[L_GCNT / 4 + 1] < 8u) { if (expired()) break; __builtin_amdgcn_s_sleep(8); }
        }
    } else if (b < 128u) {
#define CLK(c)
        __builtin_amdgcn_s_setprio(LPRIO);
        for (u32 f = wid - 9u; f < fills; f += 3u) {
            CLK(c0);
            while (LW[L_NRDY / 4] <= (f >> 1)) { if (expired()) break; __builtin_amdgcn_s_sleep(1); }
            CBAR();
            const u32 step = RFL(LW[L_STEPS / 4 + (f >> 1)]) & 63u;
            CLK(c1);
            bool ahead = false;
            while (BALLOT(LW[L_FREE / 4 + (lane & 7u)] + 2u >= f) != ~0ull && f >= 4u) {
                ahead = true;
                if (expired()) break;
                __builtin_amdgcn_s_sleep(LSLEEP);
            }
            CBAR();
            const u32 sbase = STAGE_OFF + step * STEP_BYTES + (f & 1u) * 16384u;
            LAS char* ldst = L + (f & 3u) * SLOT_BYTES;
            CLK(c2);
            auto fill = [&](auto pace) {
#pragma unroll
                for (int i = 0; i < DIAG_NDMA; ++i) {
                    const u32 so = sbase + (i < 4 ? mb : 16u + (f < 128u ? nb : nb2)) * PIECE + (i & 3) * 4096u;
                    __builtin_amdgcn_raw_ptr_buffer_load_lds(rs_ws, ldst + i * 4096, 16, lane * 16u, so, 0, SYS); pace();
                    __builtin_amdgcn_raw_ptr_buffer_load_lds(rs_ws, ldst + i * 4096, 16, lane * 16u, so, 1024, SYS); pace();
                    __builtin_amdgcn_raw_ptr_buffer_load_lds(rs_ws, ldst + i * 4096, 16, lane * 16u, so, 2048, SYS); pace();
                    __builtin_amdgcn_raw_ptr_buffer_load_lds(rs_ws, ldst + i * 4096, 16, lane * 16u, so, 3072, SYS); pace();
                }
            };
            if (ahead || !LADAPT) fill([] { asm volatile("s_nop %0" :: "n"(LNOP) : "memory"); });
            else fill([] { });
            CBAR();
            CLK(c3);
            VMWAIT(0);
            CLK(c4);
            LW[L_FULL / 4 + (f & 3u)] = f + 1u;
            if (f - 125u < 3u) STAMP(1);
        }
    } else if (b < 240u) {
        arrive(0u, 12u);
        const u32 src = (rank - rel) & 7u;
        const rsrc_t rs_src = mk_rsrc(p.ws[(8u - rel) & 7u], WS_NEED);
        auto in_unit = [&](u32 u, u32& ch, u32& r, bool& isw) -> u32 {
            if (u >= 768u) {
                ch = (u - 768u) / 14u; r = (u - 768u) % 14u; isw = true;
                const u32 os = r >> 1, rs = 28u + (r & 1u) * 2u;
                r = strip_at(os, rs);
                return E1W_OFF + ((((rel + 1u + os) & 7u) * 8u + ch) * 32u + rs) * E1_SLOT;
            }
            ch = u / 96u; r = u % 96u;
            if ((XFIRST >> ch) & 1u) r = r < 64u ? r + 32u : r - 64u;
            isw = r < 32u;
            if (!isw) r = (r - 32u + 8u * rel) & 63u;
            return !isw ? E1X_OFF + (ch * 64u + r) * E1_SLOT : PUSHED(ch, r) ? E1P_OFF + ((rel * 2u + ch) * 32u + r) * E1_SLOT
                                                                         : E1W_OFF + ((rel * 8u + ch) * 32u + r) * E1_SLOT;
        };
        auto out_unit = [&](u32 o) -> u32 {
            const u32 k = o >> 4, j = o - 768u;
            if (k >= 48u) return j % 12u * 64u + (j / 48u) * 16u + 8u + j / 12u % 4u;
            return (o & 15u) * 64u + (k < 32u ? (k >> 2) * 8u + 4u : ((k >> 2) & 3u) * 16u) + (k & 3u);
        };
        auto out_at = [&](u32 t, u32 lk) -> u32 {
            const u32 x = (t >> 3) & 7u, ix = (t >> 6) & 15u;
            return ((4u * (x >> 1) + (ix & 3u)) * 256u + (t & 3u) * 64u) * NN + ((rank - lk) & 7u) * 2048u + (4u * (x & 1u) + (ix >> 2)) * 256u + ((t >> 2) & 1u) * 128u;
        };
        const u32 rows = (lane >> 3) * NN + (lane & 7u) * 16u, base = seq << 12;
        auto in_size = [](u32 h) -> u32 {
            h = ((h >> 4) & 15u) | ((h >> 8) & 0xF0u);
            return h > 127u ? E1_SLOT / 16u : I_HDR / 16u + h;
        };
        if (wid == 0u) {
            __builtin_amdgcn_s_setprio(3);
            auto ctl = [&](u32 a) -> u32 {
                u32 v;
                asm volatile("ds_read_b32 %0, %1\n s_waitcnt lgkmcnt(0)" : "=v"(v) : "v"(a) : "memory");
                return v;
            };
            auto put = [&](u32 a, u32 n) { asm volatile("ds_write_b32 %0, %1" :: "v"(a), "v"(n) : "memory"); };
            const u32 ca = L_FULL + (lane & 15u) * 4u;
            const u32 vone = lane == 0u ? 0u : OOB;
            u32 n = 0, w6;
            auto wait = [&](int word, u32 bit) {
                auto ok = [&](u32 w) -> bool {
                    w6 = (u32)__builtin_amdgcn_readlane((int)w, 6);
                    return (((u32)__builtin_amdgcn_readlane((int)w, word) >> bit) & 1u) != 0u &&
                           (u32)__builtin_amdgcn_readlane((int)w, 8 + (n & 7u)) + (n < (word == 1 ? 16u : 8u) ? 15u : 7u) >= n;
                };
                if (!ok(ctl(ca))) {
                    VMWAIT(0);
                    put(L_FULL, n);
                    while (!ok(ctl(ca))) { if (expired()) break; __builtin_amdgcn_s_sleep(1); }
                }
            };
#define CLAIM(cl, so) asm volatile("v_mov_b32 %0, 1\n s_nop 4\n buffer_atomic_add %0, %1, %2, %3 offen sc0 sc1" \
                                   : "=&v"(cl) : "v"(vone), "s"(rs_ws), "s"(so) : "memory")
#define CLAIMED(d, cl) asm volatile("v_readfirstlane_b32 %0, %1" : "=s"(d) : "v"(cl) : "memory")
            for (u32 u = 32u + pj; u < N1; ++n) {
                u32 ch, r, cl; bool isw;
                const u32 so = in_unit(u, ch, r, isw);
                if (u >= 768u) put(L_P1, 1u);
                const u32 seg = u >= 768u ? 16u : n < 4u ? 24u + n : ch * 2u + !isw;
                wait(1, seg);
                if (n == 0) STAMP(1);
                LAS char* ld = L + RING + (n < 16u ? n : n & 7u) * E1_SLOT;
                u32 sz = 0;
                put(L_Q + (n & 15u) * 4u, u);
                CLAIM(cl, n < 3u ? OOB : OFF_CTR + (64u + rel) * 128u);
                static_assert(WIN1 <= 4, "the header, read first, has landed behind the fifth read");
#pragma unroll
                for (int i = 0; i < 9; ++i) {
                    if (i == 5) sz = in_size(RFL(ctl(RING + (n < 16u ? n : n & 7u) * E1_SLOT + I_HDR)));
                    if (i >= 6 && i * 64u >= sz) break;
                    VMWAIT(WIN1 - (i < WIN1 - 1 ? 0 : 1));
                    if (i == WIN1 - 1) put(L_FULL, n);
                    const int k = i > 4 ? i : (i + 4) % 5;
                    if (i < 5 || i * 64u + lane < sz)
                        __builtin_amdgcn_raw_ptr_buffer_load_lds(rs_src, ld + k * 1024, 16, lane * 16u, so + k * 1024u, 0, SYS);
                }
                u32 c;
                CLAIMED(c, cl);
                c -= (seq - 1u) * (N1 - 96u);
                u = n < 3u ? u + 16u : c + (c < 16u ? 96u : 112u);
            }
            put(L_P1, 1u);
            STAMP(2);
            for (u32 got = 0, i = 0; got != SGM;) {
                const u32 w = ctl(ca);
                const u32 m = (u32)__builtin_amdgcn_readlane((int)w, 5) & ~got;
                if (m == 0u || (n < 24u && (BALLOT((int)w + 8 > (int)n) >> 8 & 255u) != 255u)) {
                    VMWAIT(0);
                    put(L_FULL, n);
                    if (expired()) break;
                    __builtin_amdgcn_s_sleep(1);
                    continue;
                }
                if (got == 0u) STAMP(23);
                const u32 t = (u32)__builtin_ctz(m);
                got |= 1u << t;
#pragma unroll
                for (int e = 0; e < 4; ++e) {
                    VMWAIT(WINS - 1);
                    if (++i == WINS) { put(L_FULL, n); put(L_P1, 2u); }
                    __builtin_amdgcn_raw_ptr_buffer_load_lds(rs_src, L + L_SG + (t * 4u + e) * 1024u, 16, lane * 16u,
                                                             E3_OFF + (pj * 64u + t * 4u + e) * 1024u, 0, SYS);
                }
            }
            STAMP(24);
            put(L_FULL, n);
            put(L_P1, 2u);
            while ((BALLOT((int)ctl(ca) + 8 > (int)n) >> 8 & 255u) != 255u) if (expired()) break;
            static_assert(S2 == 48u && N2 == 60u, "claimed units");
            const char* wsrc = p.ws[(8u - rel) & 7u];
            for (u32 o = pj, cur = rel, vict = 0; o < 960u; ++n) {
                const u32 j = o + 24u * cur - (o + 24u * cur < 960u ? 0u : 192u), t = out_unit(o < 768u ? o : j);
                u32 cl, c;
                wait(2, o < 768u ? (t >> 2) & 15u : 16u);
                const bool dyn = o >= 752u, raw = o + ((int)w6 >> 8) >= 960u - NRAW || cur != rel;
                if (o == pj) STAMP(3);
                if (o >= 768u + SFROM) put(L_P1, 3u);
                u32 sz = raw ? 512u : min(RFL(ctl(L_SZ + (t >> 1) * 4u)) >> ((t & 1u) * 16u) & 0xFFFFu, 512u);
                const u32 skip = sz < 512u && o < 768u ? 1024u : 0u;
                put(L_Q + (n & 15u) * 4u, 1024u + t + skip * 2u + (raw ? 4096u : 0u) + (cur << 13) + (sz << 16) + (o < 768u ? 0u : (j - 744u) / 24u << 26));
                LAS char* ld = L + RING + (n & 7u) * E1_SLOT + skip;
                const rsrc_t rs = mk_rsrc(raw ? p.out[(8u - (o < 936u ? cur : rel)) & 7u] : wsrc, raw ? OUT_BYTES : WS_NEED);
                const u32 so = raw ? out_at(t, cur) : E2_OFF + t * E2S + skip, pitch = raw ? 8u * NN : 1024u, vo = raw ? rows : lane * 16u;
                sz -= skip >> 4;
                bool again = false;
                CLAIM(cl, dyn ? OFF_CTR + (72u + cur) * 128u : OOB);
#pragma unroll
                for (int i = 0; i < 8; ++i) {
                    if (i >= 5 && (int)(i * 64u) >= (int)sz) break;
                    VMWAIT(WIN2 - (i < WIN2 - 1 ? 0 : 1));
                    if (i == WIN2 - 1) {
                        put(L_FULL, n);
                        CLAIMED(c, cl);
                        o = !dyn ? o + 16u : 768u + min(c - base, 192u);
                        if (STEAL && o == 960u) {
                            if (cur == rel) vict = w6 & 63u;
                            cur = vict & 7u;
                            vict >>= 3;
                            again = cur != 0u;
                            if (again) CLAIM(cl, OFF_CTR + (72u + cur) * 128u);
                        }
                        if (STEAL && ((o - 768u - SMARK) & ~48u) == 0u && o != 768u + SMARK)
                            __builtin_amdgcn_raw_buffer_store_b32((u32)now(), rs_ws, lane < 8u ? lane * 128u : OOB, OFF_RF + cur * 16u + (o - 768u - SMARK) / 4u, SYS);
                    }
                    if (i < 5 || i * 64u + lane < sz)
                        __builtin_amdgcn_raw_ptr_buffer_load_lds(rs, ld + i * 1024, 16, vo, so + i * pitch, 0, SYS);
                }
                if (again) {
                    VMWAIT(2);
                    CLAIMED(c, cl);
                    o = 768u + min(c - base, 192u);
                }
            }
            put(L_P1, 4u);
            STAMPV(25, stolen);
            VMWAIT(0);
            put(L_FULL, n);
            put(L_END, 1u);
            STAMP(4);
        } else if (wid == 1u) {
            const u32 fa = (lane & 31u) < 20u ? OFF_F1 + rel * 256u + (lane & 31u) * 8u : OFF_FP + (rel * 16u + pj) * 128u + ((lane - 20u) & 7u) * 8u;
            for (u32 m = 0; m != 0xF71FFFCu;) {
                const u32x2 f = __builtin_amdgcn_raw_buffer_load_b64(rs_ws, fa, 0, SYS);
                m = (u32)BALLOT(f[0] >= seq) & 0xF71FFFCu;
                LW[L_PSZ / 4 + (lane & 31u)] = f[1];
                if (expired()) m = 0xF71FFFCu;
                LW[L_SEG / 4] = m;
                __builtin_amdgcn_s_sleep(4);
            }
            STAMP(1);
            while (LW[L_P1 / 4] == 0u) { if (expired()) break; __builtin_amdgcn_s_sleep(16); }
            u32 m = 0;
            while (m != SGM) {
                m |= (u32)BALLOT(__builtin_amdgcn_raw_buffer_load_b32(rs_ws, lane < 16u && ((SGM & ~m) >> lane & 1u) ? lane * 64u : OOB,
                                                                      OFF_F3 + (rel * 16u + pj) * 1024u, SYS) == seq);
                if (expired()) m = SGM;
                LW[L_G3 / 4] = m;
                __builtin_amdgcn_s_sleep(2);
            }
            while (LW[L_P1 / 4] < 2u) { if (expired()) break; __builtin_amdgcn_s_sleep(2); }
            if (lane == 0) __builtin_amdgcn_raw_buffer_store_b32(seq, rs_src, OFF_DONE + rank * 128u + pj * 4u, 0, SYS);
            m = 0;
            while (m != 0xFFFFu) {
                const u32x4 f = LD128(rs_ws, lane < 16u ? lane * 16u : OOB, OFF_F2 + (rel * 16u + pj) * 256u, SYS);
                m = (u32)BALLOT(f[0] == seq && f[3] == seq) & 0xFFFFu;
                if (lane < 16u) *(LAS u32x2*)(L + L_SZ + pj * 128u + lane * 8u) = u32x2{f[1], f[2]};
                if (expired()) m = 0xFFFFu;
                CBAR();
                LW[L_G2 / 4] = m;
                __builtin_amdgcn_s_sleep(2);
            }
            STAMP(2);
            for (u32 i = 0; i < 4u;) {
                const u32x4 f = LD128(rs_ws, lane * 16u, OFF_F2 + rel * 4096u + i * 1024u, SYS);
                *(LAS u32x2*)(L + L_SZ + i * 512u + lane * 8u) = u32x2{f[1], f[2]};
                if (BALLOT(f[0] == seq && f[3] == seq) == ~0ull || expired()) ++i; else __builtin_amdgcn_s_sleep(16);
            }
            CBAR();
            LW[L_G2 / 4] = 0x1FFFFu;
            STAMP(3);
            const u32 tr = (u32)now();
            while (STEAL && LW[L_P1 / 4] < 3u) { expired(); __builtin_amdgcn_s_sleep(32); }
            while (STEAL && LW[L_P1 / 4] == 3u) {
                const u32x4 v = LD128(rs_ws, lane < 8u ? lane * 16u : OOB, OFF_RF + rel * 128u, SYS);
                const u32 tn = (u32)now() - tr;
                const u32 at = lane - 1u < 7u ? min(v[1] - tr, tn) + min(v[2] - tr, tn) + min(v[3] - tr, tn) : 0u;
                const u32 key = v[0] == seq ? (at + ((lane * 5u + pj) & 15u) * JIT) * 8u + lane : 0u;
                const u32 k1 = wave_max_u32(key);
                const int late = (__builtin_amdgcn_readlane((int)at, rel) * 7 - __builtin_amdgcn_readlane((int)wave_scan_u32(at), 63)) / LDIV;
                LW[L_VICT / 4] = (k1 & 7u) + (wave_max_u32(key == k1 ? 0u : key) & 7u) * 8u + ((u32)min(max(late, rel < 7u ? -LMIN : 0), 24) << 8);
                __builtin_amdgcn_s_sleep(VSLEEP);
            }
        } else if (wid < 10u) {
            const rsrc_t rs_ol = mk_rsrc(p.out[0], OUT_BYTES);
            const u32 ovoff = (lane & 7u) * NN + (((lane >> 3) & 1u) * 4u + (lane >> 4)) * 16u;
            for (u32 n = wid - (wid >= 7u ? 10u : 2u);; n += 8u) {
                const bool pre = (int)n < 0;
                bool more = true;
                const u32 k = n + 3u;
                while (pre && ((LW[L_SEG / 4] >> (20u + k)) & 1u) == 0u) __builtin_amdgcn_s_sleep(2);
                while (!pre) {
                    const u32 end = LW[L_END / 4];
                    if (LW[L_FULL / 4] > n) break;
                    if (end != 0u) { more = false; break; }
                    expired();
                    __builtin_amdgcn_s_sleep(2);
                }
                if (!more) break;
                CBAR();
                const u32 id = pre ? (k == 2u ? 112u : k * 16u) + pj + (min(max(RFL(LW[L_PSZ / 4 + 20u + k]), 321u), E1_SLOT / 16u) << 16)
                                   : RFL(LW[L_Q / 4 + (n & 15u)]);
                const u32 sl = RING + (n & ((int)n < 16 && (id & 1024u) == 0u ? 15u : 7u)) * E1_SLOT;
                if ((id & 1024u) == 0u) {
                u32 ch, r; bool isw;
                const u32 so = in_unit(id & 1023u, ch, r, isw);
                if (pre) {
#pragma unroll 1
                    for (u32 at = 0; at < (id >> 16) * 16u; at += 1024u)
                        if (at + lane * 16u < (id >> 16) * 16u)
                            __builtin_amdgcn_raw_ptr_buffer_load_lds(rs_ws, L + sl + at, 16, lane * 16u, so + at, 0, SYS);
                    VMWAIT(0);
                }
                const u32 hd = sl + I_HDR;
                u32x4 tab = LDSV(hd);
                const u32x4 fw = LDSV(hd + I_FLD + lane * 16u);
                const u32 h0 = RFL(tab[0]), sz = in_size(h0), top = ((h0 >> 20) & 15u) | ((h0 >> 24) & 0xF0u), cnt = (RFL(tab[1]) >> 4) & 15u;
                u32x4 o[8];
                if (sz == E1_SLOT / 16u) {
#pragma unroll
                    for (int e = 0; e < 8; ++e) o[e] = LDSV(sl + (e >> 2) * (I_HDR + 128u) + (e & 3) * 1024u + lane * 16u);
                } else {
                    u32x4 mq[4];
                    u32 ex[16], es[8], at[8], d[8][3], ne = 0;
#pragma unroll
                    for (int g = 0; g < 8; ++g) {
                        es[g] = ((fw[g >> 1] & (fw[g >> 1] << 1)) << (g & 1 ? 0 : 2)) & 0x88888888u;
                        at[g] = ne;
                        ne += (u32)__builtin_popcount(es[g]);
                    }
                    const u32 v0 = (I_HDR + I_VAR) / 8u + cnt;
                    const u32 a0 = unary_list<2>(lane, sl + v0 * 8u, (max(sz * 2u, v0) - v0) * 8u, ne, L_PACK + wid * 2048u);
                    CBAR();
#pragma unroll
                    for (int g = 0; g < 8; ++g) { at[g] += a0; LIST8(d[g], at[g]); }
#pragma unroll
                    for (int e = 0; e < 4; ++e) mq[e] = LDSV(sl + e * 1024u + lane * 16u);
                    __builtin_amdgcn_sched_barrier(0);
                    tab = (tab & 0x0F0F0F0Fu) ^ u32x4{TFLIP0, TFLIP1, TFLIP0, TFLIP1};
#pragma unroll
                    for (int g = 0; g < 8; ++g) {
                        u32 c0, c1;
                        deal8(d[g], at[g], es[g], c0, c1);
                        ex[2 * g] = lut16(tab, ((fw[g >> 1] >> (2 * (g & 1))) & 0x03030303u) + c0);
                        ex[2 * g + 1] = lut16(tab, ((fw[g >> 1] >> (2 * (g & 1) + 4)) & 0x03030303u) + c1);
                    }
#pragma unroll 1
                    for (u32 k = 0; k < cnt; ++k) {
                        const u32 at = LW[(hd + I_VAR) / 4 + 2u * k] - lane * 16u, val = LW[(hd + I_VAR) / 4 + 1u + 2u * k];
#pragma unroll
                        for (int j = 0; j < 16; ++j) { ex[j] = at == (u32)j ? val : ex[j]; if ((j & 3) == 3) __builtin_amdgcn_sched_barrier(0); }
                    }
#pragma unroll
                    for (int j = 0; j < 16; ++j) {
                        const u32 e = top * 0x01010101u - ex[j], m = mq[j >> 2][j & 3];
                        const u32 r0 = PERM(e, m, 0x05010400u), r1 = PERM(e, m, 0x07030602u);
                        o[j >> 1][(2 * j) & 3] = __builtin_amdgcn_alignbit(r0, r0, 1);
                        o[j >> 1][(2 * j + 1) & 3] = __builtin_amdgcn_alignbit(r1, r1, 1);
                    }
                }
                CBAR();
                LW[L_FREE / 4 + (n & 7u)] = n + 1u;
                const u32 flag = put_unit(o, isw, rel, ch, r);
                VMWAIT(0);
                put_flag(flag, seq);
                STAMPV(26, (u64)(c1 - c0) | ((u64)((u32)__builtin_amdgcn_s_memtime() - c1) << 20) | ((u64)sz << 40));
                STAMP(pre ? 3 : 1);
                continue;
                }
                const u32 sz = (id >> 16) & 1023u, lk = (id >> 13) & 7u, osoff = out_at(id, lk);
                u32x4 qv[8];
                if (sz >= 512u) {
#pragma unroll
                    for (int e = 0; e < 8; ++e) qv[e] = LDSV(sl + e * 1024u + lane * 16u);
                } else {
                    u32x4 nib[4];
#pragma unroll
                    for (int e = 0; e < 4; ++e) nib[e] = LDSV(sl + R_NIB + e * 1024u + lane * 16u);
                    const u32x4 pl = LDSV(sl + R_BIT + lane * 16u);
                    const u32x4 sg = LDSV((id & 2048u ? L_SG + (id & 63u) * 1024u : sl) + lane * 16u);
                    u32 cnt = 0, at[16], d[16][3];
#pragma unroll
                    for (int o = 0; o < 16; ++o) {
                        at[o] = cnt;
                        cnt += (u32)__builtin_popcount(nib[o >> 2][o & 3] & (pl[o >> 2] << (o & 3)) & 0x88888888u);
                    }
                    const u32 a0 = unary_list<3>(lane, sl + R_VAR, (max(sz, R_VAR / 16u) - R_VAR / 16u) * 16u, cnt, L_PACK + (wid - 2u) * 3072u);
                    CBAR();
#pragma unroll
                    for (int o = 0; o < 16; ++o) { at[o] += a0; LIST8(d[o], at[o]); }
                    __builtin_amdgcn_sched_barrier(0);
#pragma unroll
                    for (int o = 0; o < 16; ++o) {
                        const u32 nv = nib[o >> 2][o & 3], pb = pl[o >> 2] << (o & 3), sv = sg[o >> 2] << (6 - 2 * (o & 3));
                        u32 c0, c1;
                        deal8(d[o], at[o], nv & pb & 0x88888888u, c0, c1);
                        const u32 m0 = (c0 << 3) + (bfi(0x0F0F0F0Fu, nv, pb << 1) & 0x1F1F1F1Fu);
                        const u32 m1 = (c1 << 3) + (bfi(0x0F0F0F0Fu, nv >> 4, pb >> 3) & 0x1F1F1F1Fu);
                        qv[o >> 1][(2 * o) & 3] = __builtin_amdgcn_bitop3_b32(byte_signs(sv << 1), (0x80808080u - m0) ^ 0x80808080u, m0, 0xCA);
                        qv[o >> 1][(2 * o + 1) & 3] = __builtin_amdgcn_bitop3_b32(byte_signs(sv), (0x80808080u - m1) ^ 0x80808080u, m1, 0xCA);
                    }
                }
                CBAR();
                LW[L_FREE / 4 + (n & 7u)] = n + 1u;
                if (LW[L_BAD / 4] != 0u) qv[0] = u32x4{0x80808080u, 0x80808080u, 0x80808080u, 0x80808080u};
#pragma unroll
                for (int e = 0; e < 8; ++e) ST128(qv[e], rs_ol, id & 4096u ? rows : ovoff, osoff + e * 8u * NN, AGT);
                const u32 cls = (id >> 26) & 7u, to = (cls - lk) & 7u;
                if (STEAL && cls != 0u && to - 1u < 6u) {
                    const rsrc_t rs_to = mk_rsrc(p.ws[to], WS_NEED);
                    u32 old = 0;
                    VMWAIT(0);
                    if (lane == 0) old = 1u + __hip_atomic_fetch_add((u32*)(wsl + OFF_CTR + (96u + lk * 8u + cls) * 128u), 1u, __ATOMIC_RELAXED, __HIP_MEMORY_SCOPE_AGENT);
                    if (old == seq * 24u) __builtin_amdgcn_raw_buffer_store_b32(seq, rs_to, OFF_RF + to * 128u + cls * 16u, 0, SYS);
                }
            }
            STAMP(2);
        }
    } else {
        arrive(0u, 12u);
        if (b == AUX_WG && wid == 2u) {
            u32 a = 0;
            u64* gp = (u64*)(p.ws[lane & 7u] + OFF_GAM + (lane >> 3) * 128u + rank * 8u);
            asm volatile("" : "+v"(gp));
            __builtin_amdgcn_raw_buffer_store_b32(seq << 12, rs_ws, lane < 8u ? OFF_CTR + (72u + lane) * 128u : OOB, 0, SYS);
            VMWAIT(0);
            for (;;) {
                const u32x4 g0 = LD128(rs_ws, OFF_AMX + lane * 32u, 0, AGT);
                const u32x4 g1 = LD128(rs_ws, OFF_AMX + lane * 32u + 16u, 0, AGT);
                a = max(max(g0[0], g0[2]), max(g1[0], g1[2]));
                const bool ok = g0[1] == seq && g0[3] == seq && g1[1] == seq && g1[3] == seq;
                if (BALLOT(ok) == ~0ull) break;
                if (expired()) { a = 0xFFFFFFFFu; break; }
            }
            a = wave_max_u32(a);
            __hip_atomic_store((GAS u64*)gp, ((u64)seq << 32) | a, __ATOMIC_RELAXED, __HIP_MEMORY_SCOPE_SYSTEM);
            if (lane == 0) __builtin_amdgcn_raw_buffer_store_b32(seq, rs_ws, OFF_SEQ, 0, AGT);
            STAMP(3);
            bool give_up = false;
            for (;;) {
                const u32x4 f = LD128(rs_ws, OFF_DONE + lane * 16u, 0, SYS);
                const bool ok = (lane >> 3) == rank || (lane & 4u) != 0u || (f[0] >= seq && f[1] >= seq && f[2] >= seq && f[3] >= seq);
                if (BALLOT(ok) == ~0ull) break;
                if (expired()) { give_up = true; break; }
                __builtin_amdgcn_s_sleep(8);
            }
            if (give_up || __builtin_amdgcn_raw_buffer_load_b32(rs_ws, OFF_TMO, 0, AGT) == seq)
                if (lane == 0) p.out[0][rank * 2048u] = (char)-128;
            STAMP(7);
        }
    }
}

extern "C" void kernel_launch(int rank, int nranks, ncclComm_t comm, hipStream_t stream,
                              void* const* d_in, const int* in_sizes, int n_in,
                              void* d_out, int out_size, const int* in_dtypes, int out_dtype,
                              void* const* d_out_all, void* const* d_ws_all, size_t ws_bytes) {
    if (nranks != NR || (unsigned)rank >= (unsigned)NR || n_in < 2 || ws_bytes < 3ull * WS_NEED ||
        in_sizes[0] != (int)(MM * KSH) || in_sizes[1] != (int)(KSH * NN) || out_size != (int)OUT_BYTES) {
        fprintf(stderr, "kernel_launch: unexpected problem shape\n");
        return;
    }
    static bool init = false;
    if (!init) {
        HIP_CHECK(hipFuncSetAttribute((const void*)gemm_ar_q8, hipFuncAttributeMaxDynamicSharedMemorySize, LDS_BYTES));
        init = true;
    }
    Params p;
    p.x = (const char*)d_in[0];
    p.w = (const char*)d_in[1];
    for (int r = 0; r < NR; ++r) p.ws[r] = (char*)d_ws_all[(rank + r) % NR];
    for (int r = 0; r < NR; ++r) p.out[r] = (char*)d_out_all[(rank + r) % NR];
    p.rank = rank;
    hipLaunchKernelGGL(gemm_ar_q8, dim3(256), dim3(768), LDS_BYTES, stream, p);
    HIP_CHECK(hipGetLastError());
}
```

```cpp
#include <hip/hip_runtime.h>
#include <rccl/rccl.h>
#include <cstdio>
#include <cstdint>
#include <cstddef>

#define HIP_CHECK(expr) do { hipError_t _e = (expr); if (_e != hipSuccess) { \
    fprintf(stderr, "HIP error %s at %s:%d\n", hipGetErrorString(_e), __FILE__, __LINE__); \
    return; } } while (0)

#define LAS __attribute__((address_space(3)))
#define GAS __attribute__((address_space(1)))
typedef unsigned int u32;
typedef unsigned long long u64;
typedef u32 u32x4 __attribute__((ext_vector_type(4)));
typedef u32 u32x2 __attribute__((ext_vector_type(2)));
typedef unsigned short u16x2 __attribute__((ext_vector_type(2)));
typedef float f32x16 __attribute__((ext_vector_type(16)));

#define WIN1 4
#define WIN2 4
#define WINS 4
#define E2S 8192u
#define SGM 0xBBBBu
#define PCH 2
#define P1 0u
#define PUSHED(ch, r) ((ch) == 0u || (P1 && (ch) == 1u && (r) >= 16u))
#define HMASK (P1 ? 0xF71FFFCu : 0xF31FFFCu)
#define PB1 12u
#define PSTEP 7u
#define PSTEP2 11u
#define PBACK 20u
#define PLEAD 8u
#define XFIRST 0x0Eu
#define NRAW 24u
#define STEAL 1
#define VSLEEP 8
#define SMARK 64u
#define LDIV 256
#define LMIN 12
#define JIT 15u
#define SFROM 120u
#define TDELAY 100
#define WDELAY1 50
#define XPD -1
#define SPRIO 0
#define STRY 20u
#define TOUCH 1
#define XPS 0
#define XAUX 0
#define WDELAY3 100
#define PDELAY 100
#define WDELAY2 200
#define S2 (SGM == 0xAAAAu ? 32u : 48u)
#define DIAG_NDMA 8
#define CSLEEP 1
#define POLL_SLEEP 2
#define BALANCE 1
#define LSLEEP 1
#define KPAD "s_nop 0\n"
#define FLAGW 4
#define PAHEAD 4
#define LNOP 5
#define LADAPT 0
#define CPRIO 0
#define LPRIO 0
#define PRED_PAD 20
#define TMAP 0x144444444320ull
#define FORCE_RAW1 0
#define FORCE_RAW2 0

constexpr int NR = 8;
constexpr u32 MM = 4096, NN = 16384, KSH = 512;
constexpr u32 X_BYTES = MM * KSH * 2, W_BYTES = KSH * NN * 2, OUT_BYTES = MM * NN;
constexpr u32 PIECE = 32768;
constexpr u32 STEP_BYTES = 28 * PIECE;
constexpr u32 OFF_UF = 1u << 20;
constexpr u32 OFF_F3 = OFF_UF + 64 * 112 * 128;
constexpr u32 STAGE_OFF = 2u << 20;
static_assert(OFF_F3 + 2048 * 64 <= STAGE_OFF, "flags");
constexpr u32 E1_SLOT = 8192 + 128;
constexpr u32 I_HDR = 4096, I_FLD = 16, I_VAR = 1040, I_MAX = 1900;
constexpr u32 E1X_OFF = STAGE_OFF + 64 * STEP_BYTES;
constexpr u32 E1W_OFF = E1X_OFF + 512 * E1_SLOT;
constexpr u32 E1P_OFF = E1W_OFF + 2048 * E1_SLOT;
constexpr u32 E2_OFF = E1P_OFF + 512 * E1_SLOT;
constexpr u32 E3_OFF = E2_OFF + 1024 * E2S;
constexpr u32 WS_NEED = E3_OFF + 1024 * 1024;
constexpr u32 OFF_SEQ = 0, OFF_TMO = 256;
constexpr u32 OFF_AMX = 4096;
constexpr u32 OFF_GAM = OFF_AMX + 2048;
constexpr u32 OFF_DONE = OFF_GAM + 1024;
constexpr u32 OFF_F1 = OFF_DONE + 1024;
constexpr u32 OFF_CTR = OFF_F1 + 2048;
constexpr u32 OFF_RF = OFF_CTR + 160 * 128;
constexpr u32 OFF_F2 = 65536;
constexpr u32 OFF_FP = OFF_F2 + 32768;
constexpr u32 OFF_STAMP = 131072;
static_assert(OFF_RF + 1024 <= OFF_F2 && OFF_FP + 16384 <= OFF_STAMP && OFF_STAMP + 256 * 12 * 32 * 8 <= OFF_UF, "flags");

constexpr u32 SLOT_BYTES = 32768;
constexpr u32 RING = 2048;
constexpr u32 CTRL = RING + 16 * E1_SLOT;
constexpr u32 L_FULL = CTRL, L_NEXT = CTRL + 16, L_FREE = CTRL + 32, L_NRDY = CTRL + 64, L_AMAX = CTRL + 68, L_ACNT = CTRL + 72,
              L_BAD = CTRL + 76, L_SCALE = CTRL + 80, L_GCNT = CTRL + 88, L_GSZ = CTRL + 96, L_SCNT = CTRL + 112, L_STEPS = CTRL + 128, L_PTR = CTRL + 640;
constexpr u32 L_SEG = CTRL + 4, L_G2 = CTRL + 8, L_END = CTRL + 12, L_P1 = CTRL + 16, L_G3 = CTRL + 20, L_VICT = CTRL + 24, L_Q = CTRL + 128, L_Q2 = CTRL + 320,
              L_PSZ = CTRL + 192, L_SG = RING + 8 * E1_SLOT;
constexpr u32 L_PACK = CTRL + 1024;
constexpr u32 L_SZ = L_PACK + 8 * 3072;
constexpr u32 LDS_BYTES = L_SZ + 2048;
constexpr u64 DEADLINE = 100000000ull;
constexpr u32 OOB = WS_NEED;
constexpr int SYS = 17, AGT = 16;
constexpr u32 CODE_BYTES = 49152;
constexpr u32 AUX_WG = 248;
constexpr u32 N1 = 768 + 112;
constexpr u32 N2 = 60;
constexpr u32 R_BIT = 1024, R_NIB = 2048, R_VAR = 6144, R_MAX = 2800;

struct Params {
    char* ws[NR];
    const char* x;
    const char* w;
    int rank;
    char* out[NR];
};

typedef __amdgpu_buffer_rsrc_t rsrc_t;
__device__ __forceinline__ rsrc_t mk_rsrc(const void* p, u32 bytes) {
    return __builtin_amdgcn_make_buffer_rsrc((void*)p, 0, bytes, 0x00020000);
}
__device__ __forceinline__ u64 now() { return __builtin_amdgcn_s_memrealtime(); }
#define VMWAIT(n) asm volatile("s_waitcnt vmcnt(%0)" :: "n"(n) : "memory")
#define CBAR() asm volatile("" ::: "memory")
#define ST128(v, rs, voff, soff, aux) do { const u32x4 d_ = (v); \
    __builtin_amdgcn_raw_buffer_store_b128(d_, rs, voff, soff, aux); \
    asm volatile("s_nop 1" :: "v"(d_) : "memory"); } while (0)
#define LD128(rs, voff, soff, aux) __builtin_amdgcn_raw_buffer_load_b128(rs, voff, soff, aux)
#define PERM(a, b, s) __builtin_amdgcn_perm(a, b, s)
#define BALLOT(c) __builtin_amdgcn_ballot_w64(c)
#define RFL(v) ((u32)__builtin_amdgcn_readfirstlane(v))
__device__ __forceinline__ u32 bfi(u32 m, u32 a, u32 b) { return (a & m) | (b & ~m); }
__device__ __forceinline__ u32 rank_in(u64 m) {
    return __builtin_amdgcn_mbcnt_hi((u32)(m >> 32), __builtin_amdgcn_mbcnt_lo((u32)m, 0u));
}

#define STAMPV(i, v) do { } while (0)
#define STAMP(i) STAMPV(i, now())
#define TK(i)
#define TKOUT()

#define DPP_STEPS(OP) \
    OP(0x111, 0xf, true) OP(0x112, 0xf, true) OP(0x114, 0xf, true) OP(0x118, 0xf, true) OP(0x142, 0xa, false) OP(0x143, 0xc, false)
__device__ __forceinline__ u32 wave_max_u32(u32 v) {
#define OP_(c, rm, bc) v = max(v, (u32)__builtin_amdgcn_update_dpp(0, (int)v, c, rm, 0xf, bc));
    DPP_STEPS(OP_)
#undef OP_
    return (u32)__builtin_amdgcn_readlane((int)v, 63);
}
__device__ __forceinline__ u32 wave_scan_u32(u32 v) {
#define OP_(c, rm, bc) v += (u32)__builtin_amdgcn_update_dpp(0, (int)v, c, rm, 0xf, bc);
    DPP_STEPS(OP_)
#undef OP_
    return v;
}
__device__ __forceinline__ u32 byte_signs(u32 a) { return PERM(a, a << 8, 0x0B090A08u); }

constexpr u32 TFLIP0 = 0xFFFFFFFFu, TFLIP1 = 0xFFFFFF00u;
__device__ __forceinline__ u32 lut16(u32x4 t, u32 i) { return PERM(t[1], t[0], i) ^ PERM(t[3], t[2], i ^ 0x08080808u); }
__device__ __forceinline__ u32 unary8(u32 l0, u32 l1, u32& code) {
    u32 p1 = l0 + l1;
    p1 += p1 << 8; p1 += p1 << 16;
    const u32 p0 = p1 - l1;
    code = 0;
#pragma unroll
    for (int k = 0; k < 4; ++k) code |= (1u << ((p0 >> (8 * k)) & 31u)) | (1u << ((p1 >> (8 * k)) & 31u));
    code >>= 1;
    return p1 >> 24;
}
__device__ __forceinline__ void put_bits(LAS char* L, u32 pos, u32 code) {
    LAS u32* a = (LAS u32*)(L + ((pos >> 5) << 2));
    __hip_atomic_fetch_or(a, code << (pos & 31u), __ATOMIC_RELAXED, __HIP_MEMORY_SCOPE_WORKGROUP);
    __hip_atomic_fetch_or(a + 1, (code >> 1) >> (~pos & 31u), __ATOMIC_RELAXED, __HIP_MEMORY_SCOPE_WORKGROUP);
}
__device__ __forceinline__ u32x2 table_of(u32 v) {
    u64 t = 0;
#pragma unroll 1
    for (u32 i = 0, n = 0; i < 8u; ++i) {
        if ((v >> i) & 1u) n += 8u; else t += 1ull << n;
    }
    return __builtin_bit_cast(u32x2, t);
}
#define LP(a) ((const LAS u32*)(size_t)(a))
template <u32 KB>
__device__ __forceinline__ u32 unary_list(u32 lane, u32 from, u32 bytes, u32 mine, u32 area) {
    const u32 sb = RFL((bytes + 63u) >> 6), at = from + lane * sb;
    const LAS u32* a = LP(at & ~3u);
    auto word = [&](u32 i) -> u32 { return __builtin_amdgcn_alignbyte(a[i + 1u], a[i], at); };
    auto ones = [&](u32 w, u32 i) -> u32 { return (u32)__builtin_popcount(w & RFL(sb >= 4u * i + 4u ? ~0u : sb > 4u * i ? ~(~0u << (sb & 3u) * 8u) : 0u)); };
    u32 w[KB], ix[4 * KB], cnt = 0;
    u32x2 t[4 * KB];
#pragma unroll
    for (u32 i = 0; i < KB; ++i) w[i] = word(i);
#pragma unroll
    for (u32 i = 0; i < KB; ++i) *(LAS u32x4*)(size_t)(area + i * 1024u + lane * 16u) = u32x4{0u, 0u, 0u, 0u};
#pragma unroll
    for (u32 e = 0; e < 4u * KB; ++e) { ix[e] = ((w[e >> 2] >> (8 * (e & 3))) & 255u) << 3; t[e] = *(const LAS u32x2*)(size_t)ix[e]; }
#pragma unroll
    for (u32 i = 0; i < KB; ++i) cnt += ones(w[i], i);
#pragma unroll 1
    for (u32 i = KB; 4u * i < sb; ++i) cnt += ones(word(i), i);
    const bool on = lane * sb < bytes;
    cnt = (on ? cnt << 16 : 0u) + mine;
    const u32 end = wave_scan_u32(cnt);
    u32 k = area - 1u + ((end - cnt) >> 16);
    auto add = [&](u32x2 v, u32 x) {
        LAS u32* d = (LAS u32*)(size_t)(k & ~3u);
        const u32 m = ~k;
        __hip_atomic_fetch_add(d, __builtin_amdgcn_alignbyte(v[0], 0u, m), __ATOMIC_RELAXED, __HIP_MEMORY_SCOPE_WORKGROUP);
        __hip_atomic_fetch_add(d + 1, __builtin_amdgcn_alignbyte(v[1], v[0], m), __ATOMIC_RELAXED, __HIP_MEMORY_SCOPE_WORKGROUP);
        __hip_atomic_fetch_add(d + 2, __builtin_amdgcn_alignbyte(0u, v[1], m), __ATOMIC_RELAXED, __HIP_MEMORY_SCOPE_WORKGROUP);
        k += (u32)__builtin_popcount(x);
    };
    asm volatile("" : "+v"(t[4 * KB - 1]), "+v"(k));
    if (on && (u32)__builtin_amdgcn_readlane((int)end, 63) < (KB * 1024u - 12u) << 16) do {
#define STEP_(e) if (e < 4u * KB) { add(t[e % (4u * KB)], ix[e % (4u * KB)]); if (sb == e + 1u) break; }
        STEP_(0u) STEP_(1u) STEP_(2u) STEP_(3u) STEP_(4u) STEP_(5u) STEP_(6u) STEP_(7u) STEP_(8u) STEP_(9u) STEP_(10u) STEP_(11u)
#undef STEP_
#pragma unroll 1
        for (u32 e = 4u * KB; e < sb; ++e) {
            const u32 x = (u32)*(const LAS unsigned char*)(size_t)(at + e) << 3;
            add(*(const LAS u32x2*)(size_t)x, x);
        }
    } while (0);
    return area + ((end - cnt) & 0xFFFFu);
}
#define LIST8(d, ad) do { const LAS u32* a_ = LP((ad) & ~3u); d[0] = a_[0]; d[1] = a_[1]; d[2] = a_[2]; } while (0)
__device__ __forceinline__ void deal8(const u32 (&d)[3], u32 ad, u32 es, u32& c0, u32& c1) {
    const u32 lo = __builtin_amdgcn_alignbyte(d[1], d[0], ad), hi = __builtin_amdgcn_alignbyte(d[2], d[1], ad);
    u32 pre = es << 1;
    asm("" : "+v"(pre));
    const u32 mk = pre - (es >> 3);
    pre += pre << 4;
    asm("" : "+v"(pre));
    pre += pre << 8;
    asm("" : "+v"(pre));
    pre += pre << 16;
    const u32 sel = __builtin_amdgcn_bitop3_b32(mk, pre, 0xCCCCCCCCu, 0xCA);
    c0 = PERM(hi, lo, sel & 0x0F0F0F0Fu);
    c1 = PERM(hi, lo, (sel >> 4) & 0x0F0F0F0Fu);
}

__global__ void __launch_bounds__(768, 3) gemm_ar_q8(Params p) {
    extern __shared__ __attribute__((aligned(16))) char smem[];
    char *wsl, *px, *pw;
    u32 rank;
    asm volatile("s_load_dwordx2 %0, %4, 0\n s_load_dwordx2 %1, %4, 0x40\n s_load_dwordx2 %2, %4, 0x48\n s_load_dword %3, %4, 0x50"
                 : "=&s"(wsl), "=&s"(px), "=&s"(pw), "=&s"(rank) : "s"(__builtin_amdgcn_kernarg_segment_ptr()));
    __builtin_amdgcn_sched_barrier(0);
    static_assert(offsetof(Params, x) == 0x40 && offsetof(Params, w) == 0x48 && offsetof(Params, rank) == 0x50, "arguments");
    LAS char* L = (LAS char*)smem;
    u32 lane = threadIdx.x & 63;
    const u32 wid = RFL(threadIdx.x >> 6);
    const u32 b = blockIdx.x;
    const u32 q = b & 7u, idx = b >> 3;
    const u32 mb = 4 * (q >> 1) + (idx & 3), nb = 4 * (q & 1) + ((idx >> 2) & 3);
    const u32 nb2 = 8u + 2u * (q & 1u) + ((idx >> 2) & 1u), fills = idx < 8u ? 256u : 128u;
    const u32 rel = 1u + (b - 128u) % 7u, pj = ((b - 128u) / 7u) & 15u;
    auto x_at = [](u32 l) -> u32 { return (l >> 2) * 1024u + (((l & 3u) ^ ((l >> 4) & 2u)) << 4); };
    auto w_at = [](u32 l) -> u32 { return (l >> 4) * 262144u + (l & 15u) * 16u; };
    u32 mg = (b >> 7) - 1u;
    const u32 tq = (b + (wid & 1u) * 128u) >> 3;
    u32 tr = b + tq, ts = bfi(mg, (tq & 2u) * 524288u + (tq >> 2) * 512u + (tq & 1u) * 256u, (b & 127u) << 16);
    u32 bar = (bfi(mg, 0x801u, b & 64u ? 0u : 4u) >> wid) & 1u, tv = bfi(mg, w_at(lane), x_at(lane));
    asm volatile("" : "+s"(tr), "+s"(ts), "+s"(bar), "+s"(mg));
    asm volatile("" : "+v"(tv));
    asm volatile("s_waitcnt lgkmcnt(0)" : "+s"(wsl), "+s"(px), "+s"(pw), "+s"(rank));
    rank &= 7u;
    const rsrc_t rs_ws = mk_rsrc(wsl, WS_NEED);
    const u32 seqv = __builtin_amdgcn_raw_buffer_load_b32(rs_ws, OFF_SEQ, 0, AGT);
    __builtin_amdgcn_sched_barrier(0);
    u32x4 v[8];
    auto fetch = [&](bool w, u32 vo, u32 so) {
        const rsrc_t rs = mk_rsrc(w ? pw : px, w ? W_BYTES : X_BYTES);
#pragma unroll
        for (int i = 0; i < 8; ++i) v[i] = LD128(rs, vo, so + (i & 3) * (w ? 32768u : 16384u) + (i >> 2) * (w ? 131072u : 64u), 0);
    };
    if (bar) fetch(mg != 0u, tv, ts + (((rank + tr) << 12) & mg & 0x7000u));
    __builtin_amdgcn_sched_barrier(0);
    volatile LAS u32* LW = (volatile LAS u32*)L;
#define LDSV(a) (*(const LAS u32x4*)(L + (a)))

    STAMP(22);
    if (threadIdx.x < 32) LW[CTRL / 4 + threadIdx.x] = b >= 128u && threadIdx.x - 13u < 2u + P1 ? ~7u : 0u;
    auto barrier = [] {
        asm volatile("s_waitcnt lgkmcnt(0)" ::: "memory");
        __builtin_amdgcn_s_barrier();
        CBAR();
    };
#define PEERS(d) asm volatile("global_load_dwordx2 %0, %1, %2" : "=v"(d) : "v"((lane & 7u) * 8u), "s"(__builtin_amdgcn_kernarg_segment_ptr()) : "memory")

    auto put_unit = [&](const u32x4 (&o)[8], bool isw, u32 lk, u32 ch, u32 r) -> u32 {
        const u32 so = STAGE_OFF + (lk * 8u + ch) * STEP_BYTES + (isw ? 16u * PIECE + r * 8192u : (r >> 2) * PIECE + (r & 3u) * 4096u);
        const u32 v0 = isw ? ((lane & 48u) + (lane & 7u) * 2u + ((lane >> 3) & 1u)) * 16u : lane * 16u, v1 = v0 + (isw ? 4096u : 16384u);
#pragma unroll
        for (int e = 0; e < 8; ++e) ST128(o[e], rs_ws, (e < 4 ? v0 : v1) + (e & 3) * 1024u, so, SYS);
        return OFF_UF + ((lk * 8u + ch) * 112u + (isw ? 64u : 0u) + r) * 128u;
    };
    auto put_flag = [&](u32 flag, u32 sq) { if (lane < FLAGW) ST128((u32x4{sq, sq, sq, sq}), rs_ws, flag + lane * 16u, 0, SYS); };

    auto strip_at = [](u32 os, u32 r) -> u32 { return 32u + (os >> 1) * 4u + (r & 2u) + (os & 1u); };

    {
        const u32 ma = 0u - ((b + 16u) >> 8), ml = 0u - ((wid + 6u) >> 4);
        const u32 tg = (u32)(TMAP >> (wid * 4u)) & 7u, step = bfi(ma, 192u, 224u);
        u32 t = b + bfi(mg, tg * 128u, bfi(ma, 1536u + wid * 16u, 384u + ((wid * 112u - 1056u) & ml)));
        const u32 tend = bfi(mg, min(t + 1u, 480u), bfi(ma, 3008u, bfi(ml, 1376u, wid == 2u ? 576u : 0u)));
        const u32 delay = bfi(mg, (max(tg, 1u) - 1u) * WDELAY1, bfi(ma, WDELAY2, PDELAY & ml));
        auto pause = [](u32 d) {
            const u32 ts = (u32)now();
            while ((u32)now() - ts < d) __builtin_amdgcn_s_sleep(2);
        };
        if (!bar) {
            if (b - 128u < 112u && wid - 6u < 4u) *(LAS u32x2*)(L + (threadIdx.x - 384u) * 8u) = table_of(threadIdx.x - 384u);
            barrier();
            if (t < tend) pause(delay);
        }
        if (b < 128u && wid - 4u < 4u) {
            u64 pc;
            asm volatile("s_getpc_b64 %0" : "=s"(pc));
            const u32 v = __builtin_amdgcn_raw_buffer_load_b32(mk_rsrc((void*)(pc & ~255ull), CODE_BYTES), lane * 128u, ((idx * 4u + wid) % 6u) * 8192u, 0);
            asm volatile("" :: "v"(v));
        }
        if (XPD >= 0 && b >= 128u && wid == (b < 240u ? 3u : 11u)) {
            if (b < 240u) pause(XPD);
            const rsrc_t rs_x = mk_rsrc(px, X_BYTES);
            u32 v = 0;
#pragma unroll
            for (u32 k = 1; k < 8u; ++k) {
                v |= __builtin_amdgcn_raw_buffer_load_b32(rs_x, lane * 1024u + k * 128u, ((q >> 1) * 16u + (idx & 15u)) * 65536u, 0);
                if (XPS) { asm volatile("" :: "v"(v)); }
            }
            asm volatile("" :: "v"(v));
        }
        for (; t < tend; t += step) {
            TK(0);
            u32 kind = 2u, lk = 0, ch, r;
            const u32 u = t + (t < 640u ? 512u : t < 752u ? 1024u : t < 880u ? 400u : t < 1104u ? 2128u : t < 1376u ? 176u : 0u);
            if (t < 256u) { ch = 0u; r = t >> 3; lk = (t + r) & 7u; kind = lk == 0u ? 1u : 2u; }
            else if (t < 480u) { ch = (t - 224u) >> 5; r = t & 31u; kind = 1u; }
            else if (u < 1536u) { ch = (u >> 6) & 7u; r = u & 63u; kind = 0u; }
            else if (u < 1552u) { ch = (u >> 1) & 7u; r = 28u + (u & 1u) * 2u; }
            else { ch = (0x3765421u >> ((u - 1664u) / 224u * 4u)) & 7u; r = (u - 1664u) % 224u; lk = 1u + r % 7u; r /= 7u; }
            const u32 to = (rank + lk) & 7u;
            const bool strip = kind == 2u && (r | 2u) == 30u;
            const bool push = lk != 0u && PUSHED(ch, r), both = push && strip;
            char* pto = wsl;
            u32x4 o[8];
            if (!bar) fetch(kind != 0u, kind != 0u ? w_at(lane) : x_at(lane), kind == 0u ? r * 65536u + ch * 128u
                                : (ch * 64u + (r & 2u) * 16u) * 32768u + (to * 2048u + (r >> 2) * 256u + (r & 1u) * 128u) * 2u);
            if (kind == 0u) {
                if (bar) barrier();
#pragma unroll
                for (int i = 0; i < 8; ++i) o[i] = v[i];
                asm volatile("" : "+v"(o[0]), "+v"(o[1]), "+v"(o[2]), "+v"(o[3]), "+v"(o[4]), "+v"(o[5]), "+v"(o[6]), "+v"(o[7]));
            } else {
                pto = p.ws[push ? lk : 0u];
                asm volatile("" : "+s"(pto));
                if (bar) barrier();
#pragma unroll
                for (int e = 0; e < 4; ++e)
#pragma unroll
                    for (int k = 0; k < 4; ++k) {
                        const u32 lo = PERM(v[2 * k + 1][e >> 1], v[2 * k][e >> 1], (e & 1) ? 0x07060302u : 0x05040100u);
                        const u32 hi = PERM(v[2 * k + 1][2 + (e >> 1)], v[2 * k][2 + (e >> 1)], (e & 1) ? 0x07060302u : 0x05040100u);
                        o[e][k] = (u32)__builtin_amdgcn_update_dpp((int)lo, (int)hi, 0x128, 0xf, 0xc, false);
                        o[4 + e][k] = (u32)__builtin_amdgcn_update_dpp((int)hi, (int)lo, 0x128, 0xf, 0x3, false);
                    }
            }
            bar = 0u;
            u32 flag = OOB, size = E1_SLOT, sq = RFL(seqv) + 1u;
            u32 reg = L_PACK + wid * 2048u;
            asm volatile("" : "+s"(sq), "+s"(reg), "+v"(lane));
            if (kind == 1u || (strip && lk != 0u)) flag = put_unit(o, true, 0u, ch, kind == 2u ? strip_at(lk - 1u, r) : r);
            if (kind == 1u) { TK(2); VMWAIT(0); TK(3); put_flag(flag, sq); TK(4); TKOUT(); continue; }
            u64 f1;
            PEERS(f1);
            const u32 kept = kind == 0u ? E1X_OFF + (ch * 64u + r) * E1_SLOT : E1W_OFF + ((lk * 8u + ch) * 32u + r) * E1_SLOT;
            const u32 slot = push ? E1P_OFF + ((lk * 2u + ch) * 32u + r) * E1_SLOT : kept;
            const rsrc_t rs_to = mk_rsrc(pto, WS_NEED);
#define PST(v, voff, at) do { const u32x4 p_ = (v); ST128(p_, rs_to, voff, slot + (at), SYS); if (both) ST128(p_, rs_ws, voff, kept + (at), SYS); } while (0)
            *(LAS u32x4*)(L + reg + lane * 16u) = u32x4{0u, 0u, 0u, 0u};
            *(LAS u32x4*)(L + reg + 1024u + lane * 16u) = u32x4{0u, 0u, 0u, 0u};
            u32 ex[16], mx = 0;
            const bool late = push && ch != 0u;
            if (push && !late && r > PLEAD) pause((r - PLEAD) * PSTEP);
#pragma unroll
            for (int e = 0; e < 4; ++e) {
                u32x4 mq;
#pragma unroll
                for (int k = 0; k < 4; ++k) {
                    const int j = 4 * e + k;
                    const u32 d0 = o[j >> 1][(2 * j) & 3], d1 = o[j >> 1][(2 * j + 1) & 3];
                    const u32 r0 = __builtin_amdgcn_alignbit(d0, d0, 31), r1 = __builtin_amdgcn_alignbit(d1, d1, 31);
                    mq[k] = PERM(r1, r0, 0x06040200u);
                    ex[j] = PERM(r1, r0, 0x07050301u);
                    mx = __builtin_bit_cast(u32, __builtin_elementwise_max(__builtin_bit_cast(u16x2, mx),
                             __builtin_elementwise_max(__builtin_bit_cast(u16x2, r0), __builtin_bit_cast(u16x2, r1))));
                }
                if (!late) PST(mq, lane * 16u + e * 1024u, 0u);
            }
            const u32 top = wave_max_u32(max(mx >> 24, (mx >> 8) & 255u));
            u32 any = 0, cnt = 0;
#pragma unroll
            for (int j = 0; j < 16; ++j) { ex[j] = top * 0x01010101u - ex[j]; any |= ex[j]; }
            CBAR();
#pragma unroll
            for (int j = 0; j < 16; j += 2)
                __hip_atomic_fetch_add((LAS u32*)(L + reg + 256u + (((ex[j] << 4) & 0xF0u) | ((lane & 3u) << 2))), 1u, __ATOMIC_RELAXED,
                                       __HIP_MEMORY_SCOPE_WORKGROUP);
            CBAR();
            const u32x4 hist = LDSV(reg + 256u + (lane & 15u) * 16u);
            CBAR();
            if (BALLOT((any & 0xF0F0F0F0u) != 0u) != 0ull) {
#pragma unroll
                for (int j = 0; j < 16; ++j) {
                    const bool far = (ex[j] & 0xF0F0F0F0u) != 0u;
                    const u64 m = BALLOT(far);
                    if (m != 0ull) {
                        const u32 k = cnt + rank_in(m);
                        if (far && k < 14u) *(LAS u32x2*)(L + reg + I_VAR + k * 8u) = u32x2{lane * 16u + j, ex[j]};
                        cnt += (u32)__builtin_popcountll(m);
                        ex[j] &= 0x0F0F0F0Fu;
                    }
                }
            }
            const u32 key = (((hist[0] + hist[1] + hist[2] + hist[3]) << 4) | (lane & 15u)) ^ 15u;
            u32 above = 0;
#define ROT_(n) above = __builtin_amdgcn_alignbit(above, key - (u32)__builtin_amdgcn_mov_dpp((int)key, 0x120 + n, 0xf, 0xf, false), 31);
            ROT_(1) ROT_(2) ROT_(3) ROT_(4) ROT_(5) ROT_(6) ROT_(7) ROT_(8) ROT_(9) ROT_(10) ROT_(11) ROT_(12) ROT_(13) ROT_(14) ROT_(15)
#undef ROT_
            above = (u32)__builtin_popcount(above);
            u32 tb = ((max(above, 2u) - 2u) | (min(above, 3u) << 4)) << ((lane & 3u) * 8u);
            tb |= (u32)__builtin_amdgcn_mov_dpp((int)tb, 0xB1, 0xf, 0xf, false);
            tb |= (u32)__builtin_amdgcn_mov_dpp((int)tb, 0x4E, 0xf, 0xf, false);
            const u32x4 tab = u32x4{(u32)__builtin_amdgcn_readlane((int)tb, 0) ^ TFLIP0, (u32)__builtin_amdgcn_readlane((int)tb, 4) ^ TFLIP1,
                                    (u32)__builtin_amdgcn_readlane((int)tb, 8) ^ TFLIP0, (u32)__builtin_amdgcn_readlane((int)tb, 12) ^ TFLIP1};
            u32 code[8], len[8], over = 0, tot = 0;
            u32x4 fw;
#pragma unroll
            for (int g = 0; g < 8; ++g) {
                const u32 s0 = lut16(tab, ex[2 * g]), s1 = lut16(tab, ex[2 * g + 1]), h = bfi(0x0F0F0F0Fu, s0 >> 4, s1);
                fw[g >> 1] = g & 1 ? fw[g >> 1] | (h << 2) : h;
                len[g] = unary8(s0 & 0x0F0F0F0Fu, s1 & 0x0F0F0F0Fu, code[g]);
                over |= len[g];
                tot += len[g] + ((u32)__builtin_popcount(code[g]) << 16);
            }
            const u32 end = wave_scan_u32(tot), all = (u32)__builtin_amdgcn_readlane((int)end, 63);
            const u32 bytes = ((I_VAR + cnt * 8u) * 8u + (all & 0xFFFFu) + 127u) >> 7 << 4;
            if (BALLOT(over > 31u) == 0ull && cnt < 15u && bytes <= 2032u && (all >> 16) <= I_MAX && !(FORCE_RAW1 && (r & 1u))) {
                size = I_HDR + bytes;
                *(LAS char*)(L + reg + above) = (char)((lane & 15u) | ((((bytes >> 4) | (top << 8) | (cnt << 16)) >> (above * 4u & 31u)) << 4));
                *(LAS u32x4*)(L + reg + I_FLD + lane * 16u) = fw;
                u32 pos = ((end - tot) & 0xFFFFu) + (reg + I_VAR + cnt * 8u) * 8u;
#pragma unroll
                for (int g = 0; g < 8; ++g) { put_bits(L, pos, code[g]); pos += len[g]; }
                CBAR();
                if (late) {
                    pause((r - (r < 25u ? PB1 : PBACK)) * PSTEP2);
#pragma unroll
                    for (int e = 0; e < 4; ++e) {
                        u32x4 mq;
#pragma unroll
                        for (int k = 0; k < 4; ++k) {
                            const u32 d0 = o[2 * e + (k >> 1)][(2 * k) & 3], d1 = o[2 * e + (k >> 1)][(2 * k + 1) & 3];
                            mq[k] = PERM(__builtin_amdgcn_alignbit(d1, d1, 31), __builtin_amdgcn_alignbit(d0, d0, 31), 0x06040200u);
                        }
                        PST(mq, lane * 16u + e * 1024u, 0u);
                    }
                }
                PST(LDSV(reg + lane * 16u), lane * 16u, I_HDR);
                PST(LDSV(reg + 1024u + lane * 16u), (lane & 60u) * 16u + 1024u < bytes ? lane * 16u + 1024u : OOB, I_HDR);
                CBAR();
            } else {
                PST((u32x4{~0u, ~0u, ~0u, ~0u}), lane == 0u ? 0u : OOB, I_HDR);
#pragma unroll
                for (int e = 0; e < 8; ++e) PST(o[e], lane * 16u + (e & 3) * 1024u, (e >> 2) * (I_HDR + 128u));
            }
            u32 old = 0;
            TK(2);
            VMWAIT(0);
            TK(3);
            asm volatile("" : "+v"(f1));
            put_flag(flag, sq);
            if (push) {
                if (lane == 0) __builtin_amdgcn_raw_buffer_store_b64(u32x2{sq, (size + 15u) >> 4}, rs_to, 0u, OFF_FP + (lk * 16u + (r & 15u)) * 128u + (ch + (r >> 4)) * 8u, SYS);
                __builtin_amdgcn_s_setprio(0);
                if (!strip) { TK(4); TKOUT(); continue; }
            }
            if (kind == 0u && ch == 0u) {
                const u32 j = (r - 8u * lane) & 63u;
                if (lane - 1u < 7u)
                    __hip_atomic_store((GAS u64*)(f1 + OFF_FP + (lane * 16u + (j & 15u)) * 128u + 32u + (j >> 4) * 8u), (u64)sq, __ATOMIC_RELAXED, __HIP_MEMORY_SCOPE_SYSTEM);
                TK(4); TKOUT();
                continue;
            }
            if (lane == 0 ? (kind == 0u || lk != 0u) && !push : (lane == 1u && strip))
                old = 1u + __hip_atomic_fetch_add((u32*)(wsl + OFF_CTR + (lane == 0 ? lk * 8u + ch : 80u) * 128u), 1u, __ATOMIC_RELAXED,
                                                  __HIP_MEMORY_SCOPE_AGENT);
            f1 += OFF_F1 + lane * 256u;
            const u64 fv = sq;
            if (RFL(old) == sq * (kind == 0u ? 64u : P1 && ch == 1u ? 16u : 32u) && lane - 1u < 7u && (kind == 0u || lane == lk))
                __hip_atomic_store((GAS u64*)f1 + ch * 2u + (kind == 0u), fv, __ATOMIC_RELAXED, __HIP_MEMORY_SCOPE_SYSTEM);
            if ((u32)__builtin_amdgcn_readlane((int)old, 1) == sq * 128u && lane - 1u < 7u)
                __hip_atomic_store((GAS u64*)f1 + 16, fv, __ATOMIC_RELAXED, __HIP_MEMORY_SCOPE_SYSTEM);
            TKOUT();
        }
    }
    const u32 seq = RFL(seqv) + 1u;
    lane = threadIdx.x & 63;
    STAMP(10);

    u32 spins = 0;
    u64 t0 = 0;
    auto expired = [&]() -> bool {
        if ((++spins & 63u) != 0u) return false;
        if (LW[L_BAD / 4] != 0u) return true;
        const u64 t = now();
        if (t0 == 0) t0 = t;
        if (t - t0 > DEADLINE || __builtin_amdgcn_raw_buffer_load_b32(rs_ws, OFF_TMO, 0, AGT) == seq) {
            u32 sq = seq;
            asm volatile("" : "+s"(sq));
            __builtin_amdgcn_raw_buffer_store_b32(sq, rs_ws, OFF_TMO, 0, AGT);
            LW[L_BAD / 4] = 1u;
            return true;
        }
        return false;
    };

    auto arrive = [&](u32 wm, u32 n) {
        if (lane == 0) {
            __hip_atomic_fetch_max((LAS u32*)(L + L_AMAX), wm, __ATOMIC_RELAXED, __HIP_MEMORY_SCOPE_WORKGROUP);
            const u32 old = __hip_atomic_fetch_add((LAS u32*)(L + L_ACNT), 1u, __ATOMIC_RELAXED, __HIP_MEMORY_SCOPE_WORKGROUP);
            if (old == n - 1u) {
                u32 a = LW[L_AMAX / 4];
                if (LW[L_BAD / 4] != 0u) a = 0xFFFFFFFFu;
                __builtin_amdgcn_raw_buffer_store_b64(u32x2{a, seq}, rs_ws, OFF_AMX + b * 8u, 0, AGT);
            }
        }
    };

    if (b < 128u && wid < 8u) {
        u32 c = lane & 15u, g = lane >> 4;
        const u32 mw = wid & 3u, nh = wid >> 2;
        for (u32 f0 = 0; f0 < fills; f0 += 128u) {
        f32x16 acc[8];
        u32 xa = mw * 4096u + (c * 4u + (g ^ ((c >> 2) & 2u))) * 16u, wa = 16384u + nh * 8192u + lane * 16u;
        const u32 ca = lane < 32u ? L_FULL + ((lane - 1u) & 3u) * 4u : lane < 48u ? L_FREE + (wid ^ 4u) * 4u : L_FREE + wid * 4u - 768u;
        u32 f2 = f0 + 1u, spin = 0, tv;
#define S0_(x) #x
#define S_(x) S0_(x)
#define R4(r) "v[" S_(r) ":" S_(r) "+3]"
#define MM_(j, m, w, x) "v_mfma_f32_16x16x32_bf16 " R4(16*j+4*m) ", " R4(w) ", " R4(x) ", " R4(16*j+4*m) "\n"
#define MM4(j, w, x0) MM_(j, 0, w, x0) MM_(j, 1, w, 132) MM_(j, 2, w, 136) MM_(j, 3, w, 140)
#define RW(r, o) "ds_read_b128 " R4(r) ", %[wa] offset:" S_(o) "\n"
#define RX(r, o) "ds_read_b128 " R4(r) ", %[xa] offset:" S_(o) "\n"
#define LG(n) "s_waitcnt lgkmcnt(" S_(n) ")\n" KPAD
#define KWAIT(l) \
    "8" l ":\n s_cmp_eq_u32 %[f2], %[last]\n s_cbranch_scc1 9" l "b\n ds_read_b32 %[tv], %[ca]\n s_waitcnt lgkmcnt(0)\n" \
    "v_cmp_le_u32 vcc, %[f2], %[tv]\n s_nop 4\n s_bitcmp1_b32 vcc_lo, %[f2]\n s_cbranch_scc1 9" l "b\n s_sleep 1\n" \
    "s_add_u32 %[spin], %[spin], 1\n s_cmp_lt_u32 %[spin], 0x1000000\n s_cbranch_scc1 8" l "b\n s_branch 9" l "b\n"
#define KFILL(l, x0, y0, cur, nxt, TOGGLE) \
    LG(5) MM_(0, 0, 144, x0) MM_(1, 0, 148, x0) LG(4) MM_(0, 1, 144, 132) MM_(1, 1, 148, 132) \
    LG(3) MM_(0, 2, 144, 136) MM_(1, 2, 148, 136) LG(2) MM_(0, 3, 144, 140) RW(144, cur+4096) MM_(1, 3, 148, 140) RW(148, cur+5120) \
    "ds_read_b32 %[tv], %[ca]\n" \
    "s_waitcnt lgkmcnt(4)\n s_add_u32 %[f2], %[f2], 1\n" MM4(2, 152, x0) RW(152, cur+6144) \
    LG(4) MM4(3, 156, x0) RW(156, cur+7168) \
    "s_waitcnt lgkmcnt(2)\n v_cmp_le_u32 vcc, %[f2], %[tv]\n" MM_(4, 0, 144, x0) \
    "v_mov_b32_e64 %[tv], %[f2]\n" MM_(4, 1, 144, 132) \
    "ds_write_b32 %[ca], %[tv] offset:768\n"         \
    MM_(4, 2, 144, 136) TOGGLE MM_(4, 3, 144, 140) \
      \
    "s_setprio " S_(CPRIO) "\n s_bitcmp0_b32 vcc_hi, 0\n s_cbranch_scc1 7" l "f\n s_setprio " S_(CPRIO+BALANCE) "\n7" l ":\n" \
    "s_bitcmp0_b32 vcc_lo, %[f2]\n s_cbranch_scc1 8" l "f\n9" l ":\n" \
    RW(144, nxt) RX(y0, nxt) MM4(5, 148, x0) RW(148, nxt+1024) \
    LG(4) MM_(6, 1, 152, 132) MM_(7, 1, 156, 132) RX(132, nxt+1024) MM_(6, 2, 152, 136) MM_(7, 2, 156, 136) RX(136, nxt+2048) \
    MM_(6, 3, 152, 140) MM_(7, 3, 156, 140) RX(140, nxt+3072) MM_(6, 0, 152, x0) RW(152, nxt+2048) MM_(7, 0, 156, x0) RW(156, nxt+3072)
        asm volatile(
            ".set i_, 0\n .rept 64\n v_mov_b64 v[i_:i_+1], 0\n .set i_, i_+2\n .endr\n"
            "s_branch 80f\n90:\n"
            RW(144, 0) RX(128, 0) RW(148, 1024) RX(132, 1024) RX(136, 2048) RX(140, 3072) RW(152, 2048) RW(156, 3072)
            ".p2align 3\n1:\n"
            KFILL("1", 128, 160, 0, 32768, "")
            KFILL("2", 160, 128, 32768, 0, "v_xor_b32 %[wa], 0x10000, %[wa]\n v_xor_b32 %[xa], 0x10000, %[xa]\n")
            "s_cmp_lt_u32 %[f2], %[last]\n s_cbranch_scc1 1b\n s_branch 2f\n"
            KWAIT("0") KWAIT("1") KWAIT("2")
            "2:\n s_waitcnt lgkmcnt(0)\n s_nop 7\n s_nop 7\n"
            : "={v[0:15]}"(acc[0]), "={v[16:31]}"(acc[1]), "={v[32:47]}"(acc[2]), "={v[48:63]}"(acc[3]), "={v[64:79]}"(acc[4]),
              "={v[80:95]}"(acc[5]), "={v[96:111]}"(acc[6]), "={v[112:127]}"(acc[7]),
              [xa] "+{v164}"(xa), [wa] "+{v165}"(wa), [tv] "={v166}"(tv), [f2] "+s"(f2), [spin] "+s"(spin)
            : [ca] "{v167}"(ca), [last] "s"(f0 + 129u)
            : "v128", "v129", "v130", "v131", "v132", "v133", "v134", "v135", "v136", "v137", "v138", "v139", "v140", "v141", "v142",
              "v143", "v144", "v145", "v146", "v147", "v148", "v149", "v150", "v151", "v152", "v153", "v154", "v155", "v156", "v157",
              "v158", "v159", "v160", "v161", "v162", "v163", "vcc", "scc", "memory");
        if (spin >= 0x1000000u) LW[L_BAD / 4] = 1u;
        asm volatile("v_mbcnt_lo_u32_b32 %0, -1, 0\n v_mbcnt_hi_u32_b32 %0, -1, %0" : "=v"(lane));
        c = lane & 15u; g = lane >> 4;
        STAMPV(f0 == 0u ? 14 : 15, (u64)spin << 24);
        const u32 reg = L_PACK + wid * 2048u, slot = E2_OFF + (b * 2u + nh) * 32768u + mw * R_VAR;
        auto post = [&](u32 cnt, u32 at) {
            u32 old = 0;
            VMWAIT(0);
            if (lane == 0) old = __hip_atomic_fetch_add((LAS u32*)(L + cnt + nh * 4u), 1u, __ATOMIC_RELAXED, __HIP_MEMORY_SCOPE_WORKGROUP);
            if (RFL(old) == 3u && lane - 1u < 7u) {
                const u32x4 fv = u32x4{seq, LW[L_GSZ / 4 + nh * 2u], LW[L_GSZ / 4 + nh * 2u + 1u], seq};
                u32x2 fp = *(volatile LAS u32x2*)(L + L_PTR + lane * 8u);
                asm volatile("v_add_co_u32 %0, vcc, %0, %2\n v_addc_co_u32 %1, vcc, 0, %1, vcc" : "+v"(fp[0]), "+v"(fp[1]) : "v"(at) : "vcc");
                asm volatile("global_store_dwordx4 %0, %1, off sc0 sc1\n s_nop 1" :: "v"(fp), "v"(fv) : "memory");
            }
        };
        if (f0 == 0u) {
            STAMP(2);
            STAMPV(9, __builtin_amdgcn_s_memtime());
            u32x4 sg;
            if (SPRIO == 1) { if (wid >= 4u) __builtin_amdgcn_s_setprio(2); else __builtin_amdgcn_s_setprio(0); }
            if (SPRIO == 2) __builtin_amdgcn_s_setprio(3);
#pragma unroll
            for (int m = 0; m < 4; ++m) {
                u32 d = 0;
#pragma unroll
                for (int i = 31; i >= 0; --i) d = __builtin_amdgcn_alignbit(d, __float_as_uint(acc[(i & 4) + (i >> 3)][4 * m + (i & 3)]), 31);
                const u32 o = (u32)__builtin_amdgcn_mov_dpp((int)d, 0x128, 0xf, 0xf, false);
                sg[m] = bfi(c < 8u ? 0x0F0F0F0Fu : 0xF0F0F0F0u, d, __builtin_amdgcn_alignbit(o, o, c < 8u ? 28u : 4u));
            }
            ST128(sg, rs_ws, lane * 16u, (SGM >> (q * 2u + nh)) & 1u ? E3_OFF + (b * 8u + wid) * 1024u : slot, AGT);
            if (SPRIO == 2) __builtin_amdgcn_s_setprio(0);
            float mx = 0.f;
#pragma unroll
            for (int j = 0; j < 8; ++j)
#pragma unroll
                for (int m = 0; m < 4; ++m) {
                    mx = fmaxf(fmaxf(mx, fabsf(acc[j][4 * m])), fabsf(acc[j][4 * m + 1]));
                    mx = fmaxf(fmaxf(mx, fabsf(acc[j][4 * m + 2])), fabsf(acc[j][4 * m + 3]));
                }
            arrive(wave_max_u32(__float_as_uint(mx)), 8u);
            STAMP(3);
            if (wid >= 4u) __builtin_amdgcn_s_setprio(2); else __builtin_amdgcn_s_setprio(0);
#pragma unroll
            for (int k = 0; k < 2; ++k) *(LAS u32x4*)(L + reg + k * 1024u + lane * 16u) = u32x4{0u, 0u, 0u, 0u};
            if ((SGM >> (q * 2u + nh)) & 1u) post(L_SCNT, OFF_F3 + ((lane * 16u + idx) * 16u + q * 2u + nh) * 64u);
            STAMP(23);
            while (LW[L_SCALE / 4] != 1u) { if (expired()) break; __builtin_amdgcn_s_sleep(CSLEEP); }
            CBAR();
        } else STAMP(13);
        const u32 gmax = LW[L_SCALE / 4 + 1];
        const bool bad = (gmax == 0xFFFFFFFFu) || (LW[L_BAD / 4] != 0u);
        const float amax = __uint_as_float(gmax);
        const float inv = amax > 0.f ? 127.0f / amax : 0.f;
        if (f0 == 0u) STAMP(4);
        const float MAGIC = 12582912.0f;
        const float qlo = bad ? MAGIC - 128.0f : MAGIC - 127.0f, qhi = bad ? MAGIC - 128.0f : MAGIC + 127.0f;
        const u32 ovoff = (c & 7u) * NN + ((c >> 3) * 4u + g) * 16u;
        const u32 os = (nb2 - 8u) * 2u + nh;
        const u32 osoff = (mb * 256u + mw * 64u) * NN + (f0 == 0u ? rank * 2048u + nb * 256u + nh * 128u : ((rank + 1u + os) & 7u) * 2048u + 1792u);
        u32x4 qv[8];
#pragma unroll
        for (int m = 0; m < 4; ++m) {
#pragma unroll
            for (int r = 0; r < 4; ++r) {
                u32 o[2];
#pragma unroll
                for (int h = 0; h < 2; ++h) {
                    u32 t[4];
#pragma unroll
                    for (int jj = 0; jj < 4; ++jj)
                        t[jj] = __float_as_uint(__builtin_amdgcn_fmed3f(__builtin_fmaf(acc[4 * h + jj][4 * m + r], inv, MAGIC), qlo, qhi));
                    o[h] = PERM(t[1], t[0], 0x0c0c0400u) | PERM(t[3], t[2], 0x04000c0cu);
                }
                qv[2 * m][r] = (u32)__builtin_amdgcn_update_dpp((int)o[0], (int)o[1], 0x128, 0xf, 0xc, false);
                qv[2 * m + 1][r] = (u32)__builtin_amdgcn_update_dpp((int)o[1], (int)o[0], 0x128, 0xf, 0x3, false);
            }
        }
        if (f0 == 0u) {
        STAMP(7);
        u32 size = nb == 7u && nh == 0u ? 0u : 8192u;
        u32x4 rest[2] = {qv[6], qv[7]};
        auto tell = [&] { if (lane == 0) *(volatile LAS unsigned short*)(L + L_GSZ + wid * 2u) = (unsigned short)(size >> 4); };
        if (size != 0u && !bad && !(FORCE_RAW2 && (wid & 1u))) {
            u32x4 nib[4], pl = u32x4{0u, 0u, 0u, 0u};
            u32 code[16], len[16], over = 0, tot = 0;
#pragma unroll
            for (int o = 0; o < 16; ++o) {
                u32 y[2], l[2];
#pragma unroll
                for (int e = 0; e < 2; ++e) {
                    const u32 v = qv[o >> 1][(2 * o + e) & 3], s = byte_signs(v);
                    y[e] = (v ^ s) + ((s & 0x01010101u) | 0x70707070u);
                    l[e] = (y[e] & byte_signs(y[e]) & 0x78787878u) >> 3;
                }
                const u32 h = bfi(0x0F0F0F0Fu, y[0] >> 4, y[1]);
                nib[o >> 2][o & 3] = bfi(0x0F0F0F0Fu, y[0], y[1] << 4) | (((h & 0x77777777u) + 0x77777777u) & h & 0x88888888u);
                pl[o >> 2] = bfi(0x88888888u >> (o & 3), h >> (o & 3), pl[o >> 2]);
                len[o] = unary8(l[0], l[1], code[o]);
                over |= len[o];
                tot += len[o] + ((u32)__builtin_popcount(code[o]) << 16);
            }
            const u32 end = wave_scan_u32(tot);
            const u32 all = (u32)__builtin_amdgcn_readlane((int)end, 63), bits = all & 0xFFFFu;
            if (BALLOT(over > 31u) == 0ull && bits <= (8192u - 64u - R_VAR) * 8u && (all >> 16) <= R_MAX) {
                size = (R_VAR + ((bits + 127u) >> 7) * 16u);
                tell();
                ST128(pl, rs_ws, lane * 16u, slot + R_BIT, AGT);
#pragma unroll
                for (int e = 0; e < 4; ++e) ST128(nib[e], rs_ws, lane * 16u + e * 1024u, slot + R_NIB, AGT);
                u32 pos = ((end - tot) & 0xFFFFu) + reg * 8u;
#pragma unroll
                for (int o = 0; o < 16; ++o) { put_bits(L, pos, code[o]); pos += len[o]; }
                CBAR();
                rest[0] = LDSV(reg + lane * 16u);
                rest[1] = LDSV(reg + 1024u + lane * 16u);
            }
        }
        if (size == 8192u) {
            tell();
#pragma unroll
            for (int e = 0; e < 6; ++e) ST128(qv[e], rs_ws, lane * 16u, slot + e * 1024u, AGT);
        }
        if (size != 0u) {
            u32 at = mw * 128u;
            if ((SGM >> (q * 2u + nh)) & 1u) {
                do at = lane < mw ? *(volatile LAS unsigned short*)(L + L_GSZ + (nh * 4u + lane) * 2u) : R_VAR / 16u;
                while (BALLOT(at == 0u) != 0ull && !expired());
                at = (u32)(__builtin_amdgcn_readlane((int)at, 0) + __builtin_amdgcn_readlane((int)at, 1) + __builtin_amdgcn_readlane((int)at, 2)) - 3u * R_VAR / 16u;
            }
#pragma unroll
            for (int k = 0; k < 2; ++k) {
                const u32 off = R_VAR + k * 1024u + lane * 16u;
                if (R_VAR + k * 1024u < size) ST128(rest[k], rs_ws, off < size ? off : OOB, slot + (3u - mw) * R_VAR + min(at, 384u) * 16u, AGT);
            }
        }
        STAMP(11);
        }
        const bool first = f0 == 0u && (nh != 0u || (q & 1u) == 0u);
        if (first) { post(L_GCNT, OFF_F2 + ((lane * 16u + idx) * 16u + q * 2u + nh) * 16u); STAMP(5); __builtin_amdgcn_s_setprio(0); }
        const rsrc_t rs_ol = mk_rsrc(p.out[0], f0 == 0u || os < 7u ? OUT_BYTES : 0u);
#pragma unroll
        for (int e = 0; e < 8; ++e) ST128(qv[e], rs_ol, ovoff, osoff + e * 8u * NN, AGT);
        if (f0 == 0u && !first) { post(L_GCNT, OFF_F2 + ((lane * 16u + idx) * 16u + q * 2u + nh) * 16u); STAMP(5); }
        }
    } else if (b < 128u && wid == 8u) {
        __builtin_amdgcn_s_setprio(LPRIO);
        *(volatile LAS u64*)(L + L_PTR + (lane & 7u) * 8u) = (u64)p.ws[lane & 7u];
        for (u32 n = 0; n < fills / 2u;) {
            const u32 wn = n == 0u ? nb : nb2, end = n + 64u;
            const bool hole = (wn == 11u && (lane & 5u) == 5u) || lane < 4u;
            u32 step = lane & 56u, va = OFF_UF + (step * 112u + (lane & 4u ? 64u + wn * 4u : mb * 4u) + (lane & 3u)) * 128u;
            bool seen = hole;
            while (n != end) {
                while (2u * n > LW[L_FREE / 4] + PAHEAD) { if (expired()) break; __builtin_amdgcn_s_sleep(8); }
                seen |= __builtin_amdgcn_raw_buffer_load_b32(rs_ws, seen ? OOB : va, 0, SYS) >= seq;
                u64 avail = BALLOT(seen);
                avail &= avail >> 1; avail &= avail >> 2; avail &= avail >> 4;
                avail &= 0x0101010101010101ull;
                if (avail == 0) {
                    if (!expired()) { __builtin_amdgcn_s_sleep(POLL_SLEEP); continue; }
                    avail = BALLOT(va != OOB) & 0x0101010101010101ull;
                }
                if ((avail >> lane) & 1ull) LW[L_STEPS / 4 + n + rank_in(avail)] = step;
                n += (u32)__builtin_popcountll(avail);
                LW[L_NRDY / 4] = n;
                if ((avail >> (lane & 56u)) & 1ull) {
                    seen = hole;
                    va = (++step & 7u) ? va + 112u * 128u : OOB;
                }
            }
            if (n != 64u) break;
            STAMP(1);
            while (LW[L_FREE / 4] < 128u) { if (expired()) break; __builtin_amdgcn_s_sleep(8); }
            u32 a = 0;
            for (;;) {
                const u32x2 gm = __builtin_amdgcn_raw_buffer_load_b64(rs_ws, OFF_GAM + q * 128u + (lane & 7u) * 8u, 0, SYS);
                a = gm[0];
                if (BALLOT(gm[1] == seq) == ~0ull) break;
                if (expired()) { a = 0xFFFFFFFFu; break; }
                __builtin_amdgcn_s_sleep(1);
            }
            a = wave_max_u32(a);
            LW[L_SCALE / 4 + 1] = a;
            LW[L_SCALE / 4] = 1u;
            STAMP(2);
            while (LW[L_GCNT / 4] + LW[L_GCNT / 4 + 1] < 8u) { if (expired()) break; __builtin_amdgcn_s_sleep(8); }
        }
    } else if (b < 128u) {
#define CLK(c)
        __builtin_amdgcn_s_setprio(LPRIO);
        for (;;) {
            u32 f = 0;
            if (lane == 0) f = __hip_atomic_fetch_add((LAS u32*)(L + L_NEXT), 1u, __ATOMIC_RELAXED, __HIP_MEMORY_SCOPE_WORKGROUP);
            f = RFL(f);
            if (f >= fills) break;
            CLK(c0);
            while (LW[L_NRDY / 4] <= (f >> 1)) { if (expired()) break; __builtin_amdgcn_s_sleep(1); }
            CBAR();
            const u32 step = RFL(LW[L_STEPS / 4 + (f >> 1)]) & 63u;
            CLK(c1);
            bool ahead = false;
            while (BALLOT(LW[L_FREE / 4 + (lane & 7u)] + 2u >= f) != ~0ull && f >= 4u) {
                ahead = true;
                if (expired()) break;
                __builtin_amdgcn_s_sleep(LSLEEP);
            }
            CBAR();
            const u32 sbase = STAGE_OFF + step * STEP_BYTES + (f & 1u) * 16384u;
            LAS char* ldst = L + (f & 3u) * SLOT_BYTES;
            CLK(c2);
            auto fill = [&](auto pace) {
                if (step < 8u) {
                    const rsrc_t rs_x = mk_rsrc(px, X_BYTES);
                    const u32 so = mb * 262144u + step * 128u + (f & 1u) * 64u;
#pragma unroll
                    for (int i = 0; i < 4; ++i) {
                        __builtin_amdgcn_raw_ptr_buffer_load_lds(rs_x, ldst + i * 4096, 16, x_at(lane), so + i * 65536u, 0, XAUX); pace();
                        __builtin_amdgcn_raw_ptr_buffer_load_lds(rs_x, ldst + i * 4096, 16, x_at(lane), so + i * 65536u + 15360u, 1024, XAUX); pace();
                        __builtin_amdgcn_raw_ptr_buffer_load_lds(rs_x, ldst + i * 4096, 16, x_at(lane), so + i * 65536u + 30720u, 2048, XAUX); pace();
                        __builtin_amdgcn_raw_ptr_buffer_load_lds(rs_x, ldst + i * 4096, 16, x_at(lane), so + i * 65536u + 46080u, 3072, XAUX); pace();
                    }
                }
#pragma unroll
                for (int i = 7; i >= 0; --i) {
                    if (i == 3 && step < 8u) break;
                    const u32 so = sbase + (i < 4 ? mb : 16u + (f < 128u ? nb : nb2)) * PIECE + (i & 3) * 4096u;
                    __builtin_amdgcn_raw_ptr_buffer_load_lds(rs_ws, ldst + i * 4096, 16, lane * 16u, so, 0, SYS); pace();
                    __builtin_amdgcn_raw_ptr_buffer_load_lds(rs_ws, ldst + i * 4096, 16, lane * 16u, so, 1024, SYS); pace();
                    __builtin_amdgcn_raw_ptr_buffer_load_lds(rs_ws, ldst + i * 4096, 16, lane * 16u, so, 2048, SYS); pace();
                    __builtin_amdgcn_raw_ptr_buffer_load_lds(rs_ws, ldst + i * 4096, 16, lane * 16u, so, 3072, SYS); pace();
                }
            };
            if (ahead || !LADAPT) fill([] { asm volatile("s_nop %0" :: "n"(LNOP) : "memory"); });
            else fill([] { });
            CBAR();
            CLK(c3);
            VMWAIT(0);
            CLK(c4);
            LW[L_FULL / 4 + (f & 3u)] = f + 1u;
            if (f - 125u < 3u) STAMP(1);
        }
    } else if (b < 240u) {
        arrive(0u, 12u);
        const u32 src = (rank - rel) & 7u;
        const rsrc_t rs_src = mk_rsrc(p.ws[(8u - rel) & 7u], WS_NEED);
        auto in_unit = [&](u32 u, u32& ch, u32& r, bool& isw) -> u32 {
            if (u >= 768u) {
                ch = (u - 768u) / 14u; r = (u - 768u) % 14u; isw = true;
                const u32 os = r >> 1, rs = 28u + (r & 1u) * 2u;
                r = strip_at(os, rs);
                return E1W_OFF + ((((rel + 1u + os) & 7u) * 8u + ch) * 32u + rs) * E1_SLOT;
            }
            ch = u / 96u; r = u % 96u;
            if ((XFIRST >> ch) & 1u) r = r < 64u ? r + 32u : r - 64u;
            isw = r < 32u;
            if (!isw) r = (r - 32u + 8u * rel) & 63u;
            return !isw ? E1X_OFF + (ch * 64u + r) * E1_SLOT : PUSHED(ch, r) ? E1P_OFF + ((rel * 2u + ch) * 32u + r) * E1_SLOT
                                                                         : E1W_OFF + ((rel * 8u + ch) * 32u + r) * E1_SLOT;
        };
        auto out_unit = [&](u32 o) -> u32 {
            const u32 k = o >> 4, j = o - 768u;
            if (k >= 48u) return j % 12u * 64u + (j / 48u) * 16u + 8u + j / 12u % 4u;
            return (o & 15u) * 64u + (k < 32u ? (k >> 2) * 8u + 4u : ((k >> 2) & 3u) * 16u) + (k & 3u);
        };
        auto out_at = [&](u32 t, u32 lk) -> u32 {
            const u32 x = (t >> 3) & 7u, ix = (t >> 6) & 15u;
            return ((4u * (x >> 1) + (ix & 3u)) * 256u + (t & 3u) * 64u) * NN + ((rank - lk) & 7u) * 2048u + (4u * (x & 1u) + (ix >> 2)) * 256u + ((t >> 2) & 1u) * 128u;
        };
        const u32 rows = (lane >> 3) * NN + (lane & 7u) * 16u, base = seq << 12;
        auto in_size = [](u32 h) -> u32 {
            h = ((h >> 4) & 15u) | ((h >> 8) & 0xF0u);
            return h > 127u ? E1_SLOT / 16u : I_HDR / 16u + h;
        };
        if (wid == 0u) {
            __builtin_amdgcn_s_setprio(3);
            auto ctl = [&](u32 a) -> u32 {
                u32 v;
                asm volatile("ds_read_b32 %0, %1\n s_waitcnt lgkmcnt(0)" : "=v"(v) : "v"(a) : "memory");
                return v;
            };
            auto put = [&](u32 a, u32 n) { asm volatile("ds_write_b32 %0, %1" :: "v"(a), "v"(n) : "memory"); };
            const u32 ca = L_FULL + (lane & 15u) * 4u;
            const u32 vone = lane == 0u ? 0u : OOB;
            u32 n = 0, w6;
            auto wait = [&](int word, u32 bit, bool four = false) {
                auto ok = [&](u32 w) -> bool {
                    w6 = (u32)__builtin_amdgcn_readlane((int)w, 6);
                    const u32 k = n + ((lane - n) & 7u);
                    if (four && (BALLOT(k < n + 4u && k >= 8u && (int)(w + 7u - k) < 0) & 0xFF00u) != 0u) return false;
                    return (((u32)__builtin_amdgcn_readlane((int)w, word) >> bit) & 1u) != 0u &&
                           (u32)__builtin_amdgcn_readlane((int)w, 8 + (n & 7u)) + (n < (word == 1 ? 16u : 8u) ? 15u : 7u) >= n;
                };
                if (!ok(ctl(ca))) {
                    VMWAIT(0);
                    put(L_FULL, n);
                    while (!ok(ctl(ca))) { if (expired()) break; __builtin_amdgcn_s_sleep(1); }
                }
            };
#define CLAIM(cl, so) asm volatile("v_mov_b32 %0, 1\n s_nop 4\n buffer_atomic_add %0, %1, %2, %3 offen sc0 sc1" \
                                   : "=&v"(cl) : "v"(vone), "s"(rs_ws), "s"(so) : "memory")
#define CLAIMED(d, cl) asm volatile("v_readfirstlane_b32 %0, %1" : "=s"(d) : "v"(cl) : "memory")
            for (u32 u = 32u + pj; u < N1; ++n) {
                u32 ch, r, cl; bool isw;
                const u32 so = in_unit(u, ch, r, isw);
                if (u >= 768u) put(L_P1, 1u);
                const u32 seg = u >= 768u ? 16u : n < 4u ? 24u + n : ch * 2u + !isw;
                wait(1, seg);
                if (n == 0) STAMP(1);
                LAS char* ld = L + RING + (n < 16u ? n : n & 7u) * E1_SLOT;
                u32 sz = 0;
                put(L_Q + (n & 15u) * 4u, u);
                CLAIM(cl, n < 3u ? OOB : OFF_CTR + (64u + rel) * 128u);
                static_assert(WIN1 <= 4, "the header, read first, has landed behind the fifth read");
#pragma unroll
                for (int i = 0; i < 9; ++i) {
                    if (i == 5) sz = in_size(RFL(ctl(RING + (n < 16u ? n : n & 7u) * E1_SLOT + I_HDR)));
                    if (i >= 6 && i * 64u >= sz) break;
                    VMWAIT(WIN1 - (i < WIN1 - 1 ? 0 : 1));
                    if (i == WIN1 - 1) put(L_FULL, n);
                    const int k = i > 4 ? i : (i + 4) % 5;
                    if (i < 5 || i * 64u + lane < sz)
                        __builtin_amdgcn_raw_ptr_buffer_load_lds(rs_src, ld + k * 1024, 16, lane * 16u, so + k * 1024u, 0, SYS);
                }
                u32 c;
                CLAIMED(c, cl);
                c -= (seq - 1u) * (N1 - 80u - 16u * P1);
                u = n < 3u ? u + 16u : c + (c < 16u || !P1 ? 96u : 112u);
            }
            put(L_P1, 1u);
            STAMP(2);
            for (u32 got = 0, i = 0, tries = 0; got != SGM;) {
                const u32 w = ctl(ca);
                const u32 m = (u32)__builtin_amdgcn_readlane((int)w, 5) & ~got;
                if (m == 0u || (n < 24u && (BALLOT((int)w + 8 > (int)n) >> 8 & 255u) != 255u)) {
                    if (++tries < STRY) continue;
                    VMWAIT(0);
                    put(L_FULL, n);
                    if (expired()) break;
                    __builtin_amdgcn_s_sleep(1);
                    continue;
                }
                if (got == 0u) STAMP(23);
                const u32 t = (u32)__builtin_ctz(m);
                got |= 1u << t;
                tries = 0;
#pragma unroll
                for (int e = 0; e < 4; ++e) {
                    VMWAIT(WINS - 1);
                    if (++i == WINS) { put(L_FULL, n); put(L_P1, 2u); }
                    __builtin_amdgcn_raw_ptr_buffer_load_lds(rs_src, L + L_SG + (t * 4u + e) * 1024u, 16, lane * 16u,
                                                             E3_OFF + (pj * 64u + t * 4u + e) * 1024u, 0, SYS);
                }
            }
            STAMP(24);
            put(L_FULL, n);
            put(L_P1, 2u);
            while ((BALLOT((int)ctl(ca) + 8 > (int)n) >> 8 & 255u) != 255u) if (expired()) break;
            static_assert(S2 == 48u && N2 == 60u, "claimed units");
            const char* wsrc = p.ws[(8u - rel) & 7u];
            for (u32 o = pj, cur = rel, vict = 0, m = 0; o < 960u; ++n, ++m) {
                const u32 j = o + 24u * cur - (o + 24u * cur < 960u ? 0u : 192u), t = out_unit(o < 768u ? o : j);
                u32 cl, c;
                wait(2, o < 768u ? (t >> 2) & 15u : 16u, (m & 3u) == 0u);
                const bool dyn = o >= 752u, raw = o + ((int)w6 >> 8) >= 960u - NRAW || cur != rel;
                if (o == pj) STAMP(3);
                if (o >= 768u + SFROM) put(L_P1, 3u);
                u32 vs = ctl(L_SZ + (t >> 2) * 8u + (lane & 2u) * 2u) >> ((lane & 1u) * 16u) & 0xFFFFu;
                vs = min(max(vs, R_VAR / 16u), 512u) - R_VAR / 16u;
                const u32 v0 = (u32)__builtin_amdgcn_readlane((int)vs, 0), v1 = v0 + (u32)__builtin_amdgcn_readlane((int)vs, 1),
                          v2 = v1 + (u32)__builtin_amdgcn_readlane((int)vs, 2), v3 = v2 + (u32)__builtin_amdgcn_readlane((int)vs, 3);
                const u32 own = raw ? 128u : (u32)__builtin_amdgcn_readlane((int)vs, t & 3u), sz = R_VAR / 16u + own;
                const u32 more = o >= 768u ? own : (t & 3u) == 0u ? v3 : 0u;
                const u32 lv = RING + 8u * R_VAR + (m & 4u) * 2048u + (o >= 768u ? (m & 3u) * 128u : (t & 2u ? (t & 1u ? v2 : v1) : (t & 1u ? v0 : 0u))) * 16u;
                const u32 skip = sz < 512u && o < 768u ? 1024u : 0u;
                put(L_Q + (n & 15u) * 4u, 1024u + t + skip * 2u + (raw ? 4096u : 0u) + (cur << 13) + (sz << 16) + (o < 768u ? 0u : (j - 744u) / 24u << 26));
                put(L_Q2 + (n & 15u) * 4u, lv);
                LAS char* ld = L + RING + (n & 7u) * R_VAR + skip;
                const rsrc_t rs = mk_rsrc(raw ? p.out[(8u - (o < 936u ? cur : rel)) & 7u] : wsrc, raw ? OUT_BYTES : WS_NEED);
                const u32 so = raw ? out_at(t, cur) : E2_OFF + (t >> 2) * 32768u + (t & 3u) * R_VAR + skip, pitch = raw ? 8u * NN : 1024u, vo = raw ? rows : lane * 16u;
                const u32 sv = raw ? so + 6u * pitch : E2_OFF + (t >> 2) * 32768u + 4u * R_VAR + (t & 3u) * 2048u;
                bool again = false;
                CLAIM(cl, dyn ? OFF_CTR + (72u + cur) * 128u : OOB);
#pragma unroll
                for (int i = 0; i < 6; ++i) {
                    if (i == 5 && skip != 0u) break;
                    VMWAIT(WIN2 - (i < WIN2 - 1 ? 0 : 1));
                    if (i == WIN2 - 1) {
                        put(L_FULL, n);
                        CLAIMED(c, cl);
                        o = !dyn ? o + 16u : 768u + min(c - base, 192u);
                        if (STEAL && o == 960u) {
                            if (cur == rel) vict = w6 & 63u;
                            cur = vict & 7u;
                            vict >>= 3;
                            again = cur != 0u;
                            if (again) CLAIM(cl, OFF_CTR + (72u + cur) * 128u);
                        }
                        if (STEAL && ((o - 768u - SMARK) & ~48u) == 0u && o != 768u + SMARK)
                            __builtin_amdgcn_raw_buffer_store_b32((u32)now(), rs_ws, lane < 8u ? lane * 128u : OOB, OFF_RF + cur * 16u + (o - 768u - SMARK) / 4u, SYS);
                    }
                    __builtin_amdgcn_raw_ptr_buffer_load_lds(rs, ld + i * 1024, 16, vo, so + i * pitch, 0, SYS);
                }
#pragma unroll 1
                for (u32 i = 0; i * 64u < more; ++i) {
                    VMWAIT(WIN2 - 1);
                    if (i * 64u + lane < more) __builtin_amdgcn_raw_ptr_buffer_load_lds(rs, L + lv + i * 1024u, 16, vo, sv + i * pitch, 0, SYS);
                }
                if (again) {
                    VMWAIT(2);
                    CLAIMED(c, cl);
                    o = 768u + min(c - base, 192u);
                }
            }
            put(L_P1, 4u);
            VMWAIT(0);
            put(L_FULL, n);
            put(L_END, 1u);
            STAMP(4);
        } else if (wid == 1u) {
            const u32 fa = (lane & 31u) < 20u ? OFF_F1 + rel * 256u + (lane & 31u) * 8u : OFF_FP + (rel * 16u + pj) * 128u + ((lane - 20u) & 7u) * 8u;
            for (u32 m = 0; m != HMASK;) {
                const u32x2 f = __builtin_amdgcn_raw_buffer_load_b64(rs_ws, fa, 0, SYS);
                m = (u32)BALLOT(f[0] >= seq) & HMASK;
                LW[L_PSZ / 4 + (lane & 31u)] = f[1];
                if (expired()) m = HMASK;
                LW[L_SEG / 4] = m;
                __builtin_amdgcn_s_sleep(4);
            }
            STAMP(1);
            while (LW[L_P1 / 4] == 0u) { if (expired()) break; __builtin_amdgcn_s_sleep(16); }
            u32 m = 0;
            while (m != SGM) {
                m |= (u32)BALLOT(__builtin_amdgcn_raw_buffer_load_b32(rs_ws, lane < 16u && ((SGM & ~m) >> lane & 1u) ? lane * 64u : OOB,
                                                                      OFF_F3 + (rel * 16u + pj) * 1024u, SYS) == seq);
                if (expired()) m = SGM;
                LW[L_G3 / 4] = m;
                __builtin_amdgcn_s_sleep(2);
            }
            while (LW[L_P1 / 4] < 2u) { if (expired()) break; __builtin_amdgcn_s_sleep(2); }
            if (lane == 0) __builtin_amdgcn_raw_buffer_store_b32(seq, rs_src, OFF_DONE + rank * 128u + pj * 4u, 0, SYS);
            m = 0;
            while (m != 0xFFFFu) {
                const u32x4 f = LD128(rs_ws, lane < 16u ? lane * 16u : OOB, OFF_F2 + (rel * 16u + pj) * 256u, SYS);
                m = (u32)BALLOT(f[0] == seq && f[3] == seq) & 0xFFFFu;
                if (lane < 16u) *(LAS u32x2*)(L + L_SZ + pj * 128u + lane * 8u) = u32x2{f[1], f[2]};
                if (expired()) m = 0xFFFFu;
                CBAR();
                LW[L_G2 / 4] = m;
                __builtin_amdgcn_s_sleep(2);
            }
            STAMP(2);
            for (u32 i = 0; i < 4u;) {
                const u32x4 f = LD128(rs_ws, lane * 16u, OFF_F2 + rel * 4096u + i * 1024u, SYS);
                *(LAS u32x2*)(L + L_SZ + i * 512u + lane * 8u) = u32x2{f[1], f[2]};
                if (BALLOT(f[0] == seq && f[3] == seq) == ~0ull || expired()) ++i; else __builtin_amdgcn_s_sleep(16);
            }
            CBAR();
            LW[L_G2 / 4] = 0x1FFFFu;
            STAMP(3);
            const u32 tr = (u32)now();
            while (STEAL && LW[L_P1 / 4] < 3u) { expired(); __builtin_amdgcn_s_sleep(32); }
            while (STEAL && LW[L_P1 / 4] == 3u) {
                const u32x4 v = LD128(rs_ws, lane < 8u ? lane * 16u : OOB, OFF_RF + rel * 128u, SYS);
                const u32 tn = (u32)now() - tr;
                const u32 at = lane - 1u < 7u ? min(v[1] - tr, tn) + min(v[2] - tr, tn) + min(v[3] - tr, tn) : 0u;
                const u32 key = v[0] == seq ? (at + ((lane * 5u + pj) & 15u) * JIT) * 8u + lane : 0u;
                const u32 k1 = wave_max_u32(key);
                const int late = (__builtin_amdgcn_readlane((int)at, rel) * 7 - __builtin_amdgcn_readlane((int)wave_scan_u32(at), 63)) / LDIV;
                LW[L_VICT / 4] = (k1 & 7u) + (wave_max_u32(key == k1 ? 0u : key) & 7u) * 8u + ((u32)min(max(late, rel < 7u ? -LMIN : 0), 24) << 8);
                __builtin_amdgcn_s_sleep(VSLEEP);
            }
        } else if (wid < 10u) {
            const rsrc_t rs_ol = mk_rsrc(p.out[0], OUT_BYTES);
            const u32 ovoff = (lane & 7u) * NN + (((lane >> 3) & 1u) * 4u + (lane >> 4)) * 16u;
            for (u32 n = wid - (wid - 7u < 2u + P1 ? 10u : 2u);; n += 8u) {
                const bool pre = (int)n < 0;
                bool more = true;
                const u32 k = n + 3u;
                while (pre && ((LW[L_SEG / 4] >> (20u + k)) & 1u) == 0u) __builtin_amdgcn_s_sleep(2);
                while (!pre) {
                    const u32 end = LW[L_END / 4];
                    if (LW[L_FULL / 4] > n) break;
                    if (end != 0u) { more = false; break; }
                    expired();
                    __builtin_amdgcn_s_sleep(2);
                }
                if (!more) break;
                CBAR();
                const u32 id = pre ? (k == 2u ? 112u : k * 16u) + pj + (min(max(RFL(LW[L_PSZ / 4 + 20u + k]), 321u), E1_SLOT / 16u) << 16)
                                   : RFL(LW[L_Q / 4 + (n & 15u)]);
                const u32 sl = RING + (id & 1024u ? (n & 7u) * R_VAR : (n & ((int)n < 16 ? 15u : 7u)) * E1_SLOT);
                if ((id & 1024u) == 0u) {
                u32 ch, r; bool isw;
                const u32 so = in_unit(id & 1023u, ch, r, isw);
                if (pre) {
#pragma unroll 1
                    for (u32 at = 0; at < (id >> 16) * 16u; at += 1024u)
                        if (at + lane * 16u < (id >> 16) * 16u)
                            __builtin_amdgcn_raw_ptr_buffer_load_lds(rs_ws, L + sl + at, 16, lane * 16u, so + at, 0, SYS);
                    VMWAIT(0);
                }
                const u32 hd = sl + I_HDR;
                u32x4 tab = LDSV(hd);
                const u32x4 fw = LDSV(hd + I_FLD + lane * 16u);
                const u32 h0 = RFL(tab[0]), sz = in_size(h0), top = ((h0 >> 20) & 15u) | ((h0 >> 24) & 0xF0u), cnt = (RFL(tab[1]) >> 4) & 15u;
                u32x4 o[8];
                if (sz == E1_SLOT / 16u) {
#pragma unroll
                    for (int e = 0; e < 8; ++e) o[e] = LDSV(sl + (e >> 2) * (I_HDR + 128u) + (e & 3) * 1024u + lane * 16u);
                } else {
                    u32x4 mq[4];
                    u32 ex[16], es[8], at[8], d[8][3], ne = 0;
#pragma unroll
                    for (int g = 0; g < 8; ++g) {
                        es[g] = ((fw[g >> 1] & (fw[g >> 1] << 1)) << (g & 1 ? 0 : 2)) & 0x88888888u;
                        at[g] = ne;
                        ne += (u32)__builtin_popcount(es[g]);
                    }
                    const u32 v0 = (I_HDR + I_VAR) / 8u + cnt;
                    const u32 a0 = unary_list<2>(lane, sl + v0 * 8u, (max(sz * 2u, v0) - v0) * 8u, ne, L_PACK + wid * 2048u);
                    CBAR();
#pragma unroll
                    for (int g = 0; g < 8; ++g) { at[g] += a0; LIST8(d[g], at[g]); }
#pragma unroll
                    for (int e = 0; e < 4; ++e) mq[e] = LDSV(sl + e * 1024u + lane * 16u);
                    __builtin_amdgcn_sched_barrier(0);
                    tab = (tab & 0x0F0F0F0Fu) ^ u32x4{TFLIP0, TFLIP1, TFLIP0, TFLIP1};
#pragma unroll
                    for (int g = 0; g < 8; ++g) {
                        u32 c0, c1;
                        deal8(d[g], at[g], es[g], c0, c1);
                        ex[2 * g] = lut16(tab, ((fw[g >> 1] >> (2 * (g & 1))) & 0x03030303u) + c0);
                        ex[2 * g + 1] = lut16(tab, ((fw[g >> 1] >> (2 * (g & 1) + 4)) & 0x03030303u) + c1);
                    }
#pragma unroll 1
                    for (u32 k = 0; k < cnt; ++k) {
                        const u32 at = LW[(hd + I_VAR) / 4 + 2u * k] - lane * 16u, val = LW[(hd + I_VAR) / 4 + 1u + 2u * k];
#pragma unroll
                        for (int j = 0; j < 16; ++j) { ex[j] = at == (u32)j ? val : ex[j]; if ((j & 3) == 3) __builtin_amdgcn_sched_barrier(0); }
                    }
#pragma unroll
                    for (int j = 0; j < 16; ++j) {
                        const u32 e = top * 0x01010101u - ex[j], m = mq[j >> 2][j & 3];
                        const u32 r0 = PERM(e, m, 0x05010400u), r1 = PERM(e, m, 0x07030602u);
                        o[j >> 1][(2 * j) & 3] = __builtin_amdgcn_alignbit(r0, r0, 1);
                        o[j >> 1][(2 * j + 1) & 3] = __builtin_amdgcn_alignbit(r1, r1, 1);
                    }
                }
                CBAR();
                LW[L_FREE / 4 + (n & 7u)] = n + 1u;
                const u32 flag = put_unit(o, isw, rel, ch, r);
                VMWAIT(0);
                put_flag(flag, seq);
                continue;
                }
                const u32 sz = (id >> 16) & 1023u, lk = (id >> 13) & 7u, osoff = out_at(id, lk), lv = RFL(LW[L_Q2 / 4 + (n & 15u)]);
                u32x4 qv[8];
                if (sz >= 512u) {
#pragma unroll
                    for (int e = 0; e < 8; ++e) qv[e] = LDSV((e < 6 ? sl : lv - R_VAR) + e * 1024u + lane * 16u);
                } else {
                    u32x4 nib[4];
#pragma unroll
                    for (int e = 0; e < 4; ++e) nib[e] = LDSV(sl + R_NIB + e * 1024u + lane * 16u);
                    const u32x4 pl = LDSV(sl + R_BIT + lane * 16u);
                    const u32x4 sg = LDSV((id & 2048u ? L_SG + (id & 63u) * 1024u : sl) + lane * 16u);
                    u32 cnt = 0, at[16], d[16][3];
#pragma unroll
                    for (int o = 0; o < 16; ++o) {
                        at[o] = cnt;
                        cnt += (u32)__builtin_popcount(nib[o >> 2][o & 3] & (pl[o >> 2] << (o & 3)) & 0x88888888u);
                    }
                    const u32 a0 = unary_list<3>(lane, lv, (sz - R_VAR / 16u) * 16u, cnt, L_PACK + (wid - 2u) * 3072u);
                    CBAR();
#pragma unroll
                    for (int o = 0; o < 16; ++o) { at[o] += a0; LIST8(d[o], at[o]); }
                    __builtin_amdgcn_sched_barrier(0);
#pragma unroll
                    for (int o = 0; o < 16; ++o) {
                        const u32 nv = nib[o >> 2][o & 3], pb = pl[o >> 2] << (o & 3), sv = sg[o >> 2] << (6 - 2 * (o & 3));
                        u32 c0, c1;
                        deal8(d[o], at[o], nv & pb & 0x88888888u, c0, c1);
                        const u32 m0 = (c0 << 3) + (bfi(0x0F0F0F0Fu, nv, pb << 1) & 0x1F1F1F1Fu);
                        const u32 m1 = (c1 << 3) + (bfi(0x0F0F0F0Fu, nv >> 4, pb >> 3) & 0x1F1F1F1Fu);
                        qv[o >> 1][(2 * o) & 3] = __builtin_amdgcn_bitop3_b32(byte_signs(sv << 1), (0x80808080u - m0) ^ 0x80808080u, m0, 0xCA);
                        qv[o >> 1][(2 * o + 1) & 3] = __builtin_amdgcn_bitop3_b32(byte_signs(sv), (0x80808080u - m1) ^ 0x80808080u, m1, 0xCA);
                    }
                }
                CBAR();
                LW[L_FREE / 4 + (n & 7u)] = n + 1u;
                if (LW[L_BAD / 4] != 0u) qv[0] = u32x4{0x80808080u, 0x80808080u, 0x80808080u, 0x80808080u};
#pragma unroll
                for (int e = 0; e < 8; ++e) ST128(qv[e], rs_ol, id & 4096u ? rows : ovoff, osoff + e * 8u * NN, AGT);
                const u32 cls = (id >> 26) & 7u, to = (cls - lk) & 7u;
                if (STEAL && cls != 0u && to - 1u < 6u) {
                    const rsrc_t rs_to = mk_rsrc(p.ws[to], WS_NEED);
                    u32 old = 0;
                    VMWAIT(0);
                    if (lane == 0) old = 1u + __hip_atomic_fetch_add((u32*)(wsl + OFF_CTR + (96u + lk * 8u + cls) * 128u), 1u, __ATOMIC_RELAXED, __HIP_MEMORY_SCOPE_AGENT);
                    if (old == seq * 24u) __builtin_amdgcn_raw_buffer_store_b32(seq, rs_to, OFF_RF + to * 128u + cls * 16u, 0, SYS);
                }
            }
            STAMP(2);
        }
    } else {
        arrive(0u, 12u);
        if (b == AUX_WG && wid == 2u) {
            u32 a = 0;
            u64* gp = (u64*)(p.ws[lane & 7u] + OFF_GAM + (lane >> 3) * 128u + rank * 8u);
            asm volatile("" : "+v"(gp));
            __builtin_amdgcn_raw_buffer_store_b32(seq << 12, rs_ws, lane < 8u ? OFF_CTR + (72u + lane) * 128u : OOB, 0, SYS);
            VMWAIT(0);
            for (;;) {
                const u32x4 g0 = LD128(rs_ws, OFF_AMX + lane * 32u, 0, AGT);
                const u32x4 g1 = LD128(rs_ws, OFF_AMX + lane * 32u + 16u, 0, AGT);
                a = max(max(g0[0], g0[2]), max(g1[0], g1[2]));
                const bool ok = g0[1] == seq && g0[3] == seq && g1[1] == seq && g1[3] == seq;
                if (BALLOT(ok) == ~0ull) break;
                if (expired()) { a = 0xFFFFFFFFu; break; }
            }
            a = wave_max_u32(a);
            __hip_atomic_store((GAS u64*)gp, ((u64)seq << 32) | a, __ATOMIC_RELAXED, __HIP_MEMORY_SCOPE_SYSTEM);
            if (lane == 0) __builtin_amdgcn_raw_buffer_store_b32(seq, rs_ws, OFF_SEQ, 0, AGT);
            STAMP(3);
            bool give_up = false;
            for (;;) {
                const u32x4 f = LD128(rs_ws, OFF_DONE + lane * 16u, 0, SYS);
                const bool ok = (lane >> 3) == rank || (lane & 4u) != 0u || (f[0] >= seq && f[1] >= seq && f[2] >= seq && f[3] >= seq);
                if (BALLOT(ok) == ~0ull) break;
                if (expired()) { give_up = true; break; }
                __builtin_amdgcn_s_sleep(8);
            }
            if (give_up || __builtin_amdgcn_raw_buffer_load_b32(rs_ws, OFF_TMO, 0, AGT) == seq)
                if (lane == 0) p.out[0][rank * 2048u] = (char)-128;
            STAMP(7);
        }
    }
}

extern "C" void kernel_launch(int rank, int nranks, ncclComm_t comm, hipStream_t stream,
                              void* const* d_in, const int* in_sizes, int n_in,
                              void* d_out, int out_size, const int* in_dtypes, int out_dtype,
                              void* const* d_out_all, void* const* d_ws_all, size_t ws_bytes) {
    if (nranks != NR || (unsigned)rank >= (unsigned)NR || n_in < 2 || ws_bytes < 3ull * WS_NEED ||
        in_sizes[0] != (int)(MM * KSH) || in_sizes[1] != (int)(KSH * NN) || out_size != (int)OUT_BYTES) {
        fprintf(stderr, "kernel_launch: unexpected problem shape\n");
        return;
    }
    static bool init = false;
    if (!init) {
        HIP_CHECK(hipFuncSetAttribute((const void*)gemm_ar_q8, hipFuncAttributeMaxDynamicSharedMemorySize, LDS_BYTES));
        init = true;
    }
    Params p;
    p.x = (const char*)d_in[0];
    p.w = (const char*)d_in[1];
    for (int r = 0; r < NR; ++r) p.ws[r] = (char*)d_ws_all[(rank + r) % NR];
    for (int r = 0; r < NR; ++r) p.out[r] = (char*)d_out_all[(rank + r) % NR];
    p.rank = rank;
    hipLaunchKernelGGL(gemm_ar_q8, dim3(256), dim3(768), LDS_BYTES, stream, p);
    HIP_CHECK(hipGetLastError());
}
```

```cpp
#include <hip/hip_runtime.h>
#include <rccl/rccl.h>
#include <cstdio>
#include <cstdint>
#include <cstddef>

#define HIP_CHECK(expr) do { hipError_t _e = (expr); if (_e != hipSuccess) { \
    fprintf(stderr, "HIP error %s at %s:%d\n", hipGetErrorString(_e), __FILE__, __LINE__); \
    return; } } while (0)

#define LAS __attribute__((address_space(3)))
#define GAS __attribute__((address_space(1)))
typedef unsigned int u32;
typedef unsigned long long u64;
typedef u32 u32x4 __attribute__((ext_vector_type(4)));
typedef u32 u32x2 __attribute__((ext_vector_type(2)));
typedef unsigned short u16x2 __attribute__((ext_vector_type(2)));
typedef float f32x16 __attribute__((ext_vector_type(16)));

constexpr int WIN = 4;
constexpr u32 SGM = 0xBBBBu;
constexpr u32 HMASK = 0xF31FFFCu;
constexpr u32 XFIRST = 0x0Eu;
constexpr u32 PLEAD = 8, PSTEP = 7;
constexpr u32 WDELAY1 = 50, PDELAY = 100, WDELAY2 = 200;
constexpr u64 TMAP = 0x144444444320ull;
constexpr u32 NRAW = 24;
constexpr u32 SMARK = 64, SFROM = 120, JIT = 15;
constexpr int LDIV = 256, LMIN = 12;

constexpr int NR = 8;
constexpr u32 MM = 4096, NN = 16384, KSH = 512;
constexpr u32 X_BYTES = MM * KSH * 2, W_BYTES = KSH * NN * 2, OUT_BYTES = MM * NN;
constexpr u32 PIECE = 32768;
constexpr u32 STEP_BYTES = 28 * PIECE;
constexpr u32 OFF_UF = 1u << 20;
constexpr u32 OFF_F3 = OFF_UF + 64 * 112 * 128;
constexpr u32 STAGE_OFF = 2u << 20;
static_assert(OFF_F3 + 2048 * 64 <= STAGE_OFF, "flags");
constexpr u32 E1_SLOT = 8192 + 128;
constexpr u32 I_HDR = 4096, I_FLD = 16, I_VAR = 1040, I_MAX = 1900;
constexpr u32 E1X_OFF = STAGE_OFF + 64 * STEP_BYTES;
constexpr u32 E1W_OFF = E1X_OFF + 512 * E1_SLOT;
constexpr u32 E1P_OFF = E1W_OFF + 2048 * E1_SLOT;
constexpr u32 E2_OFF = E1P_OFF + 512 * E1_SLOT;
constexpr u32 E3_OFF = E2_OFF + 1024 * 8192;
constexpr u32 WS_NEED = E3_OFF + 1024 * 1024;
constexpr u32 OFF_SEQ = 0, OFF_TMO = 256;
constexpr u32 OFF_AMX = 4096;
constexpr u32 OFF_GAM = OFF_AMX + 2048;
constexpr u32 OFF_DONE = OFF_GAM + 1024;
constexpr u32 OFF_F1 = OFF_DONE + 1024;
constexpr u32 OFF_CTR = OFF_F1 + 2048;
constexpr u32 OFF_RF = OFF_CTR + 160 * 128;
constexpr u32 OFF_F2 = 65536;
constexpr u32 OFF_FP = OFF_F2 + 32768;
static_assert(OFF_RF + 1024 <= OFF_F2 && OFF_FP + 16384 <= OFF_UF, "flags");

constexpr u32 SLOT_BYTES = 32768;
constexpr u32 RING = 2048;
constexpr u32 CTRL = RING + 16 * E1_SLOT;
constexpr u32 L_FULL = CTRL, L_NEXT = CTRL + 16, L_FREE = CTRL + 32, L_NRDY = CTRL + 64, L_AMAX = CTRL + 68, L_ACNT = CTRL + 72,
              L_BAD = CTRL + 76, L_SCALE = CTRL + 80, L_GCNT = CTRL + 88, L_GSZ = CTRL + 96, L_SCNT = CTRL + 112, L_STEPS = CTRL + 128, L_PTR = CTRL + 640;
constexpr u32 L_SEG = CTRL + 4, L_G2 = CTRL + 8, L_END = CTRL + 12, L_P1 = CTRL + 16, L_G3 = CTRL + 20, L_VICT = CTRL + 24, L_Q = CTRL + 128, L_Q2 = CTRL + 320,
              L_PSZ = CTRL + 192, L_SG = RING + 8 * E1_SLOT;
constexpr u32 L_PACK = CTRL + 1024;
constexpr u32 L_SZ = L_PACK + 8 * 3072;
constexpr u32 LDS_BYTES = L_SZ + 2048;
constexpr u64 DEADLINE = 100000000ull;
constexpr u32 OOB = WS_NEED;
constexpr int SYS = 17, AGT = 16;
constexpr u32 CODE_BYTES = 49152;
constexpr u32 AUX_WG = 248;
constexpr u32 N1 = 768 + 112;
constexpr u32 R_BIT = 1024, R_NIB = 2048, R_VAR = 6144, R_MAX = 2800;

struct Params {
    char* ws[NR];
    const char* x;
    const char* w;
    int rank;
    char* out[NR];
};

typedef __amdgpu_buffer_rsrc_t rsrc_t;
__device__ __forceinline__ rsrc_t mk_rsrc(const void* p, u32 bytes) {
    return __builtin_amdgcn_make_buffer_rsrc((void*)p, 0, bytes, 0x00020000);
}
__device__ __forceinline__ u64 now() { return __builtin_amdgcn_s_memrealtime(); }
#define VMWAIT(n) asm volatile("s_waitcnt vmcnt(%0)" :: "n"(n) : "memory")
#define CBAR() asm volatile("" ::: "memory")
#define ST128(v, rs, voff, soff, aux) do { const u32x4 d_ = (v); \
    __builtin_amdgcn_raw_buffer_store_b128(d_, rs, voff, soff, aux); \
    asm volatile("s_nop 1" :: "v"(d_) : "memory"); } while (0)
#define LD128(rs, voff, soff, aux) __builtin_amdgcn_raw_buffer_load_b128(rs, voff, soff, aux)
#define PERM(a, b, s) __builtin_amdgcn_perm(a, b, s)
#define BALLOT(c) __builtin_amdgcn_ballot_w64(c)
#define RFL(v) ((u32)__builtin_amdgcn_readfirstlane(v))
__device__ __forceinline__ u32 bfi(u32 m, u32 a, u32 b) { return (a & m) | (b & ~m); }
__device__ __forceinline__ u32 rank_in(u64 m) {
    return __builtin_amdgcn_mbcnt_hi((u32)(m >> 32), __builtin_amdgcn_mbcnt_lo((u32)m, 0u));
}

#define DPP_STEPS(OP) \
    OP(0x111, 0xf, true) OP(0x112, 0xf, true) OP(0x114, 0xf, true) OP(0x118, 0xf, true) OP(0x142, 0xa, false) OP(0x143, 0xc, false)
__device__ __forceinline__ u32 wave_max_u32(u32 v) {
#define OP_(c, rm, bc) v = max(v, (u32)__builtin_amdgcn_update_dpp(0, (int)v, c, rm, 0xf, bc));
    DPP_STEPS(OP_)
#undef OP_
    return (u32)__builtin_amdgcn_readlane((int)v, 63);
}
__device__ __forceinline__ u32 wave_scan_u32(u32 v) {
#define OP_(c, rm, bc) v += (u32)__builtin_amdgcn_update_dpp(0, (int)v, c, rm, 0xf, bc);
    DPP_STEPS(OP_)
#undef OP_
    return v;
}
__device__ __forceinline__ u32 byte_signs(u32 a) { return PERM(a, a << 8, 0x0B090A08u); }

constexpr u32 TFLIP0 = 0xFFFFFFFFu, TFLIP1 = 0xFFFFFF00u;
__device__ __forceinline__ u32 lut16(u32x4 t, u32 i) { return PERM(t[1], t[0], i) ^ PERM(t[3], t[2], i ^ 0x08080808u); }
__device__ __forceinline__ u32 unary8(u32 l0, u32 l1, u32& code) {
    u32 p1 = l0 + l1;
    p1 += p1 << 8; p1 += p1 << 16;
    const u32 p0 = p1 - l1;
    code = 0;
#pragma unroll
    for (int k = 0; k < 4; ++k) code |= (1u << ((p0 >> (8 * k)) & 31u)) | (1u << ((p1 >> (8 * k)) & 31u));
    code >>= 1;
    return p1 >> 24;
}
__device__ __forceinline__ void put_bits(LAS char* L, u32 pos, u32 code) {
    LAS u32* a = (LAS u32*)(L + ((pos >> 5) << 2));
    __hip_atomic_fetch_or(a, code << (pos & 31u), __ATOMIC_RELAXED, __HIP_MEMORY_SCOPE_WORKGROUP);
    __hip_atomic_fetch_or(a + 1, (code >> 1) >> (~pos & 31u), __ATOMIC_RELAXED, __HIP_MEMORY_SCOPE_WORKGROUP);
}
__device__ __forceinline__ u32x2 table_of(u32 v) {
    u64 t = 0;
#pragma unroll 1
    for (u32 i = 0, n = 0; i < 8u; ++i) {
        if ((v >> i) & 1u) n += 8u; else t += 1ull << n;
    }
    return __builtin_bit_cast(u32x2, t);
}
#define LP(a) ((const LAS u32*)(size_t)(a))
template <u32 KB>
__device__ __forceinline__ u32 unary_list(u32 lane, u32 from, u32 bytes, u32 mine, u32 area) {
    const u32 sb = RFL((bytes + 63u) >> 6), at = from + lane * sb;
    const LAS u32* a = LP(at & ~3u);
    auto word = [&](u32 i) -> u32 { return __builtin_amdgcn_alignbyte(a[i + 1u], a[i], at); };
    auto ones = [&](u32 w, u32 i) -> u32 { return (u32)__builtin_popcount(w & RFL(sb >= 4u * i + 4u ? ~0u : sb > 4u * i ? ~(~0u << (sb & 3u) * 8u) : 0u)); };
    u32 w[KB], ix[4 * KB], cnt = 0;
    u32x2 t[4 * KB];
#pragma unroll
    for (u32 i = 0; i < KB; ++i) w[i] = word(i);
#pragma unroll
    for (u32 i = 0; i < KB; ++i) *(LAS u32x4*)(size_t)(area + i * 1024u + lane * 16u) = u32x4{0u, 0u, 0u, 0u};
#pragma unroll
    for (u32 e = 0; e < 4u * KB; ++e) { ix[e] = ((w[e >> 2] >> (8 * (e & 3))) & 255u) << 3; t[e] = *(const LAS u32x2*)(size_t)ix[e]; }
#pragma unroll
    for (u32 i = 0; i < KB; ++i) cnt += ones(w[i], i);
#pragma unroll 1
    for (u32 i = KB; 4u * i < sb; ++i) cnt += ones(word(i), i);
    const bool on = lane * sb < bytes;
    cnt = (on ? cnt << 16 : 0u) + mine;
    const u32 end = wave_scan_u32(cnt);
    u32 k = area - 1u + ((end - cnt) >> 16);
    auto add = [&](u32x2 v, u32 x) {
        LAS u32* d = (LAS u32*)(size_t)(k & ~3u);
        const u32 m = ~k;
        __hip_atomic_fetch_add(d, __builtin_amdgcn_alignbyte(v[0], 0u, m), __ATOMIC_RELAXED, __HIP_MEMORY_SCOPE_WORKGROUP);
        __hip_atomic_fetch_add(d + 1, __builtin_amdgcn_alignbyte(v[1], v[0], m), __ATOMIC_RELAXED, __HIP_MEMORY_SCOPE_WORKGROUP);
        __hip_atomic_fetch_add(d + 2, __builtin_amdgcn_alignbyte(0u, v[1], m), __ATOMIC_RELAXED, __HIP_MEMORY_SCOPE_WORKGROUP);
        k += (u32)__builtin_popcount(x);
    };
    asm volatile("" : "+v"(t[4 * KB - 1]), "+v"(k));
    if (on && (u32)__builtin_amdgcn_readlane((int)end, 63) < (KB * 1024u - 12u) << 16) do {
#define STEP_(e) if (e < 4u * KB) { add(t[e % (4u * KB)], ix[e % (4u * KB)]); if (sb == e + 1u) break; }
        STEP_(0u) STEP_(1u) STEP_(2u) STEP_(3u) STEP_(4u) STEP_(5u) STEP_(6u) STEP_(7u) STEP_(8u) STEP_(9u) STEP_(10u) STEP_(11u)
#undef STEP_
#pragma unroll 1
        for (u32 e = 4u * KB; e < sb; ++e) {
            const u32 x = (u32)*(const LAS unsigned char*)(size_t)(at + e) << 3;
            add(*(const LAS u32x2*)(size_t)x, x);
        }
    } while (0);
    return area + ((end - cnt) & 0xFFFFu);
}
#define LIST8(d, ad) do { const LAS u32* a_ = LP((ad) & ~3u); d[0] = a_[0]; d[1] = a_[1]; d[2] = a_[2]; } while (0)
__device__ __forceinline__ void deal8(const u32 (&d)[3], u32 ad, u32 es, u32& c0, u32& c1) {
    const u32 lo = __builtin_amdgcn_alignbyte(d[1], d[0], ad), hi = __builtin_amdgcn_alignbyte(d[2], d[1], ad);
    u32 pre = es << 1;
    asm("" : "+v"(pre));
    const u32 mk = pre - (es >> 3);
    pre += pre << 4;
    asm("" : "+v"(pre));
    pre += pre << 8;
    asm("" : "+v"(pre));
    pre += pre << 16;
    const u32 sel = __builtin_amdgcn_bitop3_b32(mk, pre, 0xCCCCCCCCu, 0xCA);
    c0 = PERM(hi, lo, sel & 0x0F0F0F0Fu);
    c1 = PERM(hi, lo, (sel >> 4) & 0x0F0F0F0Fu);
}

__global__ void __launch_bounds__(768, 3) gemm_ar_q8(Params p) {
    extern __shared__ __attribute__((aligned(16))) char smem[];
    char *wsl, *px, *pw;
    u32 rank;
    asm volatile("s_load_dwordx2 %0, %4, 0\n s_load_dwordx2 %1, %4, 0x40\n s_load_dwordx2 %2, %4, 0x48\n s_load_dword %3, %4, 0x50"
                 : "=&s"(wsl), "=&s"(px), "=&s"(pw), "=&s"(rank) : "s"(__builtin_amdgcn_kernarg_segment_ptr()));
    __builtin_amdgcn_sched_barrier(0);
    static_assert(offsetof(Params, x) == 0x40 && offsetof(Params, w) == 0x48 && offsetof(Params, rank) == 0x50, "arguments");
    LAS char* L = (LAS char*)smem;
    u32 lane = threadIdx.x & 63;
    const u32 wid = RFL(threadIdx.x >> 6);
    const u32 b = blockIdx.x;
    const u32 q = b & 7u, idx = b >> 3;
    const u32 mb = 4 * (q >> 1) + (idx & 3), nb = 4 * (q & 1) + ((idx >> 2) & 3);
    const u32 nb2 = 8u + 2u * (q & 1u) + ((idx >> 2) & 1u), fills = idx < 8u ? 256u : 128u;
    const u32 rel = 1u + (b - 128u) % 7u, pj = ((b - 128u) / 7u) & 15u;
    auto x_at = [](u32 l) -> u32 { return (l >> 2) * 1024u + (((l & 3u) ^ ((l >> 4) & 2u)) << 4); };
    auto w_at = [](u32 l) -> u32 { return (l >> 4) * 262144u + (l & 15u) * 16u; };
    u32 mg = (b >> 7) - 1u;
    const u32 tq = (b + (wid & 1u) * 128u) >> 3;
    u32 tr = b + tq, ts = bfi(mg, (tq & 2u) * 524288u + (tq >> 2) * 512u + (tq & 1u) * 256u, (b & 127u) << 16);
    u32 bar = (bfi(mg, 0x801u, b & 64u ? 0u : 4u) >> wid) & 1u, tv = bfi(mg, w_at(lane), x_at(lane));
    asm volatile("" : "+s"(tr), "+s"(ts), "+s"(bar), "+s"(mg));
    asm volatile("" : "+v"(tv));
    asm volatile("s_waitcnt lgkmcnt(0)" : "+s"(wsl), "+s"(px), "+s"(pw), "+s"(rank));
    rank &= 7u;
    const rsrc_t rs_ws = mk_rsrc(wsl, WS_NEED);
    const u32 seqv = __builtin_amdgcn_raw_buffer_load_b32(rs_ws, OFF_SEQ, 0, AGT);
    __builtin_amdgcn_sched_barrier(0);
    u32x4 v[8];
    auto fetch = [&](bool w, u32 vo, u32 so) {
        const rsrc_t rs = mk_rsrc(w ? pw : px, w ? W_BYTES : X_BYTES);
#pragma unroll
        for (int i = 0; i < 8; ++i) v[i] = LD128(rs, vo, so + (i & 3) * (w ? 32768u : 16384u) + (i >> 2) * (w ? 131072u : 64u), 0);
    };
    if (bar) fetch(mg != 0u, tv, ts + (((rank + tr) << 12) & mg & 0x7000u));
    __builtin_amdgcn_sched_barrier(0);
    volatile LAS u32* LW = (volatile LAS u32*)L;
#define LDSV(a) (*(const LAS u32x4*)(L + (a)))

    if (threadIdx.x < 32) LW[CTRL / 4 + threadIdx.x] = b >= 128u && threadIdx.x - 13u < 2u ? ~7u : 0u;
    auto barrier = [] {
        asm volatile("s_waitcnt lgkmcnt(0)" ::: "memory");
        __builtin_amdgcn_s_barrier();
        CBAR();
    };
#define PEERS(d) asm volatile("global_load_dwordx2 %0, %1, %2" : "=v"(d) : "v"((lane & 7u) * 8u), "s"(__builtin_amdgcn_kernarg_segment_ptr()) : "memory")

    auto put_unit = [&](const u32x4 (&o)[8], bool isw, u32 lk, u32 ch, u32 r) -> u32 {
        const u32 so = STAGE_OFF + (lk * 8u + ch) * STEP_BYTES + (isw ? 16u * PIECE + r * 8192u : (r >> 2) * PIECE + (r & 3u) * 4096u);
        const u32 v0 = isw ? ((lane & 48u) + (lane & 7u) * 2u + ((lane >> 3) & 1u)) * 16u : lane * 16u, v1 = v0 + (isw ? 4096u : 16384u);
#pragma unroll
        for (int e = 0; e < 8; ++e) ST128(o[e], rs_ws, (e < 4 ? v0 : v1) + (e & 3) * 1024u, so, SYS);
        return OFF_UF + ((lk * 8u + ch) * 112u + (isw ? 64u : 0u) + r) * 128u;
    };
    auto put_flag = [&](u32 flag, u32 sq) { if (lane < 4u) ST128((u32x4{sq, sq, sq, sq}), rs_ws, flag + lane * 16u, 0, SYS); };

    auto strip_at = [](u32 os, u32 r) -> u32 { return 32u + (os >> 1) * 4u + (r & 2u) + (os & 1u); };

    {
        const u32 ma = 0u - ((b + 16u) >> 8), ml = 0u - ((wid + 6u) >> 4);
        const u32 tg = (u32)(TMAP >> (wid * 4u)) & 7u, step = bfi(ma, 192u, 224u);
        u32 t = b + bfi(mg, tg * 128u, bfi(ma, 1536u + wid * 16u, 384u + ((wid * 112u - 1056u) & ml)));
        const u32 tend = bfi(mg, min(t + 1u, 480u), bfi(ma, 3008u, bfi(ml, 1376u, wid == 2u ? 576u : 0u)));
        const u32 delay = bfi(mg, (max(tg, 1u) - 1u) * WDELAY1, bfi(ma, WDELAY2, PDELAY & ml));
        auto pause = [](u32 d) {
            const u32 ts = (u32)now();
            while ((u32)now() - ts < d) __builtin_amdgcn_s_sleep(2);
        };
        if (!bar) {
            if (b - 128u < 112u && wid - 6u < 4u) *(LAS u32x2*)(L + (threadIdx.x - 384u) * 8u) = table_of(threadIdx.x - 384u);
            barrier();
            if (t < tend) pause(delay);
        }
        if (b < 128u && wid - 4u < 4u) {
            u64 pc;
            asm volatile("s_getpc_b64 %0" : "=s"(pc));
            const u32 v = __builtin_amdgcn_raw_buffer_load_b32(mk_rsrc((void*)(pc & ~255ull), CODE_BYTES), lane * 128u, ((idx * 4u + wid) % 6u) * 8192u, 0);
            asm volatile("" :: "v"(v));
        }
        for (; t < tend; t += step) {
            u32 kind = 2u, lk = 0, ch, r;
            const u32 u = t + (t < 640u ? 512u : t < 752u ? 1024u : t < 880u ? 400u : t < 1104u ? 2128u : t < 1376u ? 176u : 0u);
            if (t < 256u) { ch = 0u; r = t >> 3; lk = (t + r) & 7u; kind = lk == 0u ? 1u : 2u; }
            else if (t < 480u) { ch = (t - 224u) >> 5; r = t & 31u; kind = 1u; }
            else if (u < 1536u) { ch = (u >> 6) & 7u; r = u & 63u; kind = 0u; }
            else if (u < 1552u) { ch = (u >> 1) & 7u; r = 28u + (u & 1u) * 2u; }
            else { ch = (0x3765421u >> ((u - 1664u) / 224u * 4u)) & 7u; r = (u - 1664u) % 224u; lk = 1u + r % 7u; r /= 7u; }
            const u32 to = (rank + lk) & 7u;
            const bool strip = kind == 2u && (r | 2u) == 30u;
            const bool push = lk != 0u && ch == 0u, both = push && strip;
            char* pto = wsl;
            u32x4 o[8];
            if (!bar) fetch(kind != 0u, kind != 0u ? w_at(lane) : x_at(lane), kind == 0u ? r * 65536u + ch * 128u
                                : (ch * 64u + (r & 2u) * 16u) * 32768u + (to * 2048u + (r >> 2) * 256u + (r & 1u) * 128u) * 2u);
            if (kind == 0u) {
                if (bar) barrier();
#pragma unroll
                for (int i = 0; i < 8; ++i) o[i] = v[i];
                asm volatile("" : "+v"(o[0]), "+v"(o[1]), "+v"(o[2]), "+v"(o[3]), "+v"(o[4]), "+v"(o[5]), "+v"(o[6]), "+v"(o[7]));
            } else {
                pto = p.ws[push ? lk : 0u];
                asm volatile("" : "+s"(pto));
                if (bar) barrier();
#pragma unroll
                for (int e = 0; e < 4; ++e)
#pragma unroll
                    for (int k = 0; k < 4; ++k) {
                        const u32 lo = PERM(v[2 * k + 1][e >> 1], v[2 * k][e >> 1], (e & 1) ? 0x07060302u : 0x05040100u);
                        const u32 hi = PERM(v[2 * k + 1][2 + (e >> 1)], v[2 * k][2 + (e >> 1)], (e & 1) ? 0x07060302u : 0x05040100u);
                        o[e][k] = (u32)__builtin_amdgcn_update_dpp((int)lo, (int)hi, 0x128, 0xf, 0xc, false);
                        o[4 + e][k] = (u32)__builtin_amdgcn_update_dpp((int)hi, (int)lo, 0x128, 0xf, 0x3, false);
                    }
            }
            bar = 0u;
            u32 flag = OOB, size = E1_SLOT, sq = RFL(seqv) + 1u;
            u32 reg = L_PACK + wid * 2048u;
            asm volatile("" : "+s"(sq), "+s"(reg), "+v"(lane));
            if (kind == 1u || (strip && lk != 0u)) flag = put_unit(o, true, 0u, ch, kind == 2u ? strip_at(lk - 1u, r) : r);
            if (kind == 1u) { VMWAIT(0); put_flag(flag, sq); continue; }
            u64 f1;
            PEERS(f1);
            const u32 kept = kind == 0u ? E1X_OFF + (ch * 64u + r) * E1_SLOT : E1W_OFF + ((lk * 8u + ch) * 32u + r) * E1_SLOT;
            const u32 slot = push ? E1P_OFF + ((lk * 2u + ch) * 32u + r) * E1_SLOT : kept;
            const rsrc_t rs_to = mk_rsrc(pto, WS_NEED);
#define PST(v, voff, at) do { const u32x4 p_ = (v); ST128(p_, rs_to, voff, slot + (at), SYS); if (both) ST128(p_, rs_ws, voff, kept + (at), SYS); } while (0)
            *(LAS u32x4*)(L + reg + lane * 16u) = u32x4{0u, 0u, 0u, 0u};
            *(LAS u32x4*)(L + reg + 1024u + lane * 16u) = u32x4{0u, 0u, 0u, 0u};
            u32 ex[16], mx = 0;
            if (push && r > PLEAD) pause((r - PLEAD) * PSTEP);
#pragma unroll
            for (int e = 0; e < 4; ++e) {
                u32x4 mq;
#pragma unroll
                for (int k = 0; k < 4; ++k) {
                    const int j = 4 * e + k;
                    const u32 d0 = o[j >> 1][(2 * j) & 3], d1 = o[j >> 1][(2 * j + 1) & 3];
                    const u32 r0 = __builtin_amdgcn_alignbit(d0, d0, 31), r1 = __builtin_amdgcn_alignbit(d1, d1, 31);
                    mq[k] = PERM(r1, r0, 0x06040200u);
                    ex[j] = PERM(r1, r0, 0x07050301u);
                    mx = __builtin_bit_cast(u32, __builtin_elementwise_max(__builtin_bit_cast(u16x2, mx),
                             __builtin_elementwise_max(__builtin_bit_cast(u16x2, r0), __builtin_bit_cast(u16x2, r1))));
                }
                PST(mq, lane * 16u + e * 1024u, 0u);
            }
            const u32 top = wave_max_u32(max(mx >> 24, (mx >> 8) & 255u));
            u32 any = 0, cnt = 0;
#pragma unroll
            for (int j = 0; j < 16; ++j) { ex[j] = top * 0x01010101u - ex[j]; any |= ex[j]; }
            CBAR();
#pragma unroll
            for (int j = 0; j < 16; j += 2)
                __hip_atomic_fetch_add((LAS u32*)(L + reg + 256u + (((ex[j] << 4) & 0xF0u) | ((lane & 3u) << 2))), 1u, __ATOMIC_RELAXED,
                                       __HIP_MEMORY_SCOPE_WORKGROUP);
            CBAR();
            const u32x4 hist = LDSV(reg + 256u + (lane & 15u) * 16u);
            CBAR();
            if (BALLOT((any & 0xF0F0F0F0u) != 0u) != 0ull) {
#pragma unroll
                for (int j = 0; j < 16; ++j) {
                    const bool far = (ex[j] & 0xF0F0F0F0u) != 0u;
                    const u64 m = BALLOT(far);
                    if (m != 0ull) {
                        const u32 k = cnt + rank_in(m);
                        if (far && k < 14u) *(LAS u32x2*)(L + reg + I_VAR + k * 8u) = u32x2{lane * 16u + j, ex[j]};
                        cnt += (u32)__builtin_popcountll(m);
                        ex[j] &= 0x0F0F0F0Fu;
                    }
                }
            }
            const u32 key = (((hist[0] + hist[1] + hist[2] + hist[3]) << 4) | (lane & 15u)) ^ 15u;
            u32 above = 0;
#define ROT_(n) above = __builtin_amdgcn_alignbit(above, key - (u32)__builtin_amdgcn_mov_dpp((int)key, 0x120 + n, 0xf, 0xf, false), 31);
            ROT_(1) ROT_(2) ROT_(3) ROT_(4) ROT_(5) ROT_(6) ROT_(7) ROT_(8) ROT_(9) ROT_(10) ROT_(11) ROT_(12) ROT_(13) ROT_(14) ROT_(15)
#undef ROT_
            above = (u32)__builtin_popcount(above);
            u32 tb = ((max(above, 2u) - 2u) | (min(above, 3u) << 4)) << ((lane & 3u) * 8u);
            tb |= (u32)__builtin_amdgcn_mov_dpp((int)tb, 0xB1, 0xf, 0xf, false);
            tb |= (u32)__builtin_amdgcn_mov_dpp((int)tb, 0x4E, 0xf, 0xf, false);
            const u32x4 tab = u32x4{(u32)__builtin_amdgcn_readlane((int)tb, 0) ^ TFLIP0, (u32)__builtin_amdgcn_readlane((int)tb, 4) ^ TFLIP1,
                                    (u32)__builtin_amdgcn_readlane((int)tb, 8) ^ TFLIP0, (u32)__builtin_amdgcn_readlane((int)tb, 12) ^ TFLIP1};
            u32 code[8], len[8], over = 0, tot = 0;
            u32x4 fw;
#pragma unroll
            for (int g = 0; g < 8; ++g) {
                const u32 s0 = lut16(tab, ex[2 * g]), s1 = lut16(tab, ex[2 * g + 1]), h = bfi(0x0F0F0F0Fu, s0 >> 4, s1);
                fw[g >> 1] = g & 1 ? fw[g >> 1] | (h << 2) : h;
                len[g] = unary8(s0 & 0x0F0F0F0Fu, s1 & 0x0F0F0F0Fu, code[g]);
                over |= len[g];
                tot += len[g] + ((u32)__builtin_popcount(code[g]) << 16);
            }
            const u32 end = wave_scan_u32(tot), all = (u32)__builtin_amdgcn_readlane((int)end, 63);
            const u32 bytes = ((I_VAR + cnt * 8u) * 8u + (all & 0xFFFFu) + 127u) >> 7 << 4;
            if (BALLOT(over > 31u) == 0ull && cnt < 15u && bytes <= 2032u && (all >> 16) <= I_MAX) {
                size = I_HDR + bytes;
                *(LAS char*)(L + reg + above) = (char)((lane & 15u) | ((((bytes >> 4) | (top << 8) | (cnt << 16)) >> (above * 4u & 31u)) << 4));
                *(LAS u32x4*)(L + reg + I_FLD + lane * 16u) = fw;
                u32 pos = ((end - tot) & 0xFFFFu) + (reg + I_VAR + cnt * 8u) * 8u;
#pragma unroll
                for (int g = 0; g < 8; ++g) { put_bits(L, pos, code[g]); pos += len[g]; }
                CBAR();
                PST(LDSV(reg + lane * 16u), lane * 16u, I_HDR);
                PST(LDSV(reg + 1024u + lane * 16u), (lane & 60u) * 16u + 1024u < bytes ? lane * 16u + 1024u : OOB, I_HDR);
                CBAR();
            } else {
                PST((u32x4{~0u, ~0u, ~0u, ~0u}), lane == 0u ? 0u : OOB, I_HDR);
#pragma unroll
                for (int e = 0; e < 8; ++e) PST(o[e], lane * 16u + (e & 3) * 1024u, (e >> 2) * (I_HDR + 128u));
            }
            u32 old = 0;
            VMWAIT(0);
            asm volatile("" : "+v"(f1));
            put_flag(flag, sq);
            if (push) {
                if (lane == 0) __builtin_amdgcn_raw_buffer_store_b64(u32x2{sq, (size + 15u) >> 4}, rs_to, 0u, OFF_FP + (lk * 16u + (r & 15u)) * 128u + (ch + (r >> 4)) * 8u, SYS);
                __builtin_amdgcn_s_setprio(0);
                if (!strip) continue;
            }
            if (kind == 0u && ch == 0u) {
                const u32 j = (r - 8u * lane) & 63u;
                if (lane - 1u < 7u)
                    __hip_atomic_store((GAS u64*)(f1 + OFF_FP + (lane * 16u + (j & 15u)) * 128u + 32u + (j >> 4) * 8u), (u64)sq, __ATOMIC_RELAXED, __HIP_MEMORY_SCOPE_SYSTEM);
                continue;
            }
            if (lane == 0 ? (kind == 0u || lk != 0u) && !push : (lane == 1u && strip))
                old = 1u + __hip_atomic_fetch_add((u32*)(wsl + OFF_CTR + (lane == 0 ? lk * 8u + ch : 80u) * 128u), 1u, __ATOMIC_RELAXED,
                                                  __HIP_MEMORY_SCOPE_AGENT);
            f1 += OFF_F1 + lane * 256u;
            const u64 fv = sq;
            if (RFL(old) == sq * (kind == 0u ? 64u : 32u) && lane - 1u < 7u && (kind == 0u || lane == lk))
                __hip_atomic_store((GAS u64*)f1 + ch * 2u + (kind == 0u), fv, __ATOMIC_RELAXED, __HIP_MEMORY_SCOPE_SYSTEM);
            if ((u32)__builtin_amdgcn_readlane((int)old, 1) == sq * 128u && lane - 1u < 7u)
                __hip_atomic_store((GAS u64*)f1 + 16, fv, __ATOMIC_RELAXED, __HIP_MEMORY_SCOPE_SYSTEM);
        }
    }
    const u32 seq = RFL(seqv) + 1u;
    lane = threadIdx.x & 63;

    u32 spins = 0;
    u64 t0 = 0;
    auto expired = [&]() -> bool {
        if ((++spins & 63u) != 0u) return false;
        if (LW[L_BAD / 4] != 0u) return true;
        const u64 t = now();
        if (t0 == 0) t0 = t;
        if (t - t0 > DEADLINE || __builtin_amdgcn_raw_buffer_load_b32(rs_ws, OFF_TMO, 0, AGT) == seq) {
            u32 sq = seq;
            asm volatile("" : "+s"(sq));
            __builtin_amdgcn_raw_buffer_store_b32(sq, rs_ws, OFF_TMO, 0, AGT);
            LW[L_BAD / 4] = 1u;
            return true;
        }
        return false;
    };

    auto arrive = [&](u32 wm, u32 n) {
        if (lane == 0) {
            __hip_atomic_fetch_max((LAS u32*)(L + L_AMAX), wm, __ATOMIC_RELAXED, __HIP_MEMORY_SCOPE_WORKGROUP);
            const u32 old = __hip_atomic_fetch_add((LAS u32*)(L + L_ACNT), 1u, __ATOMIC_RELAXED, __HIP_MEMORY_SCOPE_WORKGROUP);
            if (old == n - 1u) {
                u32 a = LW[L_AMAX / 4];
                if (LW[L_BAD / 4] != 0u) a = 0xFFFFFFFFu;
                __builtin_amdgcn_raw_buffer_store_b64(u32x2{a, seq}, rs_ws, OFF_AMX + b * 8u, 0, AGT);
            }
        }
    };

    if (b < 128u && wid < 8u) {
        u32 c = lane & 15u, g = lane >> 4;
        const u32 mw = wid & 3u, nh = wid >> 2;
        for (u32 f0 = 0; f0 < fills; f0 += 128u) {
        f32x16 acc[8];
        u32 xa = mw * 4096u + (c * 4u + (g ^ ((c >> 2) & 2u))) * 16u, wa = 16384u + nh * 8192u + lane * 16u;
        const u32 ca = lane < 32u ? L_FULL + ((lane - 1u) & 3u) * 4u : lane < 48u ? L_FREE + (wid ^ 4u) * 4u : L_FREE + wid * 4u - 768u;
        u32 f2 = f0 + 1u, spin = 0, tv;
#define S0_(x) #x
#define S_(x) S0_(x)
#define R4(r) "v[" S_(r) ":" S_(r) "+3]"
#define MM_(j, m, w, x) "v_mfma_f32_16x16x32_bf16 " R4(16*j+4*m) ", " R4(w) ", " R4(x) ", " R4(16*j+4*m) "\n"
#define MM4(j, w, x0) MM_(j, 0, w, x0) MM_(j, 1, w, 132) MM_(j, 2, w, 136) MM_(j, 3, w, 140)
#define RW(r, o) "ds_read_b128 " R4(r) ", %[wa] offset:" S_(o) "\n"
#define RX(r, o) "ds_read_b128 " R4(r) ", %[xa] offset:" S_(o) "\n"
#define LG(n) "s_waitcnt lgkmcnt(" S_(n) ")\n s_nop 0\n"
#define KWAIT(l) \
    "8" l ":\n s_cmp_eq_u32 %[f2], %[last]\n s_cbranch_scc1 9" l "b\n ds_read_b32 %[tv], %[ca]\n s_waitcnt lgkmcnt(0)\n" \
    "v_cmp_le_u32 vcc, %[f2], %[tv]\n s_nop 4\n s_bitcmp1_b32 vcc_lo, %[f2]\n s_cbranch_scc1 9" l "b\n s_sleep 1\n" \
    "s_add_u32 %[spin], %[spin], 1\n s_cmp_lt_u32 %[spin], 0x1000000\n s_cbranch_scc1 8" l "b\n s_branch 9" l "b\n"
#define KFILL(l, x0, y0, cur, nxt, TOGGLE) \
    LG(5) MM_(0, 0, 144, x0) MM_(1, 0, 148, x0) LG(4) MM_(0, 1, 144, 132) MM_(1, 1, 148, 132) \
    LG(3) MM_(0, 2, 144, 136) MM_(1, 2, 148, 136) LG(2) MM_(0, 3, 144, 140) RW(144, cur+4096) MM_(1, 3, 148, 140) RW(148, cur+5120) \
    "ds_read_b32 %[tv], %[ca]\n" \
    "s_waitcnt lgkmcnt(4)\n s_add_u32 %[f2], %[f2], 1\n" MM4(2, 152, x0) RW(152, cur+6144) \
    LG(4) MM4(3, 156, x0) RW(156, cur+7168) \
    "s_waitcnt lgkmcnt(2)\n v_cmp_le_u32 vcc, %[f2], %[tv]\n" MM_(4, 0, 144, x0) \
    "v_mov_b32_e64 %[tv], %[f2]\n" MM_(4, 1, 144, 132) \
    "ds_write_b32 %[ca], %[tv] offset:768\n"         \
    MM_(4, 2, 144, 136) TOGGLE MM_(4, 3, 144, 140) \
      \
    "s_setprio 0\n s_bitcmp0_b32 vcc_hi, 0\n s_cbranch_scc1 7" l "f\n s_setprio 1\n7" l ":\n" \
    "s_bitcmp0_b32 vcc_lo, %[f2]\n s_cbranch_scc1 8" l "f\n9" l ":\n" \
    RW(144, nxt) RX(y0, nxt) MM4(5, 148, x0) RW(148, nxt+1024) \
    LG(4) MM_(6, 1, 152, 132) MM_(7, 1, 156, 132) RX(132, nxt+1024) MM_(6, 2, 152, 136) MM_(7, 2, 156, 136) RX(136, nxt+2048) \
    MM_(6, 3, 152, 140) MM_(7, 3, 156, 140) RX(140, nxt+3072) MM_(6, 0, 152, x0) RW(152, nxt+2048) MM_(7, 0, 156, x0) RW(156, nxt+3072)
        asm volatile(
            ".set i_, 0\n .rept 64\n v_mov_b64 v[i_:i_+1], 0\n .set i_, i_+2\n .endr\n"
            "s_branch 80f\n90:\n"
            RW(144, 0) RX(128, 0) RW(148, 1024) RX(132, 1024) RX(136, 2048) RX(140, 3072) RW(152, 2048) RW(156, 3072)
            ".p2align 3\n1:\n"
            KFILL("1", 128, 160, 0, 32768, "")
            KFILL("2", 160, 128, 32768, 0, "v_xor_b32 %[wa], 0x10000, %[wa]\n v_xor_b32 %[xa], 0x10000, %[xa]\n")
            "s_cmp_lt_u32 %[f2], %[last]\n s_cbranch_scc1 1b\n s_branch 2f\n"
            KWAIT("0") KWAIT("1") KWAIT("2")
            "2:\n s_waitcnt lgkmcnt(0)\n s_nop 7\n s_nop 7\n"
            : "={v[0:15]}"(acc[0]), "={v[16:31]}"(acc[1]), "={v[32:47]}"(acc[2]), "={v[48:63]}"(acc[3]), "={v[64:79]}"(acc[4]),
              "={v[80:95]}"(acc[5]), "={v[96:111]}"(acc[6]), "={v[112:127]}"(acc[7]),
              [xa] "+{v164}"(xa), [wa] "+{v165}"(wa), [tv] "={v166}"(tv), [f2] "+s"(f2), [spin] "+s"(spin)
            : [ca] "{v167}"(ca), [last] "s"(f0 + 129u)
            : "v128", "v129", "v130", "v131", "v132", "v133", "v134", "v135", "v136", "v137", "v138", "v139", "v140", "v141", "v142",
              "v143", "v144", "v145", "v146", "v147", "v148", "v149", "v150", "v151", "v152", "v153", "v154", "v155", "v156", "v157",
              "v158", "v159", "v160", "v161", "v162", "v163", "vcc", "scc", "memory");
        if (spin >= 0x1000000u) LW[L_BAD / 4] = 1u;
        asm volatile("v_mbcnt_lo_u32_b32 %0, -1, 0\n v_mbcnt_hi_u32_b32 %0, -1, %0" : "=v"(lane));
        c = lane & 15u; g = lane >> 4;
        const u32 reg = L_PACK + wid * 2048u, slot = E2_OFF + (b * 2u + nh) * 32768u + mw * R_VAR;
        auto post = [&](u32 cnt, u32 at) {
            u32 old = 0;
            VMWAIT(0);
            if (lane == 0) old = __hip_atomic_fetch_add((LAS u32*)(L + cnt + nh * 4u), 1u, __ATOMIC_RELAXED, __HIP_MEMORY_SCOPE_WORKGROUP);
            if (RFL(old) == 3u && lane - 1u < 7u) {
                const u32x4 fv = u32x4{seq, LW[L_GSZ / 4 + nh * 2u], LW[L_GSZ / 4 + nh * 2u + 1u], seq};
                u32x2 fp = *(volatile LAS u32x2*)(L + L_PTR + lane * 8u);
                asm volatile("v_add_co_u32 %0, vcc, %0, %2\n v_addc_co_u32 %1, vcc, 0, %1, vcc" : "+v"(fp[0]), "+v"(fp[1]) : "v"(at) : "vcc");
                asm volatile("global_store_dwordx4 %0, %1, off sc0 sc1\n s_nop 1" :: "v"(fp), "v"(fv) : "memory");
            }
        };
        if (f0 == 0u) {
            u32x4 sg;
#pragma unroll
            for (int m = 0; m < 4; ++m) {
                u32 d = 0;
#pragma unroll
                for (int i = 31; i >= 0; --i) d = __builtin_amdgcn_alignbit(d, __float_as_uint(acc[(i & 4) + (i >> 3)][4 * m + (i & 3)]), 31);
                const u32 o = (u32)__builtin_amdgcn_mov_dpp((int)d, 0x128, 0xf, 0xf, false);
                sg[m] = bfi(c < 8u ? 0x0F0F0F0Fu : 0xF0F0F0F0u, d, __builtin_amdgcn_alignbit(o, o, c < 8u ? 28u : 4u));
            }
            ST128(sg, rs_ws, lane * 16u, (SGM >> (q * 2u + nh)) & 1u ? E3_OFF + (b * 8u + wid) * 1024u : slot, AGT);
            float mx = 0.f;
#pragma unroll
            for (int j = 0; j < 8; ++j)
#pragma unroll
                for (int m = 0; m < 4; ++m) {
                    mx = fmaxf(fmaxf(mx, fabsf(acc[j][4 * m])), fabsf(acc[j][4 * m + 1]));
                    mx = fmaxf(fmaxf(mx, fabsf(acc[j][4 * m + 2])), fabsf(acc[j][4 * m + 3]));
                }
            arrive(wave_max_u32(__float_as_uint(mx)), 8u);
            if (wid >= 4u) __builtin_amdgcn_s_setprio(2); else __builtin_amdgcn_s_setprio(0);
#pragma unroll
            for (int k = 0; k < 2; ++k) *(LAS u32x4*)(L + reg + k * 1024u + lane * 16u) = u32x4{0u, 0u, 0u, 0u};
            if ((SGM >> (q * 2u + nh)) & 1u) post(L_SCNT, OFF_F3 + ((lane * 16u + idx) * 16u + q * 2u + nh) * 64u);
            while (LW[L_SCALE / 4] != 1u) { if (expired()) break; __builtin_amdgcn_s_sleep(1); }
            CBAR();
        }
        const u32 gmax = LW[L_SCALE / 4 + 1];
        const bool bad = (gmax == 0xFFFFFFFFu) || (LW[L_BAD / 4] != 0u);
        const float amax = __uint_as_float(gmax);
        const float inv = amax > 0.f ? 127.0f / amax : 0.f;
        const float MAGIC = 12582912.0f;
        const float qlo = bad ? MAGIC - 128.0f : MAGIC - 127.0f, qhi = bad ? MAGIC - 128.0f : MAGIC + 127.0f;
        const u32 ovoff = (c & 7u) * NN + ((c >> 3) * 4u + g) * 16u;
        const u32 os = (nb2 - 8u) * 2u + nh;
        const u32 osoff = (mb * 256u + mw * 64u) * NN + (f0 == 0u ? rank * 2048u + nb * 256u + nh * 128u : ((rank + 1u + os) & 7u) * 2048u + 1792u);
        u32x4 qv[8];
#pragma unroll
        for (int m = 0; m < 4; ++m) {
#pragma unroll
            for (int r = 0; r < 4; ++r) {
                u32 o[2];
#pragma unroll
                for (int h = 0; h < 2; ++h) {
                    u32 t[4];
#pragma unroll
                    for (int jj = 0; jj < 4; ++jj)
                        t[jj] = __float_as_uint(__builtin_amdgcn_fmed3f(__builtin_fmaf(acc[4 * h + jj][4 * m + r], inv, MAGIC), qlo, qhi));
                    o[h] = PERM(t[1], t[0], 0x0c0c0400u) | PERM(t[3], t[2], 0x04000c0cu);
                }
                qv[2 * m][r] = (u32)__builtin_amdgcn_update_dpp((int)o[0], (int)o[1], 0x128, 0xf, 0xc, false);
                qv[2 * m + 1][r] = (u32)__builtin_amdgcn_update_dpp((int)o[1], (int)o[0], 0x128, 0xf, 0x3, false);
            }
        }
        if (f0 == 0u) {
        u32 size = nb == 7u && nh == 0u ? 0u : 8192u;
        u32x4 rest[2] = {qv[6], qv[7]};
        auto tell = [&] { if (lane == 0) *(volatile LAS unsigned short*)(L + L_GSZ + wid * 2u) = (unsigned short)(size >> 4); };
        if (size != 0u && !bad) {
            u32x4 nib[4], pl = u32x4{0u, 0u, 0u, 0u};
            u32 code[16], len[16], over = 0, tot = 0;
#pragma unroll
            for (int o = 0; o < 16; ++o) {
                u32 y[2], l[2];
#pragma unroll
                for (int e = 0; e < 2; ++e) {
                    const u32 v = qv[o >> 1][(2 * o + e) & 3], s = byte_signs(v);
                    y[e] = (v ^ s) + ((s & 0x01010101u) | 0x70707070u);
                    l[e] = (y[e] & byte_signs(y[e]) & 0x78787878u) >> 3;
                }
                const u32 h = bfi(0x0F0F0F0Fu, y[0] >> 4, y[1]);
                nib[o >> 2][o & 3] = bfi(0x0F0F0F0Fu, y[0], y[1] << 4) | (((h & 0x77777777u) + 0x77777777u) & h & 0x88888888u);
                pl[o >> 2] = bfi(0x88888888u >> (o & 3), h >> (o & 3), pl[o >> 2]);
                len[o] = unary8(l[0], l[1], code[o]);
                over |= len[o];
                tot += len[o] + ((u32)__builtin_popcount(code[o]) << 16);
            }
            const u32 end = wave_scan_u32(tot);
            const u32 all = (u32)__builtin_amdgcn_readlane((int)end, 63), bits = all & 0xFFFFu;
            if (BALLOT(over > 31u) == 0ull && bits <= (8192u - 64u - R_VAR) * 8u && (all >> 16) <= R_MAX) {
                size = (R_VAR + ((bits + 127u) >> 7) * 16u);
                tell();
                ST128(pl, rs_ws, lane * 16u, slot + R_BIT, AGT);
#pragma unroll
                for (int e = 0; e < 4; ++e) ST128(nib[e], rs_ws, lane * 16u + e * 1024u, slot + R_NIB, AGT);
                u32 pos = ((end - tot) & 0xFFFFu) + reg * 8u;
#pragma unroll
                for (int o = 0; o < 16; ++o) { put_bits(L, pos, code[o]); pos += len[o]; }
                CBAR();
                rest[0] = LDSV(reg + lane * 16u);
                rest[1] = LDSV(reg + 1024u + lane * 16u);
            }
        }
        if (size == 8192u) {
            tell();
#pragma unroll
            for (int e = 0; e < 6; ++e) ST128(qv[e], rs_ws, lane * 16u, slot + e * 1024u, AGT);
        }
        if (size != 0u) {
            u32 at = mw * 128u;
            if ((SGM >> (q * 2u + nh)) & 1u) {
                do at = lane < mw ? *(volatile LAS unsigned short*)(L + L_GSZ + (nh * 4u + lane) * 2u) : R_VAR / 16u;
                while (BALLOT(at == 0u) != 0ull && !expired());
                at = (u32)(__builtin_amdgcn_readlane((int)at, 0) + __builtin_amdgcn_readlane((int)at, 1) + __builtin_amdgcn_readlane((int)at, 2)) - 3u * R_VAR / 16u;
            }
#pragma unroll
            for (int k = 0; k < 2; ++k) {
                const u32 off = R_VAR + k * 1024u + lane * 16u;
                if (R_VAR + k * 1024u < size) ST128(rest[k], rs_ws, off < size ? off : OOB, slot + (3u - mw) * R_VAR + min(at, 384u) * 16u, AGT);
            }
        }
        }
        const bool first = f0 == 0u && (nh != 0u || (q & 1u) == 0u);
        if (first) { post(L_GCNT, OFF_F2 + ((lane * 16u + idx) * 16u + q * 2u + nh) * 16u); __builtin_amdgcn_s_setprio(0); }
        const rsrc_t rs_ol = mk_rsrc(p.out[0], f0 == 0u || os < 7u ? OUT_BYTES : 0u);
#pragma unroll
        for (int e = 0; e < 8; ++e) ST128(qv[e], rs_ol, ovoff, osoff + e * 8u * NN, AGT);
        if (f0 == 0u && !first) post(L_GCNT, OFF_F2 + ((lane * 16u + idx) * 16u + q * 2u + nh) * 16u);
        }
    } else if (b < 128u && wid == 8u) {
        __builtin_amdgcn_s_setprio(0);
        *(volatile LAS u64*)(L + L_PTR + (lane & 7u) * 8u) = (u64)p.ws[lane & 7u];
        for (u32 n = 0; n < fills / 2u;) {
            const u32 wn = n == 0u ? nb : nb2, end = n + 64u;
            const bool hole = (wn == 11u && (lane & 5u) == 5u) || lane < 4u;
            u32 step = lane & 56u, va = OFF_UF + (step * 112u + (lane & 4u ? 64u + wn * 4u : mb * 4u) + (lane & 3u)) * 128u;
            bool seen = hole;
            while (n != end) {
                while (2u * n > LW[L_FREE / 4] + 4u) { if (expired()) break; __builtin_amdgcn_s_sleep(8); }
                seen |= __builtin_amdgcn_raw_buffer_load_b32(rs_ws, seen ? OOB : va, 0, SYS) >= seq;
                u64 avail = BALLOT(seen);
                avail &= avail >> 1; avail &= avail >> 2; avail &= avail >> 4;
                avail &= 0x0101010101010101ull;
                if (avail == 0) {
                    if (!expired()) { __builtin_amdgcn_s_sleep(2); continue; }
                    avail = BALLOT(va != OOB) & 0x0101010101010101ull;
                }
                if ((avail >> lane) & 1ull) LW[L_STEPS / 4 + n + rank_in(avail)] = step;
                n += (u32)__builtin_popcountll(avail);
                LW[L_NRDY / 4] = n;
                if ((avail >> (lane & 56u)) & 1ull) {
                    seen = hole;
                    va = (++step & 7u) ? va + 112u * 128u : OOB;
                }
            }
            if (n != 64u) break;
            while (LW[L_FREE / 4] < 128u) { if (expired()) break; __builtin_amdgcn_s_sleep(8); }
            u32 a = 0;
            for (;;) {
                const u32x2 gm = __builtin_amdgcn_raw_buffer_load_b64(rs_ws, OFF_GAM + q * 128u + (lane & 7u) * 8u, 0, SYS);
                a = gm[0];
                if (BALLOT(gm[1] == seq) == ~0ull) break;
                if (expired()) { a = 0xFFFFFFFFu; break; }
                __builtin_amdgcn_s_sleep(1);
            }
            a = wave_max_u32(a);
            LW[L_SCALE / 4 + 1] = a;
            LW[L_SCALE / 4] = 1u;
            while (LW[L_GCNT / 4] + LW[L_GCNT / 4 + 1] < 8u) { if (expired()) break; __builtin_amdgcn_s_sleep(8); }
        }
    } else if (b < 128u) {
        __builtin_amdgcn_s_setprio(0);
        for (;;) {
            u32 f = 0;
            if (lane == 0) f = __hip_atomic_fetch_add((LAS u32*)(L + L_NEXT), 1u, __ATOMIC_RELAXED, __HIP_MEMORY_SCOPE_WORKGROUP);
            f = RFL(f);
            if (f >= fills) break;
            while (LW[L_NRDY / 4] <= (f >> 1)) { if (expired()) break; __builtin_amdgcn_s_sleep(1); }
            CBAR();
            const u32 step = RFL(LW[L_STEPS / 4 + (f >> 1)]) & 63u;
            while (BALLOT(LW[L_FREE / 4 + (lane & 7u)] + 2u >= f) != ~0ull && f >= 4u) { if (expired()) break; __builtin_amdgcn_s_sleep(1); }
            CBAR();
            const u32 sbase = STAGE_OFF + step * STEP_BYTES + (f & 1u) * 16384u;
            LAS char* ldst = L + (f & 3u) * SLOT_BYTES;
            auto pace = [] { asm volatile("s_nop 5" ::: "memory"); };
            if (step < 8u) {
                const rsrc_t rs_x = mk_rsrc(px, X_BYTES);
                const u32 so = mb * 262144u + step * 128u + (f & 1u) * 64u;
#pragma unroll
                for (int i = 0; i < 4; ++i) {
                    __builtin_amdgcn_raw_ptr_buffer_load_lds(rs_x, ldst + i * 4096, 16, x_at(lane), so + i * 65536u, 0, 0); pace();
                    __builtin_amdgcn_raw_ptr_buffer_load_lds(rs_x, ldst + i * 4096, 16, x_at(lane), so + i * 65536u + 15360u, 1024, 0); pace();
                    __builtin_amdgcn_raw_ptr_buffer_load_lds(rs_x, ldst + i * 4096, 16, x_at(lane), so + i * 65536u + 30720u, 2048, 0); pace();
                    __builtin_amdgcn_raw_ptr_buffer_load_lds(rs_x, ldst + i * 4096, 16, x_at(lane), so + i * 65536u + 46080u, 3072, 0); pace();
                }
            }
#pragma unroll
            for (int i = 7; i >= 0; --i) {
                if (i == 3 && step < 8u) break;
                const u32 so = sbase + (i < 4 ? mb : 16u + (f < 128u ? nb : nb2)) * PIECE + (i & 3) * 4096u;
                __builtin_amdgcn_raw_ptr_buffer_load_lds(rs_ws, ldst + i * 4096, 16, lane * 16u, so, 0, SYS); pace();
                __builtin_amdgcn_raw_ptr_buffer_load_lds(rs_ws, ldst + i * 4096, 16, lane * 16u, so, 1024, SYS); pace();
                __builtin_amdgcn_raw_ptr_buffer_load_lds(rs_ws, ldst + i * 4096, 16, lane * 16u, so, 2048, SYS); pace();
                __builtin_amdgcn_raw_ptr_buffer_load_lds(rs_ws, ldst + i * 4096, 16, lane * 16u, so, 3072, SYS); pace();
            }
            CBAR();
            VMWAIT(0);
            LW[L_FULL / 4 + (f & 3u)] = f + 1u;
        }
    } else if (b < 240u) {
        arrive(0u, 12u);
        const rsrc_t rs_src = mk_rsrc(p.ws[(8u - rel) & 7u], WS_NEED);
        auto in_unit = [&](u32 u, u32& ch, u32& r, bool& isw) -> u32 {
            if (u >= 768u) {
                ch = (u - 768u) / 14u; r = (u - 768u) % 14u; isw = true;
                const u32 os = r >> 1, rs = 28u + (r & 1u) * 2u;
                r = strip_at(os, rs);
                return E1W_OFF + ((((rel + 1u + os) & 7u) * 8u + ch) * 32u + rs) * E1_SLOT;
            }
            ch = u / 96u; r = u % 96u;
            if ((XFIRST >> ch) & 1u) r = r < 64u ? r + 32u : r - 64u;
            isw = r < 32u;
            if (!isw) r = (r - 32u + 8u * rel) & 63u;
            return !isw ? E1X_OFF + (ch * 64u + r) * E1_SLOT : ch == 0u ? E1P_OFF + ((rel * 2u + ch) * 32u + r) * E1_SLOT
                                                                         : E1W_OFF + ((rel * 8u + ch) * 32u + r) * E1_SLOT;
        };
        auto out_unit = [&](u32 o) -> u32 {
            const u32 k = o >> 4, j = o - 768u;
            if (k >= 48u) return j % 12u * 64u + (j / 48u) * 16u + 8u + j / 12u % 4u;
            return (o & 15u) * 64u + (k < 32u ? (k >> 2) * 8u + 4u : ((k >> 2) & 3u) * 16u) + (k & 3u);
        };
        auto out_at = [&](u32 t, u32 lk) -> u32 {
            const u32 x = (t >> 3) & 7u, ix = (t >> 6) & 15u;
            return ((4u * (x >> 1) + (ix & 3u)) * 256u + (t & 3u) * 64u) * NN + ((rank - lk) & 7u) * 2048u + (4u * (x & 1u) + (ix >> 2)) * 256u + ((t >> 2) & 1u) * 128u;
        };
        const u32 rows = (lane >> 3) * NN + (lane & 7u) * 16u, base = seq << 12;
        auto in_size = [](u32 h) -> u32 {
            h = ((h >> 4) & 15u) | ((h >> 8) & 0xF0u);
            return h > 127u ? E1_SLOT / 16u : I_HDR / 16u + h;
        };
        if (wid == 0u) {
            __builtin_amdgcn_s_setprio(3);
            auto ctl = [&](u32 a) -> u32 {
                u32 v;
                asm volatile("ds_read_b32 %0, %1\n s_waitcnt lgkmcnt(0)" : "=v"(v) : "v"(a) : "memory");
                return v;
            };
            auto put = [&](u32 a, u32 n) { asm volatile("ds_write_b32 %0, %1" :: "v"(a), "v"(n) : "memory"); };
            const u32 ca = L_FULL + (lane & 15u) * 4u;
            const u32 vone = lane == 0u ? 0u : OOB;
            u32 n = 0, w6;
            auto wait = [&](int word, u32 bit, bool four = false) {
                auto ok = [&](u32 w) -> bool {
                    w6 = (u32)__builtin_amdgcn_readlane((int)w, 6);
                    const u32 k = n + ((lane - n) & 7u);
                    if (four && (BALLOT(k < n + 4u && k >= 8u && (int)(w + 7u - k) < 0) & 0xFF00u) != 0u) return false;
                    return (((u32)__builtin_amdgcn_readlane((int)w, word) >> bit) & 1u) != 0u &&
                           (u32)__builtin_amdgcn_readlane((int)w, 8 + (n & 7u)) + (n < (word == 1 ? 16u : 8u) ? 15u : 7u) >= n;
                };
                if (!ok(ctl(ca))) {
                    VMWAIT(0);
                    put(L_FULL, n);
                    while (!ok(ctl(ca))) { if (expired()) break; __builtin_amdgcn_s_sleep(1); }
                }
            };
#define CLAIM(cl, so) asm volatile("v_mov_b32 %0, 1\n s_nop 4\n buffer_atomic_add %0, %1, %2, %3 offen sc0 sc1" \
                                   : "=&v"(cl) : "v"(vone), "s"(rs_ws), "s"(so) : "memory")
#define CLAIMED(d, cl) asm volatile("v_readfirstlane_b32 %0, %1" : "=s"(d) : "v"(cl) : "memory")
            for (u32 u = 32u + pj; u < N1; ++n) {
                u32 ch, r, cl; bool isw;
                const u32 so = in_unit(u, ch, r, isw);
                if (u >= 768u) put(L_P1, 1u);
                const u32 seg = u >= 768u ? 16u : n < 4u ? 24u + n : ch * 2u + !isw;
                wait(1, seg);
                LAS char* ld = L + RING + (n < 16u ? n : n & 7u) * E1_SLOT;
                u32 sz = 0;
                put(L_Q + (n & 15u) * 4u, u);
                CLAIM(cl, n < 3u ? OOB : OFF_CTR + (64u + rel) * 128u);
                static_assert(WIN <= 4, "the header, read first, has landed behind the fifth read");
#pragma unroll
                for (int i = 0; i < 9; ++i) {
                    if (i == 5) sz = in_size(RFL(ctl(RING + (n < 16u ? n : n & 7u) * E1_SLOT + I_HDR)));
                    if (i >= 6 && i * 64u >= sz) break;
                    VMWAIT(WIN - (i < WIN - 1 ? 0 : 1));
                    if (i == WIN - 1) put(L_FULL, n);
                    const int k = i > 4 ? i : (i + 4) % 5;
                    if (i < 5 || i * 64u + lane < sz)
                        __builtin_amdgcn_raw_ptr_buffer_load_lds(rs_src, ld + k * 1024, 16, lane * 16u, so + k * 1024u, 0, SYS);
                }
                u32 c;
                CLAIMED(c, cl);
                c -= (seq - 1u) * (N1 - 80u);
                u = n < 3u ? u + 16u : c + 96u;
            }
            put(L_P1, 1u);
            for (u32 got = 0, i = 0, tries = 0; got != SGM;) {
                const u32 w = ctl(ca);
                const u32 m = (u32)__builtin_amdgcn_readlane((int)w, 5) & ~got;
                if (m == 0u || (n < 24u && (BALLOT((int)w + 8 > (int)n) >> 8 & 255u) != 255u)) {
                    if (++tries < 20u) continue;
                    VMWAIT(0);
                    put(L_FULL, n);
                    if (expired()) break;
                    __builtin_amdgcn_s_sleep(1);
                    continue;
                }
                const u32 t = (u32)__builtin_ctz(m);
                got |= 1u << t;
                tries = 0;
#pragma unroll
                for (int e = 0; e < 4; ++e) {
                    VMWAIT(WIN - 1);
                    if (++i == WIN) { put(L_FULL, n); put(L_P1, 2u); }
                    __builtin_amdgcn_raw_ptr_buffer_load_lds(rs_src, L + L_SG + (t * 4u + e) * 1024u, 16, lane * 16u,
                                                             E3_OFF + (pj * 64u + t * 4u + e) * 1024u, 0, SYS);
                }
            }
            put(L_FULL, n);
            put(L_P1, 2u);
            while ((BALLOT((int)ctl(ca) + 8 > (int)n) >> 8 & 255u) != 255u) if (expired()) break;
            const char* wsrc = p.ws[(8u - rel) & 7u];
            for (u32 o = pj, cur = rel, vict = 0, m = 0; o < 960u; ++n, ++m) {
                const u32 j = o + 24u * cur - (o + 24u * cur < 960u ? 0u : 192u), t = out_unit(o < 768u ? o : j);
                u32 cl, c;
                wait(2, o < 768u ? (t >> 2) & 15u : 16u, (m & 3u) == 0u);
                const bool dyn = o >= 752u, raw = o + ((int)w6 >> 8) >= 960u - NRAW || cur != rel;
                if (o >= 768u + SFROM) put(L_P1, 3u);
                u32 vs = ctl(L_SZ + (t >> 2) * 8u + (lane & 2u) * 2u) >> ((lane & 1u) * 16u) & 0xFFFFu;
                vs = min(max(vs, R_VAR / 16u), 512u) - R_VAR / 16u;
                const u32 v0 = (u32)__builtin_amdgcn_readlane((int)vs, 0), v1 = v0 + (u32)__builtin_amdgcn_readlane((int)vs, 1),
                          v2 = v1 + (u32)__builtin_amdgcn_readlane((int)vs, 2), v3 = v2 + (u32)__builtin_amdgcn_readlane((int)vs, 3);
                const u32 own = raw ? 128u : (u32)__builtin_amdgcn_readlane((int)vs, t & 3u), sz = R_VAR / 16u + own;
                const u32 more = o >= 768u ? own : (t & 3u) == 0u ? v3 : 0u;
                const u32 lv = RING + 8u * R_VAR + (m & 4u) * 2048u + (o >= 768u ? (m & 3u) * 128u : (t & 2u ? (t & 1u ? v2 : v1) : (t & 1u ? v0 : 0u))) * 16u;
                const u32 skip = sz < 512u && o < 768u ? 1024u : 0u;
                put(L_Q + (n & 15u) * 4u, 1024u + t + skip * 2u + (raw ? 4096u : 0u) + (cur << 13) + (sz << 16) + (o < 768u ? 0u : (j - 744u) / 24u << 26));
                put(L_Q2 + (n & 15u) * 4u, lv);
                LAS char* ld = L + RING + (n & 7u) * R_VAR + skip;
                const rsrc_t rs = mk_rsrc(raw ? p.out[(8u - (o < 936u ? cur : rel)) & 7u] : wsrc, raw ? OUT_BYTES : WS_NEED);
                const u32 so = raw ? out_at(t, cur) : E2_OFF + (t >> 2) * 32768u + (t & 3u) * R_VAR + skip, pitch = raw ? 8u * NN : 1024u, vo = raw ? rows : lane * 16u;
                const u32 sv = raw ? so + 6u * pitch : E2_OFF + (t >> 2) * 32768u + 4u * R_VAR + (t & 3u) * 2048u;
                bool again = false;
                CLAIM(cl, dyn ? OFF_CTR + (72u + cur) * 128u : OOB);
#pragma unroll
                for (int i = 0; i < 6; ++i) {
                    if (i == 5 && skip != 0u) break;
                    VMWAIT(WIN - (i < WIN - 1 ? 0 : 1));
                    if (i == WIN - 1) {
                        put(L_FULL, n);
                        CLAIMED(c, cl);
                        o = !dyn ? o + 16u : 768u + min(c - base, 192u);
                        if (o == 960u) {
                            if (cur == rel) vict = w6 & 63u;
                            cur = vict & 7u;
                            vict >>= 3;
                            again = cur != 0u;
                            if (again) CLAIM(cl, OFF_CTR + (72u + cur) * 128u);
                        }
                        if (((o - 768u - SMARK) & ~48u) == 0u && o != 768u + SMARK)
                            __builtin_amdgcn_raw_buffer_store_b32((u32)now(), rs_ws, lane < 8u ? lane * 128u : OOB, OFF_RF + cur * 16u + (o - 768u - SMARK) / 4u, SYS);
                    }
                    __builtin_amdgcn_raw_ptr_buffer_load_lds(rs, ld + i * 1024, 16, vo, so + i * pitch, 0, SYS);
                }
#pragma unroll 1
                for (u32 i = 0; i * 64u < more; ++i) {
                    VMWAIT(WIN - 1);
                    if (i * 64u + lane < more) __builtin_amdgcn_raw_ptr_buffer_load_lds(rs, L + lv + i * 1024u, 16, vo, sv + i * pitch, 0, SYS);
                }
                if (again) {
                    VMWAIT(2);
                    CLAIMED(c, cl);
                    o = 768u + min(c - base, 192u);
                }
            }
            put(L_P1, 4u);
            VMWAIT(0);
            put(L_FULL, n);
            put(L_END, 1u);
        } else if (wid == 1u) {
            const u32 fa = (lane & 31u) < 20u ? OFF_F1 + rel * 256u + (lane & 31u) * 8u : OFF_FP + (rel * 16u + pj) * 128u + ((lane - 20u) & 7u) * 8u;
            for (u32 m = 0; m != HMASK;) {
                const u32x2 f = __builtin_amdgcn_raw_buffer_load_b64(rs_ws, fa, 0, SYS);
                m = (u32)BALLOT(f[0] >= seq) & HMASK;
                LW[L_PSZ / 4 + (lane & 31u)] = f[1];
                if (expired()) m = HMASK;
                LW[L_SEG / 4] = m;
                __builtin_amdgcn_s_sleep(4);
            }
            while (LW[L_P1 / 4] == 0u) { if (expired()) break; __builtin_amdgcn_s_sleep(16); }
            u32 m = 0;
            while (m != SGM) {
                m |= (u32)BALLOT(__builtin_amdgcn_raw_buffer_load_b32(rs_ws, lane < 16u && ((SGM & ~m) >> lane & 1u) ? lane * 64u : OOB,
                                                                      OFF_F3 + (rel * 16u + pj) * 1024u, SYS) == seq);
                if (expired()) m = SGM;
                LW[L_G3 / 4] = m;
                __builtin_amdgcn_s_sleep(2);
            }
            while (LW[L_P1 / 4] < 2u) { if (expired()) break; __builtin_amdgcn_s_sleep(2); }
            if (lane == 0) __builtin_amdgcn_raw_buffer_store_b32(seq, rs_src, OFF_DONE + rank * 128u + pj * 4u, 0, SYS);
            m = 0;
            while (m != 0xFFFFu) {
                const u32x4 f = LD128(rs_ws, lane < 16u ? lane * 16u : OOB, OFF_F2 + (rel * 16u + pj) * 256u, SYS);
                m = (u32)BALLOT(f[0] == seq && f[3] == seq) & 0xFFFFu;
                if (lane < 16u) *(LAS u32x2*)(L + L_SZ + pj * 128u + lane * 8u) = u32x2{f[1], f[2]};
                if (expired()) m = 0xFFFFu;
                CBAR();
                LW[L_G2 / 4] = m;
                __builtin_amdgcn_s_sleep(2);
            }
            for (u32 i = 0; i < 4u;) {
                const u32x4 f = LD128(rs_ws, lane * 16u, OFF_F2 + rel * 4096u + i * 1024u, SYS);
                *(LAS u32x2*)(L + L_SZ + i * 512u + lane * 8u) = u32x2{f[1], f[2]};
                if (BALLOT(f[0] == seq && f[3] == seq) == ~0ull || expired()) ++i; else __builtin_amdgcn_s_sleep(16);
            }
            CBAR();
            LW[L_G2 / 4] = 0x1FFFFu;
            const u32 tr = (u32)now();
            while (LW[L_P1 / 4] < 3u) { expired(); __builtin_amdgcn_s_sleep(32); }
            while (LW[L_P1 / 4] == 3u) {
                const u32x4 v = LD128(rs_ws, lane < 8u ? lane * 16u : OOB, OFF_RF + rel * 128u, SYS);
                const u32 tn = (u32)now() - tr;
                const u32 at = lane - 1u < 7u ? min(v[1] - tr, tn) + min(v[2] - tr, tn) + min(v[3] - tr, tn) : 0u;
                const u32 key = v[0] == seq ? (at + ((lane * 5u + pj) & 15u) * JIT) * 8u + lane : 0u;
                const u32 k1 = wave_max_u32(key);
                const int late = (__builtin_amdgcn_readlane((int)at, rel) * 7 - __builtin_amdgcn_readlane((int)wave_scan_u32(at), 63)) / LDIV;
                LW[L_VICT / 4] = (k1 & 7u) + (wave_max_u32(key == k1 ? 0u : key) & 7u) * 8u + ((u32)min(max(late, rel < 7u ? -LMIN : 0), 24) << 8);
                __builtin_amdgcn_s_sleep(8);
            }
        } else if (wid < 10u) {
            const rsrc_t rs_ol = mk_rsrc(p.out[0], OUT_BYTES);
            const u32 ovoff = (lane & 7u) * NN + (((lane >> 3) & 1u) * 4u + (lane >> 4)) * 16u;
            for (u32 n = wid - (wid - 7u < 2u ? 10u : 2u);; n += 8u) {
                const bool pre = (int)n < 0;
                bool more = true;
                const u32 k = n + 3u;
                while (pre && ((LW[L_SEG / 4] >> (20u + k)) & 1u) == 0u) __builtin_amdgcn_s_sleep(2);
                while (!pre) {
                    const u32 end = LW[L_END / 4];
                    if (LW[L_FULL / 4] > n) break;
                    if (end != 0u) { more = false; break; }
                    expired();
                    __builtin_amdgcn_s_sleep(2);
                }
                if (!more) break;
                CBAR();
                const u32 id = pre ? k * 16u + pj + (min(max(RFL(LW[L_PSZ / 4 + 20u + k]), 321u), E1_SLOT / 16u) << 16)
                                   : RFL(LW[L_Q / 4 + (n & 15u)]);
                const u32 sl = RING + (id & 1024u ? (n & 7u) * R_VAR : (n & ((int)n < 16 ? 15u : 7u)) * E1_SLOT);
                if ((id & 1024u) == 0u) {
                u32 ch, r; bool isw;
                const u32 so = in_unit(id & 1023u, ch, r, isw);
                if (pre) {
#pragma unroll 1
                    for (u32 at = 0; at < (id >> 16) * 16u; at += 1024u)
                        if (at + lane * 16u < (id >> 16) * 16u)
                            __builtin_amdgcn_raw_ptr_buffer_load_lds(rs_ws, L + sl + at, 16, lane * 16u, so + at, 0, SYS);
                    VMWAIT(0);
                }
                const u32 hd = sl + I_HDR;
                u32x4 tab = LDSV(hd);
                const u32x4 fw = LDSV(hd + I_FLD + lane * 16u);
                const u32 h0 = RFL(tab[0]), sz = in_size(h0), top = ((h0 >> 20) & 15u) | ((h0 >> 24) & 0xF0u), cnt = (RFL(tab[1]) >> 4) & 15u;
                u32x4 o[8];
                if (sz == E1_SLOT / 16u) {
#pragma unroll
                    for (int e = 0; e < 8; ++e) o[e] = LDSV(sl + (e >> 2) * (I_HDR + 128u) + (e & 3) * 1024u + lane * 16u);
                } else {
                    u32x4 mq[4];
                    u32 ex[16], es[8], at[8], d[8][3], ne = 0;
#pragma unroll
                    for (int g = 0; g < 8; ++g) {
                        es[g] = ((fw[g >> 1] & (fw[g >> 1] << 1)) << (g & 1 ? 0 : 2)) & 0x88888888u;
                        at[g] = ne;
                        ne += (u32)__builtin_popcount(es[g]);
                    }
                    const u32 v0 = (I_HDR + I_VAR) / 8u + cnt;
                    const u32 a0 = unary_list<2>(lane, sl + v0 * 8u, (max(sz * 2u, v0) - v0) * 8u, ne, L_PACK + wid * 2048u);
                    CBAR();
#pragma unroll
                    for (int g = 0; g < 8; ++g) { at[g] += a0; LIST8(d[g], at[g]); }
#pragma unroll
                    for (int e = 0; e < 4; ++e) mq[e] = LDSV(sl + e * 1024u + lane * 16u);
                    __builtin_amdgcn_sched_barrier(0);
                    tab = (tab & 0x0F0F0F0Fu) ^ u32x4{TFLIP0, TFLIP1, TFLIP0, TFLIP1};
#pragma unroll
                    for (int g = 0; g < 8; ++g) {
                        u32 c0, c1;
                        deal8(d[g], at[g], es[g], c0, c1);
                        ex[2 * g] = lut16(tab, ((fw[g >> 1] >> (2 * (g & 1))) & 0x03030303u) + c0);
                        ex[2 * g + 1] = lut16(tab, ((fw[g >> 1] >> (2 * (g & 1) + 4)) & 0x03030303u) + c1);
                    }
#pragma unroll 1
                    for (u32 k = 0; k < cnt; ++k) {
                        const u32 at = LW[(hd + I_VAR) / 4 + 2u * k] - lane * 16u, val = LW[(hd + I_VAR) / 4 + 1u + 2u * k];
#pragma unroll
                        for (int j = 0; j < 16; ++j) { ex[j] = at == (u32)j ? val : ex[j]; if ((j & 3) == 3) __builtin_amdgcn_sched_barrier(0); }
                    }
#pragma unroll
                    for (int j = 0; j < 16; ++j) {
                        const u32 e = top * 0x01010101u - ex[j], m = mq[j >> 2][j & 3];
                        const u32 r0 = PERM(e, m, 0x05010400u), r1 = PERM(e, m, 0x07030602u);
                        o[j >> 1][(2 * j) & 3] = __builtin_amdgcn_alignbit(r0, r0, 1);
                        o[j >> 1][(2 * j + 1) & 3] = __builtin_amdgcn_alignbit(r1, r1, 1);
                    }
                }
                CBAR();
                LW[L_FREE / 4 + (n & 7u)] = n + 1u;
                const u32 flag = put_unit(o, isw, rel, ch, r);
                VMWAIT(0);
                put_flag(flag, seq);
                continue;
                }
                const u32 sz = (id >> 16) & 1023u, lk = (id >> 13) & 7u, osoff = out_at(id, lk), lv = RFL(LW[L_Q2 / 4 + (n & 15u)]);
                u32x4 qv[8];
                if (sz >= 512u) {
#pragma unroll
                    for (int e = 0; e < 8; ++e) qv[e] = LDSV((e < 6 ? sl : lv - R_VAR) + e * 1024u + lane * 16u);
                } else {
                    u32x4 nib[4];
#pragma unroll
                    for (int e = 0; e < 4; ++e) nib[e] = LDSV(sl + R_NIB + e * 1024u + lane * 16u);
                    const u32x4 pl = LDSV(sl + R_BIT + lane * 16u);
                    const u32x4 sg = LDSV((id & 2048u ? L_SG + (id & 63u) * 1024u : sl) + lane * 16u);
                    u32 cnt = 0, at[16], d[16][3];
#pragma unroll
                    for (int o = 0; o < 16; ++o) {
                        at[o] = cnt;
                        cnt += (u32)__builtin_popcount(nib[o >> 2][o & 3] & (pl[o >> 2] << (o & 3)) & 0x88888888u);
                    }
                    const u32 a0 = unary_list<3>(lane, lv, (sz - R_VAR / 16u) * 16u, cnt, L_PACK + (wid - 2u) * 3072u);
                    CBAR();
#pragma unroll
                    for (int o = 0; o < 16; ++o) { at[o] += a0; LIST8(d[o], at[o]); }
                    __builtin_amdgcn_sched_barrier(0);
#pragma unroll
                    for (int o = 0; o < 16; ++o) {
                        const u32 nv = nib[o >> 2][o & 3], pb = pl[o >> 2] << (o & 3), sv = sg[o >> 2] << (6 - 2 * (o & 3));
                        u32 c0, c1;
                        deal8(d[o], at[o], nv & pb & 0x88888888u, c0, c1);
                        const u32 m0 = (c0 << 3) + (bfi(0x0F0F0F0Fu, nv, pb << 1) & 0x1F1F1F1Fu);
                        const u32 m1 = (c1 << 3) + (bfi(0x0F0F0F0Fu, nv >> 4, pb >> 3) & 0x1F1F1F1Fu);
                        qv[o >> 1][(2 * o) & 3] = __builtin_amdgcn_bitop3_b32(byte_signs(sv << 1), (0x80808080u - m0) ^ 0x80808080u, m0, 0xCA);
                        qv[o >> 1][(2 * o + 1) & 3] = __builtin_amdgcn_bitop3_b32(byte_signs(sv), (0x80808080u - m1) ^ 0x80808080u, m1, 0xCA);
                    }
                }
                CBAR();
                LW[L_FREE / 4 + (n & 7u)] = n + 1u;
                if (LW[L_BAD / 4] != 0u) qv[0] = u32x4{0x80808080u, 0x80808080u, 0x80808080u, 0x80808080u};
#pragma unroll
                for (int e = 0; e < 8; ++e) ST128(qv[e], rs_ol, id & 4096u ? rows : ovoff, osoff + e * 8u * NN, AGT);
                const u32 cls = (id >> 26) & 7u, to = (cls - lk) & 7u;
                if (cls != 0u && to - 1u < 6u) {
                    const rsrc_t rs_to = mk_rsrc(p.ws[to], WS_NEED);
                    u32 old = 0;
                    VMWAIT(0);
                    if (lane == 0) old = 1u + __hip_atomic_fetch_add((u32*)(wsl + OFF_CTR + (96u + lk * 8u + cls) * 128u), 1u, __ATOMIC_RELAXED, __HIP_MEMORY_SCOPE_AGENT);
                    if (old == seq * 24u) __builtin_amdgcn_raw_buffer_store_b32(seq, rs_to, OFF_RF + to * 128u + cls * 16u, 0, SYS);
                }
            }
        }
    } else {
        arrive(0u, 12u);
        if (b == AUX_WG && wid == 2u) {
            u32 a = 0;
            u64* gp = (u64*)(p.ws[lane & 7u] + OFF_GAM + (lane >> 3) * 128u + rank * 8u);
            asm volatile("" : "+v"(gp));
            __builtin_amdgcn_raw_buffer_store_b32(seq << 12, rs_ws, lane < 8u ? OFF_CTR + (72u + lane) * 128u : OOB, 0, SYS);
            VMWAIT(0);
            for (;;) {
                const u32x4 g0 = LD128(rs_ws, OFF_AMX + lane * 32u, 0, AGT);
                const u32x4 g1 = LD128(rs_ws, OFF_AMX + lane * 32u + 16u, 0, AGT);
                a = max(max(g0[0], g0[2]), max(g1[0], g1[2]));
                const bool ok = g0[1] == seq && g0[3] == seq && g1[1] == seq && g1[3] == seq;
                if (BALLOT(ok) == ~0ull) break;
                if (expired()) { a = 0xFFFFFFFFu; break; }
            }
            a = wave_max_u32(a);
            __hip_atomic_store((GAS u64*)gp, ((u64)seq << 32) | a, __ATOMIC_RELAXED, __HIP_MEMORY_SCOPE_SYSTEM);
            if (lane == 0) __builtin_amdgcn_raw_buffer_store_b32(seq, rs_ws, OFF_SEQ, 0, AGT);
            bool give_up = false;
            for (;;) {
                const u32x4 f = LD128(rs_ws, OFF_DONE + lane * 16u, 0, SYS);
                const bool ok = (lane >> 3) == rank || (lane & 4u) != 0u || (f[0] >= seq && f[1] >= seq && f[2] >= seq && f[3] >= seq);
                if (BALLOT(ok) == ~0ull) break;
                if (expired()) { give_up = true; break; }
                __builtin_amdgcn_s_sleep(8);
            }
            if (give_up || __builtin_amdgcn_raw_buffer_load_b32(rs_ws, OFF_TMO, 0, AGT) == seq)
                if (lane == 0) p.out[0][rank * 2048u] = (char)-128;
        }
    }
}

extern "C" void kernel_launch(int rank, int nranks, ncclComm_t comm, hipStream_t stream,
                              void* const* d_in, const int* in_sizes, int n_in,
                              void* d_out, int out_size, const int* in_dtypes, int out_dtype,
                              void* const* d_out_all, void* const* d_ws_all, size_t ws_bytes) {
    if (nranks != NR || (unsigned)rank >= (unsigned)NR || n_in < 2 || ws_bytes < 3ull * WS_NEED ||
        in_sizes[0] != (int)(MM * KSH) || in_sizes[1] != (int)(KSH * NN) || out_size != (int)OUT_BYTES) {
        fprintf(stderr, "kernel_launch: unexpected problem shape\n");
        return;
    }
    static bool init = false;
    if (!init) {
        HIP_CHECK(hipFuncSetAttribute((const void*)gemm_ar_q8, hipFuncAttributeMaxDynamicSharedMemorySize, LDS_BYTES));
        init = true;
    }
    Params p;
    p.x = (const char*)d_in[0];
    p.w = (const char*)d_in[1];
    for (int r = 0; r < NR; ++r) p.ws[r] = (char*)d_ws_all[(rank + r) % NR];
    for (int r = 0; r < NR; ++r) p.out[r] = (char*)d_out_all[(rank + r) % NR];
    p.rank = rank;
    hipLaunchKernelGGL(gemm_ar_q8, dim3(256), dim3(768), LDS_BYTES, stream, p);
    HIP_CHECK(hipGetLastError());
}
```
